```python
import numpy as np

kernel_name = "gpu_a2a_gemm_m8192_k8192_n8192_bf16_relu"


ABSMAX_RTOL = 0.008

COMPUTE_DTYPE = "bfloat16"
EPILOGUE = "relu"
IN_QUANT = None
OUT_QUANT = None
IN_DTYPES = {"x": "bfloat16", "w_mat": "bfloat16"}
WORLD_SIZE = 8

def _to_bf16(x):
    u = np.ascontiguousarray(x, dtype=np.float32).view(np.uint32)
    return ((u + 0x7FFF + ((u >> 16) & 1)) & 0xFFFF0000).view(np.float32)

def setup_inputs():

    x = _to_bf16(np.random.randn(8192, 8192).astype(np.float32))
    w_mat = _to_bf16((np.random.randn(8192, 8192) / np.sqrt(8192)).astype(np.float32))
    return {"x": x, "w_mat": w_mat}

def _epilogue(y):
    return np.maximum(y, 0.0).astype(np.float32)

def reference(x, w_mat):
    y = (x @ w_mat).astype(np.float32)
    return _epilogue(y)
```

```cpp
#include <hip/hip_runtime.h>
#include <rccl/rccl.h>
#include <cstdio>

#define HIP_CHECK(expr) do { hipError_t _e = (expr); if (_e != hipSuccess) { \
    fprintf(stderr, "HIP error %s at %s:%d\n", hipGetErrorString(_e), __FILE__, __LINE__); \
    return; } } while (0)

constexpr unsigned SLOT = 16u << 20;
constexpr unsigned XCH = 32u << 20;
constexpr unsigned CTRL = 64u << 20;
constexpr unsigned BAL = 512u << 10;
constexpr unsigned XFL = 576u << 10;
constexpr unsigned TMO_OFF = CTRL + (768u << 10), CNT_OFF = CTRL + (1u << 20), DBG_OFF = CTRL + (2u << 20);
constexpr unsigned FS = 1, ALT = (72u << 20) - 128u;
constexpr int LDS_BYTES = 163840;
constexpr int PNUM = 7, POFF = 0;
typedef unsigned u32x4 __attribute__((ext_vector_type(4)));
typedef unsigned u32x16 __attribute__((ext_vector_type(16)));
struct Args { unsigned rX[4], rW[4]; char* myws; char* out; unsigned rank, pad[3]; char* p[8]; };

__device__ __forceinline__ unsigned peer_of(unsigned j) { return (j + ((j * 9363u) >> 16)) & 7u; }
__device__ __forceinline__ unsigned slice_of(unsigned j) { return (((j * 9363u) >> 16) + 0u * peer_of(j)) & 15u; }
#define LD4(M, I, R, O, V0, V1, V2, V3, PRE, ...) \
  asm volatile("s_or_b32 m0, %[m], " #I "\n\t" PRE "\n\tbuffer_load_dwordx4 %[v0], %[r], %[o] offen lds\n\tbuffer_load_dwordx4 %[v1], %[r], %[o] offen offset:1024 lds\n\t" \
               "buffer_load_dwordx4 %[v2], %[r], %[o] offen offset:2048 lds\n\tbuffer_load_dwordx4 %[v3], %[r], %[o] offen offset:3072 lds" \
               : __VA_ARGS__ : [m] "s"(M), [r] "s"(R), [o] "s"(O), [v0] "v"(V0), [v1] "v"(V1), [v2] "v"(V2), [v3] "v"(V3), [y] "s"(myws), [c] "s"(cnto), [k] "s"(ka) : "scc", "memory"); \
  __builtin_amdgcn_sched_barrier(0)

__global__ void __launch_bounds__(256, 1)
a2a_gemm(Args args) {
  extern __shared__ __attribute__((aligned(16))) char lds[];
  u32x4 rX, rW;
  char *myws, *out;
  unsigned urank;
  asm volatile("s_load_dwordx4 %0, %5, 0x0\n\ts_load_dwordx4 %1, %5, 0x10\n\ts_load_dwordx2 %2, %5, 0x20\n\ts_load_dwordx2 %3, %5, 0x28\n\ts_load_dword %4, %5, 0x30"
               : "=&s"(rX), "=&s"(rW), "=&s"(myws), "=&s"(out), "=&s"(urank) : "s"(__builtin_amdgcn_kernarg_segment_ptr()));
  __builtin_amdgcn_sched_barrier(0);
  const unsigned tid = threadIdx.x;
  unsigned l = tid & 63u;
  const unsigned w = __builtin_amdgcn_readfirstlane(tid >> 6), g = w >> 1, wm = w & 1u;
  unsigned bid = blockIdx.x;
  const unsigned xcd = bid & 7u, kh = xcd & 1u, idx = bid >> 3, mt = idx & 3u, nt = (xcd >> 1) * 8u + (idx >> 2);
  const unsigned r = l >> 3, cpos = l & 7u, rg = wm + g * 4u, hh = ((r >> 1) & 1u) | (wm << 1);
  const unsigned c0 = 0 ? (nt >> 1) * 1024u + (nt & 1u) * 128u : nt * 512u, gs = 0 == 1 ? 512u : 256u, ks = 128u << 0;
  const unsigned wrow = (rg * 8u + r) * 16384u + (cpos ^ hh) * 16u + c0 + 3072u;
  unsigned ws0 = wrow + (kh ^ 1u) * ks, ws1 = wrow + kh * ks - 1024u, ws2 = ws0 + gs - 2048u, ws3 = ws1 + gs - 2048u;
  unsigned ws4 = ws0 + 0x40000u, ws5 = ws1 + 0x40000u, ws6 = ws2 + 0x40000u, ws7 = ws3 + 0x40000u;
  unsigned av0 = r * 2048u + (cpos ^ (r >> 1)) * 16u, av1 = (av0 ^ 64u) + 15360u, av2 = av0 + 30720u, av3 = av1 + 30720u;
  unsigned av4 = av0 + 65536u, av5 = av1 + 65536u, av6 = av2 + 65536u, av7 = av3 + 65536u;
  unsigned wa = w * 8192u, ww = rg * 4096u, rowo = (mt * 4u + w) << 17, fx = ((mt & 1u) << 1) ^ ((xcd & 0u) << 1), s0 = kh ^ fx, cnto = CNT_OFF + bid * 128u;
  unsigned wof = s0 << 20, aof = rowo + (s0 << 7);
  asm volatile("s_waitcnt lgkmcnt(0)" : "+v"(ws0), "+v"(ws1), "+v"(ws2), "+v"(ws3), "+v"(ws4), "+v"(ws5), "+v"(ws6), "+v"(ws7),
               "+v"(av0), "+v"(av1), "+v"(av2), "+v"(av3), "+v"(av4), "+v"(av5), "+v"(av6), "+v"(av7),
               "+s"(wa), "+s"(ww), "+s"(wof), "+s"(aof), "+s"(cnto), "+s"(rX), "+s"(rW), "+s"(myws), "+s"(out), "+s"(urank));
  __builtin_amdgcn_sched_barrier(0);
  wof += urank << 24; aof += urank << 21;
  unsigned call;
  u32x16 pt;
  const auto ka = __builtin_amdgcn_kernarg_segment_ptr();
  LD4(ww, 0x10000, rW, wof, ws0, ws1, ws2, ws3, "s_load_dword %0, %[y], %[c]", "=&s"(call), "+v"(l), "+s"(bid));
  const unsigned q = min(l, 6u) * 256u + bid, qb = q & 15u, qu = q >> 4;
  const unsigned qj = qu ^ (qu < 104u - 16u ? ((qb >> 2) & 1u) << 1 : 0u);
  unsigned qi = peer_of(qj), qs = slice_of(qj);
  unsigned tabE = (((q & ~1u) * 58u + 0u) >> 11) + 1u;
  LD4(ww, 0x12000, rW, wof, ws4, ws5, ws6, ws7, "s_load_dwordx16 %0, %[k], 0x40", "=&{s[48:63]}"(pt), "+v"(l), "+v"(qi), "+v"(qs), "+v"(tabE));
  const unsigned qlow = (qb << 17) + (w << 15) + (qs << 7);
  unsigned tabS = (((urank - 1u - qi) & 7u) << 21) + qlow, tabD = (urank << 21) + qlow + 2u * qi + 0x80000002u + ((qs & 7u) == 3u ? ALT : 0u);
  unsigned tabF = CTRL + ((qb * 128u + urank * 16u + qs) << (7u + FS)) + w * 4u + 2u * qi + 2u;
  tabE = l > 6u ? 255u : tabE;
  const unsigned g4 = l >> 4, lq = (l & 15u) >> 2, h = ((lq >> 1) & 1u) | ((g4 & 1u) << 1);
  const unsigned bb = 0x10000u + g * 2048u + g4 * 4096u + lq * 128u + (l & 1u) * 8u + (l & 2u) * 32u;
  unsigned b0 = bb + (0u ^ h) * 16u, b1 = bb + (2u ^ h) * 16u, b2 = bb + (1u ^ h) * 16u, b3 = bb + (3u ^ h) * 16u;
  LD4(wa, 0, rX, aof, av0, av1, av2, av3, "s_nop 0", "+v"(l), "+v"(tabS), "+v"(tabD), "+v"(tabF), "+v"(tabE), "+v"(b0), "+v"(b1), "+v"(b2), "+v"(b3), "+s"(bid));
  const unsigned v = 2u * l + 4u + kh, vp = l < 6u ? v + 128u : v, t = vp < 128u ? vp : 255u - vp, j = (t < 104u ? t ^ ((mt & 1u) << 1) : t) - 16u;
  unsigned own = urank * 16u + (v ^ fx), peer = ((urank + 1u + peer_of(j)) & 7u) * 16u + slice_of(j);
  unsigned fb = (mt * 4u + w) << (14u + FS), xfo = XFL + (bid * 4u + w) * 128u - fb, xfs = XFL + ((bid ^ 1u) * 4u + w) * 128u - fb;
  unsigned fo = CTRL + fb, oo = (mt * 256u + wm * 128u) * 16384u + c0 + g * gs + kh * ks;
  unsigned xo = XCH + (bid >> 1) * 131072u + w * 16384u, xs = kh * 65536u, bal = BAL + (xcd >> 1) * 4096u - fb;
  unsigned bas = w ? 0x80000000u : bal + idx * 16u + kh * 8u - 0u;
  bal += 0u * 16u;
  LD4(wa, 0x1000, rX, aof, av4, av5, av6, av7, "s_nop 0", "+v"(l), "+v"(own), "+v"(peer), "+s"(xfo), "+s"(xfs), "+s"(fo), "+s"(oo), "+s"(xo), "+s"(xs), "+s"(bal), "+s"(bas));
  const unsigned sa = l < 6u ? own : peer;
  unsigned tabW = sa << 20, tabA = ((sa >> 4) << 21) + ((sa & 15u) << 7) + rowo + (l >= 6u && (sa & 7u) == 3u ? ALT : 0u), tabT = peer << 20;
  const unsigned pd = (l - 8u) / 8u, pk = l - 8u - pd * 8u, pi = pk < 2u ? pk : 2u * pk - 2u + (mt >> 1);
  const unsigned pv = 2u * (60u - 2u - 3u + pd + pi) + kh, pj = (pv < 128u ? pv : 255u - pv) - 16u;
  const unsigned pf = ((((urank + 1u + peer_of(pj)) & 7u) * 16u + slice_of(pj)) << 20) + ((mt & 1u) << 18) + (pk < 2u && mt >= 2u ? (kh ^ 1u) * 256u : 128u)
                      + ((255u >> pk) & 1u ? (~nt & 1u) * 1024u : 512u);
  tabD = l >= 8u && pd <= 2u * 3u && pi < 9u + 2u ? pf : tabD;
  unsigned a0 = wm * 16384u + (l & 15u) * 128u + ((l >> 4) ^ ((l & 15u) >> 1)) * 16u, a1 = a0 ^ 64u;
  unsigned foff = (l & 3u) * 4u, lane = l << 14;
  asm volatile(
      "s_mov_b64 s[40:41], s[36:37]\n"
      "s_mov_b64 s[42:43], s[38:39]\n"
      "s_xor_b32 s64, %34, 0x100\n"
      "s_or_b32 m0, %41, 0x8000\n"
      "s_xor_b32 s90, %33, 0x200000\n"
      "buffer_load_dwordx4 v14, s[40:43], s64 offen offset:0 sc0 sc1 lds\n"
      "v_accvgpr_write_b32 a252, 0\n"
      "v_accvgpr_write_b32 a253, 0\n"
      "v_accvgpr_write_b32 a254, 0\n"
      "buffer_load_dwordx4 v15, s[40:43], s64 offen offset:1024 sc0 sc1 lds\n"
      "v_accvgpr_write_b32 a255, 0\n"
      "s_or_b32 s68, %42, 0x10000\n"
      "s_mov_b32 s67, %41\n"
      "buffer_load_dwordx4 v16, s[40:43], s64 offen offset:2048 sc0 sc1 lds\n"
      "v_mfma_f32_32x32x16_bf16 v[128:143], a[252:255], a[252:255], 0\n"
      "s_mov_b32 s69, 0x8000\n"
      "s_mov_b32 s70, 0x30000\n"
      "buffer_load_dwordx4 v17, s[40:43], s64 offen offset:3072 sc0 sc1 lds\n"
      "v_mfma_f32_32x32x16_bf16 v[144:159], a[252:255], a[252:255], 0\n"
      "s_mov_b32 s71, -1\n"
      "s_mov_b32 s75, 0\n"
      "s_or_b32 m0, %41, 0x9000\n"
      "v_mfma_f32_32x32x16_bf16 v[160:175], a[252:255], a[252:255], 0\n"
      "buffer_load_dwordx4 v18, s[40:43], s64 offen offset:0 sc0 sc1 lds\n"
      "s_brev_b32 s33, 1\n"
      "v_readlane_b32 s73, v27, 0\n"
      "v_mfma_f32_32x32x16_bf16 v[176:191], a[252:255], a[252:255], 0\n"
      "buffer_load_dwordx4 v19, s[40:43], s64 offen offset:1024 sc0 sc1 lds\n"
      "v_accvgpr_write_b32 a104, v29\n"
      "v_mov_b32 v29, %44\n"
      "buffer_load_dwordx4 v20, s[40:43], s64 offen offset:2048 sc0 sc1 lds\n"
      "v_mfma_f32_32x32x16_bf16 v[192:207], a[252:255], a[252:255], 0\n"
      "s_mov_b32 s30, 0xb0000\n"
      "buffer_load_dwordx4 v21, s[40:43], s64 offen offset:3072 sc0 sc1 lds\n"
      "s_mov_b32 s31, 0x20000\n"
      "v_mfma_f32_32x32x16_bf16 v[208:223], a[252:255], a[252:255], 0\n"
      "s_or_b32 m0, %42, 0x18000\n"
      "s_mov_b32 s77, 0\n"
      "buffer_load_dwordx4 v6, %36, s90 offen offset:0 lds\n"
      "v_mfma_f32_32x32x16_bf16 v[224:239], a[252:255], a[252:255], 0\n"
      "v_mfma_f32_32x32x16_bf16 v[240:255], a[252:255], a[252:255], 0\n"
      "buffer_load_dwordx4 v7, %36, s90 offen offset:1024 lds\n"
      "s_xor_b32 s65, %33, 0x400000\n"
      "s_waitcnt lgkmcnt(0)\n"
      "s_add_u32 %31, %31, 1\n"
      "s_and_b32 s35, %31, 1\n"
      "s_lshl_b32 s35, s35, 24\n"
      "s_add_u32 s24, s48, s35\n"
      "s_addc_u32 s25, s49, 0\n"
      "s_bitset1_b32 s35, 31\n"
      "v_readlane_b32 s86, v25, s77\n"
      "s_and_b32 m0, s86, 14\n"
      "s_add_u32 s86, s86, s35\n"
      "s_movrels_b64 s[82:83], s[48:49]\n"
      "s_add_u32 s84, s86, s82\n"
      "s_addc_u32 s85, s83, 0\n"
      "s_mov_b64 s[80:81], s[82:83]\n"
      "v_mov_b32 v31, %31\n"
      "v_accvgpr_write_b32 a115, %31\n"
      "s_add_u32 s28, s48, %32\n"
      "s_addc_u32 s29, s49, 0\n"
      "s_call_b64 s[46:47], 30f\n"
      "s_cmp_lg_u32 %40, 0\n"
      "s_cbranch_scc1 28f\n"
      "s_memrealtime s[86:87]\n"
      "s_mov_b64 exec, 1\n"
      "s_waitcnt lgkmcnt(0)\n"
      "v_accvgpr_write_b32 a114, s86\n"
      "buffer_store_dwordx2 a[114:115], off, s[28:31], %35 sc0 sc1\n"
      "s_mov_b64 exec, -1\n"
      "28:\n"
      "s_mov_b32 s74, 46\n"
      "s_min_u32 s72, s73, s74\n"
      "s_call_b64 s[44:45], 27f\n"
      "v_accvgpr_write_b32 a105, v29\n"
      "v_lshrrev_b32 v29, 11, v30\n"
      "buffer_load_dwordx2 a[106:107], v29, s[28:31], %49 offen sc0 sc1\n"
      "v_accvgpr_read_b32 v29, a105\n"
      "s_add_u32 s74, s71, 1\n"
      "s_min_u32 s72, s73, s74\n"
      "s_call_b64 s[44:45], 27f\n"
      "v_accvgpr_write_b32 a105, v29\n"
      "v_accvgpr_read_b32 v29, a107\n"
      "s_lshr_b32 s91, %48, 15\n"
      "v_cmp_gt_u32 vcc, s75, v29\n"
      "v_accvgpr_read_b32 v29, a106\n"
      "s_cbranch_vccnz 31f\n"
      "28:\n"
      "s_sub_u32 s90, 1, s91\n"
      "v_sub_u32_dpp v29, v29, v29 quad_perm:[1,0,3,2] row_mask:0xf bank_mask:0xf\n"
      "s_add_u32 s22, s71, 10\n"
      "s_nop 0\n"
      "v_add_u32_dpp v29, v29, v29 row_ror:2 row_mask:0xf bank_mask:0xf\n"
      "s_nop 1\n"
      "v_add_u32_dpp v29, v29, v29 row_ror:4 row_mask:0xf bank_mask:0xf\n"
      "s_nop 1\n"
      "v_add_u32_dpp v29, v29, v29 row_ror:8 row_mask:0xf bank_mask:0xf\n"
      "s_nop 0\n"
      "v_readlane_b32 s86, v29, 0\n"
      "v_readlane_b32 s87, v29, 16\n"
      "v_readlane_b32 s88, v29, 32\n"
      "v_readlane_b32 s89, v29, 48\n"
      "v_accvgpr_read_b32 v29, a105\n"
      "s_add_u32 s86, s86, s87\n"
      "s_add_u32 s88, s88, s89\n"
      "s_add_u32 s86, s86, s88\n"
      "s_mul_i32 s86, s86, 420\n"
      "s_add_u32 s86, s86, 0x33334\n"
      "s_ashr_i32 s86, s86, 20\n"
      "s_max_i32 s86, s86, -6\n"
      "s_min_i32 s86, s86, 6\n"
      "s_mul_i32 s25, s86, s90\n"
      "s_ashr_i32 s86, s25, 1\n"
      "s_add_u32 s74, s86, 58\n"
      "s_mul_i32 s86, s86, 8\n"
      "s_add_u32 s24, s86, 31\n"
      "s_sub_u32 s22, s22, s74\n"
      "s_add_u32 s22, s22, s24\n"
      "s_min_u32 s72, s73, s74\n"
      "s_branch 57f\n"
      "30:\n"
      "s_mov_b64 s[44:45], s[46:47]\n"
      "v_readlane_b32 s87, v24, 0\n"
      "s_xor_b32 s90, s65, 0x600000\n"
      "s_mov_b32 s71, 1\n"
      "s_mov_b32 s74, 40\n"
      "s_min_u32 s72, s73, s74\n"
      "s_memrealtime s[0:1]\n"
      "s_memtime s[2:3]\n"
      "s_mov_b32 s19, 0\n"
      "s_mov_b32 s92, 0\n"
      "s_waitcnt lgkmcnt(0)\n"
      "s_mul_i32 s1, s0, 14\n"
      "s_sub_u32 s46, s2, s1\n"
      "s_or_b32 m0, %42, 0x18000\n"
      "s_waitcnt vmcnt(10)\n"
      "s_barrier\n"
      "ds_read_b64_tr_b16 a[128:129], v0 offset:0\n"
      "ds_read_b64_tr_b16 a[130:131], v0 offset:512\n"
      "ds_read_b64_tr_b16 a[132:133], v0 offset:16384\n"
      "ds_read_b64_tr_b16 a[134:135], v0 offset:16896\n"
      "ds_read_b128 a[192:195], v4 offset:0\n"
      "ds_read_b128 a[196:199], v5 offset:0\n"
      "ds_read_b128 a[200:203], v4 offset:2048\n"
      "ds_read_b128 a[204:207], v5 offset:2048\n"
      "ds_read_b64_tr_b16 a[136:137], v1 offset:0\n"
      "ds_read_b64_tr_b16 a[138:139], v1 offset:512\n"
      "ds_read_b64_tr_b16 a[140:141], v1 offset:16384\n"
      "ds_read_b64_tr_b16 a[142:143], v1 offset:16896\n"
      "ds_read_b128 a[208:211], v4 offset:4096\n"
      "ds_read_b128 a[212:215], v5 offset:4096\n"
      "ds_read_b64_tr_b16 a[144:145], v2 offset:0\n"
      "ds_read_b64_tr_b16 a[146:147], v2 offset:512\n"
      "ds_read_b64_tr_b16 a[148:149], v2 offset:16384\n"
      "ds_read_b64_tr_b16 a[150:151], v2 offset:16896\n"
      "s_waitcnt lgkmcnt(13)\n"
      "v_mfma_f32_16x16x32_bf16 a[0:3], a[128:131], a[192:195], 0\n"
      "ds_read_b128 a[216:219], v4 offset:6144\n"
      "s_waitcnt lgkmcnt(13)\n"
      "v_mfma_f32_16x16x32_bf16 a[0:3], a[132:135], a[196:199], a[0:3]\n"
      "buffer_load_dwordx4 v8, %36, s90 offen offset:2048 lds\n"
      "s_waitcnt lgkmcnt(11)\n"
      "v_mfma_f32_16x16x32_bf16 a[4:7], a[132:135], a[204:207], 0\n"
      "ds_read_b128 a[220:223], v5 offset:6144\n"
      "s_xor_b32 s64, %34, 0x300\n"
      "v_mfma_f32_16x16x32_bf16 a[4:7], a[128:131], a[200:203], a[4:7]\n"
      "ds_read_b64_tr_b16 a[152:153], v3 offset:0\n"
      "s_waitcnt lgkmcnt(11)\n"
      "v_mfma_f32_16x16x32_bf16 v[32:35], a[136:139], a[192:195], 0\n"
      "buffer_load_dwordx4 v9, %36, s90 offen offset:3072 lds\n"
      "s_waitcnt lgkmcnt(9)\n"
      "v_mfma_f32_16x16x32_bf16 v[32:35], a[140:143], a[196:199], v[32:35]\n"
      "s_or_b32 m0, %42, 0x1a000\n"
      "ds_read_b64_tr_b16 a[154:155], v3 offset:512\n"
      "v_mfma_f32_16x16x32_bf16 v[36:39], a[140:143], a[204:207], 0\n"
      "ds_read_b64_tr_b16 a[156:157], v3 offset:16384\n"
      "v_mfma_f32_16x16x32_bf16 v[36:39], a[136:139], a[200:203], v[36:39]\n"
      "buffer_load_dwordx4 v10, %36, s90 offen offset:0 lds\n"
      "s_waitcnt lgkmcnt(10)\n"
      "v_mfma_f32_16x16x32_bf16 a[8:11], a[128:131], a[208:211], 0\n"
      "ds_read_b64_tr_b16 a[158:159], v3 offset:16896\n"
      "s_waitcnt lgkmcnt(10)\n"
      "v_mfma_f32_16x16x32_bf16 a[8:11], a[132:135], a[212:215], a[8:11]\n"
      "ds_read_b128 a[224:227], v4 offset:8192\n"
      "v_mfma_f32_16x16x32_bf16 v[40:43], a[140:143], a[212:215], 0\n"
      "ds_read_b128 a[228:231], v5 offset:8192\n"
      "v_mfma_f32_16x16x32_bf16 v[40:43], a[136:139], a[208:211], v[40:43]\n"
      "buffer_load_dwordx4 v11, %36, s90 offen offset:1024 lds\n"
      "s_waitcnt lgkmcnt(10)\n"
      "v_mfma_f32_16x16x32_bf16 v[64:67], a[144:147], a[192:195], 0\n"
      "ds_read_b128 a[232:235], v4 offset:10240\n"
      "s_waitcnt lgkmcnt(9)\n"
      "v_mfma_f32_16x16x32_bf16 v[64:67], a[148:151], a[196:199], v[64:67]\n"
      "ds_read_b128 a[236:239], v5 offset:10240\n"
      "v_mfma_f32_16x16x32_bf16 v[68:71], a[148:151], a[204:207], 0\n"
      "buffer_load_dwordx4 v12, %36, s90 offen offset:2048 lds\n"
      "v_mfma_f32_16x16x32_bf16 v[68:71], a[144:147], a[200:203], v[68:71]\n"
      "ds_read_b128 a[240:243], v4 offset:12288\n"
      "v_mfma_f32_16x16x32_bf16 v[72:75], a[144:147], a[208:211], 0\n"
      "ds_read_b128 a[244:247], v5 offset:12288\n"
      "v_mfma_f32_16x16x32_bf16 v[72:75], a[148:151], a[212:215], v[72:75]\n"
      "ds_read_b128 a[248:251], v4 offset:14336\n"
      "s_waitcnt lgkmcnt(11)\n"
      "v_mfma_f32_16x16x32_bf16 a[12:15], a[132:135], a[220:223], 0\n"
      "buffer_load_dwordx4 v13, %36, s90 offen offset:3072 lds\n"
      "v_mfma_f32_16x16x32_bf16 a[12:15], a[128:131], a[216:219], a[12:15]\n"
      "ds_read_b128 a[252:255], v5 offset:14336\n"
      "v_mfma_f32_16x16x32_bf16 v[44:47], a[136:139], a[216:219], 0\n"
      "v_xor_b32 v4, 0x8000, v4\n"
      "v_xor_b32 v5, 0x8000, v5\n"
      "v_mfma_f32_16x16x32_bf16 v[44:47], a[140:143], a[220:223], v[44:47]\n"
      "buffer_load_dwordx4 a[96:99], v14, %37, s87 offen offset:0\n"
      "v_mfma_f32_16x16x32_bf16 v[76:79], a[148:151], a[220:223], 0\n"
      "ds_read_b64_tr_b16 a[160:161], v0 offset:1024\n"
      "v_mfma_f32_16x16x32_bf16 v[76:79], a[144:147], a[216:219], v[76:79]\n"
      "ds_read_b64_tr_b16 a[162:163], v0 offset:1536\n"
      "s_waitcnt lgkmcnt(10)\n"
      "v_mfma_f32_16x16x32_bf16 v[96:99], a[152:155], a[192:195], 0\n"
      "buffer_load_dwordx4 a[100:103], v15, %37, s87 offen offset:1024\n"
      "v_mfma_f32_16x16x32_bf16 v[96:99], a[156:159], a[196:199], v[96:99]\n"
      "s_or_b32 m0, %42, 0x20000\n"
      "ds_read_b64_tr_b16 a[164:165], v0 offset:17408\n"
      "v_mfma_f32_16x16x32_bf16 v[100:103], a[156:159], a[204:207], 0\n"
      "ds_read_b64_tr_b16 a[166:167], v0 offset:17920\n"
      "v_mfma_f32_16x16x32_bf16 v[100:103], a[152:155], a[200:203], v[100:103]\n"
      "v_xor_b32 v0, 0x8000, v0\n"
      "v_mfma_f32_16x16x32_bf16 v[104:107], a[152:155], a[208:211], 0\n"
      "buffer_load_dwordx4 v6, %36, s65 offen offset:0 lds\n"
      "v_mfma_f32_16x16x32_bf16 v[104:107], a[156:159], a[212:215], v[104:107]\n"
      "ds_read_b64_tr_b16 a[168:169], v1 offset:1024\n"
      "v_mfma_f32_16x16x32_bf16 v[108:111], a[156:159], a[220:223], 0\n"
      "ds_read_b64_tr_b16 a[170:171], v1 offset:1536\n"
      "v_mfma_f32_16x16x32_bf16 v[108:111], a[152:155], a[216:219], v[108:111]\n"
      "buffer_load_dwordx4 v7, %36, s65 offen offset:1024 lds\n"
      "s_waitcnt lgkmcnt(8)\n"
      "v_mfma_f32_16x16x32_bf16 a[16:19], a[128:131], a[224:227], 0\n"
      "ds_read_b64_tr_b16 a[172:173], v1 offset:17408\n"
      "v_mfma_f32_16x16x32_bf16 a[16:19], a[132:135], a[228:231], a[16:19]\n"
      "ds_read_b64_tr_b16 a[174:175], v1 offset:17920\n"
      "v_mfma_f32_16x16x32_bf16 v[48:51], a[140:143], a[228:231], 0\n"
      "v_xor_b32 v1, 0x8000, v1\n"
      "v_mfma_f32_16x16x32_bf16 v[48:51], a[136:139], a[224:227], v[48:51]\n"
      "buffer_load_dwordx4 v8, %36, s65 offen offset:2048 lds\n"
      "v_mfma_f32_16x16x32_bf16 v[80:83], a[144:147], a[224:227], 0\n"
      "ds_read_b64_tr_b16 a[176:177], v2 offset:1024\n"
      "v_mfma_f32_16x16x32_bf16 v[80:83], a[148:151], a[228:231], v[80:83]\n"
      "ds_read_b64_tr_b16 a[178:179], v2 offset:1536\n"
      "v_mfma_f32_16x16x32_bf16 v[112:115], a[156:159], a[228:231], 0\n"
      "buffer_load_dwordx4 v9, %36, s65 offen offset:3072 lds\n"
      "v_mfma_f32_16x16x32_bf16 v[112:115], a[152:155], a[224:227], v[112:115]\n"
      "s_or_b32 m0, %42, 0x22000\n"
      "ds_read_b64_tr_b16 a[180:181], v2 offset:17408\n"
      "v_mfma_f32_16x16x32_bf16 a[20:23], a[128:131], a[232:235], 0\n"
      "ds_read_b64_tr_b16 a[182:183], v2 offset:17920\n"
      "v_mfma_f32_16x16x32_bf16 a[20:23], a[132:135], a[236:239], a[20:23]\n"
      "buffer_load_dwordx4 v10, %36, s65 offen offset:0 lds\n"
      "v_mfma_f32_16x16x32_bf16 v[52:55], a[140:143], a[236:239], 0\n"
      "v_xor_b32 v2, 0x8000, v2\n"
      "v_mfma_f32_16x16x32_bf16 v[52:55], a[136:139], a[232:235], v[52:55]\n"
      "ds_read_b64_tr_b16 a[184:185], v3 offset:1024\n"
      "v_mfma_f32_16x16x32_bf16 v[84:87], a[144:147], a[232:235], 0\n"
      "ds_read_b64_tr_b16 a[186:187], v3 offset:1536\n"
      "v_mfma_f32_16x16x32_bf16 v[84:87], a[148:151], a[236:239], v[84:87]\n"
      "buffer_load_dwordx4 v11, %36, s65 offen offset:1024 lds\n"
      "v_mfma_f32_16x16x32_bf16 v[116:119], a[156:159], a[236:239], 0\n"
      "ds_read_b64_tr_b16 a[188:189], v3 offset:17408\n"
      "v_mfma_f32_16x16x32_bf16 v[116:119], a[152:155], a[232:235], v[116:119]\n"
      "ds_read_b64_tr_b16 a[190:191], v3 offset:17920\n"
      "v_mfma_f32_16x16x32_bf16 a[24:27], a[128:131], a[240:243], 0\n"
      "buffer_load_dwordx4 v12, %36, s65 offen offset:2048 lds\n"
      "v_mfma_f32_16x16x32_bf16 a[24:27], a[132:135], a[244:247], a[24:27]\n"
      "v_xor_b32 v3, 0x8000, v3\n"
      "s_cmp_eq_u32 %40, 3\n"
      "s_cselect_b64 exec, 1, 0\n"
      "global_store_dword v29, v31, s[48:49] sc1\n"
      "s_mov_b64 exec, -1\n"
      "v_mfma_f32_16x16x32_bf16 v[56:59], a[140:143], a[244:247], 0\n"
      "v_mfma_f32_16x16x32_bf16 v[56:59], a[136:139], a[240:243], v[56:59]\n"
      "v_mfma_f32_16x16x32_bf16 v[88:91], a[144:147], a[240:243], 0\n"
      "buffer_load_dwordx4 v13, %36, s65 offen offset:3072 lds\n"
      "v_mfma_f32_16x16x32_bf16 v[88:91], a[148:151], a[244:247], v[88:91]\n"
      "v_readlane_b32 s65, v22, 1\n"
      "v_mfma_f32_16x16x32_bf16 v[120:123], a[156:159], a[244:247], 0\n"
      "v_mfma_f32_16x16x32_bf16 v[120:123], a[152:155], a[240:243], v[120:123]\n"
      "s_waitcnt lgkmcnt(4)\n"
      "v_mfma_f32_16x16x32_bf16 a[28:31], a[128:131], a[248:251], 0\n"
      "v_mfma_f32_16x16x32_bf16 a[28:31], a[132:135], a[252:255], a[28:31]\n"
      "v_mfma_f32_16x16x32_bf16 v[60:63], a[140:143], a[252:255], 0\n"
      "v_mfma_f32_16x16x32_bf16 v[60:63], a[136:139], a[248:251], v[60:63]\n"
      "v_mfma_f32_16x16x32_bf16 v[92:95], a[144:147], a[248:251], 0\n"
      "s_waitcnt vmcnt(8) lgkmcnt(0)\n"
      "s_barrier\n"
      "v_mfma_f32_16x16x32_bf16 v[92:95], a[148:151], a[252:255], v[92:95]\n"
      "s_cmp_eq_u32 s73, 1\n"
      "v_mfma_f32_16x16x32_bf16 v[124:127], a[156:159], a[252:255], 0\n"
      "s_mov_b32 m0, s67\n"
      "v_mfma_f32_16x16x32_bf16 v[124:127], a[152:155], a[248:251], v[124:127]\n"
      "s_cbranch_scc0 20f\n"
      "s_branch 23f\n"
      ".p2align 6\n"
      "20:\n"
      "v_mfma_f32_16x16x32_bf16 v[128:131], a[160:163], a[192:195], v[128:131]\n"
      "v_mfma_f32_16x16x32_bf16 v[128:131], a[164:167], a[196:199], v[128:131]\n"
      "v_mfma_f32_16x16x32_bf16 v[160:163], a[172:175], a[196:199], v[160:163]\n"
      "v_mfma_f32_16x16x32_bf16 v[160:163], a[168:171], a[192:195], v[160:163]\n"
      "buffer_load_dwordx4 v14, s[40:43], s64 offen offset:0 sc0 sc1 lds\n"
      "v_mfma_f32_16x16x32_bf16 v[192:195], a[176:179], a[192:195], v[192:195]\n"
      "ds_read_b64_tr_b16 a[128:129], v0 offset:0\n"
      "v_mfma_f32_16x16x32_bf16 v[192:195], a[180:183], a[196:199], v[192:195]\n"
      "s_lshr_b32 s66, s65, 12\n"
      "v_mfma_f32_16x16x32_bf16 v[224:227], a[188:191], a[196:199], v[224:227]\n"
      "s_xor_b32 s69, s69, s70\n"
      "v_mfma_f32_16x16x32_bf16 v[224:227], a[184:187], a[192:195], v[224:227]\n"
      "ds_read_b64_tr_b16 a[130:131], v0 offset:512\n"
      "v_mfma_f32_16x16x32_bf16 v[228:231], a[184:187], a[200:203], v[228:231]\n"
      "buffer_load_dwordx4 v15, s[40:43], s64 offen offset:1024 sc0 sc1 lds\n"
      "v_mfma_f32_16x16x32_bf16 v[228:231], a[188:191], a[204:207], v[228:231]\n"
      "ds_read_b128 a[192:195], v4 offset:0\n"
      "v_mfma_f32_16x16x32_bf16 v[196:199], a[180:183], a[204:207], v[196:199]\n"
      "ds_read_b128 a[196:199], v5 offset:0\n"
      "v_mfma_f32_16x16x32_bf16 v[196:199], a[176:179], a[200:203], v[196:199]\n"
      "ds_read_b64_tr_b16 a[132:133], v0 offset:16384\n"
      "v_mfma_f32_16x16x32_bf16 v[164:167], a[168:171], a[200:203], v[164:167]\n"
      "s_xor_b32 s70, s70, s69\n"
      "v_mfma_f32_16x16x32_bf16 v[164:167], a[172:175], a[204:207], v[164:167]\n"
      "buffer_load_dwordx4 v16, s[40:43], s64 offen offset:2048 sc0 sc1 lds\n"
      "v_mfma_f32_16x16x32_bf16 v[132:135], a[164:167], a[204:207], v[132:135]\n"
      "v_mfma_f32_16x16x32_bf16 v[132:135], a[160:163], a[200:203], v[132:135]\n"
      "ds_read_b64_tr_b16 a[134:135], v0 offset:16896\n"
      "v_mfma_f32_16x16x32_bf16 v[136:139], a[160:163], a[208:211], v[136:139]\n"
      "ds_read_b128 a[200:203], v4 offset:2048\n"
      "v_mfma_f32_16x16x32_bf16 v[136:139], a[164:167], a[212:215], v[136:139]\n"
      "ds_read_b128 a[204:207], v5 offset:2048\n"
      "v_mfma_f32_16x16x32_bf16 v[168:171], a[172:175], a[212:215], v[168:171]\n"
      "buffer_load_dwordx4 v17, s[40:43], s64 offen offset:3072 sc0 sc1 lds\n"
      "v_mfma_f32_16x16x32_bf16 v[168:171], a[168:171], a[208:211], v[168:171]\n"
      "s_bitset1_b32 m0, 12\n"
      "v_mfma_f32_16x16x32_bf16 v[200:203], a[176:179], a[208:211], v[200:203]\n"
      "ds_read_b64_tr_b16 a[136:137], v1 offset:0\n"
      "v_mfma_f32_16x16x32_bf16 v[200:203], a[180:183], a[212:215], v[200:203]\n"
      "v_mfma_f32_16x16x32_bf16 v[232:235], a[188:191], a[212:215], v[232:235]\n"
      "ds_read_b64_tr_b16 a[138:139], v1 offset:512\n"
      "v_mfma_f32_16x16x32_bf16 v[232:235], a[184:187], a[208:211], v[232:235]\n"
      "buffer_load_dwordx4 v18, s[40:43], s64 offen offset:0 sc0 sc1 lds\n"
      "v_mfma_f32_16x16x32_bf16 v[236:239], a[184:187], a[216:219], v[236:239]\n"
      "ds_read_b128 a[208:211], v4 offset:4096\n"
      "v_mfma_f32_16x16x32_bf16 v[236:239], a[188:191], a[220:223], v[236:239]\n"
      "ds_read_b128 a[212:215], v5 offset:4096\n"
      "v_mfma_f32_16x16x32_bf16 v[204:207], a[180:183], a[220:223], v[204:207]\n"
      "ds_read_b64_tr_b16 a[140:141], v1 offset:16384\n"
      "v_mfma_f32_16x16x32_bf16 v[204:207], a[176:179], a[216:219], v[204:207]\n"
      "v_mfma_f32_16x16x32_bf16 v[172:175], a[168:171], a[216:219], v[172:175]\n"
      "buffer_load_dwordx4 v19, s[40:43], s64 offen offset:1024 sc0 sc1 lds\n"
      "v_mfma_f32_16x16x32_bf16 v[172:175], a[172:175], a[220:223], v[172:175]\n"
      "ds_read_b64_tr_b16 a[142:143], v1 offset:16896\n"
      "v_mfma_f32_16x16x32_bf16 v[140:143], a[164:167], a[220:223], v[140:143]\n"
      "v_mfma_f32_16x16x32_bf16 v[140:143], a[160:163], a[216:219], v[140:143]\n"
      "v_mfma_f32_16x16x32_bf16 v[144:147], a[160:163], a[224:227], v[144:147]\n"
      "ds_read_b128 a[216:219], v4 offset:6144\n"
      "v_mfma_f32_16x16x32_bf16 v[144:147], a[164:167], a[228:231], v[144:147]\n"
      "buffer_load_dwordx4 v20, s[40:43], s64 offen offset:2048 sc0 sc1 lds\n"
      "v_mfma_f32_16x16x32_bf16 v[176:179], a[172:175], a[228:231], v[176:179]\n"
      "ds_read_b128 a[220:223], v5 offset:6144\n"
      "v_mfma_f32_16x16x32_bf16 v[176:179], a[168:171], a[224:227], v[176:179]\n"
      "ds_read_b64_tr_b16 a[144:145], v2 offset:0\n"
      "v_mfma_f32_16x16x32_bf16 v[208:211], a[176:179], a[224:227], v[208:211]\n"
      "buffer_load_dword v29, v28, s[28:31], s66 offen sc0 sc1\n"
      "v_mfma_f32_16x16x32_bf16 v[208:211], a[180:183], a[228:231], v[208:211]\n"
      "ds_read_b64_tr_b16 a[146:147], v2 offset:512\n"
      "v_mfma_f32_16x16x32_bf16 v[240:243], a[188:191], a[228:231], v[240:243]\n"
      "buffer_load_dwordx4 v21, s[40:43], s64 offen offset:3072 sc0 sc1 lds\n"
      "v_mfma_f32_16x16x32_bf16 v[240:243], a[184:187], a[224:227], v[240:243]\n"
      "s_mov_b32 m0, s68\n"
      "v_mfma_f32_16x16x32_bf16 v[244:247], a[184:187], a[232:235], v[244:247]\n"
      "ds_read_b128 a[224:227], v4 offset:8192\n"
      "v_mfma_f32_16x16x32_bf16 v[244:247], a[188:191], a[236:239], v[244:247]\n"
      "ds_read_b128 a[228:231], v5 offset:8192\n"
      "v_mfma_f32_16x16x32_bf16 v[212:215], a[180:183], a[236:239], v[212:215]\n"
      "ds_read_b64_tr_b16 a[148:149], v2 offset:16384\n"
      "v_mfma_f32_16x16x32_bf16 v[212:215], a[176:179], a[232:235], v[212:215]\n"
      "v_readlane_b32 s64, v23, s71\n"
      "v_mfma_f32_16x16x32_bf16 v[180:183], a[168:171], a[232:235], v[180:183]\n"
      "ds_read_b64_tr_b16 a[150:151], v2 offset:16896\n"
      "v_mfma_f32_16x16x32_bf16 v[180:183], a[172:175], a[236:239], v[180:183]\n"
      "s_xor_b32 s67, s67, 0x8000\n"
      "v_mfma_f32_16x16x32_bf16 v[148:151], a[164:167], a[236:239], v[148:151]\n"
      "buffer_load_dwordx4 v6, %36, s65 offen offset:0 lds\n"
      "v_mfma_f32_16x16x32_bf16 v[148:151], a[160:163], a[232:235], v[148:151]\n"
      "v_mfma_f32_16x16x32_bf16 v[152:155], a[160:163], a[240:243], v[152:155]\n"
      "ds_read_b128 a[232:235], v4 offset:10240\n"
      "v_mfma_f32_16x16x32_bf16 v[152:155], a[164:167], a[244:247], v[152:155]\n"
      "ds_read_b128 a[236:239], v5 offset:10240\n"
      "v_mfma_f32_16x16x32_bf16 v[184:187], a[172:175], a[244:247], v[184:187]\n"
      "ds_read_b64_tr_b16 a[152:153], v3 offset:0\n"
      "v_mfma_f32_16x16x32_bf16 v[184:187], a[168:171], a[240:243], v[184:187]\n"
      "v_mfma_f32_16x16x32_bf16 v[216:219], a[176:179], a[240:243], v[216:219]\n"
      "ds_read_b64_tr_b16 a[154:155], v3 offset:512\n"
      "v_mfma_f32_16x16x32_bf16 v[216:219], a[180:183], a[244:247], v[216:219]\n"
      "v_mfma_f32_16x16x32_bf16 v[248:251], a[188:191], a[244:247], v[248:251]\n"
      "buffer_load_dwordx4 v7, %36, s65 offen offset:1024 lds\n"
      "v_mfma_f32_16x16x32_bf16 v[248:251], a[184:187], a[240:243], v[248:251]\n"
      "ds_read_b64_tr_b16 a[156:157], v3 offset:16384\n"
      "v_mfma_f32_16x16x32_bf16 v[252:255], a[184:187], a[248:251], v[252:255]\n"
      "ds_read_b128 a[240:243], v4 offset:12288\n"
      "v_mfma_f32_16x16x32_bf16 v[252:255], a[188:191], a[252:255], v[252:255]\n"
      "ds_read_b128 a[244:247], v5 offset:12288\n"
      "v_mfma_f32_16x16x32_bf16 v[220:223], a[180:183], a[252:255], v[220:223]\n"
      "v_mfma_f32_16x16x32_bf16 v[220:223], a[176:179], a[248:251], v[220:223]\n"
      "ds_read_b64_tr_b16 a[158:159], v3 offset:16896\n"
      "v_mfma_f32_16x16x32_bf16 v[188:191], a[168:171], a[248:251], v[188:191]\n"
      "s_mul_i32 s1, s0, 14\n"
      "v_mfma_f32_16x16x32_bf16 v[188:191], a[172:175], a[252:255], v[188:191]\n"
      "s_sub_u32 s1, s2, s1\n"
      "v_mfma_f32_16x16x32_bf16 v[156:159], a[164:167], a[252:255], v[156:159]\n"
      "buffer_load_dwordx4 v8, %36, s65 offen offset:2048 lds\n"
      "v_mfma_f32_16x16x32_bf16 v[156:159], a[160:163], a[248:251], v[156:159]\n"
      "s_cmp_lt_i32 s1, s46\n"
      "s_cbranch_scc1 160f\n"
      "60:\n"
      "s_waitcnt lgkmcnt(7)\n"
      "v_mfma_f32_16x16x32_bf16 a[0:3], a[128:131], a[192:195], a[0:3]\n"
      "ds_read_b128 a[248:251], v4 offset:14336\n"
      "v_mfma_f32_16x16x32_bf16 a[0:3], a[132:135], a[196:199], a[0:3]\n"
      "ds_read_b128 a[252:255], v5 offset:14336\n"
      "v_mfma_f32_16x16x32_bf16 a[4:7], a[132:135], a[204:207], a[4:7]\n"
      "ds_read_b64_tr_b16 a[160:161], v0 offset:1024\n"
      "v_mfma_f32_16x16x32_bf16 a[4:7], a[128:131], a[200:203], a[4:7]\n"
      "v_xor_b32 v4, 0x8000, v4\n"
      "v_mfma_f32_16x16x32_bf16 a[8:11], a[128:131], a[208:211], a[8:11]\n"
      "ds_read_b64_tr_b16 a[162:163], v0 offset:1536\n"
      "v_mfma_f32_16x16x32_bf16 a[8:11], a[132:135], a[212:215], a[8:11]\n"
      "v_xor_b32 v5, 0x8000, v5\n"
      "v_mfma_f32_16x16x32_bf16 a[12:15], a[132:135], a[220:223], a[12:15]\n"
      "buffer_load_dwordx4 v9, %36, s65 offen offset:3072 lds\n"
      "v_mfma_f32_16x16x32_bf16 a[12:15], a[128:131], a[216:219], a[12:15]\n"
      "s_bitset1_b32 m0, 13\n"
      "v_mfma_f32_16x16x32_bf16 a[16:19], a[128:131], a[224:227], a[16:19]\n"
      "ds_read_b64_tr_b16 a[164:165], v0 offset:17408\n"
      "v_mfma_f32_16x16x32_bf16 a[16:19], a[132:135], a[228:231], a[16:19]\n"
      "s_xor_b32 s68, s68, s70\n"
      "s_waitcnt lgkmcnt(6)\n"
      "v_mfma_f32_16x16x32_bf16 a[20:23], a[132:135], a[236:239], a[20:23]\n"
      "ds_read_b64_tr_b16 a[166:167], v0 offset:17920\n"
      "v_mfma_f32_16x16x32_bf16 a[20:23], a[128:131], a[232:235], a[20:23]\n"
      "v_xor_b32 v0, s69, v0\n"
      "v_mfma_f32_16x16x32_bf16 a[24:27], a[128:131], a[240:243], a[24:27]\n"
      "s_mov_b32 s46, s1\n"
      "v_mfma_f32_16x16x32_bf16 a[24:27], a[132:135], a[244:247], a[24:27]\n"
      "s_waitcnt lgkmcnt(4)\n"
      "v_mfma_f32_16x16x32_bf16 a[28:31], a[132:135], a[252:255], a[28:31]\n"
      "buffer_load_dwordx4 v10, %36, s65 offen offset:0 lds\n"
      "v_mfma_f32_16x16x32_bf16 a[28:31], a[128:131], a[248:251], a[28:31]\n"
      "ds_read_b64_tr_b16 a[168:169], v1 offset:1024\n"
      "v_mfma_f32_16x16x32_bf16 v[60:63], a[136:139], a[248:251], v[60:63]\n"
      "s_memrealtime s[0:1]\n"
      "s_memtime s[2:3]\n"
      "v_mfma_f32_16x16x32_bf16 v[60:63], a[140:143], a[252:255], v[60:63]\n"
      "ds_read_b64_tr_b16 a[170:171], v1 offset:1536\n"
      "v_mfma_f32_16x16x32_bf16 v[56:59], a[140:143], a[244:247], v[56:59]\n"
      "v_mfma_f32_16x16x32_bf16 v[56:59], a[136:139], a[240:243], v[56:59]\n"
      "v_mfma_f32_16x16x32_bf16 v[52:55], a[136:139], a[232:235], v[52:55]\n"
      "ds_read_b64_tr_b16 a[172:173], v1 offset:17408\n"
      "v_mfma_f32_16x16x32_bf16 v[52:55], a[140:143], a[236:239], v[52:55]\n"
      "v_mfma_f32_16x16x32_bf16 v[48:51], a[140:143], a[228:231], v[48:51]\n"
      "buffer_load_dwordx4 v11, %36, s65 offen offset:1024 lds\n"
      "v_mfma_f32_16x16x32_bf16 v[48:51], a[136:139], a[224:227], v[48:51]\n"
      "v_mfma_f32_16x16x32_bf16 v[44:47], a[136:139], a[216:219], v[44:47]\n"
      "ds_read_b64_tr_b16 a[174:175], v1 offset:17920\n"
      "v_mfma_f32_16x16x32_bf16 v[44:47], a[140:143], a[220:223], v[44:47]\n"
      "v_xor_b32 v1, s69, v1\n"
      "v_mfma_f32_16x16x32_bf16 v[40:43], a[140:143], a[212:215], v[40:43]\n"
      "v_mfma_f32_16x16x32_bf16 v[40:43], a[136:139], a[208:211], v[40:43]\n"
      "ds_read_b64_tr_b16 a[176:177], v2 offset:1024\n"
      "v_mfma_f32_16x16x32_bf16 v[36:39], a[136:139], a[200:203], v[36:39]\n"
      "v_mfma_f32_16x16x32_bf16 v[36:39], a[140:143], a[204:207], v[36:39]\n"
      "v_mfma_f32_16x16x32_bf16 v[32:35], a[140:143], a[196:199], v[32:35]\n"
      "buffer_load_dwordx4 v12, %36, s65 offen offset:2048 lds\n"
      "v_mfma_f32_16x16x32_bf16 v[32:35], a[136:139], a[192:195], v[32:35]\n"
      "ds_read_b64_tr_b16 a[178:179], v2 offset:1536\n"
      "v_mfma_f32_16x16x32_bf16 v[64:67], a[144:147], a[192:195], v[64:67]\n"
      "v_mfma_f32_16x16x32_bf16 v[64:67], a[148:151], a[196:199], v[64:67]\n"
      "v_mfma_f32_16x16x32_bf16 v[68:71], a[148:151], a[204:207], v[68:71]\n"
      "ds_read_b64_tr_b16 a[180:181], v2 offset:17408\n"
      "v_mfma_f32_16x16x32_bf16 v[68:71], a[144:147], a[200:203], v[68:71]\n"
      "v_mfma_f32_16x16x32_bf16 v[72:75], a[144:147], a[208:211], v[72:75]\n"
      "v_mfma_f32_16x16x32_bf16 v[72:75], a[148:151], a[212:215], v[72:75]\n"
      "ds_read_b64_tr_b16 a[182:183], v2 offset:17920\n"
      "v_mfma_f32_16x16x32_bf16 v[76:79], a[148:151], a[220:223], v[76:79]\n"
      "buffer_load_dwordx4 v13, %36, s65 offen offset:3072 lds\n"
      "v_mfma_f32_16x16x32_bf16 v[76:79], a[144:147], a[216:219], v[76:79]\n"
      "v_xor_b32 v2, s69, v2\n"
      "v_mfma_f32_16x16x32_bf16 v[80:83], a[144:147], a[224:227], v[80:83]\n"
      "ds_read_b64_tr_b16 a[184:185], v3 offset:1024\n"
      "v_mfma_f32_16x16x32_bf16 v[80:83], a[148:151], a[228:231], v[80:83]\n"
      "s_add_u32 s71, s71, 1\n"
      "v_mfma_f32_16x16x32_bf16 v[84:87], a[148:151], a[236:239], v[84:87]\n"
      "v_readlane_b32 s65, v22, s71\n"
      "v_mfma_f32_16x16x32_bf16 v[84:87], a[144:147], a[232:235], v[84:87]\n"
      "ds_read_b64_tr_b16 a[186:187], v3 offset:1536\n"
      "v_mfma_f32_16x16x32_bf16 v[88:91], a[144:147], a[240:243], v[88:91]\n"
      "s_cmp_eq_u32 s71, 7\n"
      "v_mfma_f32_16x16x32_bf16 v[88:91], a[148:151], a[244:247], v[88:91]\n"
      "s_cmov_b32 s42, 0x5800000\n"
      "v_mfma_f32_16x16x32_bf16 v[92:95], a[148:151], a[252:255], v[92:95]\n"
      "s_cmov_b64 s[40:41], s[24:25]\n"
      "v_mfma_f32_16x16x32_bf16 v[92:95], a[144:147], a[248:251], v[92:95]\n"
      "ds_read_b64_tr_b16 a[188:189], v3 offset:17408\n"
      "v_mfma_f32_16x16x32_bf16 v[124:127], a[152:155], a[248:251], v[124:127]\n"
      "s_cmov_b32 s75, %31\n"
      "v_mfma_f32_16x16x32_bf16 v[124:127], a[156:159], a[252:255], v[124:127]\n"
      "s_cmp_ge_u32 s71, s72\n"
      "v_mfma_f32_16x16x32_bf16 v[120:123], a[156:159], a[244:247], v[120:123]\n"
      "ds_read_b64_tr_b16 a[190:191], v3 offset:17920\n"
      "v_mfma_f32_16x16x32_bf16 v[120:123], a[152:155], a[240:243], v[120:123]\n"
      "v_xor_b32 v3, s69, v3\n"
      "v_mfma_f32_16x16x32_bf16 v[116:119], a[152:155], a[232:235], v[116:119]\n"
      "s_cselect_b32 s76, -1, s75\n"
      "v_mfma_f32_16x16x32_bf16 v[116:119], a[156:159], a[236:239], v[116:119]\n"
      "v_mfma_f32_16x16x32_bf16 v[112:115], a[156:159], a[228:231], v[112:115]\n"
      "v_mfma_f32_16x16x32_bf16 v[112:115], a[152:155], a[224:227], v[112:115]\n"
      "v_mfma_f32_16x16x32_bf16 v[108:111], a[152:155], a[216:219], v[108:111]\n"
      "v_mfma_f32_16x16x32_bf16 v[108:111], a[156:159], a[220:223], v[108:111]\n"
      "v_mfma_f32_16x16x32_bf16 v[104:107], a[156:159], a[212:215], v[104:107]\n"
      "v_mfma_f32_16x16x32_bf16 v[104:107], a[152:155], a[208:211], v[104:107]\n"
      "v_mfma_f32_16x16x32_bf16 v[100:103], a[152:155], a[200:203], v[100:103]\n"
      "s_waitcnt vmcnt(8) lgkmcnt(0)\n"
      "s_barrier\n"
      "v_mfma_f32_16x16x32_bf16 v[100:103], a[156:159], a[204:207], v[100:103]\n"
      "v_cmp_gt_u32 vcc, s76, v29\n"
      "v_mfma_f32_16x16x32_bf16 v[96:99], a[156:159], a[196:199], v[96:99]\n"
      "s_mov_b32 m0, s67\n"
      "v_mfma_f32_16x16x32_bf16 v[96:99], a[152:155], a[192:195], v[96:99]\n"
      "s_cbranch_vccz 20b\n"
      "22:\n"
      "s_cmp_ge_u32 s71, s74\n"
      "s_cbranch_scc1 25f\n"
      "27:\n"
      "s_cmp_eq_u32 s71, s73\n"
      "s_cbranch_scc0 26f\n"
      "23:\n"
      "v_mfma_f32_16x16x32_bf16 v[128:131], a[160:163], a[192:195], v[128:131]\n"
      "v_cmp_gt_u32 vcc, s75, v29\n"
      "v_mfma_f32_16x16x32_bf16 v[128:131], a[164:167], a[196:199], v[128:131]\n"
      "v_mfma_f32_16x16x32_bf16 v[160:163], a[172:175], a[196:199], v[160:163]\n"
      "s_cbranch_vccnz 140f\n"
      "40:\n"
      "v_mfma_f32_16x16x32_bf16 v[160:163], a[168:171], a[192:195], v[160:163]\n"
      "buffer_load_dwordx4 v14, s[40:43], s64 offen offset:0 sc0 sc1 lds\n"
      "v_mfma_f32_16x16x32_bf16 v[192:195], a[176:179], a[192:195], v[192:195]\n"
      "ds_read_b64_tr_b16 a[128:129], v0 offset:0\n"
      "v_mfma_f32_16x16x32_bf16 v[192:195], a[180:183], a[196:199], v[192:195]\n"
      "s_lshr_b32 s66, s65, 12\n"
      "v_mfma_f32_16x16x32_bf16 v[224:227], a[188:191], a[196:199], v[224:227]\n"
      "s_xor_b32 s69, s69, s70\n"
      "v_mfma_f32_16x16x32_bf16 v[224:227], a[184:187], a[192:195], v[224:227]\n"
      "ds_read_b64_tr_b16 a[130:131], v0 offset:512\n"
      "v_mfma_f32_16x16x32_bf16 v[228:231], a[184:187], a[200:203], v[228:231]\n"
      "buffer_load_dwordx4 v15, s[40:43], s64 offen offset:1024 sc0 sc1 lds\n"
      "v_mfma_f32_16x16x32_bf16 v[228:231], a[188:191], a[204:207], v[228:231]\n"
      "ds_read_b128 a[192:195], v4 offset:0\n"
      "v_mfma_f32_16x16x32_bf16 v[196:199], a[180:183], a[204:207], v[196:199]\n"
      "ds_read_b128 a[196:199], v5 offset:0\n"
      "v_mfma_f32_16x16x32_bf16 v[196:199], a[176:179], a[200:203], v[196:199]\n"
      "ds_read_b64_tr_b16 a[132:133], v0 offset:16384\n"
      "v_mfma_f32_16x16x32_bf16 v[164:167], a[168:171], a[200:203], v[164:167]\n"
      "s_xor_b32 s70, s70, s69\n"
      "v_mfma_f32_16x16x32_bf16 v[164:167], a[172:175], a[204:207], v[164:167]\n"
      "buffer_load_dwordx4 v16, s[40:43], s64 offen offset:2048 sc0 sc1 lds\n"
      "v_mfma_f32_16x16x32_bf16 v[132:135], a[164:167], a[204:207], v[132:135]\n"
      "s_add_u32 s77, s77, 1\n"
      "v_mfma_f32_16x16x32_bf16 v[132:135], a[160:163], a[200:203], v[132:135]\n"
      "ds_read_b64_tr_b16 a[134:135], v0 offset:16896\n"
      "v_mfma_f32_16x16x32_bf16 v[136:139], a[160:163], a[208:211], v[136:139]\n"
      "ds_read_b128 a[200:203], v4 offset:2048\n"
      "v_mfma_f32_16x16x32_bf16 v[136:139], a[164:167], a[212:215], v[136:139]\n"
      "ds_read_b128 a[204:207], v5 offset:2048\n"
      "v_mfma_f32_16x16x32_bf16 v[168:171], a[172:175], a[212:215], v[168:171]\n"
      "buffer_load_dwordx4 v17, s[40:43], s64 offen offset:3072 sc0 sc1 lds\n"
      "v_mfma_f32_16x16x32_bf16 v[168:171], a[168:171], a[208:211], v[168:171]\n"
      "s_bitset1_b32 m0, 12\n"
      "v_mfma_f32_16x16x32_bf16 v[200:203], a[176:179], a[208:211], v[200:203]\n"
      "ds_read_b64_tr_b16 a[136:137], v1 offset:0\n"
      "v_mfma_f32_16x16x32_bf16 v[200:203], a[180:183], a[212:215], v[200:203]\n"
      "v_mfma_f32_16x16x32_bf16 v[232:235], a[188:191], a[212:215], v[232:235]\n"
      "ds_read_b64_tr_b16 a[138:139], v1 offset:512\n"
      "v_mfma_f32_16x16x32_bf16 v[232:235], a[184:187], a[208:211], v[232:235]\n"
      "buffer_load_dwordx4 v18, s[40:43], s64 offen offset:0 sc0 sc1 lds\n"
      "v_mfma_f32_16x16x32_bf16 v[236:239], a[184:187], a[216:219], v[236:239]\n"
      "ds_read_b128 a[208:211], v4 offset:4096\n"
      "v_mfma_f32_16x16x32_bf16 v[236:239], a[188:191], a[220:223], v[236:239]\n"
      "ds_read_b128 a[212:215], v5 offset:4096\n"
      "v_mfma_f32_16x16x32_bf16 v[204:207], a[180:183], a[220:223], v[204:207]\n"
      "ds_read_b64_tr_b16 a[140:141], v1 offset:16384\n"
      "v_mfma_f32_16x16x32_bf16 v[204:207], a[176:179], a[216:219], v[204:207]\n"
      "v_mfma_f32_16x16x32_bf16 v[172:175], a[168:171], a[216:219], v[172:175]\n"
      "buffer_load_dwordx4 v19, s[40:43], s64 offen offset:1024 sc0 sc1 lds\n"
      "v_mfma_f32_16x16x32_bf16 v[172:175], a[172:175], a[220:223], v[172:175]\n"
      "ds_read_b64_tr_b16 a[142:143], v1 offset:16896\n"
      "v_mfma_f32_16x16x32_bf16 v[140:143], a[164:167], a[220:223], v[140:143]\n"
      "v_mfma_f32_16x16x32_bf16 v[140:143], a[160:163], a[216:219], v[140:143]\n"
      "v_mfma_f32_16x16x32_bf16 v[144:147], a[160:163], a[224:227], v[144:147]\n"
      "ds_read_b128 a[216:219], v4 offset:6144\n"
      "v_mfma_f32_16x16x32_bf16 v[144:147], a[164:167], a[228:231], v[144:147]\n"
      "buffer_load_dwordx4 v20, s[40:43], s64 offen offset:2048 sc0 sc1 lds\n"
      "v_mfma_f32_16x16x32_bf16 v[176:179], a[172:175], a[228:231], v[176:179]\n"
      "ds_read_b128 a[220:223], v5 offset:6144\n"
      "v_mfma_f32_16x16x32_bf16 v[176:179], a[168:171], a[224:227], v[176:179]\n"
      "ds_read_b64_tr_b16 a[144:145], v2 offset:0\n"
      "v_mfma_f32_16x16x32_bf16 v[208:211], a[176:179], a[224:227], v[208:211]\n"
      "buffer_load_dword v29, v28, s[28:31], s66 offen sc0 sc1\n"
      "v_mfma_f32_16x16x32_bf16 v[208:211], a[180:183], a[228:231], v[208:211]\n"
      "ds_read_b64_tr_b16 a[146:147], v2 offset:512\n"
      "v_mfma_f32_16x16x32_bf16 v[240:243], a[188:191], a[228:231], v[240:243]\n"
      "buffer_load_dwordx4 v21, s[40:43], s64 offen offset:3072 sc0 sc1 lds\n"
      "v_mfma_f32_16x16x32_bf16 v[240:243], a[184:187], a[224:227], v[240:243]\n"
      "s_mov_b32 m0, s68\n"
      "v_mfma_f32_16x16x32_bf16 v[244:247], a[184:187], a[232:235], v[244:247]\n"
      "ds_read_b128 a[224:227], v4 offset:8192\n"
      "v_mfma_f32_16x16x32_bf16 v[244:247], a[188:191], a[236:239], v[244:247]\n"
      "ds_read_b128 a[228:231], v5 offset:8192\n"
      "v_mfma_f32_16x16x32_bf16 v[212:215], a[180:183], a[236:239], v[212:215]\n"
      "global_store_dwordx4 v14, a[96:99], s[84:85] offset:0 sc0 sc1\n"
      "v_mfma_f32_16x16x32_bf16 v[212:215], a[176:179], a[232:235], v[212:215]\n"
      "ds_read_b64_tr_b16 a[148:149], v2 offset:16384\n"
      "v_mfma_f32_16x16x32_bf16 v[180:183], a[168:171], a[232:235], v[180:183]\n"
      "ds_read_b64_tr_b16 a[150:151], v2 offset:16896\n"
      "v_mfma_f32_16x16x32_bf16 v[180:183], a[172:175], a[236:239], v[180:183]\n"
      "v_readlane_b32 s64, v23, s71\n"
      "v_mfma_f32_16x16x32_bf16 v[148:151], a[164:167], a[236:239], v[148:151]\n"
      "buffer_load_dwordx4 v6, %36, s65 offen offset:0 lds\n"
      "v_mfma_f32_16x16x32_bf16 v[148:151], a[160:163], a[232:235], v[148:151]\n"
      "s_xor_b32 s67, s67, 0x8000\n"
      "v_mfma_f32_16x16x32_bf16 v[152:155], a[160:163], a[240:243], v[152:155]\n"
      "ds_read_b128 a[232:235], v4 offset:10240\n"
      "v_mfma_f32_16x16x32_bf16 v[152:155], a[164:167], a[244:247], v[152:155]\n"
      "ds_read_b128 a[236:239], v5 offset:10240\n"
      "v_mfma_f32_16x16x32_bf16 v[184:187], a[172:175], a[244:247], v[184:187]\n"
      "global_store_dwordx4 v15, a[100:103], s[84:85] offset:1024 sc0 sc1\n"
      "v_mfma_f32_16x16x32_bf16 v[184:187], a[168:171], a[240:243], v[184:187]\n"
      "ds_read_b64_tr_b16 a[152:153], v3 offset:0\n"
      "v_mfma_f32_16x16x32_bf16 v[216:219], a[176:179], a[240:243], v[216:219]\n"
      "ds_read_b64_tr_b16 a[154:155], v3 offset:512\n"
      "v_mfma_f32_16x16x32_bf16 v[216:219], a[180:183], a[244:247], v[216:219]\n"
      "v_mfma_f32_16x16x32_bf16 v[248:251], a[188:191], a[244:247], v[248:251]\n"
      "buffer_load_dwordx4 v7, %36, s65 offen offset:1024 lds\n"
      "v_mfma_f32_16x16x32_bf16 v[248:251], a[184:187], a[240:243], v[248:251]\n"
      "ds_read_b64_tr_b16 a[156:157], v3 offset:16384\n"
      "v_mfma_f32_16x16x32_bf16 v[252:255], a[184:187], a[248:251], v[252:255]\n"
      "ds_read_b128 a[240:243], v4 offset:12288\n"
      "v_mfma_f32_16x16x32_bf16 v[252:255], a[188:191], a[252:255], v[252:255]\n"
      "ds_read_b128 a[244:247], v5 offset:12288\n"
      "v_mfma_f32_16x16x32_bf16 v[220:223], a[180:183], a[252:255], v[220:223]\n"
      "v_mfma_f32_16x16x32_bf16 v[220:223], a[176:179], a[248:251], v[220:223]\n"
      "ds_read_b64_tr_b16 a[158:159], v3 offset:16896\n"
      "v_mfma_f32_16x16x32_bf16 v[188:191], a[168:171], a[248:251], v[188:191]\n"
      "s_mul_i32 s1, s0, 14\n"
      "v_mfma_f32_16x16x32_bf16 v[188:191], a[172:175], a[252:255], v[188:191]\n"
      "s_sub_u32 s1, s2, s1\n"
      "v_mfma_f32_16x16x32_bf16 v[156:159], a[164:167], a[252:255], v[156:159]\n"
      "buffer_load_dwordx4 v8, %36, s65 offen offset:2048 lds\n"
      "v_mfma_f32_16x16x32_bf16 v[156:159], a[160:163], a[248:251], v[156:159]\n"
      "s_cmp_lt_i32 s1, s46\n"
      "s_cbranch_scc1 161f\n"
      "61:\n"
      "s_waitcnt lgkmcnt(7)\n"
      "v_mfma_f32_16x16x32_bf16 a[0:3], a[128:131], a[192:195], a[0:3]\n"
      "ds_read_b128 a[248:251], v4 offset:14336\n"
      "v_mfma_f32_16x16x32_bf16 a[0:3], a[132:135], a[196:199], a[0:3]\n"
      "ds_read_b128 a[252:255], v5 offset:14336\n"
      "v_mfma_f32_16x16x32_bf16 a[4:7], a[132:135], a[204:207], a[4:7]\n"
      "ds_read_b64_tr_b16 a[160:161], v0 offset:1024\n"
      "v_mfma_f32_16x16x32_bf16 a[4:7], a[128:131], a[200:203], a[4:7]\n"
      "v_xor_b32 v4, 0x8000, v4\n"
      "v_mfma_f32_16x16x32_bf16 a[8:11], a[128:131], a[208:211], a[8:11]\n"
      "ds_read_b64_tr_b16 a[162:163], v0 offset:1536\n"
      "v_mfma_f32_16x16x32_bf16 a[8:11], a[132:135], a[212:215], a[8:11]\n"
      "v_xor_b32 v5, 0x8000, v5\n"
      "v_mfma_f32_16x16x32_bf16 a[12:15], a[132:135], a[220:223], a[12:15]\n"
      "buffer_load_dwordx4 v9, %36, s65 offen offset:3072 lds\n"
      "v_mfma_f32_16x16x32_bf16 a[12:15], a[128:131], a[216:219], a[12:15]\n"
      "s_bitset1_b32 m0, 13\n"
      "v_mfma_f32_16x16x32_bf16 a[16:19], a[128:131], a[224:227], a[16:19]\n"
      "ds_read_b64_tr_b16 a[164:165], v0 offset:17408\n"
      "v_mfma_f32_16x16x32_bf16 a[16:19], a[132:135], a[228:231], a[16:19]\n"
      "s_xor_b32 s68, s68, s70\n"
      "s_waitcnt lgkmcnt(6)\n"
      "v_mfma_f32_16x16x32_bf16 a[20:23], a[132:135], a[236:239], a[20:23]\n"
      "ds_read_b64_tr_b16 a[166:167], v0 offset:17920\n"
      "v_mfma_f32_16x16x32_bf16 a[20:23], a[128:131], a[232:235], a[20:23]\n"
      "v_xor_b32 v0, s69, v0\n"
      "v_mfma_f32_16x16x32_bf16 a[24:27], a[128:131], a[240:243], a[24:27]\n"
      "s_mov_b32 s46, s1\n"
      "v_mfma_f32_16x16x32_bf16 a[24:27], a[132:135], a[244:247], a[24:27]\n"
      "s_waitcnt lgkmcnt(4)\n"
      "v_mfma_f32_16x16x32_bf16 a[28:31], a[132:135], a[252:255], a[28:31]\n"
      "buffer_load_dwordx4 v10, %36, s65 offen offset:0 lds\n"
      "v_mfma_f32_16x16x32_bf16 a[28:31], a[128:131], a[248:251], a[28:31]\n"
      "ds_read_b64_tr_b16 a[168:169], v1 offset:1024\n"
      "v_mfma_f32_16x16x32_bf16 v[60:63], a[136:139], a[248:251], v[60:63]\n"
      "s_memrealtime s[0:1]\n"
      "s_memtime s[2:3]\n"
      "v_mfma_f32_16x16x32_bf16 v[60:63], a[140:143], a[252:255], v[60:63]\n"
      "ds_read_b64_tr_b16 a[170:171], v1 offset:1536\n"
      "v_mfma_f32_16x16x32_bf16 v[56:59], a[140:143], a[244:247], v[56:59]\n"
      "v_mfma_f32_16x16x32_bf16 v[56:59], a[136:139], a[240:243], v[56:59]\n"
      "v_mfma_f32_16x16x32_bf16 v[52:55], a[136:139], a[232:235], v[52:55]\n"
      "ds_read_b64_tr_b16 a[172:173], v1 offset:17408\n"
      "v_mfma_f32_16x16x32_bf16 v[52:55], a[140:143], a[236:239], v[52:55]\n"
      "v_mfma_f32_16x16x32_bf16 v[48:51], a[140:143], a[228:231], v[48:51]\n"
      "buffer_load_dwordx4 v11, %36, s65 offen offset:1024 lds\n"
      "v_mfma_f32_16x16x32_bf16 v[48:51], a[136:139], a[224:227], v[48:51]\n"
      "v_mfma_f32_16x16x32_bf16 v[44:47], a[136:139], a[216:219], v[44:47]\n"
      "ds_read_b64_tr_b16 a[174:175], v1 offset:17920\n"
      "v_mfma_f32_16x16x32_bf16 v[44:47], a[140:143], a[220:223], v[44:47]\n"
      "v_xor_b32 v1, s69, v1\n"
      "v_mfma_f32_16x16x32_bf16 v[40:43], a[140:143], a[212:215], v[40:43]\n"
      "v_mfma_f32_16x16x32_bf16 v[40:43], a[136:139], a[208:211], v[40:43]\n"
      "ds_read_b64_tr_b16 a[176:177], v2 offset:1024\n"
      "v_mfma_f32_16x16x32_bf16 v[36:39], a[136:139], a[200:203], v[36:39]\n"
      "v_mfma_f32_16x16x32_bf16 v[36:39], a[140:143], a[204:207], v[36:39]\n"
      "v_mfma_f32_16x16x32_bf16 v[32:35], a[140:143], a[196:199], v[32:35]\n"
      "buffer_load_dwordx4 v12, %36, s65 offen offset:2048 lds\n"
      "v_mfma_f32_16x16x32_bf16 v[32:35], a[136:139], a[192:195], v[32:35]\n"
      "ds_read_b64_tr_b16 a[178:179], v2 offset:1536\n"
      "v_mfma_f32_16x16x32_bf16 v[64:67], a[144:147], a[192:195], v[64:67]\n"
      "v_mfma_f32_16x16x32_bf16 v[64:67], a[148:151], a[196:199], v[64:67]\n"
      "v_mfma_f32_16x16x32_bf16 v[68:71], a[148:151], a[204:207], v[68:71]\n"
      "ds_read_b64_tr_b16 a[180:181], v2 offset:17408\n"
      "v_mfma_f32_16x16x32_bf16 v[68:71], a[144:147], a[200:203], v[68:71]\n"
      "v_mfma_f32_16x16x32_bf16 v[72:75], a[144:147], a[208:211], v[72:75]\n"
      "v_mfma_f32_16x16x32_bf16 v[72:75], a[148:151], a[212:215], v[72:75]\n"
      "ds_read_b64_tr_b16 a[182:183], v2 offset:17920\n"
      "v_mfma_f32_16x16x32_bf16 v[76:79], a[148:151], a[220:223], v[76:79]\n"
      "buffer_load_dwordx4 v13, %36, s65 offen offset:3072 lds\n"
      "v_mfma_f32_16x16x32_bf16 v[76:79], a[144:147], a[216:219], v[76:79]\n"
      "v_xor_b32 v2, s69, v2\n"
      "v_mfma_f32_16x16x32_bf16 v[80:83], a[144:147], a[224:227], v[80:83]\n"
      "ds_read_b64_tr_b16 a[184:185], v3 offset:1024\n"
      "v_mfma_f32_16x16x32_bf16 v[80:83], a[148:151], a[228:231], v[80:83]\n"
      "s_add_u32 s71, s71, 1\n"
      "v_mfma_f32_16x16x32_bf16 v[84:87], a[148:151], a[236:239], v[84:87]\n"
      "v_readlane_b32 s65, v22, s71\n"
      "v_mfma_f32_16x16x32_bf16 v[84:87], a[144:147], a[232:235], v[84:87]\n"
      "ds_read_b64_tr_b16 a[186:187], v3 offset:1536\n"
      "v_mfma_f32_16x16x32_bf16 v[88:91], a[144:147], a[240:243], v[88:91]\n"
      "s_cmp_eq_u32 s71, 7\n"
      "v_mfma_f32_16x16x32_bf16 v[88:91], a[148:151], a[244:247], v[88:91]\n"
      "s_cmov_b32 s42, 0x5800000\n"
      "v_mfma_f32_16x16x32_bf16 v[92:95], a[148:151], a[252:255], v[92:95]\n"
      "s_cmov_b64 s[40:41], s[24:25]\n"
      "v_mfma_f32_16x16x32_bf16 v[92:95], a[144:147], a[248:251], v[92:95]\n"
      "ds_read_b64_tr_b16 a[188:189], v3 offset:17408\n"
      "v_mfma_f32_16x16x32_bf16 v[124:127], a[152:155], a[248:251], v[124:127]\n"
      "s_cmov_b32 s75, %31\n"
      "v_mfma_f32_16x16x32_bf16 v[124:127], a[156:159], a[252:255], v[124:127]\n"
      "v_readlane_b32 s86, v25, s77\n"
      "v_mfma_f32_16x16x32_bf16 v[120:123], a[156:159], a[244:247], v[120:123]\n"
      "ds_read_b64_tr_b16 a[190:191], v3 offset:17920\n"
      "v_mfma_f32_16x16x32_bf16 v[120:123], a[152:155], a[240:243], v[120:123]\n"
      "v_xor_b32 v3, s69, v3\n"
      "v_mfma_f32_16x16x32_bf16 v[116:119], a[152:155], a[232:235], v[116:119]\n"
      "s_and_b32 m0, s86, 14\n"
      "v_mfma_f32_16x16x32_bf16 v[116:119], a[156:159], a[236:239], v[116:119]\n"
      "s_add_u32 s86, s86, s35\n"
      "v_mfma_f32_16x16x32_bf16 v[112:115], a[156:159], a[228:231], v[112:115]\n"
      "s_movrels_b64 s[82:83], s[48:49]\n"
      "v_mfma_f32_16x16x32_bf16 v[112:115], a[152:155], a[224:227], v[112:115]\n"
      "s_add_u32 s84, s86, s82\n"
      "s_addc_u32 s85, s83, 0\n"
      "v_mfma_f32_16x16x32_bf16 v[108:111], a[152:155], a[216:219], v[108:111]\n"
      "v_mfma_f32_16x16x32_bf16 v[108:111], a[156:159], a[220:223], v[108:111]\n"
      "v_mfma_f32_16x16x32_bf16 v[104:107], a[156:159], a[212:215], v[104:107]\n"
      "v_mfma_f32_16x16x32_bf16 v[104:107], a[152:155], a[208:211], v[104:107]\n"
      "v_mfma_f32_16x16x32_bf16 v[100:103], a[152:155], a[200:203], v[100:103]\n"
      "s_waitcnt vmcnt(10) lgkmcnt(0)\n"
      "s_barrier\n"
      "v_mfma_f32_16x16x32_bf16 v[100:103], a[156:159], a[204:207], v[100:103]\n"
      "v_mfma_f32_16x16x32_bf16 v[96:99], a[156:159], a[196:199], v[96:99]\n"
      "s_mov_b32 m0, s67\n"
      "v_mfma_f32_16x16x32_bf16 v[96:99], a[152:155], a[192:195], v[96:99]\n"
      "v_mfma_f32_16x16x32_bf16 v[128:131], a[160:163], a[192:195], v[128:131]\n"
      "v_cmp_gt_u32 vcc, s75, v29\n"
      "v_mfma_f32_16x16x32_bf16 v[128:131], a[164:167], a[196:199], v[128:131]\n"
      "v_mfma_f32_16x16x32_bf16 v[160:163], a[172:175], a[196:199], v[160:163]\n"
      "s_cbranch_vccnz 141f\n"
      "41:\n"
      "v_mfma_f32_16x16x32_bf16 v[160:163], a[168:171], a[192:195], v[160:163]\n"
      "buffer_load_dwordx4 v14, s[40:43], s64 offen offset:0 sc0 sc1 lds\n"
      "v_mfma_f32_16x16x32_bf16 v[192:195], a[176:179], a[192:195], v[192:195]\n"
      "ds_read_b64_tr_b16 a[128:129], v0 offset:0\n"
      "v_mfma_f32_16x16x32_bf16 v[192:195], a[180:183], a[196:199], v[192:195]\n"
      "s_lshr_b32 s66, s65, 12\n"
      "v_mfma_f32_16x16x32_bf16 v[224:227], a[188:191], a[196:199], v[224:227]\n"
      "s_xor_b32 s69, s69, s70\n"
      "v_mfma_f32_16x16x32_bf16 v[224:227], a[184:187], a[192:195], v[224:227]\n"
      "ds_read_b64_tr_b16 a[130:131], v0 offset:512\n"
      "v_mfma_f32_16x16x32_bf16 v[228:231], a[184:187], a[200:203], v[228:231]\n"
      "buffer_load_dwordx4 v15, s[40:43], s64 offen offset:1024 sc0 sc1 lds\n"
      "v_mfma_f32_16x16x32_bf16 v[228:231], a[188:191], a[204:207], v[228:231]\n"
      "ds_read_b128 a[192:195], v4 offset:0\n"
      "v_mfma_f32_16x16x32_bf16 v[196:199], a[180:183], a[204:207], v[196:199]\n"
      "ds_read_b128 a[196:199], v5 offset:0\n"
      "v_mfma_f32_16x16x32_bf16 v[196:199], a[176:179], a[200:203], v[196:199]\n"
      "ds_read_b64_tr_b16 a[132:133], v0 offset:16384\n"
      "v_mfma_f32_16x16x32_bf16 v[164:167], a[168:171], a[200:203], v[164:167]\n"
      "s_xor_b32 s70, s70, s69\n"
      "v_mfma_f32_16x16x32_bf16 v[164:167], a[172:175], a[204:207], v[164:167]\n"
      "buffer_load_dwordx4 v16, s[40:43], s64 offen offset:2048 sc0 sc1 lds\n"
      "v_mfma_f32_16x16x32_bf16 v[132:135], a[164:167], a[204:207], v[132:135]\n"
      "v_readlane_b32 s86, v24, s77\n"
      "v_mfma_f32_16x16x32_bf16 v[132:135], a[160:163], a[200:203], v[132:135]\n"
      "ds_read_b64_tr_b16 a[134:135], v0 offset:16896\n"
      "v_mfma_f32_16x16x32_bf16 v[136:139], a[160:163], a[208:211], v[136:139]\n"
      "ds_read_b128 a[200:203], v4 offset:2048\n"
      "v_mfma_f32_16x16x32_bf16 v[136:139], a[164:167], a[212:215], v[136:139]\n"
      "ds_read_b128 a[204:207], v5 offset:2048\n"
      "v_mfma_f32_16x16x32_bf16 v[168:171], a[172:175], a[212:215], v[168:171]\n"
      "buffer_load_dwordx4 v17, s[40:43], s64 offen offset:3072 sc0 sc1 lds\n"
      "v_mfma_f32_16x16x32_bf16 v[168:171], a[168:171], a[208:211], v[168:171]\n"
      "s_bitset1_b32 m0, 12\n"
      "v_mfma_f32_16x16x32_bf16 v[200:203], a[176:179], a[208:211], v[200:203]\n"
      "ds_read_b64_tr_b16 a[136:137], v1 offset:0\n"
      "v_mfma_f32_16x16x32_bf16 v[200:203], a[180:183], a[212:215], v[200:203]\n"
      "s_sub_u32 s90, s77, 1\n"
      "v_mfma_f32_16x16x32_bf16 v[232:235], a[188:191], a[212:215], v[232:235]\n"
      "ds_read_b64_tr_b16 a[138:139], v1 offset:512\n"
      "v_mfma_f32_16x16x32_bf16 v[232:235], a[184:187], a[208:211], v[232:235]\n"
      "buffer_load_dwordx4 v18, s[40:43], s64 offen offset:0 sc0 sc1 lds\n"
      "v_mfma_f32_16x16x32_bf16 v[236:239], a[184:187], a[216:219], v[236:239]\n"
      "ds_read_b128 a[208:211], v4 offset:4096\n"
      "v_mfma_f32_16x16x32_bf16 v[236:239], a[188:191], a[220:223], v[236:239]\n"
      "ds_read_b128 a[212:215], v5 offset:4096\n"
      "v_mfma_f32_16x16x32_bf16 v[204:207], a[180:183], a[220:223], v[204:207]\n"
      "ds_read_b64_tr_b16 a[140:141], v1 offset:16384\n"
      "v_mfma_f32_16x16x32_bf16 v[204:207], a[176:179], a[216:219], v[204:207]\n"
      "v_readlane_b32 s73, v27, s77\n"
      "v_mfma_f32_16x16x32_bf16 v[172:175], a[168:171], a[216:219], v[172:175]\n"
      "buffer_load_dwordx4 v19, s[40:43], s64 offen offset:1024 sc0 sc1 lds\n"
      "v_mfma_f32_16x16x32_bf16 v[172:175], a[172:175], a[220:223], v[172:175]\n"
      "ds_read_b64_tr_b16 a[142:143], v1 offset:16896\n"
      "v_mfma_f32_16x16x32_bf16 v[140:143], a[164:167], a[220:223], v[140:143]\n"
      "s_min_u32 s72, s73, s74\n"
      "v_mfma_f32_16x16x32_bf16 v[140:143], a[160:163], a[216:219], v[140:143]\n"
      "v_mfma_f32_16x16x32_bf16 v[144:147], a[160:163], a[224:227], v[144:147]\n"
      "ds_read_b128 a[216:219], v4 offset:6144\n"
      "v_mfma_f32_16x16x32_bf16 v[144:147], a[164:167], a[228:231], v[144:147]\n"
      "buffer_load_dwordx4 v20, s[40:43], s64 offen offset:2048 sc0 sc1 lds\n"
      "v_mfma_f32_16x16x32_bf16 v[176:179], a[172:175], a[228:231], v[176:179]\n"
      "ds_read_b128 a[220:223], v5 offset:6144\n"
      "v_mfma_f32_16x16x32_bf16 v[176:179], a[168:171], a[224:227], v[176:179]\n"
      "ds_read_b64_tr_b16 a[144:145], v2 offset:0\n"
      "v_mfma_f32_16x16x32_bf16 v[208:211], a[176:179], a[224:227], v[208:211]\n"
      "buffer_load_dword v29, v28, s[28:31], s66 offen sc0 sc1\n"
      "v_mfma_f32_16x16x32_bf16 v[208:211], a[180:183], a[228:231], v[208:211]\n"
      "ds_read_b64_tr_b16 a[146:147], v2 offset:512\n"
      "v_mfma_f32_16x16x32_bf16 v[240:243], a[188:191], a[228:231], v[240:243]\n"
      "buffer_load_dwordx4 v21, s[40:43], s64 offen offset:3072 sc0 sc1 lds\n"
      "v_mfma_f32_16x16x32_bf16 v[240:243], a[184:187], a[224:227], v[240:243]\n"
      "s_mov_b32 m0, s68\n"
      "v_mfma_f32_16x16x32_bf16 v[244:247], a[184:187], a[232:235], v[244:247]\n"
      "ds_read_b128 a[224:227], v4 offset:8192\n"
      "v_mfma_f32_16x16x32_bf16 v[244:247], a[188:191], a[236:239], v[244:247]\n"
      "ds_read_b128 a[228:231], v5 offset:8192\n"
      "v_mfma_f32_16x16x32_bf16 v[212:215], a[180:183], a[236:239], v[212:215]\n"
      "buffer_load_dwordx4 a[96:99], v14, %37, s86 offen offset:0\n"
      "v_mfma_f32_16x16x32_bf16 v[212:215], a[176:179], a[232:235], v[212:215]\n"
      "ds_read_b64_tr_b16 a[148:149], v2 offset:16384\n"
      "v_mfma_f32_16x16x32_bf16 v[180:183], a[168:171], a[232:235], v[180:183]\n"
      "ds_read_b64_tr_b16 a[150:151], v2 offset:16896\n"
      "v_mfma_f32_16x16x32_bf16 v[180:183], a[172:175], a[236:239], v[180:183]\n"
      "v_readlane_b32 s64, v23, s71\n"
      "v_mfma_f32_16x16x32_bf16 v[148:151], a[164:167], a[236:239], v[148:151]\n"
      "buffer_load_dwordx4 v6, %36, s65 offen offset:0 lds\n"
      "v_mfma_f32_16x16x32_bf16 v[148:151], a[160:163], a[232:235], v[148:151]\n"
      "s_xor_b32 s67, s67, 0x8000\n"
      "v_mfma_f32_16x16x32_bf16 v[152:155], a[160:163], a[240:243], v[152:155]\n"
      "ds_read_b128 a[232:235], v4 offset:10240\n"
      "v_mfma_f32_16x16x32_bf16 v[152:155], a[164:167], a[244:247], v[152:155]\n"
      "ds_read_b128 a[236:239], v5 offset:10240\n"
      "v_mfma_f32_16x16x32_bf16 v[184:187], a[172:175], a[244:247], v[184:187]\n"
      "buffer_load_dwordx4 a[100:103], v15, %37, s86 offen offset:1024\n"
      "v_mfma_f32_16x16x32_bf16 v[184:187], a[168:171], a[240:243], v[184:187]\n"
      "ds_read_b64_tr_b16 a[152:153], v3 offset:0\n"
      "v_mfma_f32_16x16x32_bf16 v[216:219], a[176:179], a[240:243], v[216:219]\n"
      "ds_read_b64_tr_b16 a[154:155], v3 offset:512\n"
      "v_mfma_f32_16x16x32_bf16 v[216:219], a[180:183], a[244:247], v[216:219]\n"
      "v_mfma_f32_16x16x32_bf16 v[248:251], a[188:191], a[244:247], v[248:251]\n"
      "buffer_load_dwordx4 v7, %36, s65 offen offset:1024 lds\n"
      "v_mfma_f32_16x16x32_bf16 v[248:251], a[184:187], a[240:243], v[248:251]\n"
      "ds_read_b64_tr_b16 a[156:157], v3 offset:16384\n"
      "v_mfma_f32_16x16x32_bf16 v[252:255], a[184:187], a[248:251], v[252:255]\n"
      "ds_read_b128 a[240:243], v4 offset:12288\n"
      "v_mfma_f32_16x16x32_bf16 v[252:255], a[188:191], a[252:255], v[252:255]\n"
      "ds_read_b128 a[244:247], v5 offset:12288\n"
      "v_mfma_f32_16x16x32_bf16 v[220:223], a[180:183], a[252:255], v[220:223]\n"
      "v_mfma_f32_16x16x32_bf16 v[220:223], a[176:179], a[248:251], v[220:223]\n"
      "ds_read_b64_tr_b16 a[158:159], v3 offset:16896\n"
      "v_mfma_f32_16x16x32_bf16 v[188:191], a[168:171], a[248:251], v[188:191]\n"
      "s_mul_i32 s1, s0, 14\n"
      "v_mfma_f32_16x16x32_bf16 v[188:191], a[172:175], a[252:255], v[188:191]\n"
      "s_sub_u32 s1, s2, s1\n"
      "v_mfma_f32_16x16x32_bf16 v[156:159], a[164:167], a[252:255], v[156:159]\n"
      "buffer_load_dwordx4 v8, %36, s65 offen offset:2048 lds\n"
      "v_mfma_f32_16x16x32_bf16 v[156:159], a[160:163], a[248:251], v[156:159]\n"
      "s_cmp_lt_i32 s1, s46\n"
      "s_cbranch_scc1 162f\n"
      "62:\n"
      "s_waitcnt lgkmcnt(7)\n"
      "v_mfma_f32_16x16x32_bf16 a[0:3], a[128:131], a[192:195], a[0:3]\n"
      "ds_read_b128 a[248:251], v4 offset:14336\n"
      "v_mfma_f32_16x16x32_bf16 a[0:3], a[132:135], a[196:199], a[0:3]\n"
      "ds_read_b128 a[252:255], v5 offset:14336\n"
      "v_mfma_f32_16x16x32_bf16 a[4:7], a[132:135], a[204:207], a[4:7]\n"
      "ds_read_b64_tr_b16 a[160:161], v0 offset:1024\n"
      "v_mfma_f32_16x16x32_bf16 a[4:7], a[128:131], a[200:203], a[4:7]\n"
      "v_xor_b32 v4, 0x8000, v4\n"
      "v_mfma_f32_16x16x32_bf16 a[8:11], a[128:131], a[208:211], a[8:11]\n"
      "ds_read_b64_tr_b16 a[162:163], v0 offset:1536\n"
      "v_mfma_f32_16x16x32_bf16 a[8:11], a[132:135], a[212:215], a[8:11]\n"
      "v_xor_b32 v5, 0x8000, v5\n"
      "v_mfma_f32_16x16x32_bf16 a[12:15], a[132:135], a[220:223], a[12:15]\n"
      "buffer_load_dwordx4 v9, %36, s65 offen offset:3072 lds\n"
      "v_mfma_f32_16x16x32_bf16 a[12:15], a[128:131], a[216:219], a[12:15]\n"
      "s_bitset1_b32 m0, 13\n"
      "v_mfma_f32_16x16x32_bf16 a[16:19], a[128:131], a[224:227], a[16:19]\n"
      "ds_read_b64_tr_b16 a[164:165], v0 offset:17408\n"
      "v_mfma_f32_16x16x32_bf16 a[16:19], a[132:135], a[228:231], a[16:19]\n"
      "s_xor_b32 s68, s68, s70\n"
      "s_waitcnt lgkmcnt(6)\n"
      "v_mfma_f32_16x16x32_bf16 a[20:23], a[132:135], a[236:239], a[20:23]\n"
      "ds_read_b64_tr_b16 a[166:167], v0 offset:17920\n"
      "v_mfma_f32_16x16x32_bf16 a[20:23], a[128:131], a[232:235], a[20:23]\n"
      "v_xor_b32 v0, s69, v0\n"
      "v_mfma_f32_16x16x32_bf16 a[24:27], a[128:131], a[240:243], a[24:27]\n"
      "s_mov_b32 s46, s1\n"
      "v_mfma_f32_16x16x32_bf16 a[24:27], a[132:135], a[244:247], a[24:27]\n"
      "s_waitcnt lgkmcnt(4)\n"
      "v_mfma_f32_16x16x32_bf16 a[28:31], a[132:135], a[252:255], a[28:31]\n"
      "buffer_load_dwordx4 v10, %36, s65 offen offset:0 lds\n"
      "v_mfma_f32_16x16x32_bf16 a[28:31], a[128:131], a[248:251], a[28:31]\n"
      "ds_read_b64_tr_b16 a[168:169], v1 offset:1024\n"
      "v_mfma_f32_16x16x32_bf16 v[60:63], a[136:139], a[248:251], v[60:63]\n"
      "s_memrealtime s[0:1]\n"
      "s_memtime s[2:3]\n"
      "v_mfma_f32_16x16x32_bf16 v[60:63], a[140:143], a[252:255], v[60:63]\n"
      "ds_read_b64_tr_b16 a[170:171], v1 offset:1536\n"
      "v_mfma_f32_16x16x32_bf16 v[56:59], a[140:143], a[244:247], v[56:59]\n"
      "v_mfma_f32_16x16x32_bf16 v[56:59], a[136:139], a[240:243], v[56:59]\n"
      "v_mfma_f32_16x16x32_bf16 v[52:55], a[136:139], a[232:235], v[52:55]\n"
      "ds_read_b64_tr_b16 a[172:173], v1 offset:17408\n"
      "v_mfma_f32_16x16x32_bf16 v[52:55], a[140:143], a[236:239], v[52:55]\n"
      "v_mfma_f32_16x16x32_bf16 v[48:51], a[140:143], a[228:231], v[48:51]\n"
      "buffer_load_dwordx4 v11, %36, s65 offen offset:1024 lds\n"
      "v_mfma_f32_16x16x32_bf16 v[48:51], a[136:139], a[224:227], v[48:51]\n"
      "v_mfma_f32_16x16x32_bf16 v[44:47], a[136:139], a[216:219], v[44:47]\n"
      "ds_read_b64_tr_b16 a[174:175], v1 offset:17920\n"
      "v_mfma_f32_16x16x32_bf16 v[44:47], a[140:143], a[220:223], v[44:47]\n"
      "v_xor_b32 v1, s69, v1\n"
      "v_mfma_f32_16x16x32_bf16 v[40:43], a[140:143], a[212:215], v[40:43]\n"
      "v_mfma_f32_16x16x32_bf16 v[40:43], a[136:139], a[208:211], v[40:43]\n"
      "ds_read_b64_tr_b16 a[176:177], v2 offset:1024\n"
      "v_mfma_f32_16x16x32_bf16 v[36:39], a[136:139], a[200:203], v[36:39]\n"
      "v_mfma_f32_16x16x32_bf16 v[36:39], a[140:143], a[204:207], v[36:39]\n"
      "v_mfma_f32_16x16x32_bf16 v[32:35], a[140:143], a[196:199], v[32:35]\n"
      "buffer_load_dwordx4 v12, %36, s65 offen offset:2048 lds\n"
      "v_mfma_f32_16x16x32_bf16 v[32:35], a[136:139], a[192:195], v[32:35]\n"
      "ds_read_b64_tr_b16 a[178:179], v2 offset:1536\n"
      "v_mfma_f32_16x16x32_bf16 v[64:67], a[144:147], a[192:195], v[64:67]\n"
      "v_mfma_f32_16x16x32_bf16 v[64:67], a[148:151], a[196:199], v[64:67]\n"
      "v_mfma_f32_16x16x32_bf16 v[68:71], a[148:151], a[204:207], v[68:71]\n"
      "ds_read_b64_tr_b16 a[180:181], v2 offset:17408\n"
      "v_mfma_f32_16x16x32_bf16 v[68:71], a[144:147], a[200:203], v[68:71]\n"
      "v_mfma_f32_16x16x32_bf16 v[72:75], a[144:147], a[208:211], v[72:75]\n"
      "v_mfma_f32_16x16x32_bf16 v[72:75], a[148:151], a[212:215], v[72:75]\n"
      "ds_read_b64_tr_b16 a[182:183], v2 offset:17920\n"
      "v_mfma_f32_16x16x32_bf16 v[76:79], a[148:151], a[220:223], v[76:79]\n"
      "buffer_load_dwordx4 v13, %36, s65 offen offset:3072 lds\n"
      "v_mfma_f32_16x16x32_bf16 v[76:79], a[144:147], a[216:219], v[76:79]\n"
      "v_xor_b32 v2, s69, v2\n"
      "v_mfma_f32_16x16x32_bf16 v[80:83], a[144:147], a[224:227], v[80:83]\n"
      "ds_read_b64_tr_b16 a[184:185], v3 offset:1024\n"
      "v_mfma_f32_16x16x32_bf16 v[80:83], a[148:151], a[228:231], v[80:83]\n"
      "s_add_u32 s71, s71, 1\n"
      "v_mfma_f32_16x16x32_bf16 v[84:87], a[148:151], a[236:239], v[84:87]\n"
      "s_waitcnt vmcnt(26)\n"
      "s_lshl_b64 exec, 1, s90\n"
      "global_store_dword v26, v31, s[80:81] sc0 sc1\n"
      "s_mov_b64 exec, -1\n"
      "v_mfma_f32_16x16x32_bf16 v[84:87], a[144:147], a[232:235], v[84:87]\n"
      "ds_read_b64_tr_b16 a[186:187], v3 offset:1536\n"
      "v_mfma_f32_16x16x32_bf16 v[88:91], a[144:147], a[240:243], v[88:91]\n"
      "v_readlane_b32 s65, v22, s71\n"
      "v_mfma_f32_16x16x32_bf16 v[88:91], a[148:151], a[244:247], v[88:91]\n"
      "s_cmp_eq_u32 s71, 7\n"
      "v_mfma_f32_16x16x32_bf16 v[92:95], a[148:151], a[252:255], v[92:95]\n"
      "s_cmov_b32 s42, 0x5800000\n"
      "v_mfma_f32_16x16x32_bf16 v[92:95], a[144:147], a[248:251], v[92:95]\n"
      "ds_read_b64_tr_b16 a[188:189], v3 offset:17408\n"
      "v_mfma_f32_16x16x32_bf16 v[124:127], a[152:155], a[248:251], v[124:127]\n"
      "s_cmov_b64 s[40:41], s[24:25]\n"
      "v_mfma_f32_16x16x32_bf16 v[124:127], a[156:159], a[252:255], v[124:127]\n"
      "s_cmov_b32 s75, %31\n"
      "v_mfma_f32_16x16x32_bf16 v[120:123], a[156:159], a[244:247], v[120:123]\n"
      "ds_read_b64_tr_b16 a[190:191], v3 offset:17920\n"
      "v_mfma_f32_16x16x32_bf16 v[120:123], a[152:155], a[240:243], v[120:123]\n"
      "v_xor_b32 v3, s69, v3\n"
      "v_mfma_f32_16x16x32_bf16 v[116:119], a[152:155], a[232:235], v[116:119]\n"
      "s_mov_b64 s[80:81], s[82:83]\n"
      "v_mfma_f32_16x16x32_bf16 v[116:119], a[156:159], a[236:239], v[116:119]\n"
      "s_cmp_ge_u32 s71, s72\n"
      "v_mfma_f32_16x16x32_bf16 v[112:115], a[156:159], a[228:231], v[112:115]\n"
      "s_cselect_b32 s76, -1, s75\n"
      "v_mfma_f32_16x16x32_bf16 v[112:115], a[152:155], a[224:227], v[112:115]\n"
      "v_mfma_f32_16x16x32_bf16 v[108:111], a[152:155], a[216:219], v[108:111]\n"
      "v_mfma_f32_16x16x32_bf16 v[108:111], a[156:159], a[220:223], v[108:111]\n"
      "v_mfma_f32_16x16x32_bf16 v[104:107], a[156:159], a[212:215], v[104:107]\n"
      "v_mfma_f32_16x16x32_bf16 v[104:107], a[152:155], a[208:211], v[104:107]\n"
      "v_mfma_f32_16x16x32_bf16 v[100:103], a[152:155], a[200:203], v[100:103]\n"
      "s_waitcnt vmcnt(11) lgkmcnt(0)\n"
      "s_barrier\n"
      "v_mfma_f32_16x16x32_bf16 v[100:103], a[156:159], a[204:207], v[100:103]\n"
      "v_cmp_gt_u32 vcc, s76, v29\n"
      "v_mfma_f32_16x16x32_bf16 v[96:99], a[156:159], a[196:199], v[96:99]\n"
      "s_mov_b32 m0, s67\n"
      "v_mfma_f32_16x16x32_bf16 v[96:99], a[152:155], a[192:195], v[96:99]\n"
      "s_cbranch_vccz 20b\n"
      "s_branch 22b\n"
      "25:\n"
      "s_setpc_b64 s[44:45]\n"
      "26:\n"
      "v_cmp_gt_u32 vcc, s75, v29\n"
      "s_cbranch_vccz 20b\n"
      "s_getpc_b64 s[78:79]\n"
      "s_branch 73f\n"
      "s_branch 20b\n"
      "57:\n"
      "s_cmp_eq_u32 s71, s73\n"
      "s_cbranch_scc0 56f\n"
      "53:\n"
      "v_mfma_f32_16x16x32_bf16 v[128:131], a[160:163], a[192:195], v[128:131]\n"
      "v_cmp_gt_u32 vcc, s75, v29\n"
      "v_mfma_f32_16x16x32_bf16 v[128:131], a[164:167], a[196:199], v[128:131]\n"
      "v_mfma_f32_16x16x32_bf16 v[160:163], a[172:175], a[196:199], v[160:163]\n"
      "s_cbranch_vccnz 142f\n"
      "42:\n"
      "v_mfma_f32_16x16x32_bf16 v[160:163], a[168:171], a[192:195], v[160:163]\n"
      "buffer_load_dwordx4 v14, s[40:43], s64 offen offset:0 sc0 sc1 lds\n"
      "v_mfma_f32_16x16x32_bf16 v[192:195], a[176:179], a[192:195], v[192:195]\n"
      "ds_read_b64_tr_b16 a[128:129], v0 offset:0\n"
      "v_mfma_f32_16x16x32_bf16 v[192:195], a[180:183], a[196:199], v[192:195]\n"
      "s_lshr_b32 s66, s65, 12\n"
      "v_mfma_f32_16x16x32_bf16 v[224:227], a[188:191], a[196:199], v[224:227]\n"
      "s_xor_b32 s69, s69, s70\n"
      "v_mfma_f32_16x16x32_bf16 v[224:227], a[184:187], a[192:195], v[224:227]\n"
      "ds_read_b64_tr_b16 a[130:131], v0 offset:512\n"
      "v_mfma_f32_16x16x32_bf16 v[228:231], a[184:187], a[200:203], v[228:231]\n"
      "buffer_load_dwordx4 v15, s[40:43], s64 offen offset:1024 sc0 sc1 lds\n"
      "v_mfma_f32_16x16x32_bf16 v[228:231], a[188:191], a[204:207], v[228:231]\n"
      "ds_read_b128 a[192:195], v4 offset:0\n"
      "v_mfma_f32_16x16x32_bf16 v[196:199], a[180:183], a[204:207], v[196:199]\n"
      "ds_read_b128 a[196:199], v5 offset:0\n"
      "v_mfma_f32_16x16x32_bf16 v[196:199], a[176:179], a[200:203], v[196:199]\n"
      "ds_read_b64_tr_b16 a[132:133], v0 offset:16384\n"
      "v_mfma_f32_16x16x32_bf16 v[164:167], a[168:171], a[200:203], v[164:167]\n"
      "s_xor_b32 s70, s70, s69\n"
      "v_mfma_f32_16x16x32_bf16 v[164:167], a[172:175], a[204:207], v[164:167]\n"
      "buffer_load_dwordx4 v16, s[40:43], s64 offen offset:2048 sc0 sc1 lds\n"
      "v_mfma_f32_16x16x32_bf16 v[132:135], a[164:167], a[204:207], v[132:135]\n"
      "s_add_u32 s77, s77, 1\n"
      "v_mfma_f32_16x16x32_bf16 v[132:135], a[160:163], a[200:203], v[132:135]\n"
      "ds_read_b64_tr_b16 a[134:135], v0 offset:16896\n"
      "v_mfma_f32_16x16x32_bf16 v[136:139], a[160:163], a[208:211], v[136:139]\n"
      "ds_read_b128 a[200:203], v4 offset:2048\n"
      "v_mfma_f32_16x16x32_bf16 v[136:139], a[164:167], a[212:215], v[136:139]\n"
      "ds_read_b128 a[204:207], v5 offset:2048\n"
      "v_mfma_f32_16x16x32_bf16 v[168:171], a[172:175], a[212:215], v[168:171]\n"
      "buffer_load_dwordx4 v17, s[40:43], s64 offen offset:3072 sc0 sc1 lds\n"
      "v_mfma_f32_16x16x32_bf16 v[168:171], a[168:171], a[208:211], v[168:171]\n"
      "s_bitset1_b32 m0, 12\n"
      "v_mfma_f32_16x16x32_bf16 v[200:203], a[176:179], a[208:211], v[200:203]\n"
      "ds_read_b64_tr_b16 a[136:137], v1 offset:0\n"
      "v_mfma_f32_16x16x32_bf16 v[200:203], a[180:183], a[212:215], v[200:203]\n"
      "v_mfma_f32_16x16x32_bf16 v[232:235], a[188:191], a[212:215], v[232:235]\n"
      "ds_read_b64_tr_b16 a[138:139], v1 offset:512\n"
      "v_mfma_f32_16x16x32_bf16 v[232:235], a[184:187], a[208:211], v[232:235]\n"
      "buffer_load_dwordx4 v18, s[40:43], s64 offen offset:0 sc0 sc1 lds\n"
      "v_mfma_f32_16x16x32_bf16 v[236:239], a[184:187], a[216:219], v[236:239]\n"
      "ds_read_b128 a[208:211], v4 offset:4096\n"
      "v_mfma_f32_16x16x32_bf16 v[236:239], a[188:191], a[220:223], v[236:239]\n"
      "ds_read_b128 a[212:215], v5 offset:4096\n"
      "v_mfma_f32_16x16x32_bf16 v[204:207], a[180:183], a[220:223], v[204:207]\n"
      "ds_read_b64_tr_b16 a[140:141], v1 offset:16384\n"
      "v_mfma_f32_16x16x32_bf16 v[204:207], a[176:179], a[216:219], v[204:207]\n"
      "v_mfma_f32_16x16x32_bf16 v[172:175], a[168:171], a[216:219], v[172:175]\n"
      "buffer_load_dwordx4 v19, s[40:43], s64 offen offset:1024 sc0 sc1 lds\n"
      "v_mfma_f32_16x16x32_bf16 v[172:175], a[172:175], a[220:223], v[172:175]\n"
      "ds_read_b64_tr_b16 a[142:143], v1 offset:16896\n"
      "v_mfma_f32_16x16x32_bf16 v[140:143], a[164:167], a[220:223], v[140:143]\n"
      "v_mfma_f32_16x16x32_bf16 v[140:143], a[160:163], a[216:219], v[140:143]\n"
      "v_mfma_f32_16x16x32_bf16 v[144:147], a[160:163], a[224:227], v[144:147]\n"
      "ds_read_b128 a[216:219], v4 offset:6144\n"
      "v_mfma_f32_16x16x32_bf16 v[144:147], a[164:167], a[228:231], v[144:147]\n"
      "buffer_load_dwordx4 v20, s[40:43], s64 offen offset:2048 sc0 sc1 lds\n"
      "v_mfma_f32_16x16x32_bf16 v[176:179], a[172:175], a[228:231], v[176:179]\n"
      "ds_read_b128 a[220:223], v5 offset:6144\n"
      "v_mfma_f32_16x16x32_bf16 v[176:179], a[168:171], a[224:227], v[176:179]\n"
      "ds_read_b64_tr_b16 a[144:145], v2 offset:0\n"
      "v_mfma_f32_16x16x32_bf16 v[208:211], a[176:179], a[224:227], v[208:211]\n"
      "buffer_load_dword v29, v28, s[28:31], s66 offen sc0 sc1\n"
      "v_mfma_f32_16x16x32_bf16 v[208:211], a[180:183], a[228:231], v[208:211]\n"
      "ds_read_b64_tr_b16 a[146:147], v2 offset:512\n"
      "v_mfma_f32_16x16x32_bf16 v[240:243], a[188:191], a[228:231], v[240:243]\n"
      "buffer_load_dwordx4 v21, s[40:43], s64 offen offset:3072 sc0 sc1 lds\n"
      "v_mfma_f32_16x16x32_bf16 v[240:243], a[184:187], a[224:227], v[240:243]\n"
      "s_mov_b32 m0, s68\n"
      "v_mfma_f32_16x16x32_bf16 v[244:247], a[184:187], a[232:235], v[244:247]\n"
      "ds_read_b128 a[224:227], v4 offset:8192\n"
      "v_mfma_f32_16x16x32_bf16 v[244:247], a[188:191], a[236:239], v[244:247]\n"
      "ds_read_b128 a[228:231], v5 offset:8192\n"
      "v_mfma_f32_16x16x32_bf16 v[212:215], a[180:183], a[236:239], v[212:215]\n"
      "global_store_dwordx4 v14, a[96:99], s[84:85] offset:0 sc0 sc1\n"
      "v_mfma_f32_16x16x32_bf16 v[212:215], a[176:179], a[232:235], v[212:215]\n"
      "ds_read_b64_tr_b16 a[148:149], v2 offset:16384\n"
      "v_mfma_f32_16x16x32_bf16 v[180:183], a[168:171], a[232:235], v[180:183]\n"
      "ds_read_b64_tr_b16 a[150:151], v2 offset:16896\n"
      "v_mfma_f32_16x16x32_bf16 v[180:183], a[172:175], a[236:239], v[180:183]\n"
      "v_readlane_b32 s64, v23, s71\n"
      "v_mfma_f32_16x16x32_bf16 v[148:151], a[164:167], a[236:239], v[148:151]\n"
      "buffer_load_dwordx4 v6, %36, s65 offen offset:0 lds\n"
      "v_mfma_f32_16x16x32_bf16 v[148:151], a[160:163], a[232:235], v[148:151]\n"
      "s_xor_b32 s67, s67, 0x8000\n"
      "v_mfma_f32_16x16x32_bf16 v[152:155], a[160:163], a[240:243], v[152:155]\n"
      "ds_read_b128 a[232:235], v4 offset:10240\n"
      "v_mfma_f32_16x16x32_bf16 v[152:155], a[164:167], a[244:247], v[152:155]\n"
      "ds_read_b128 a[236:239], v5 offset:10240\n"
      "v_mfma_f32_16x16x32_bf16 v[184:187], a[172:175], a[244:247], v[184:187]\n"
      "global_store_dwordx4 v15, a[100:103], s[84:85] offset:1024 sc0 sc1\n"
      "v_mfma_f32_16x16x32_bf16 v[184:187], a[168:171], a[240:243], v[184:187]\n"
      "ds_read_b64_tr_b16 a[152:153], v3 offset:0\n"
      "v_mfma_f32_16x16x32_bf16 v[216:219], a[176:179], a[240:243], v[216:219]\n"
      "ds_read_b64_tr_b16 a[154:155], v3 offset:512\n"
      "v_mfma_f32_16x16x32_bf16 v[216:219], a[180:183], a[244:247], v[216:219]\n"
      "v_mfma_f32_16x16x32_bf16 v[248:251], a[188:191], a[244:247], v[248:251]\n"
      "buffer_load_dwordx4 v7, %36, s65 offen offset:1024 lds\n"
      "v_mfma_f32_16x16x32_bf16 v[248:251], a[184:187], a[240:243], v[248:251]\n"
      "ds_read_b64_tr_b16 a[156:157], v3 offset:16384\n"
      "v_mfma_f32_16x16x32_bf16 v[252:255], a[184:187], a[248:251], v[252:255]\n"
      "ds_read_b128 a[240:243], v4 offset:12288\n"
      "v_mfma_f32_16x16x32_bf16 v[252:255], a[188:191], a[252:255], v[252:255]\n"
      "ds_read_b128 a[244:247], v5 offset:12288\n"
      "v_mfma_f32_16x16x32_bf16 v[220:223], a[180:183], a[252:255], v[220:223]\n"
      "buffer_load_dword a108, v7, %36, s33 offen offset:384\n"
      "v_mfma_f32_16x16x32_bf16 v[220:223], a[176:179], a[248:251], v[220:223]\n"
      "ds_read_b64_tr_b16 a[158:159], v3 offset:16896\n"
      "v_mfma_f32_16x16x32_bf16 v[188:191], a[168:171], a[248:251], v[188:191]\n"
      "s_mul_i32 s1, s0, 14\n"
      "v_mfma_f32_16x16x32_bf16 v[188:191], a[172:175], a[252:255], v[188:191]\n"
      "s_sub_u32 s1, s2, s1\n"
      "v_mfma_f32_16x16x32_bf16 v[156:159], a[164:167], a[252:255], v[156:159]\n"
      "buffer_load_dwordx4 v8, %36, s65 offen offset:2048 lds\n"
      "v_mfma_f32_16x16x32_bf16 v[156:159], a[160:163], a[248:251], v[156:159]\n"
      "s_cmp_lt_i32 s1, s46\n"
      "s_cbranch_scc1 164f\n"
      "64:\n"
      "s_waitcnt lgkmcnt(7)\n"
      "v_mfma_f32_16x16x32_bf16 a[0:3], a[128:131], a[192:195], a[0:3]\n"
      "ds_read_b128 a[248:251], v4 offset:14336\n"
      "v_mfma_f32_16x16x32_bf16 a[0:3], a[132:135], a[196:199], a[0:3]\n"
      "ds_read_b128 a[252:255], v5 offset:14336\n"
      "v_mfma_f32_16x16x32_bf16 a[4:7], a[132:135], a[204:207], a[4:7]\n"
      "buffer_load_dword a108, v7, %36, s33 offen offset:640\n"
      "v_mfma_f32_16x16x32_bf16 a[4:7], a[128:131], a[200:203], a[4:7]\n"
      "ds_read_b64_tr_b16 a[160:161], v0 offset:1024\n"
      "v_mfma_f32_16x16x32_bf16 a[8:11], a[128:131], a[208:211], a[8:11]\n"
      "ds_read_b64_tr_b16 a[162:163], v0 offset:1536\n"
      "v_mfma_f32_16x16x32_bf16 a[8:11], a[132:135], a[212:215], a[8:11]\n"
      "v_xor_b32 v4, 0x8000, v4\n"
      "v_mfma_f32_16x16x32_bf16 a[12:15], a[132:135], a[220:223], a[12:15]\n"
      "buffer_load_dwordx4 v9, %36, s65 offen offset:3072 lds\n"
      "v_mfma_f32_16x16x32_bf16 a[12:15], a[128:131], a[216:219], a[12:15]\n"
      "s_bitset1_b32 m0, 13\n"
      "v_mfma_f32_16x16x32_bf16 a[16:19], a[128:131], a[224:227], a[16:19]\n"
      "ds_read_b64_tr_b16 a[164:165], v0 offset:17408\n"
      "v_mfma_f32_16x16x32_bf16 a[16:19], a[132:135], a[228:231], a[16:19]\n"
      "v_xor_b32 v5, 0x8000, v5\n"
      "s_waitcnt lgkmcnt(6)\n"
      "v_mfma_f32_16x16x32_bf16 a[20:23], a[132:135], a[236:239], a[20:23]\n"
      "ds_read_b64_tr_b16 a[166:167], v0 offset:17920\n"
      "v_mfma_f32_16x16x32_bf16 a[20:23], a[128:131], a[232:235], a[20:23]\n"
      "s_xor_b32 s68, s68, s70\n"
      "v_mfma_f32_16x16x32_bf16 a[24:27], a[128:131], a[240:243], a[24:27]\n"
      "v_xor_b32 v0, s69, v0\n"
      "v_mfma_f32_16x16x32_bf16 a[24:27], a[132:135], a[244:247], a[24:27]\n"
      "s_max_i32 s91, s22, s24\n"
      "s_waitcnt lgkmcnt(4)\n"
      "v_mfma_f32_16x16x32_bf16 a[28:31], a[132:135], a[252:255], a[28:31]\n"
      "buffer_load_dwordx4 v10, %36, s65 offen offset:0 lds\n"
      "v_mfma_f32_16x16x32_bf16 a[28:31], a[128:131], a[248:251], a[28:31]\n"
      "ds_read_b64_tr_b16 a[168:169], v1 offset:1024\n"
      "v_mfma_f32_16x16x32_bf16 v[60:63], a[136:139], a[248:251], v[60:63]\n"
      "v_readlane_b32 s33, v25, s91\n"
      "v_mfma_f32_16x16x32_bf16 v[60:63], a[140:143], a[252:255], v[60:63]\n"
      "ds_read_b64_tr_b16 a[170:171], v1 offset:1536\n"
      "v_mfma_f32_16x16x32_bf16 v[56:59], a[140:143], a[244:247], v[56:59]\n"
      "s_add_u32 s22, s22, 1\n"
      "v_mfma_f32_16x16x32_bf16 v[56:59], a[136:139], a[240:243], v[56:59]\n"
      "s_mov_b32 s46, s1\n"
      "v_mfma_f32_16x16x32_bf16 v[52:55], a[136:139], a[232:235], v[52:55]\n"
      "ds_read_b64_tr_b16 a[172:173], v1 offset:17408\n"
      "v_mfma_f32_16x16x32_bf16 v[52:55], a[140:143], a[236:239], v[52:55]\n"
      "s_memrealtime s[0:1]\n"
      "s_memtime s[2:3]\n"
      "v_mfma_f32_16x16x32_bf16 v[48:51], a[140:143], a[228:231], v[48:51]\n"
      "buffer_load_dwordx4 v11, %36, s65 offen offset:1024 lds\n"
      "v_mfma_f32_16x16x32_bf16 v[48:51], a[136:139], a[224:227], v[48:51]\n"
      "v_mfma_f32_16x16x32_bf16 v[44:47], a[136:139], a[216:219], v[44:47]\n"
      "ds_read_b64_tr_b16 a[174:175], v1 offset:17920\n"
      "v_mfma_f32_16x16x32_bf16 v[44:47], a[140:143], a[220:223], v[44:47]\n"
      "v_xor_b32 v1, s69, v1\n"
      "v_mfma_f32_16x16x32_bf16 v[40:43], a[140:143], a[212:215], v[40:43]\n"
      "v_mfma_f32_16x16x32_bf16 v[40:43], a[136:139], a[208:211], v[40:43]\n"
      "ds_read_b64_tr_b16 a[176:177], v2 offset:1024\n"
      "v_mfma_f32_16x16x32_bf16 v[36:39], a[136:139], a[200:203], v[36:39]\n"
      "v_mfma_f32_16x16x32_bf16 v[36:39], a[140:143], a[204:207], v[36:39]\n"
      "v_mfma_f32_16x16x32_bf16 v[32:35], a[140:143], a[196:199], v[32:35]\n"
      "buffer_load_dwordx4 v12, %36, s65 offen offset:2048 lds\n"
      "v_mfma_f32_16x16x32_bf16 v[32:35], a[136:139], a[192:195], v[32:35]\n"
      "ds_read_b64_tr_b16 a[178:179], v2 offset:1536\n"
      "v_mfma_f32_16x16x32_bf16 v[64:67], a[144:147], a[192:195], v[64:67]\n"
      "v_mfma_f32_16x16x32_bf16 v[64:67], a[148:151], a[196:199], v[64:67]\n"
      "v_mfma_f32_16x16x32_bf16 v[68:71], a[148:151], a[204:207], v[68:71]\n"
      "ds_read_b64_tr_b16 a[180:181], v2 offset:17408\n"
      "v_mfma_f32_16x16x32_bf16 v[68:71], a[144:147], a[200:203], v[68:71]\n"
      "v_mfma_f32_16x16x32_bf16 v[72:75], a[144:147], a[208:211], v[72:75]\n"
      "v_mfma_f32_16x16x32_bf16 v[72:75], a[148:151], a[212:215], v[72:75]\n"
      "ds_read_b64_tr_b16 a[182:183], v2 offset:17920\n"
      "v_mfma_f32_16x16x32_bf16 v[76:79], a[148:151], a[220:223], v[76:79]\n"
      "buffer_load_dwordx4 v13, %36, s65 offen offset:3072 lds\n"
      "v_mfma_f32_16x16x32_bf16 v[76:79], a[144:147], a[216:219], v[76:79]\n"
      "v_xor_b32 v2, s69, v2\n"
      "v_mfma_f32_16x16x32_bf16 v[80:83], a[144:147], a[224:227], v[80:83]\n"
      "ds_read_b64_tr_b16 a[184:185], v3 offset:1024\n"
      "v_mfma_f32_16x16x32_bf16 v[80:83], a[148:151], a[228:231], v[80:83]\n"
      "s_add_u32 s71, s71, 1\n"
      "v_mfma_f32_16x16x32_bf16 v[84:87], a[148:151], a[236:239], v[84:87]\n"
      "v_readlane_b32 s65, v22, s71\n"
      "v_mfma_f32_16x16x32_bf16 v[84:87], a[144:147], a[232:235], v[84:87]\n"
      "ds_read_b64_tr_b16 a[186:187], v3 offset:1536\n"
      "v_mfma_f32_16x16x32_bf16 v[88:91], a[144:147], a[240:243], v[88:91]\n"
      "v_readlane_b32 s86, v25, s77\n"
      "v_mfma_f32_16x16x32_bf16 v[88:91], a[148:151], a[244:247], v[88:91]\n"
      "s_and_b32 m0, s86, 14\n"
      "v_mfma_f32_16x16x32_bf16 v[92:95], a[148:151], a[252:255], v[92:95]\n"
      "s_add_u32 s86, s86, s35\n"
      "v_mfma_f32_16x16x32_bf16 v[92:95], a[144:147], a[248:251], v[92:95]\n"
      "ds_read_b64_tr_b16 a[188:189], v3 offset:17408\n"
      "v_mfma_f32_16x16x32_bf16 v[124:127], a[152:155], a[248:251], v[124:127]\n"
      "s_movrels_b64 s[82:83], s[48:49]\n"
      "v_mfma_f32_16x16x32_bf16 v[124:127], a[156:159], a[252:255], v[124:127]\n"
      "s_add_u32 s84, s86, s82\n"
      "s_addc_u32 s85, s83, 0\n"
      "v_mfma_f32_16x16x32_bf16 v[120:123], a[156:159], a[244:247], v[120:123]\n"
      "ds_read_b64_tr_b16 a[190:191], v3 offset:17920\n"
      "v_mfma_f32_16x16x32_bf16 v[120:123], a[152:155], a[240:243], v[120:123]\n"
      "v_xor_b32 v3, s69, v3\n"
      "v_mfma_f32_16x16x32_bf16 v[116:119], a[152:155], a[232:235], v[116:119]\n"
      "v_mfma_f32_16x16x32_bf16 v[116:119], a[156:159], a[236:239], v[116:119]\n"
      "v_mfma_f32_16x16x32_bf16 v[112:115], a[156:159], a[228:231], v[112:115]\n"
      "v_mfma_f32_16x16x32_bf16 v[112:115], a[152:155], a[224:227], v[112:115]\n"
      "v_mfma_f32_16x16x32_bf16 v[108:111], a[152:155], a[216:219], v[108:111]\n"
      "v_mfma_f32_16x16x32_bf16 v[108:111], a[156:159], a[220:223], v[108:111]\n"
      "v_mfma_f32_16x16x32_bf16 v[104:107], a[156:159], a[212:215], v[104:107]\n"
      "v_mfma_f32_16x16x32_bf16 v[104:107], a[152:155], a[208:211], v[104:107]\n"
      "v_mfma_f32_16x16x32_bf16 v[100:103], a[152:155], a[200:203], v[100:103]\n"
      "s_waitcnt vmcnt(12) lgkmcnt(0)\n"
      "s_barrier\n"
      "v_mfma_f32_16x16x32_bf16 v[100:103], a[156:159], a[204:207], v[100:103]\n"
      "v_mfma_f32_16x16x32_bf16 v[96:99], a[156:159], a[196:199], v[96:99]\n"
      "s_mov_b32 m0, s67\n"
      "v_mfma_f32_16x16x32_bf16 v[96:99], a[152:155], a[192:195], v[96:99]\n"
      "v_mfma_f32_16x16x32_bf16 v[128:131], a[160:163], a[192:195], v[128:131]\n"
      "v_cmp_gt_u32 vcc, s75, v29\n"
      "v_mfma_f32_16x16x32_bf16 v[128:131], a[164:167], a[196:199], v[128:131]\n"
      "v_mfma_f32_16x16x32_bf16 v[160:163], a[172:175], a[196:199], v[160:163]\n"
      "s_cbranch_vccnz 143f\n"
      "43:\n"
      "v_mfma_f32_16x16x32_bf16 v[160:163], a[168:171], a[192:195], v[160:163]\n"
      "buffer_load_dwordx4 v14, s[40:43], s64 offen offset:0 sc0 sc1 lds\n"
      "v_mfma_f32_16x16x32_bf16 v[192:195], a[176:179], a[192:195], v[192:195]\n"
      "ds_read_b64_tr_b16 a[128:129], v0 offset:0\n"
      "v_mfma_f32_16x16x32_bf16 v[192:195], a[180:183], a[196:199], v[192:195]\n"
      "s_lshr_b32 s66, s65, 12\n"
      "v_mfma_f32_16x16x32_bf16 v[224:227], a[188:191], a[196:199], v[224:227]\n"
      "s_xor_b32 s69, s69, s70\n"
      "v_mfma_f32_16x16x32_bf16 v[224:227], a[184:187], a[192:195], v[224:227]\n"
      "ds_read_b64_tr_b16 a[130:131], v0 offset:512\n"
      "v_mfma_f32_16x16x32_bf16 v[228:231], a[184:187], a[200:203], v[228:231]\n"
      "buffer_load_dwordx4 v15, s[40:43], s64 offen offset:1024 sc0 sc1 lds\n"
      "v_mfma_f32_16x16x32_bf16 v[228:231], a[188:191], a[204:207], v[228:231]\n"
      "ds_read_b128 a[192:195], v4 offset:0\n"
      "v_mfma_f32_16x16x32_bf16 v[196:199], a[180:183], a[204:207], v[196:199]\n"
      "ds_read_b128 a[196:199], v5 offset:0\n"
      "v_mfma_f32_16x16x32_bf16 v[196:199], a[176:179], a[200:203], v[196:199]\n"
      "ds_read_b64_tr_b16 a[132:133], v0 offset:16384\n"
      "v_mfma_f32_16x16x32_bf16 v[164:167], a[168:171], a[200:203], v[164:167]\n"
      "s_xor_b32 s70, s70, s69\n"
      "v_mfma_f32_16x16x32_bf16 v[164:167], a[172:175], a[204:207], v[164:167]\n"
      "buffer_load_dwordx4 v16, s[40:43], s64 offen offset:2048 sc0 sc1 lds\n"
      "v_mfma_f32_16x16x32_bf16 v[132:135], a[164:167], a[204:207], v[132:135]\n"
      "v_readlane_b32 s86, v24, s77\n"
      "v_mfma_f32_16x16x32_bf16 v[132:135], a[160:163], a[200:203], v[132:135]\n"
      "ds_read_b64_tr_b16 a[134:135], v0 offset:16896\n"
      "v_mfma_f32_16x16x32_bf16 v[136:139], a[160:163], a[208:211], v[136:139]\n"
      "ds_read_b128 a[200:203], v4 offset:2048\n"
      "v_mfma_f32_16x16x32_bf16 v[136:139], a[164:167], a[212:215], v[136:139]\n"
      "ds_read_b128 a[204:207], v5 offset:2048\n"
      "v_mfma_f32_16x16x32_bf16 v[168:171], a[172:175], a[212:215], v[168:171]\n"
      "buffer_load_dwordx4 v17, s[40:43], s64 offen offset:3072 sc0 sc1 lds\n"
      "v_mfma_f32_16x16x32_bf16 v[168:171], a[168:171], a[208:211], v[168:171]\n"
      "s_bitset1_b32 m0, 12\n"
      "v_mfma_f32_16x16x32_bf16 v[200:203], a[176:179], a[208:211], v[200:203]\n"
      "ds_read_b64_tr_b16 a[136:137], v1 offset:0\n"
      "v_mfma_f32_16x16x32_bf16 v[200:203], a[180:183], a[212:215], v[200:203]\n"
      "s_sub_u32 s90, s77, 1\n"
      "v_mfma_f32_16x16x32_bf16 v[232:235], a[188:191], a[212:215], v[232:235]\n"
      "ds_read_b64_tr_b16 a[138:139], v1 offset:512\n"
      "v_mfma_f32_16x16x32_bf16 v[232:235], a[184:187], a[208:211], v[232:235]\n"
      "buffer_load_dwordx4 v18, s[40:43], s64 offen offset:0 sc0 sc1 lds\n"
      "v_mfma_f32_16x16x32_bf16 v[236:239], a[184:187], a[216:219], v[236:239]\n"
      "ds_read_b128 a[208:211], v4 offset:4096\n"
      "v_mfma_f32_16x16x32_bf16 v[236:239], a[188:191], a[220:223], v[236:239]\n"
      "ds_read_b128 a[212:215], v5 offset:4096\n"
      "v_mfma_f32_16x16x32_bf16 v[204:207], a[180:183], a[220:223], v[204:207]\n"
      "ds_read_b64_tr_b16 a[140:141], v1 offset:16384\n"
      "v_mfma_f32_16x16x32_bf16 v[204:207], a[176:179], a[216:219], v[204:207]\n"
      "v_readlane_b32 s73, v27, s77\n"
      "v_mfma_f32_16x16x32_bf16 v[172:175], a[168:171], a[216:219], v[172:175]\n"
      "buffer_load_dwordx4 v19, s[40:43], s64 offen offset:1024 sc0 sc1 lds\n"
      "v_mfma_f32_16x16x32_bf16 v[172:175], a[172:175], a[220:223], v[172:175]\n"
      "ds_read_b64_tr_b16 a[142:143], v1 offset:16896\n"
      "v_mfma_f32_16x16x32_bf16 v[140:143], a[164:167], a[220:223], v[140:143]\n"
      "s_min_u32 s72, s73, s74\n"
      "v_mfma_f32_16x16x32_bf16 v[140:143], a[160:163], a[216:219], v[140:143]\n"
      "v_mfma_f32_16x16x32_bf16 v[144:147], a[160:163], a[224:227], v[144:147]\n"
      "ds_read_b128 a[216:219], v4 offset:6144\n"
      "v_mfma_f32_16x16x32_bf16 v[144:147], a[164:167], a[228:231], v[144:147]\n"
      "buffer_load_dwordx4 v20, s[40:43], s64 offen offset:2048 sc0 sc1 lds\n"
      "v_mfma_f32_16x16x32_bf16 v[176:179], a[172:175], a[228:231], v[176:179]\n"
      "ds_read_b128 a[220:223], v5 offset:6144\n"
      "v_mfma_f32_16x16x32_bf16 v[176:179], a[168:171], a[224:227], v[176:179]\n"
      "ds_read_b64_tr_b16 a[144:145], v2 offset:0\n"
      "v_mfma_f32_16x16x32_bf16 v[208:211], a[176:179], a[224:227], v[208:211]\n"
      "buffer_load_dword v29, v28, s[28:31], s66 offen sc0 sc1\n"
      "v_mfma_f32_16x16x32_bf16 v[208:211], a[180:183], a[228:231], v[208:211]\n"
      "ds_read_b64_tr_b16 a[146:147], v2 offset:512\n"
      "v_mfma_f32_16x16x32_bf16 v[240:243], a[188:191], a[228:231], v[240:243]\n"
      "buffer_load_dwordx4 v21, s[40:43], s64 offen offset:3072 sc0 sc1 lds\n"
      "v_mfma_f32_16x16x32_bf16 v[240:243], a[184:187], a[224:227], v[240:243]\n"
      "s_mov_b32 m0, s68\n"
      "v_mfma_f32_16x16x32_bf16 v[244:247], a[184:187], a[232:235], v[244:247]\n"
      "ds_read_b128 a[224:227], v4 offset:8192\n"
      "v_mfma_f32_16x16x32_bf16 v[244:247], a[188:191], a[236:239], v[244:247]\n"
      "ds_read_b128 a[228:231], v5 offset:8192\n"
      "v_mfma_f32_16x16x32_bf16 v[212:215], a[180:183], a[236:239], v[212:215]\n"
      "buffer_load_dwordx4 a[96:99], v14, %37, s86 offen offset:0\n"
      "v_mfma_f32_16x16x32_bf16 v[212:215], a[176:179], a[232:235], v[212:215]\n"
      "ds_read_b64_tr_b16 a[148:149], v2 offset:16384\n"
      "v_mfma_f32_16x16x32_bf16 v[180:183], a[168:171], a[232:235], v[180:183]\n"
      "ds_read_b64_tr_b16 a[150:151], v2 offset:16896\n"
      "v_mfma_f32_16x16x32_bf16 v[180:183], a[172:175], a[236:239], v[180:183]\n"
      "v_readlane_b32 s64, v23, s71\n"
      "v_mfma_f32_16x16x32_bf16 v[148:151], a[164:167], a[236:239], v[148:151]\n"
      "buffer_load_dwordx4 v6, %36, s65 offen offset:0 lds\n"
      "v_mfma_f32_16x16x32_bf16 v[148:151], a[160:163], a[232:235], v[148:151]\n"
      "s_xor_b32 s67, s67, 0x8000\n"
      "v_mfma_f32_16x16x32_bf16 v[152:155], a[160:163], a[240:243], v[152:155]\n"
      "ds_read_b128 a[232:235], v4 offset:10240\n"
      "v_mfma_f32_16x16x32_bf16 v[152:155], a[164:167], a[244:247], v[152:155]\n"
      "ds_read_b128 a[236:239], v5 offset:10240\n"
      "v_mfma_f32_16x16x32_bf16 v[184:187], a[172:175], a[244:247], v[184:187]\n"
      "buffer_load_dwordx4 a[100:103], v15, %37, s86 offen offset:1024\n"
      "v_mfma_f32_16x16x32_bf16 v[184:187], a[168:171], a[240:243], v[184:187]\n"
      "ds_read_b64_tr_b16 a[152:153], v3 offset:0\n"
      "v_mfma_f32_16x16x32_bf16 v[216:219], a[176:179], a[240:243], v[216:219]\n"
      "ds_read_b64_tr_b16 a[154:155], v3 offset:512\n"
      "v_mfma_f32_16x16x32_bf16 v[216:219], a[180:183], a[244:247], v[216:219]\n"
      "v_mfma_f32_16x16x32_bf16 v[248:251], a[188:191], a[244:247], v[248:251]\n"
      "buffer_load_dwordx4 v7, %36, s65 offen offset:1024 lds\n"
      "v_mfma_f32_16x16x32_bf16 v[248:251], a[184:187], a[240:243], v[248:251]\n"
      "ds_read_b64_tr_b16 a[156:157], v3 offset:16384\n"
      "v_mfma_f32_16x16x32_bf16 v[252:255], a[184:187], a[248:251], v[252:255]\n"
      "ds_read_b128 a[240:243], v4 offset:12288\n"
      "v_mfma_f32_16x16x32_bf16 v[252:255], a[188:191], a[252:255], v[252:255]\n"
      "ds_read_b128 a[244:247], v5 offset:12288\n"
      "v_mfma_f32_16x16x32_bf16 v[220:223], a[180:183], a[252:255], v[220:223]\n"
      "buffer_load_dword a108, v7, %36, s33 offen offset:384\n"
      "v_mfma_f32_16x16x32_bf16 v[220:223], a[176:179], a[248:251], v[220:223]\n"
      "ds_read_b64_tr_b16 a[158:159], v3 offset:16896\n"
      "v_mfma_f32_16x16x32_bf16 v[188:191], a[168:171], a[248:251], v[188:191]\n"
      "s_mul_i32 s1, s0, 14\n"
      "v_mfma_f32_16x16x32_bf16 v[188:191], a[172:175], a[252:255], v[188:191]\n"
      "s_sub_u32 s1, s2, s1\n"
      "v_mfma_f32_16x16x32_bf16 v[156:159], a[164:167], a[252:255], v[156:159]\n"
      "buffer_load_dwordx4 v8, %36, s65 offen offset:2048 lds\n"
      "v_mfma_f32_16x16x32_bf16 v[156:159], a[160:163], a[248:251], v[156:159]\n"
      "s_cmp_lt_i32 s1, s46\n"
      "s_cbranch_scc1 165f\n"
      "65:\n"
      "s_waitcnt lgkmcnt(7)\n"
      "v_mfma_f32_16x16x32_bf16 a[0:3], a[128:131], a[192:195], a[0:3]\n"
      "ds_read_b128 a[248:251], v4 offset:14336\n"
      "v_mfma_f32_16x16x32_bf16 a[0:3], a[132:135], a[196:199], a[0:3]\n"
      "ds_read_b128 a[252:255], v5 offset:14336\n"
      "v_mfma_f32_16x16x32_bf16 a[4:7], a[132:135], a[204:207], a[4:7]\n"
      "buffer_load_dword a108, v7, %36, s33 offen offset:640\n"
      "v_mfma_f32_16x16x32_bf16 a[4:7], a[128:131], a[200:203], a[4:7]\n"
      "ds_read_b64_tr_b16 a[160:161], v0 offset:1024\n"
      "v_mfma_f32_16x16x32_bf16 a[8:11], a[128:131], a[208:211], a[8:11]\n"
      "ds_read_b64_tr_b16 a[162:163], v0 offset:1536\n"
      "v_mfma_f32_16x16x32_bf16 a[8:11], a[132:135], a[212:215], a[8:11]\n"
      "v_xor_b32 v4, 0x8000, v4\n"
      "v_mfma_f32_16x16x32_bf16 a[12:15], a[132:135], a[220:223], a[12:15]\n"
      "buffer_load_dwordx4 v9, %36, s65 offen offset:3072 lds\n"
      "v_mfma_f32_16x16x32_bf16 a[12:15], a[128:131], a[216:219], a[12:15]\n"
      "s_bitset1_b32 m0, 13\n"
      "v_mfma_f32_16x16x32_bf16 a[16:19], a[128:131], a[224:227], a[16:19]\n"
      "ds_read_b64_tr_b16 a[164:165], v0 offset:17408\n"
      "v_mfma_f32_16x16x32_bf16 a[16:19], a[132:135], a[228:231], a[16:19]\n"
      "v_xor_b32 v5, 0x8000, v5\n"
      "s_waitcnt lgkmcnt(6)\n"
      "v_mfma_f32_16x16x32_bf16 a[20:23], a[132:135], a[236:239], a[20:23]\n"
      "ds_read_b64_tr_b16 a[166:167], v0 offset:17920\n"
      "v_mfma_f32_16x16x32_bf16 a[20:23], a[128:131], a[232:235], a[20:23]\n"
      "s_xor_b32 s68, s68, s70\n"
      "v_mfma_f32_16x16x32_bf16 a[24:27], a[128:131], a[240:243], a[24:27]\n"
      "v_xor_b32 v0, s69, v0\n"
      "v_mfma_f32_16x16x32_bf16 a[24:27], a[132:135], a[244:247], a[24:27]\n"
      "s_max_i32 s91, s22, s24\n"
      "s_waitcnt lgkmcnt(4)\n"
      "v_mfma_f32_16x16x32_bf16 a[28:31], a[132:135], a[252:255], a[28:31]\n"
      "buffer_load_dwordx4 v10, %36, s65 offen offset:0 lds\n"
      "v_mfma_f32_16x16x32_bf16 a[28:31], a[128:131], a[248:251], a[28:31]\n"
      "ds_read_b64_tr_b16 a[168:169], v1 offset:1024\n"
      "v_mfma_f32_16x16x32_bf16 v[60:63], a[136:139], a[248:251], v[60:63]\n"
      "v_readlane_b32 s33, v25, s91\n"
      "v_mfma_f32_16x16x32_bf16 v[60:63], a[140:143], a[252:255], v[60:63]\n"
      "ds_read_b64_tr_b16 a[170:171], v1 offset:1536\n"
      "v_mfma_f32_16x16x32_bf16 v[56:59], a[140:143], a[244:247], v[56:59]\n"
      "s_add_u32 s22, s22, 1\n"
      "v_mfma_f32_16x16x32_bf16 v[56:59], a[136:139], a[240:243], v[56:59]\n"
      "s_mov_b32 s46, s1\n"
      "v_mfma_f32_16x16x32_bf16 v[52:55], a[136:139], a[232:235], v[52:55]\n"
      "ds_read_b64_tr_b16 a[172:173], v1 offset:17408\n"
      "v_mfma_f32_16x16x32_bf16 v[52:55], a[140:143], a[236:239], v[52:55]\n"
      "s_memrealtime s[0:1]\n"
      "s_memtime s[2:3]\n"
      "v_mfma_f32_16x16x32_bf16 v[48:51], a[140:143], a[228:231], v[48:51]\n"
      "buffer_load_dwordx4 v11, %36, s65 offen offset:1024 lds\n"
      "v_mfma_f32_16x16x32_bf16 v[48:51], a[136:139], a[224:227], v[48:51]\n"
      "v_mfma_f32_16x16x32_bf16 v[44:47], a[136:139], a[216:219], v[44:47]\n"
      "ds_read_b64_tr_b16 a[174:175], v1 offset:17920\n"
      "v_mfma_f32_16x16x32_bf16 v[44:47], a[140:143], a[220:223], v[44:47]\n"
      "v_xor_b32 v1, s69, v1\n"
      "v_mfma_f32_16x16x32_bf16 v[40:43], a[140:143], a[212:215], v[40:43]\n"
      "v_mfma_f32_16x16x32_bf16 v[40:43], a[136:139], a[208:211], v[40:43]\n"
      "ds_read_b64_tr_b16 a[176:177], v2 offset:1024\n"
      "v_mfma_f32_16x16x32_bf16 v[36:39], a[136:139], a[200:203], v[36:39]\n"
      "v_mfma_f32_16x16x32_bf16 v[36:39], a[140:143], a[204:207], v[36:39]\n"
      "v_mfma_f32_16x16x32_bf16 v[32:35], a[140:143], a[196:199], v[32:35]\n"
      "buffer_load_dwordx4 v12, %36, s65 offen offset:2048 lds\n"
      "v_mfma_f32_16x16x32_bf16 v[32:35], a[136:139], a[192:195], v[32:35]\n"
      "ds_read_b64_tr_b16 a[178:179], v2 offset:1536\n"
      "v_mfma_f32_16x16x32_bf16 v[64:67], a[144:147], a[192:195], v[64:67]\n"
      "v_mfma_f32_16x16x32_bf16 v[64:67], a[148:151], a[196:199], v[64:67]\n"
      "v_mfma_f32_16x16x32_bf16 v[68:71], a[148:151], a[204:207], v[68:71]\n"
      "ds_read_b64_tr_b16 a[180:181], v2 offset:17408\n"
      "v_mfma_f32_16x16x32_bf16 v[68:71], a[144:147], a[200:203], v[68:71]\n"
      "v_mfma_f32_16x16x32_bf16 v[72:75], a[144:147], a[208:211], v[72:75]\n"
      "v_mfma_f32_16x16x32_bf16 v[72:75], a[148:151], a[212:215], v[72:75]\n"
      "ds_read_b64_tr_b16 a[182:183], v2 offset:17920\n"
      "v_mfma_f32_16x16x32_bf16 v[76:79], a[148:151], a[220:223], v[76:79]\n"
      "buffer_load_dwordx4 v13, %36, s65 offen offset:3072 lds\n"
      "v_mfma_f32_16x16x32_bf16 v[76:79], a[144:147], a[216:219], v[76:79]\n"
      "v_xor_b32 v2, s69, v2\n"
      "v_mfma_f32_16x16x32_bf16 v[80:83], a[144:147], a[224:227], v[80:83]\n"
      "ds_read_b64_tr_b16 a[184:185], v3 offset:1024\n"
      "v_mfma_f32_16x16x32_bf16 v[80:83], a[148:151], a[228:231], v[80:83]\n"
      "s_add_u32 s71, s71, 1\n"
      "v_mfma_f32_16x16x32_bf16 v[84:87], a[148:151], a[236:239], v[84:87]\n"
      "s_waitcnt vmcnt(30)\n"
      "s_lshl_b64 exec, 1, s90\n"
      "global_store_dword v26, v31, s[80:81] sc0 sc1\n"
      "s_mov_b64 exec, -1\n"
      "v_mfma_f32_16x16x32_bf16 v[84:87], a[144:147], a[232:235], v[84:87]\n"
      "ds_read_b64_tr_b16 a[186:187], v3 offset:1536\n"
      "v_mfma_f32_16x16x32_bf16 v[88:91], a[144:147], a[240:243], v[88:91]\n"
      "v_readlane_b32 s65, v22, s71\n"
      "v_mfma_f32_16x16x32_bf16 v[88:91], a[148:151], a[244:247], v[88:91]\n"
      "s_mov_b64 s[80:81], s[82:83]\n"
      "v_mfma_f32_16x16x32_bf16 v[92:95], a[148:151], a[252:255], v[92:95]\n"
      "s_cmp_ge_u32 s71, s72\n"
      "v_mfma_f32_16x16x32_bf16 v[92:95], a[144:147], a[248:251], v[92:95]\n"
      "ds_read_b64_tr_b16 a[188:189], v3 offset:17408\n"
      "v_mfma_f32_16x16x32_bf16 v[124:127], a[152:155], a[248:251], v[124:127]\n"
      "s_cselect_b32 s76, -1, s75\n"
      "v_mfma_f32_16x16x32_bf16 v[124:127], a[156:159], a[252:255], v[124:127]\n"
      "v_mfma_f32_16x16x32_bf16 v[120:123], a[156:159], a[244:247], v[120:123]\n"
      "ds_read_b64_tr_b16 a[190:191], v3 offset:17920\n"
      "v_mfma_f32_16x16x32_bf16 v[120:123], a[152:155], a[240:243], v[120:123]\n"
      "v_xor_b32 v3, s69, v3\n"
      "v_mfma_f32_16x16x32_bf16 v[116:119], a[152:155], a[232:235], v[116:119]\n"
      "v_mfma_f32_16x16x32_bf16 v[116:119], a[156:159], a[236:239], v[116:119]\n"
      "v_mfma_f32_16x16x32_bf16 v[112:115], a[156:159], a[228:231], v[112:115]\n"
      "v_mfma_f32_16x16x32_bf16 v[112:115], a[152:155], a[224:227], v[112:115]\n"
      "v_mfma_f32_16x16x32_bf16 v[108:111], a[152:155], a[216:219], v[108:111]\n"
      "v_mfma_f32_16x16x32_bf16 v[108:111], a[156:159], a[220:223], v[108:111]\n"
      "v_mfma_f32_16x16x32_bf16 v[104:107], a[156:159], a[212:215], v[104:107]\n"
      "v_mfma_f32_16x16x32_bf16 v[104:107], a[152:155], a[208:211], v[104:107]\n"
      "v_mfma_f32_16x16x32_bf16 v[100:103], a[152:155], a[200:203], v[100:103]\n"
      "s_waitcnt vmcnt(13) lgkmcnt(0)\n"
      "s_barrier\n"
      "v_mfma_f32_16x16x32_bf16 v[100:103], a[156:159], a[204:207], v[100:103]\n"
      "v_cmp_gt_u32 vcc, s76, v29\n"
      "v_mfma_f32_16x16x32_bf16 v[96:99], a[156:159], a[196:199], v[96:99]\n"
      "s_mov_b32 m0, s67\n"
      "v_mfma_f32_16x16x32_bf16 v[96:99], a[152:155], a[192:195], v[96:99]\n"
      "s_cbranch_vccz 50f\n"
      "s_branch 52f\n"
      "56:\n"
      "v_cmp_gt_u32 vcc, s75, v29\n"
      "s_cbranch_vccz 50f\n"
      "s_getpc_b64 s[78:79]\n"
      "s_branch 73f\n"
      "s_branch 50f\n"
      ".p2align 6\n"
      "50:\n"
      "v_mfma_f32_16x16x32_bf16 v[128:131], a[160:163], a[192:195], v[128:131]\n"
      "v_mfma_f32_16x16x32_bf16 v[128:131], a[164:167], a[196:199], v[128:131]\n"
      "v_mfma_f32_16x16x32_bf16 v[160:163], a[172:175], a[196:199], v[160:163]\n"
      "v_mfma_f32_16x16x32_bf16 v[160:163], a[168:171], a[192:195], v[160:163]\n"
      "buffer_load_dwordx4 v14, s[40:43], s64 offen offset:0 sc0 sc1 lds\n"
      "v_mfma_f32_16x16x32_bf16 v[192:195], a[176:179], a[192:195], v[192:195]\n"
      "ds_read_b64_tr_b16 a[128:129], v0 offset:0\n"
      "v_mfma_f32_16x16x32_bf16 v[192:195], a[180:183], a[196:199], v[192:195]\n"
      "s_lshr_b32 s66, s65, 12\n"
      "v_mfma_f32_16x16x32_bf16 v[224:227], a[188:191], a[196:199], v[224:227]\n"
      "s_xor_b32 s69, s69, s70\n"
      "v_mfma_f32_16x16x32_bf16 v[224:227], a[184:187], a[192:195], v[224:227]\n"
      "ds_read_b64_tr_b16 a[130:131], v0 offset:512\n"
      "v_mfma_f32_16x16x32_bf16 v[228:231], a[184:187], a[200:203], v[228:231]\n"
      "buffer_load_dwordx4 v15, s[40:43], s64 offen offset:1024 sc0 sc1 lds\n"
      "v_mfma_f32_16x16x32_bf16 v[228:231], a[188:191], a[204:207], v[228:231]\n"
      "ds_read_b128 a[192:195], v4 offset:0\n"
      "v_mfma_f32_16x16x32_bf16 v[196:199], a[180:183], a[204:207], v[196:199]\n"
      "ds_read_b128 a[196:199], v5 offset:0\n"
      "v_mfma_f32_16x16x32_bf16 v[196:199], a[176:179], a[200:203], v[196:199]\n"
      "ds_read_b64_tr_b16 a[132:133], v0 offset:16384\n"
      "v_mfma_f32_16x16x32_bf16 v[164:167], a[168:171], a[200:203], v[164:167]\n"
      "s_xor_b32 s70, s70, s69\n"
      "v_mfma_f32_16x16x32_bf16 v[164:167], a[172:175], a[204:207], v[164:167]\n"
      "buffer_load_dwordx4 v16, s[40:43], s64 offen offset:2048 sc0 sc1 lds\n"
      "v_mfma_f32_16x16x32_bf16 v[132:135], a[164:167], a[204:207], v[132:135]\n"
      "v_mfma_f32_16x16x32_bf16 v[132:135], a[160:163], a[200:203], v[132:135]\n"
      "ds_read_b64_tr_b16 a[134:135], v0 offset:16896\n"
      "v_mfma_f32_16x16x32_bf16 v[136:139], a[160:163], a[208:211], v[136:139]\n"
      "ds_read_b128 a[200:203], v4 offset:2048\n"
      "v_mfma_f32_16x16x32_bf16 v[136:139], a[164:167], a[212:215], v[136:139]\n"
      "ds_read_b128 a[204:207], v5 offset:2048\n"
      "v_mfma_f32_16x16x32_bf16 v[168:171], a[172:175], a[212:215], v[168:171]\n"
      "buffer_load_dwordx4 v17, s[40:43], s64 offen offset:3072 sc0 sc1 lds\n"
      "v_mfma_f32_16x16x32_bf16 v[168:171], a[168:171], a[208:211], v[168:171]\n"
      "s_bitset1_b32 m0, 12\n"
      "v_mfma_f32_16x16x32_bf16 v[200:203], a[176:179], a[208:211], v[200:203]\n"
      "ds_read_b64_tr_b16 a[136:137], v1 offset:0\n"
      "v_mfma_f32_16x16x32_bf16 v[200:203], a[180:183], a[212:215], v[200:203]\n"
      "v_mfma_f32_16x16x32_bf16 v[232:235], a[188:191], a[212:215], v[232:235]\n"
      "ds_read_b64_tr_b16 a[138:139], v1 offset:512\n"
      "v_mfma_f32_16x16x32_bf16 v[232:235], a[184:187], a[208:211], v[232:235]\n"
      "buffer_load_dwordx4 v18, s[40:43], s64 offen offset:0 sc0 sc1 lds\n"
      "v_mfma_f32_16x16x32_bf16 v[236:239], a[184:187], a[216:219], v[236:239]\n"
      "ds_read_b128 a[208:211], v4 offset:4096\n"
      "v_mfma_f32_16x16x32_bf16 v[236:239], a[188:191], a[220:223], v[236:239]\n"
      "ds_read_b128 a[212:215], v5 offset:4096\n"
      "v_mfma_f32_16x16x32_bf16 v[204:207], a[180:183], a[220:223], v[204:207]\n"
      "ds_read_b64_tr_b16 a[140:141], v1 offset:16384\n"
      "v_mfma_f32_16x16x32_bf16 v[204:207], a[176:179], a[216:219], v[204:207]\n"
      "v_mfma_f32_16x16x32_bf16 v[172:175], a[168:171], a[216:219], v[172:175]\n"
      "buffer_load_dwordx4 v19, s[40:43], s64 offen offset:1024 sc0 sc1 lds\n"
      "v_mfma_f32_16x16x32_bf16 v[172:175], a[172:175], a[220:223], v[172:175]\n"
      "ds_read_b64_tr_b16 a[142:143], v1 offset:16896\n"
      "v_mfma_f32_16x16x32_bf16 v[140:143], a[164:167], a[220:223], v[140:143]\n"
      "v_mfma_f32_16x16x32_bf16 v[140:143], a[160:163], a[216:219], v[140:143]\n"
      "v_mfma_f32_16x16x32_bf16 v[144:147], a[160:163], a[224:227], v[144:147]\n"
      "ds_read_b128 a[216:219], v4 offset:6144\n"
      "v_mfma_f32_16x16x32_bf16 v[144:147], a[164:167], a[228:231], v[144:147]\n"
      "buffer_load_dwordx4 v20, s[40:43], s64 offen offset:2048 sc0 sc1 lds\n"
      "v_mfma_f32_16x16x32_bf16 v[176:179], a[172:175], a[228:231], v[176:179]\n"
      "ds_read_b128 a[220:223], v5 offset:6144\n"
      "v_mfma_f32_16x16x32_bf16 v[176:179], a[168:171], a[224:227], v[176:179]\n"
      "ds_read_b64_tr_b16 a[144:145], v2 offset:0\n"
      "v_mfma_f32_16x16x32_bf16 v[208:211], a[176:179], a[224:227], v[208:211]\n"
      "buffer_load_dword v29, v28, s[28:31], s66 offen sc0 sc1\n"
      "v_mfma_f32_16x16x32_bf16 v[208:211], a[180:183], a[228:231], v[208:211]\n"
      "ds_read_b64_tr_b16 a[146:147], v2 offset:512\n"
      "v_mfma_f32_16x16x32_bf16 v[240:243], a[188:191], a[228:231], v[240:243]\n"
      "buffer_load_dwordx4 v21, s[40:43], s64 offen offset:3072 sc0 sc1 lds\n"
      "v_mfma_f32_16x16x32_bf16 v[240:243], a[184:187], a[224:227], v[240:243]\n"
      "s_mov_b32 m0, s68\n"
      "v_mfma_f32_16x16x32_bf16 v[244:247], a[184:187], a[232:235], v[244:247]\n"
      "ds_read_b128 a[224:227], v4 offset:8192\n"
      "v_mfma_f32_16x16x32_bf16 v[244:247], a[188:191], a[236:239], v[244:247]\n"
      "ds_read_b128 a[228:231], v5 offset:8192\n"
      "v_mfma_f32_16x16x32_bf16 v[212:215], a[180:183], a[236:239], v[212:215]\n"
      "ds_read_b64_tr_b16 a[148:149], v2 offset:16384\n"
      "v_mfma_f32_16x16x32_bf16 v[212:215], a[176:179], a[232:235], v[212:215]\n"
      "v_readlane_b32 s64, v23, s71\n"
      "v_mfma_f32_16x16x32_bf16 v[180:183], a[168:171], a[232:235], v[180:183]\n"
      "ds_read_b64_tr_b16 a[150:151], v2 offset:16896\n"
      "v_mfma_f32_16x16x32_bf16 v[180:183], a[172:175], a[236:239], v[180:183]\n"
      "s_xor_b32 s67, s67, 0x8000\n"
      "v_mfma_f32_16x16x32_bf16 v[148:151], a[164:167], a[236:239], v[148:151]\n"
      "buffer_load_dwordx4 v6, %36, s65 offen offset:0 lds\n"
      "v_mfma_f32_16x16x32_bf16 v[148:151], a[160:163], a[232:235], v[148:151]\n"
      "v_mfma_f32_16x16x32_bf16 v[152:155], a[160:163], a[240:243], v[152:155]\n"
      "ds_read_b128 a[232:235], v4 offset:10240\n"
      "v_mfma_f32_16x16x32_bf16 v[152:155], a[164:167], a[244:247], v[152:155]\n"
      "ds_read_b128 a[236:239], v5 offset:10240\n"
      "v_mfma_f32_16x16x32_bf16 v[184:187], a[172:175], a[244:247], v[184:187]\n"
      "ds_read_b64_tr_b16 a[152:153], v3 offset:0\n"
      "v_mfma_f32_16x16x32_bf16 v[184:187], a[168:171], a[240:243], v[184:187]\n"
      "v_mfma_f32_16x16x32_bf16 v[216:219], a[176:179], a[240:243], v[216:219]\n"
      "ds_read_b64_tr_b16 a[154:155], v3 offset:512\n"
      "v_mfma_f32_16x16x32_bf16 v[216:219], a[180:183], a[244:247], v[216:219]\n"
      "v_mfma_f32_16x16x32_bf16 v[248:251], a[188:191], a[244:247], v[248:251]\n"
      "buffer_load_dwordx4 v7, %36, s65 offen offset:1024 lds\n"
      "v_mfma_f32_16x16x32_bf16 v[248:251], a[184:187], a[240:243], v[248:251]\n"
      "ds_read_b64_tr_b16 a[156:157], v3 offset:16384\n"
      "v_mfma_f32_16x16x32_bf16 v[252:255], a[184:187], a[248:251], v[252:255]\n"
      "ds_read_b128 a[240:243], v4 offset:12288\n"
      "v_mfma_f32_16x16x32_bf16 v[252:255], a[188:191], a[252:255], v[252:255]\n"
      "ds_read_b128 a[244:247], v5 offset:12288\n"
      "v_mfma_f32_16x16x32_bf16 v[220:223], a[180:183], a[252:255], v[220:223]\n"
      "buffer_load_dword a108, v7, %36, s33 offen offset:384\n"
      "v_mfma_f32_16x16x32_bf16 v[220:223], a[176:179], a[248:251], v[220:223]\n"
      "ds_read_b64_tr_b16 a[158:159], v3 offset:16896\n"
      "v_mfma_f32_16x16x32_bf16 v[188:191], a[168:171], a[248:251], v[188:191]\n"
      "s_mul_i32 s1, s0, 14\n"
      "v_mfma_f32_16x16x32_bf16 v[188:191], a[172:175], a[252:255], v[188:191]\n"
      "s_sub_u32 s1, s2, s1\n"
      "v_mfma_f32_16x16x32_bf16 v[156:159], a[164:167], a[252:255], v[156:159]\n"
      "buffer_load_dwordx4 v8, %36, s65 offen offset:2048 lds\n"
      "v_mfma_f32_16x16x32_bf16 v[156:159], a[160:163], a[248:251], v[156:159]\n"
      "s_cmp_lt_i32 s1, s46\n"
      "s_cbranch_scc1 163f\n"
      "63:\n"
      "s_waitcnt lgkmcnt(7)\n"
      "v_mfma_f32_16x16x32_bf16 a[0:3], a[128:131], a[192:195], a[0:3]\n"
      "ds_read_b128 a[248:251], v4 offset:14336\n"
      "v_mfma_f32_16x16x32_bf16 a[0:3], a[132:135], a[196:199], a[0:3]\n"
      "ds_read_b128 a[252:255], v5 offset:14336\n"
      "v_mfma_f32_16x16x32_bf16 a[4:7], a[132:135], a[204:207], a[4:7]\n"
      "buffer_load_dword a108, v7, %36, s33 offen offset:640\n"
      "v_mfma_f32_16x16x32_bf16 a[4:7], a[128:131], a[200:203], a[4:7]\n"
      "ds_read_b64_tr_b16 a[160:161], v0 offset:1024\n"
      "v_mfma_f32_16x16x32_bf16 a[8:11], a[128:131], a[208:211], a[8:11]\n"
      "ds_read_b64_tr_b16 a[162:163], v0 offset:1536\n"
      "v_mfma_f32_16x16x32_bf16 a[8:11], a[132:135], a[212:215], a[8:11]\n"
      "v_xor_b32 v4, 0x8000, v4\n"
      "v_mfma_f32_16x16x32_bf16 a[12:15], a[132:135], a[220:223], a[12:15]\n"
      "buffer_load_dwordx4 v9, %36, s65 offen offset:3072 lds\n"
      "v_mfma_f32_16x16x32_bf16 a[12:15], a[128:131], a[216:219], a[12:15]\n"
      "s_bitset1_b32 m0, 13\n"
      "v_mfma_f32_16x16x32_bf16 a[16:19], a[128:131], a[224:227], a[16:19]\n"
      "ds_read_b64_tr_b16 a[164:165], v0 offset:17408\n"
      "v_mfma_f32_16x16x32_bf16 a[16:19], a[132:135], a[228:231], a[16:19]\n"
      "v_xor_b32 v5, 0x8000, v5\n"
      "s_waitcnt lgkmcnt(6)\n"
      "v_mfma_f32_16x16x32_bf16 a[20:23], a[132:135], a[236:239], a[20:23]\n"
      "ds_read_b64_tr_b16 a[166:167], v0 offset:17920\n"
      "v_mfma_f32_16x16x32_bf16 a[20:23], a[128:131], a[232:235], a[20:23]\n"
      "s_xor_b32 s68, s68, s70\n"
      "v_mfma_f32_16x16x32_bf16 a[24:27], a[128:131], a[240:243], a[24:27]\n"
      "v_xor_b32 v0, s69, v0\n"
      "v_mfma_f32_16x16x32_bf16 a[24:27], a[132:135], a[244:247], a[24:27]\n"
      "s_max_i32 s91, s22, s24\n"
      "s_waitcnt lgkmcnt(4)\n"
      "v_mfma_f32_16x16x32_bf16 a[28:31], a[132:135], a[252:255], a[28:31]\n"
      "buffer_load_dwordx4 v10, %36, s65 offen offset:0 lds\n"
      "v_mfma_f32_16x16x32_bf16 a[28:31], a[128:131], a[248:251], a[28:31]\n"
      "ds_read_b64_tr_b16 a[168:169], v1 offset:1024\n"
      "v_mfma_f32_16x16x32_bf16 v[60:63], a[136:139], a[248:251], v[60:63]\n"
      "v_readlane_b32 s33, v25, s91\n"
      "v_mfma_f32_16x16x32_bf16 v[60:63], a[140:143], a[252:255], v[60:63]\n"
      "ds_read_b64_tr_b16 a[170:171], v1 offset:1536\n"
      "v_mfma_f32_16x16x32_bf16 v[56:59], a[140:143], a[244:247], v[56:59]\n"
      "s_add_u32 s22, s22, 1\n"
      "v_mfma_f32_16x16x32_bf16 v[56:59], a[136:139], a[240:243], v[56:59]\n"
      "s_mov_b32 s46, s1\n"
      "v_mfma_f32_16x16x32_bf16 v[52:55], a[136:139], a[232:235], v[52:55]\n"
      "ds_read_b64_tr_b16 a[172:173], v1 offset:17408\n"
      "v_mfma_f32_16x16x32_bf16 v[52:55], a[140:143], a[236:239], v[52:55]\n"
      "s_memrealtime s[0:1]\n"
      "s_memtime s[2:3]\n"
      "v_mfma_f32_16x16x32_bf16 v[48:51], a[140:143], a[228:231], v[48:51]\n"
      "buffer_load_dwordx4 v11, %36, s65 offen offset:1024 lds\n"
      "v_mfma_f32_16x16x32_bf16 v[48:51], a[136:139], a[224:227], v[48:51]\n"
      "v_mfma_f32_16x16x32_bf16 v[44:47], a[136:139], a[216:219], v[44:47]\n"
      "ds_read_b64_tr_b16 a[174:175], v1 offset:17920\n"
      "v_mfma_f32_16x16x32_bf16 v[44:47], a[140:143], a[220:223], v[44:47]\n"
      "v_xor_b32 v1, s69, v1\n"
      "v_mfma_f32_16x16x32_bf16 v[40:43], a[140:143], a[212:215], v[40:43]\n"
      "v_mfma_f32_16x16x32_bf16 v[40:43], a[136:139], a[208:211], v[40:43]\n"
      "ds_read_b64_tr_b16 a[176:177], v2 offset:1024\n"
      "v_mfma_f32_16x16x32_bf16 v[36:39], a[136:139], a[200:203], v[36:39]\n"
      "v_mfma_f32_16x16x32_bf16 v[36:39], a[140:143], a[204:207], v[36:39]\n"
      "v_mfma_f32_16x16x32_bf16 v[32:35], a[140:143], a[196:199], v[32:35]\n"
      "buffer_load_dwordx4 v12, %36, s65 offen offset:2048 lds\n"
      "v_mfma_f32_16x16x32_bf16 v[32:35], a[136:139], a[192:195], v[32:35]\n"
      "ds_read_b64_tr_b16 a[178:179], v2 offset:1536\n"
      "v_mfma_f32_16x16x32_bf16 v[64:67], a[144:147], a[192:195], v[64:67]\n"
      "v_mfma_f32_16x16x32_bf16 v[64:67], a[148:151], a[196:199], v[64:67]\n"
      "v_mfma_f32_16x16x32_bf16 v[68:71], a[148:151], a[204:207], v[68:71]\n"
      "ds_read_b64_tr_b16 a[180:181], v2 offset:17408\n"
      "v_mfma_f32_16x16x32_bf16 v[68:71], a[144:147], a[200:203], v[68:71]\n"
      "v_mfma_f32_16x16x32_bf16 v[72:75], a[144:147], a[208:211], v[72:75]\n"
      "v_mfma_f32_16x16x32_bf16 v[72:75], a[148:151], a[212:215], v[72:75]\n"
      "ds_read_b64_tr_b16 a[182:183], v2 offset:17920\n"
      "v_mfma_f32_16x16x32_bf16 v[76:79], a[148:151], a[220:223], v[76:79]\n"
      "buffer_load_dwordx4 v13, %36, s65 offen offset:3072 lds\n"
      "v_mfma_f32_16x16x32_bf16 v[76:79], a[144:147], a[216:219], v[76:79]\n"
      "v_xor_b32 v2, s69, v2\n"
      "v_mfma_f32_16x16x32_bf16 v[80:83], a[144:147], a[224:227], v[80:83]\n"
      "ds_read_b64_tr_b16 a[184:185], v3 offset:1024\n"
      "v_mfma_f32_16x16x32_bf16 v[80:83], a[148:151], a[228:231], v[80:83]\n"
      "s_add_u32 s71, s71, 1\n"
      "v_mfma_f32_16x16x32_bf16 v[84:87], a[148:151], a[236:239], v[84:87]\n"
      "v_readlane_b32 s65, v22, s71\n"
      "v_mfma_f32_16x16x32_bf16 v[84:87], a[144:147], a[232:235], v[84:87]\n"
      "ds_read_b64_tr_b16 a[186:187], v3 offset:1536\n"
      "v_mfma_f32_16x16x32_bf16 v[88:91], a[144:147], a[240:243], v[88:91]\n"
      "s_cmp_ge_u32 s71, s72\n"
      "v_mfma_f32_16x16x32_bf16 v[88:91], a[148:151], a[244:247], v[88:91]\n"
      "s_cselect_b32 s76, -1, s75\n"
      "v_mfma_f32_16x16x32_bf16 v[92:95], a[148:151], a[252:255], v[92:95]\n"
      "v_mfma_f32_16x16x32_bf16 v[92:95], a[144:147], a[248:251], v[92:95]\n"
      "ds_read_b64_tr_b16 a[188:189], v3 offset:17408\n"
      "v_mfma_f32_16x16x32_bf16 v[124:127], a[152:155], a[248:251], v[124:127]\n"
      "v_mfma_f32_16x16x32_bf16 v[124:127], a[156:159], a[252:255], v[124:127]\n"
      "v_mfma_f32_16x16x32_bf16 v[120:123], a[156:159], a[244:247], v[120:123]\n"
      "ds_read_b64_tr_b16 a[190:191], v3 offset:17920\n"
      "v_mfma_f32_16x16x32_bf16 v[120:123], a[152:155], a[240:243], v[120:123]\n"
      "v_xor_b32 v3, s69, v3\n"
      "v_mfma_f32_16x16x32_bf16 v[116:119], a[152:155], a[232:235], v[116:119]\n"
      "v_mfma_f32_16x16x32_bf16 v[116:119], a[156:159], a[236:239], v[116:119]\n"
      "v_mfma_f32_16x16x32_bf16 v[112:115], a[156:159], a[228:231], v[112:115]\n"
      "v_mfma_f32_16x16x32_bf16 v[112:115], a[152:155], a[224:227], v[112:115]\n"
      "v_mfma_f32_16x16x32_bf16 v[108:111], a[152:155], a[216:219], v[108:111]\n"
      "v_mfma_f32_16x16x32_bf16 v[108:111], a[156:159], a[220:223], v[108:111]\n"
      "v_mfma_f32_16x16x32_bf16 v[104:107], a[156:159], a[212:215], v[104:107]\n"
      "v_mfma_f32_16x16x32_bf16 v[104:107], a[152:155], a[208:211], v[104:107]\n"
      "v_mfma_f32_16x16x32_bf16 v[100:103], a[152:155], a[200:203], v[100:103]\n"
      "s_waitcnt vmcnt(10) lgkmcnt(0)\n"
      "s_barrier\n"
      "v_mfma_f32_16x16x32_bf16 v[100:103], a[156:159], a[204:207], v[100:103]\n"
      "v_cmp_gt_u32 vcc, s76, v29\n"
      "v_mfma_f32_16x16x32_bf16 v[96:99], a[156:159], a[196:199], v[96:99]\n"
      "s_mov_b32 m0, s67\n"
      "v_mfma_f32_16x16x32_bf16 v[96:99], a[152:155], a[192:195], v[96:99]\n"
      "s_cbranch_vccz 50b\n"
      "52:\n"
      "s_cmp_ge_u32 s71, s74\n"
      "s_cbranch_scc0 57b\n"
      "24:\n"
      "s_bitcmp1_b32 s25, 0\n"
      "s_cbranch_scc1 58f\n"
      "v_mfma_f32_16x16x32_bf16 v[128:131], a[160:163], a[192:195], v[128:131]\n"
      "v_cmp_gt_u32 vcc, s75, v29\n"
      "v_mfma_f32_16x16x32_bf16 v[128:131], a[164:167], a[196:199], v[128:131]\n"
      "v_mfma_f32_16x16x32_bf16 v[160:163], a[172:175], a[196:199], v[160:163]\n"
      "s_cbranch_vccnz 144f\n"
      "44:\n"
      "v_mfma_f32_16x16x32_bf16 v[160:163], a[168:171], a[192:195], v[160:163]\n"
      "buffer_load_dwordx4 v14, s[40:43], s64 offen offset:0 sc0 sc1 lds\n"
      "v_mfma_f32_16x16x32_bf16 v[192:195], a[176:179], a[192:195], v[192:195]\n"
      "v_accvgpr_read_b32 v22, a104\n"
      "v_mfma_f32_16x16x32_bf16 v[192:195], a[180:183], a[196:199], v[192:195]\n"
      "v_lshrrev_b32 v28, 12, v22\n"
      "v_mfma_f32_16x16x32_bf16 v[224:227], a[188:191], a[196:199], v[224:227]\n"
      "s_xor_b32 s69, s69, s70\n"
      "v_mfma_f32_16x16x32_bf16 v[224:227], a[184:187], a[192:195], v[224:227]\n"
      "s_xor_b32 s70, s70, s69\n"
      "v_mfma_f32_16x16x32_bf16 v[228:231], a[184:187], a[200:203], v[228:231]\n"
      "buffer_load_dwordx4 v15, s[40:43], s64 offen offset:1024 sc0 sc1 lds\n"
      "v_mfma_f32_16x16x32_bf16 v[228:231], a[188:191], a[204:207], v[228:231]\n"
      "s_and_b32 s52, s68, 0x38000\n"
      "v_mfma_f32_16x16x32_bf16 v[196:199], a[180:183], a[204:207], v[196:199]\n"
      "s_xor_b32 s53, s52, s70\n"
      "v_mfma_f32_16x16x32_bf16 v[196:199], a[176:179], a[200:203], v[196:199]\n"
      "s_xor_b32 s54, s53, s69\n"
      "v_mfma_f32_16x16x32_bf16 v[164:167], a[168:171], a[200:203], v[164:167]\n"
      "s_and_b32 s50, s67, 0x8000\n"
      "v_mfma_f32_16x16x32_bf16 v[164:167], a[172:175], a[204:207], v[164:167]\n"
      "buffer_load_dwordx4 v16, s[40:43], s64 offen offset:2048 sc0 sc1 lds\n"
      "v_mfma_f32_16x16x32_bf16 v[132:135], a[164:167], a[204:207], v[132:135]\n"
      "s_xor_b32 s51, s50, 0x8000\n"
      "v_mfma_f32_16x16x32_bf16 v[132:135], a[160:163], a[200:203], v[132:135]\n"
      "s_mov_b32 s66, 0\n"
      "v_mfma_f32_16x16x32_bf16 v[136:139], a[160:163], a[208:211], v[136:139]\n"
      "v_lshrrev_b32 v30, 10, v30\n"
      "v_mfma_f32_16x16x32_bf16 v[136:139], a[164:167], a[212:215], v[136:139]\n"
      "ds_read_b128 a[192:195], v4 offset:0\n"
      "v_mfma_f32_16x16x32_bf16 v[168:171], a[172:175], a[212:215], v[168:171]\n"
      "buffer_load_dwordx4 v17, s[40:43], s64 offen offset:3072 sc0 sc1 lds\n"
      "v_mfma_f32_16x16x32_bf16 v[168:171], a[168:171], a[208:211], v[168:171]\n"
      "s_bitset1_b32 m0, 12\n"
      "v_mfma_f32_16x16x32_bf16 v[200:203], a[176:179], a[208:211], v[200:203]\n"
      "ds_read_b128 a[196:199], v5 offset:0\n"
      "v_mfma_f32_16x16x32_bf16 v[200:203], a[180:183], a[212:215], v[200:203]\n"
      "ds_read_b128 a[200:203], v4 offset:2048\n"
      "v_mfma_f32_16x16x32_bf16 v[232:235], a[188:191], a[212:215], v[232:235]\n"
      "ds_read_b128 a[204:207], v5 offset:2048\n"
      "v_mfma_f32_16x16x32_bf16 v[232:235], a[184:187], a[208:211], v[232:235]\n"
      "buffer_load_dwordx4 v18, s[40:43], s64 offen offset:0 sc0 sc1 lds\n"
      "v_mfma_f32_16x16x32_bf16 v[236:239], a[184:187], a[216:219], v[236:239]\n"
      "ds_read_b128 a[208:211], v4 offset:4096\n"
      "v_mfma_f32_16x16x32_bf16 v[236:239], a[188:191], a[220:223], v[236:239]\n"
      "ds_read_b128 a[212:215], v5 offset:4096\n"
      "v_mfma_f32_16x16x32_bf16 v[204:207], a[180:183], a[220:223], v[204:207]\n"
      "ds_read_b64_tr_b16 a[128:129], v0 offset:0\n"
      "v_mfma_f32_16x16x32_bf16 v[204:207], a[176:179], a[216:219], v[204:207]\n"
      "ds_read_b64_tr_b16 a[130:131], v0 offset:512\n"
      "v_mfma_f32_16x16x32_bf16 v[172:175], a[168:171], a[216:219], v[172:175]\n"
      "buffer_load_dwordx4 v19, s[40:43], s64 offen offset:1024 sc0 sc1 lds\n"
      "v_mfma_f32_16x16x32_bf16 v[172:175], a[172:175], a[220:223], v[172:175]\n"
      "ds_read_b64_tr_b16 a[132:133], v0 offset:16384\n"
      "v_mfma_f32_16x16x32_bf16 v[140:143], a[164:167], a[220:223], v[140:143]\n"
      "ds_read_b64_tr_b16 a[134:135], v0 offset:16896\n"
      "v_mfma_f32_16x16x32_bf16 v[140:143], a[160:163], a[216:219], v[140:143]\n"
      "ds_read_b64_tr_b16 a[136:137], v1 offset:0\n"
      "v_mfma_f32_16x16x32_bf16 v[144:147], a[160:163], a[224:227], v[144:147]\n"
      "ds_read_b128 a[216:219], v4 offset:6144\n"
      "v_mfma_f32_16x16x32_bf16 v[144:147], a[164:167], a[228:231], v[144:147]\n"
      "buffer_load_dwordx4 v20, s[40:43], s64 offen offset:2048 sc0 sc1 lds\n"
      "v_mfma_f32_16x16x32_bf16 v[176:179], a[172:175], a[228:231], v[176:179]\n"
      "ds_read_b128 a[220:223], v5 offset:6144\n"
      "v_mfma_f32_16x16x32_bf16 v[176:179], a[168:171], a[224:227], v[176:179]\n"
      "ds_read_b64_tr_b16 a[138:139], v1 offset:512\n"
      "v_mfma_f32_16x16x32_bf16 v[208:211], a[176:179], a[224:227], v[208:211]\n"
      "buffer_load_dwordx4 v[24:27], v28, s[28:31], 0 offen sc0 sc1\n"
      "v_mfma_f32_16x16x32_bf16 v[208:211], a[180:183], a[228:231], v[208:211]\n"
      "ds_read_b64_tr_b16 a[140:141], v1 offset:16384\n"
      "v_mfma_f32_16x16x32_bf16 v[240:243], a[188:191], a[228:231], v[240:243]\n"
      "buffer_load_dwordx4 v21, s[40:43], s64 offen offset:3072 sc0 sc1 lds\n"
      "v_mfma_f32_16x16x32_bf16 v[240:243], a[184:187], a[224:227], v[240:243]\n"
      "ds_read_b64_tr_b16 a[142:143], v1 offset:16896\n"
      "v_mfma_f32_16x16x32_bf16 v[244:247], a[184:187], a[232:235], v[244:247]\n"
      "ds_read_b128 a[224:227], v4 offset:8192\n"
      "v_mfma_f32_16x16x32_bf16 v[244:247], a[188:191], a[236:239], v[244:247]\n"
      "ds_read_b128 a[228:231], v5 offset:8192\n"
      "v_mfma_f32_16x16x32_bf16 v[212:215], a[180:183], a[236:239], v[212:215]\n"
      "ds_read_b64_tr_b16 a[144:145], v2 offset:0\n"
      "v_mfma_f32_16x16x32_bf16 v[212:215], a[176:179], a[232:235], v[212:215]\n"
      "ds_read_b64_tr_b16 a[146:147], v2 offset:512\n"
      "v_mfma_f32_16x16x32_bf16 v[180:183], a[168:171], a[232:235], v[180:183]\n"
      "ds_read_b64_tr_b16 a[148:149], v2 offset:16384\n"
      "v_mfma_f32_16x16x32_bf16 v[180:183], a[172:175], a[236:239], v[180:183]\n"
      "s_or_b32 m0, s52, %42\n"
      "v_mfma_f32_16x16x32_bf16 v[148:151], a[164:167], a[236:239], v[148:151]\n"
      "buffer_load_dwordx4 v7, %36, s65 offen offset:1024 lds\n"
      "v_mfma_f32_16x16x32_bf16 v[148:151], a[160:163], a[232:235], v[148:151]\n"
      "ds_read_b64_tr_b16 a[150:151], v2 offset:16896\n"
      "v_mfma_f32_16x16x32_bf16 v[152:155], a[160:163], a[240:243], v[152:155]\n"
      "ds_read_b128 a[232:235], v4 offset:10240\n"
      "v_mfma_f32_16x16x32_bf16 v[152:155], a[164:167], a[244:247], v[152:155]\n"
      "ds_read_b128 a[236:239], v5 offset:10240\n"
      "v_mfma_f32_16x16x32_bf16 v[184:187], a[172:175], a[244:247], v[184:187]\n"
      "ds_read_b64_tr_b16 a[152:153], v3 offset:0\n"
      "v_mfma_f32_16x16x32_bf16 v[184:187], a[168:171], a[240:243], v[184:187]\n"
      "s_xor_b32 s55, s50, s51\n"
      "v_mfma_f32_16x16x32_bf16 v[216:219], a[176:179], a[240:243], v[216:219]\n"
      "ds_read_b64_tr_b16 a[154:155], v3 offset:512\n"
      "v_mfma_f32_16x16x32_bf16 v[216:219], a[180:183], a[244:247], v[216:219]\n"
      "s_xor_b32 s56, s54, s53\n"
      "v_mfma_f32_16x16x32_bf16 v[248:251], a[188:191], a[244:247], v[248:251]\n"
      "buffer_load_dwordx4 v9, %36, s65 offen offset:3072 lds\n"
      "v_mfma_f32_16x16x32_bf16 v[248:251], a[184:187], a[240:243], v[248:251]\n"
      "s_bitset1_b32 m0, 13\n"
      "v_mfma_f32_16x16x32_bf16 v[252:255], a[184:187], a[248:251], v[252:255]\n"
      "ds_read_b128 a[240:243], v4 offset:12288\n"
      "v_mfma_f32_16x16x32_bf16 v[252:255], a[188:191], a[252:255], v[252:255]\n"
      "ds_read_b128 a[244:247], v5 offset:12288\n"
      "v_mfma_f32_16x16x32_bf16 v[220:223], a[180:183], a[252:255], v[220:223]\n"
      "ds_read_b64_tr_b16 a[156:157], v3 offset:16384\n"
      "v_mfma_f32_16x16x32_bf16 v[220:223], a[176:179], a[248:251], v[220:223]\n"
      "ds_read_b64_tr_b16 a[158:159], v3 offset:16896\n"
      "v_mfma_f32_16x16x32_bf16 v[188:191], a[168:171], a[248:251], v[188:191]\n"
      "v_mfma_f32_16x16x32_bf16 v[188:191], a[172:175], a[252:255], v[188:191]\n"
      "v_mfma_f32_16x16x32_bf16 v[156:159], a[164:167], a[252:255], v[156:159]\n"
      "buffer_load_dwordx4 v11, %36, s65 offen offset:1024 lds\n"
      "v_mfma_f32_16x16x32_bf16 v[156:159], a[160:163], a[248:251], v[156:159]\n"
      "s_waitcnt lgkmcnt(7)\n"
      "v_mfma_f32_16x16x32_bf16 a[0:3], a[128:131], a[192:195], a[0:3]\n"
      "ds_read_b128 a[248:251], v4 offset:14336\n"
      "v_mfma_f32_16x16x32_bf16 a[0:3], a[132:135], a[196:199], a[0:3]\n"
      "ds_read_b128 a[252:255], v5 offset:14336\n"
      "v_mfma_f32_16x16x32_bf16 a[4:7], a[132:135], a[204:207], a[4:7]\n"
      "ds_read_b64_tr_b16 a[160:161], v0 offset:1024\n"
      "v_xor_b32 v4, s55, v4\n"
      "v_mfma_f32_16x16x32_bf16 a[4:7], a[128:131], a[200:203], a[4:7]\n"
      "v_xor_b32 v5, s55, v5\n"
      "v_mfma_f32_16x16x32_bf16 a[8:11], a[128:131], a[208:211], a[8:11]\n"
      "ds_read_b64_tr_b16 a[162:163], v0 offset:1536\n"
      "v_mfma_f32_16x16x32_bf16 a[8:11], a[132:135], a[212:215], a[8:11]\n"
      "v_mfma_f32_16x16x32_bf16 a[12:15], a[132:135], a[220:223], a[12:15]\n"
      "buffer_load_dwordx4 v13, %36, s65 offen offset:3072 lds\n"
      "v_mfma_f32_16x16x32_bf16 a[12:15], a[128:131], a[216:219], a[12:15]\n"
      "ds_read_b64_tr_b16 a[164:165], v0 offset:17408\n"
      "v_mfma_f32_16x16x32_bf16 a[16:19], a[128:131], a[224:227], a[16:19]\n"
      "s_add_u32 s80, s26, %39\n"
      "s_addc_u32 s81, s27, 0\n"
      "v_mfma_f32_16x16x32_bf16 a[16:19], a[132:135], a[228:231], a[16:19]\n"
      "s_mov_b32 s82, 0x200000\n"
      "s_waitcnt lgkmcnt(6)\n"
      "v_mfma_f32_16x16x32_bf16 a[20:23], a[132:135], a[236:239], a[20:23]\n"
      "ds_read_b64_tr_b16 a[166:167], v0 offset:17920\n"
      "s_add_u32 s90, s71, 1\n"
      "v_mfma_f32_16x16x32_bf16 a[20:23], a[128:131], a[232:235], a[20:23]\n"
      "v_xor_b32 v0, s56, v0\n"
      "v_readlane_b32 s57, v22, s90\n"
      "v_mfma_f32_16x16x32_bf16 a[24:27], a[128:131], a[240:243], a[24:27]\n"
      "s_add_u32 s65, s57, 1024\n"
      "v_mfma_f32_16x16x32_bf16 a[24:27], a[132:135], a[244:247], a[24:27]\n"
      "s_or_b32 m0, s52, %42\n"
      "s_waitcnt lgkmcnt(4)\n"
      "v_mfma_f32_16x16x32_bf16 a[28:31], a[132:135], a[252:255], a[28:31]\n"
      "buffer_load_dwordx4 v7, %36, s65 offen offset:0 lds\n"
      "v_mfma_f32_16x16x32_bf16 a[28:31], a[128:131], a[248:251], a[28:31]\n"
      "ds_read_b64_tr_b16 a[168:169], v1 offset:1024\n"
      "v_mfma_f32_16x16x32_bf16 v[60:63], a[136:139], a[248:251], v[60:63]\n"
      "s_mov_b32 s83, 0x20000\n"
      "v_mfma_f32_16x16x32_bf16 v[60:63], a[140:143], a[252:255], v[60:63]\n"
      "ds_read_b64_tr_b16 a[170:171], v1 offset:1536\n"
      "v_mfma_f32_16x16x32_bf16 v[56:59], a[140:143], a[244:247], v[56:59]\n"
      "v_mfma_f32_16x16x32_bf16 v[56:59], a[136:139], a[240:243], v[56:59]\n"
      "v_mfma_f32_16x16x32_bf16 v[52:55], a[136:139], a[232:235], v[52:55]\n"
      "ds_read_b64_tr_b16 a[172:173], v1 offset:17408\n"
      "v_mfma_f32_16x16x32_bf16 v[52:55], a[140:143], a[236:239], v[52:55]\n"
      "v_mfma_f32_16x16x32_bf16 v[48:51], a[140:143], a[228:231], v[48:51]\n"
      "buffer_load_dwordx4 v9, %36, s65 offen offset:2048 lds\n"
      "v_mfma_f32_16x16x32_bf16 v[48:51], a[136:139], a[224:227], v[48:51]\n"
      "s_bitset1_b32 m0, 13\n"
      "v_mfma_f32_16x16x32_bf16 v[44:47], a[136:139], a[216:219], v[44:47]\n"
      "ds_read_b64_tr_b16 a[174:175], v1 offset:17920\n"
      "v_mfma_f32_16x16x32_bf16 v[44:47], a[140:143], a[220:223], v[44:47]\n"
      "v_xor_b32 v1, s56, v1\n"
      "v_mfma_f32_16x16x32_bf16 v[40:43], a[140:143], a[212:215], v[40:43]\n"
      "v_mfma_f32_16x16x32_bf16 v[40:43], a[136:139], a[208:211], v[40:43]\n"
      "ds_read_b64_tr_b16 a[176:177], v2 offset:1024\n"
      "v_mfma_f32_16x16x32_bf16 v[36:39], a[136:139], a[200:203], v[36:39]\n"
      "v_mfma_f32_16x16x32_bf16 v[36:39], a[140:143], a[204:207], v[36:39]\n"
      "v_mfma_f32_16x16x32_bf16 v[32:35], a[140:143], a[196:199], v[32:35]\n"
      "buffer_load_dwordx4 v11, %36, s65 offen offset:0 lds\n"
      "v_mfma_f32_16x16x32_bf16 v[32:35], a[136:139], a[192:195], v[32:35]\n"
      "ds_read_b64_tr_b16 a[178:179], v2 offset:1536\n"
      "v_mfma_f32_16x16x32_bf16 v[64:67], a[144:147], a[192:195], v[64:67]\n"
      "v_mfma_f32_16x16x32_bf16 v[64:67], a[148:151], a[196:199], v[64:67]\n"
      "v_mfma_f32_16x16x32_bf16 v[68:71], a[148:151], a[204:207], v[68:71]\n"
      "ds_read_b64_tr_b16 a[180:181], v2 offset:17408\n"
      "v_mfma_f32_16x16x32_bf16 v[68:71], a[144:147], a[200:203], v[68:71]\n"
      "v_mfma_f32_16x16x32_bf16 v[72:75], a[144:147], a[208:211], v[72:75]\n"
      "v_mfma_f32_16x16x32_bf16 v[72:75], a[148:151], a[212:215], v[72:75]\n"
      "ds_read_b64_tr_b16 a[182:183], v2 offset:17920\n"
      "v_mfma_f32_16x16x32_bf16 v[76:79], a[148:151], a[220:223], v[76:79]\n"
      "buffer_load_dwordx4 v13, %36, s65 offen offset:2048 lds\n"
      "v_mfma_f32_16x16x32_bf16 v[76:79], a[144:147], a[216:219], v[76:79]\n"
      "v_xor_b32 v2, s56, v2\n"
      "v_mfma_f32_16x16x32_bf16 v[80:83], a[144:147], a[224:227], v[80:83]\n"
      "ds_read_b64_tr_b16 a[184:185], v3 offset:1024\n"
      "v_mfma_f32_16x16x32_bf16 v[80:83], a[148:151], a[228:231], v[80:83]\n"
      "v_mfma_f32_16x16x32_bf16 v[84:87], a[148:151], a[236:239], v[84:87]\n"
      "v_mfma_f32_16x16x32_bf16 v[84:87], a[144:147], a[232:235], v[84:87]\n"
      "ds_read_b64_tr_b16 a[186:187], v3 offset:1536\n"
      "v_mfma_f32_16x16x32_bf16 v[88:91], a[144:147], a[240:243], v[88:91]\n"
      "v_mfma_f32_16x16x32_bf16 v[88:91], a[148:151], a[244:247], v[88:91]\n"
      "v_mfma_f32_16x16x32_bf16 v[92:95], a[148:151], a[252:255], v[92:95]\n"
      "v_mfma_f32_16x16x32_bf16 v[92:95], a[144:147], a[248:251], v[92:95]\n"
      "ds_read_b64_tr_b16 a[188:189], v3 offset:17408\n"
      "v_mfma_f32_16x16x32_bf16 v[124:127], a[152:155], a[248:251], v[124:127]\n"
      "v_mfma_f32_16x16x32_bf16 v[124:127], a[156:159], a[252:255], v[124:127]\n"
      "v_mfma_f32_16x16x32_bf16 v[120:123], a[156:159], a[244:247], v[120:123]\n"
      "ds_read_b64_tr_b16 a[190:191], v3 offset:17920\n"
      "v_mfma_f32_16x16x32_bf16 v[120:123], a[152:155], a[240:243], v[120:123]\n"
      "v_xor_b32 v3, s56, v3\n"
      "v_mfma_f32_16x16x32_bf16 v[116:119], a[152:155], a[232:235], v[116:119]\n"
      "v_mfma_f32_16x16x32_bf16 v[116:119], a[156:159], a[236:239], v[116:119]\n"
      "v_mfma_f32_16x16x32_bf16 v[112:115], a[156:159], a[228:231], v[112:115]\n"
      "v_mfma_f32_16x16x32_bf16 v[112:115], a[152:155], a[224:227], v[112:115]\n"
      "v_mfma_f32_16x16x32_bf16 v[108:111], a[152:155], a[216:219], v[108:111]\n"
      "v_mfma_f32_16x16x32_bf16 v[108:111], a[156:159], a[220:223], v[108:111]\n"
      "s_add_u32 s90, s71, 0\n"
      "v_mfma_f32_16x16x32_bf16 v[104:107], a[156:159], a[212:215], v[104:107]\n"
      "v_readlane_b32 s86, v22, s90\n"
      "v_mfma_f32_16x16x32_bf16 v[104:107], a[152:155], a[208:211], v[104:107]\n"
      "s_lshr_b32 s86, s86, 13\n"
      "v_mfma_f32_16x16x32_bf16 v[100:103], a[152:155], a[200:203], v[100:103]\n"
      "s_waitcnt vmcnt(8) lgkmcnt(0)\n"
      "s_barrier\n"
      "v_mfma_f32_16x16x32_bf16 v[100:103], a[156:159], a[204:207], v[100:103]\n"
      "s_mul_i32 s86, s86, 0x401\n"
      "v_mfma_f32_16x16x32_bf16 v[96:99], a[156:159], a[196:199], v[96:99]\n"
      "s_and_b32 s86, s86, 0xe00780\n"
      "v_mfma_f32_16x16x32_bf16 v[96:99], a[152:155], a[192:195], v[96:99]\n"
      "s_or_b32 s64, s86, %43\n"
      "v_mfma_f32_16x16x32_bf16 v[128:131], a[160:163], a[192:195], v[128:131]\n"
      "v_min_u32 v29, v24, v25\n"
      "v_min3_u32 v29, v29, v26, v27\n"
      "v_cmp_gt_u32 vcc, s75, v29\n"
      "v_mfma_f32_16x16x32_bf16 v[128:131], a[164:167], a[196:199], v[128:131]\n"
      "s_or_b32 m0, s51, %41\n"
      "v_mfma_f32_16x16x32_bf16 v[160:163], a[172:175], a[196:199], v[160:163]\n"
      "s_cbranch_vccnz 145f\n"
      "45:\n"
      "v_mfma_f32_16x16x32_bf16 v[160:163], a[168:171], a[192:195], v[160:163]\n"
      "buffer_load_dwordx4 v14, s[40:43], s64 offen offset:0 sc0 sc1 lds\n"
      "v_mfma_f32_16x16x32_bf16 v[192:195], a[176:179], a[192:195], v[192:195]\n"
      "ds_read_b64_tr_b16 a[128:129], v0 offset:0\n"
      "v_mfma_f32_16x16x32_bf16 v[192:195], a[180:183], a[196:199], v[192:195]\n"
      "s_add_u32 s44, s48, %47\n"
      "s_addc_u32 s45, s49, 0\n"
      "v_mfma_f32_16x16x32_bf16 v[224:227], a[188:191], a[196:199], v[224:227]\n"
      "s_mov_b32 s58, %48\n"
      "v_mfma_f32_16x16x32_bf16 v[224:227], a[184:187], a[192:195], v[224:227]\n"
      "buffer_load_dwordx4 v15, s[40:43], s64 offen offset:1024 sc0 sc1 lds\n"
      "v_mfma_f32_16x16x32_bf16 v[228:231], a[184:187], a[200:203], v[228:231]\n"
      "ds_read_b128 a[192:195], v4 offset:0\n"
      "v_mfma_f32_16x16x32_bf16 v[228:231], a[188:191], a[204:207], v[228:231]\n"
      "ds_read_b128 a[196:199], v5 offset:0\n"
      "v_mfma_f32_16x16x32_bf16 v[196:199], a[180:183], a[204:207], v[196:199]\n"
      "ds_read_b64_tr_b16 a[130:131], v0 offset:512\n"
      "v_mfma_f32_16x16x32_bf16 v[196:199], a[176:179], a[200:203], v[196:199]\n"
      "buffer_load_dwordx4 v16, s[40:43], s64 offen offset:2048 sc0 sc1 lds\n"
      "v_mfma_f32_16x16x32_bf16 v[164:167], a[168:171], a[200:203], v[164:167]\n"
      "ds_read_b64_tr_b16 a[132:133], v0 offset:16384\n"
      "v_mfma_f32_16x16x32_bf16 v[164:167], a[172:175], a[204:207], v[164:167]\n"
      "s_min_u32 s46, s42, 0x20000\n"
      "v_mfma_f32_16x16x32_bf16 v[132:135], a[164:167], a[204:207], v[132:135]\n"
      "s_mov_b32 s47, 0x20000\n"
      "v_mfma_f32_16x16x32_bf16 v[132:135], a[160:163], a[200:203], v[132:135]\n"
      "buffer_load_dwordx4 v17, s[40:43], s64 offen offset:3072 sc0 sc1 lds\n"
      "v_mfma_f32_16x16x32_bf16 v[136:139], a[160:163], a[208:211], v[136:139]\n"
      "s_bitset1_b32 m0, 12\n"
      "v_mfma_f32_16x16x32_bf16 v[136:139], a[164:167], a[212:215], v[136:139]\n"
      "ds_read_b128 a[200:203], v4 offset:2048\n"
      "v_mfma_f32_16x16x32_bf16 v[168:171], a[172:175], a[212:215], v[168:171]\n"
      "ds_read_b128 a[204:207], v5 offset:2048\n"
      "v_mfma_f32_16x16x32_bf16 v[168:171], a[168:171], a[208:211], v[168:171]\n"
      "buffer_load_dwordx4 v18, s[40:43], s64 offen offset:0 sc0 sc1 lds\n"
      "v_mfma_f32_16x16x32_bf16 v[200:203], a[176:179], a[208:211], v[200:203]\n"
      "ds_read_b64_tr_b16 a[134:135], v0 offset:16896\n"
      "v_mfma_f32_16x16x32_bf16 v[200:203], a[180:183], a[212:215], v[200:203]\n"
      "ds_read_b64_tr_b16 a[136:137], v1 offset:0\n"
      "v_mfma_f32_16x16x32_bf16 v[232:235], a[188:191], a[212:215], v[232:235]\n"
      "ds_read_b64_tr_b16 a[138:139], v1 offset:512\n"
      "v_mfma_f32_16x16x32_bf16 v[232:235], a[184:187], a[208:211], v[232:235]\n"
      "buffer_load_dwordx4 v19, s[40:43], s64 offen offset:1024 sc0 sc1 lds\n"
      "v_mfma_f32_16x16x32_bf16 v[236:239], a[184:187], a[216:219], v[236:239]\n"
      "ds_read_b128 a[208:211], v4 offset:4096\n"
      "v_mfma_f32_16x16x32_bf16 v[236:239], a[188:191], a[220:223], v[236:239]\n"
      "ds_read_b128 a[212:215], v5 offset:4096\n"
      "v_mfma_f32_16x16x32_bf16 v[204:207], a[180:183], a[220:223], v[204:207]\n"
      "ds_read_b64_tr_b16 a[140:141], v1 offset:16384\n"
      "s_add_u32 s90, s71, 1\n"
      "v_mfma_f32_16x16x32_bf16 v[204:207], a[176:179], a[216:219], v[204:207]\n"
      "buffer_load_dwordx4 v20, s[40:43], s64 offen offset:2048 sc0 sc1 lds\n"
      "v_mfma_f32_16x16x32_bf16 v[172:175], a[168:171], a[216:219], v[172:175]\n"
      "v_readlane_b32 s86, v22, s90\n"
      "v_mfma_f32_16x16x32_bf16 v[172:175], a[172:175], a[220:223], v[172:175]\n"
      "ds_read_b64_tr_b16 a[142:143], v1 offset:16896\n"
      "s_lshr_b32 s86, s86, 13\n"
      "v_mfma_f32_16x16x32_bf16 v[140:143], a[164:167], a[220:223], v[140:143]\n"
      "s_mul_i32 s86, s86, 0x401\n"
      "v_mfma_f32_16x16x32_bf16 v[140:143], a[160:163], a[216:219], v[140:143]\n"
      "buffer_load_dwordx4 v21, s[40:43], s64 offen offset:3072 sc0 sc1 lds\n"
      "v_mfma_f32_16x16x32_bf16 v[144:147], a[160:163], a[224:227], v[144:147]\n"
      "ds_read_b128 a[216:219], v4 offset:6144\n"
      "s_and_b32 s86, s86, 0xe00780\n"
      "v_mfma_f32_16x16x32_bf16 v[144:147], a[164:167], a[228:231], v[144:147]\n"
      "s_or_b32 s64, s86, %43\n"
      "v_mfma_f32_16x16x32_bf16 v[176:179], a[172:175], a[228:231], v[176:179]\n"
      "s_or_b32 m0, s53, %41\n"
      "v_mfma_f32_16x16x32_bf16 v[176:179], a[168:171], a[224:227], v[176:179]\n"
      "buffer_load_dwordx4 v14, s[40:43], s64 offen offset:0 sc0 sc1 lds\n"
      "v_mfma_f32_16x16x32_bf16 v[208:211], a[176:179], a[224:227], v[208:211]\n"
      "ds_read_b128 a[220:223], v5 offset:6144\n"
      "v_mfma_f32_16x16x32_bf16 v[208:211], a[180:183], a[228:231], v[208:211]\n"
      "ds_read_b64_tr_b16 a[144:145], v2 offset:0\n"
      "v_mfma_f32_16x16x32_bf16 v[240:243], a[188:191], a[228:231], v[240:243]\n"
      "ds_read_b64_tr_b16 a[146:147], v2 offset:512\n"
      "v_mfma_f32_16x16x32_bf16 v[240:243], a[184:187], a[224:227], v[240:243]\n"
      "buffer_load_dwordx4 v15, s[40:43], s64 offen offset:1024 sc0 sc1 lds\n"
      "v_mfma_f32_16x16x32_bf16 v[244:247], a[184:187], a[232:235], v[244:247]\n"
      "ds_read_b128 a[224:227], v4 offset:8192\n"
      "v_mfma_f32_16x16x32_bf16 v[244:247], a[188:191], a[236:239], v[244:247]\n"
      "ds_read_b128 a[228:231], v5 offset:8192\n"
      "v_mfma_f32_16x16x32_bf16 v[212:215], a[180:183], a[236:239], v[212:215]\n"
      "ds_read_b64_tr_b16 a[148:149], v2 offset:16384\n"
      "v_mfma_f32_16x16x32_bf16 v[212:215], a[176:179], a[232:235], v[212:215]\n"
      "buffer_load_dwordx4 v16, s[40:43], s64 offen offset:2048 sc0 sc1 lds\n"
      "v_mfma_f32_16x16x32_bf16 v[180:183], a[168:171], a[232:235], v[180:183]\n"
      "ds_read_b64_tr_b16 a[150:151], v2 offset:16896\n"
      "v_mfma_f32_16x16x32_bf16 v[180:183], a[172:175], a[236:239], v[180:183]\n"
      "s_xor_b32 s55, s50, s51\n"
      "v_mfma_f32_16x16x32_bf16 v[148:151], a[164:167], a[236:239], v[148:151]\n"
      "s_xor_b32 s56, s54, s52\n"
      "v_mfma_f32_16x16x32_bf16 v[148:151], a[160:163], a[232:235], v[148:151]\n"
      "buffer_load_dwordx4 v17, s[40:43], s64 offen offset:3072 sc0 sc1 lds\n"
      "v_mfma_f32_16x16x32_bf16 v[152:155], a[160:163], a[240:243], v[152:155]\n"
      "s_bitset1_b32 m0, 12\n"
      "v_mfma_f32_16x16x32_bf16 v[152:155], a[164:167], a[244:247], v[152:155]\n"
      "ds_read_b128 a[232:235], v4 offset:10240\n"
      "v_mfma_f32_16x16x32_bf16 v[184:187], a[172:175], a[244:247], v[184:187]\n"
      "ds_read_b128 a[236:239], v5 offset:10240\n"
      "v_mfma_f32_16x16x32_bf16 v[184:187], a[168:171], a[240:243], v[184:187]\n"
      "buffer_load_dwordx4 v18, s[40:43], s64 offen offset:0 sc0 sc1 lds\n"
      "v_mfma_f32_16x16x32_bf16 v[216:219], a[176:179], a[240:243], v[216:219]\n"
      "ds_read_b64_tr_b16 a[152:153], v3 offset:0\n"
      "v_mfma_f32_16x16x32_bf16 v[216:219], a[180:183], a[244:247], v[216:219]\n"
      "ds_read_b64_tr_b16 a[154:155], v3 offset:512\n"
      "v_mfma_f32_16x16x32_bf16 v[248:251], a[188:191], a[244:247], v[248:251]\n"
      "v_mfma_f32_16x16x32_bf16 v[248:251], a[184:187], a[240:243], v[248:251]\n"
      "buffer_load_dwordx4 v19, s[40:43], s64 offen offset:1024 sc0 sc1 lds\n"
      "v_mfma_f32_16x16x32_bf16 v[252:255], a[184:187], a[248:251], v[252:255]\n"
      "ds_read_b128 a[240:243], v4 offset:12288\n"
      "v_mfma_f32_16x16x32_bf16 v[252:255], a[188:191], a[252:255], v[252:255]\n"
      "ds_read_b128 a[244:247], v5 offset:12288\n"
      "v_mfma_f32_16x16x32_bf16 v[220:223], a[180:183], a[252:255], v[220:223]\n"
      "ds_read_b64_tr_b16 a[156:157], v3 offset:16384\n"
      "v_mfma_f32_16x16x32_bf16 v[220:223], a[176:179], a[248:251], v[220:223]\n"
      "buffer_load_dwordx4 v20, s[40:43], s64 offen offset:2048 sc0 sc1 lds\n"
      "v_mfma_f32_16x16x32_bf16 v[188:191], a[168:171], a[248:251], v[188:191]\n"
      "ds_read_b64_tr_b16 a[158:159], v3 offset:16896\n"
      "v_mfma_f32_16x16x32_bf16 v[188:191], a[172:175], a[252:255], v[188:191]\n"
      "v_mfma_f32_16x16x32_bf16 v[156:159], a[164:167], a[252:255], v[156:159]\n"
      "v_mfma_f32_16x16x32_bf16 v[156:159], a[160:163], a[248:251], v[156:159]\n"
      "buffer_load_dwordx4 v21, s[40:43], s64 offen offset:3072 sc0 sc1 lds\n"
      "s_waitcnt lgkmcnt(8)\n"
      "v_mfma_f32_16x16x32_bf16 a[0:3], a[128:131], a[192:195], a[0:3]\n"
      "ds_read_b128 a[248:251], v4 offset:14336\n"
      "v_mfma_f32_16x16x32_bf16 a[0:3], a[132:135], a[196:199], a[0:3]\n"
      "ds_read_b128 a[252:255], v5 offset:14336\n"
      "v_mfma_f32_16x16x32_bf16 v[32:35], a[140:143], a[196:199], v[32:35]\n"
      "ds_read_b64_tr_b16 a[160:161], v0 offset:1024\n"
      "v_xor_b32 v4, s55, v4\n"
      "v_mfma_f32_16x16x32_bf16 v[32:35], a[136:139], a[192:195], v[32:35]\n"
      "v_xor_b32 v5, s55, v5\n"
      "v_mfma_f32_16x16x32_bf16 v[64:67], a[144:147], a[192:195], v[64:67]\n"
      "ds_read_b64_tr_b16 a[162:163], v0 offset:1536\n"
      "v_mfma_f32_16x16x32_bf16 v[64:67], a[148:151], a[196:199], v[64:67]\n"
      "s_waitcnt lgkmcnt(4)\n"
      "v_mfma_f32_16x16x32_bf16 v[96:99], a[156:159], a[196:199], v[96:99]\n"
      "v_accvgpr_read_b32 v24, a0\n"
      "v_accvgpr_read_b32 v25, a1\n"
      "v_mfma_f32_16x16x32_bf16 v[96:99], a[152:155], a[192:195], v[96:99]\n"
      "ds_read_b64_tr_b16 a[164:165], v0 offset:17408\n"
      "v_accvgpr_read_b32 v26, a2\n"
      "v_mfma_f32_16x16x32_bf16 a[4:7], a[128:131], a[200:203], a[4:7]\n"
      "v_accvgpr_read_b32 v27, a3\n"
      "v_cvt_pk_bf16_f32 v24, v24, v25\n"
      "v_mfma_f32_16x16x32_bf16 a[4:7], a[132:135], a[204:207], a[4:7]\n"
      "v_cvt_pk_bf16_f32 v25, v26, v27\n"
      "v_cvt_pk_bf16_f32 v26, v32, v33\n"
      "v_mfma_f32_16x16x32_bf16 v[36:39], a[140:143], a[204:207], v[36:39]\n"
      "ds_read_b64_tr_b16 a[166:167], v0 offset:17920\n"
      "v_cvt_pk_bf16_f32 v27, v34, v35\n"
      "v_mfma_f32_16x16x32_bf16 v[36:39], a[136:139], a[200:203], v[36:39]\n"
      "v_xor_b32 v0, s56, v0\n"
      "v_cvt_pk_bf16_f32 v64, v64, v65\n"
      "v_mfma_f32_16x16x32_bf16 v[68:71], a[144:147], a[200:203], v[68:71]\n"
      "v_cvt_pk_bf16_f32 v65, v66, v67\n"
      "v_cvt_pk_bf16_f32 v66, v96, v97\n"
      "v_mfma_f32_16x16x32_bf16 v[68:71], a[148:151], a[204:207], v[68:71]\n"
      "v_cvt_pk_bf16_f32 v67, v98, v99\n"
      "v_mfma_f32_16x16x32_bf16 v[100:103], a[156:159], a[204:207], v[100:103]\n"
      "buffer_store_dwordx4 v[24:27], v30, s[44:47], s58 offen offset:0 sc0 sc1\n"
      "v_mfma_f32_16x16x32_bf16 v[100:103], a[152:155], a[200:203], v[100:103]\n"
      "ds_read_b64_tr_b16 a[168:169], v1 offset:1024\n"
      "v_accvgpr_read_b32 v32, a4\n"
      "v_mfma_f32_16x16x32_bf16 a[8:11], a[128:131], a[208:211], a[8:11]\n"
      "v_accvgpr_read_b32 v33, a5\n"
      "v_accvgpr_read_b32 v34, a6\n"
      "v_mfma_f32_16x16x32_bf16 a[8:11], a[132:135], a[212:215], a[8:11]\n"
      "ds_read_b64_tr_b16 a[170:171], v1 offset:1536\n"
      "v_accvgpr_read_b32 v35, a7\n"
      "v_mfma_f32_16x16x32_bf16 v[40:43], a[140:143], a[212:215], v[40:43]\n"
      "v_cvt_pk_bf16_f32 v32, v32, v33\n"
      "v_cvt_pk_bf16_f32 v33, v34, v35\n"
      "v_mfma_f32_16x16x32_bf16 v[40:43], a[136:139], a[208:211], v[40:43]\n"
      "buffer_store_dwordx4 v[64:67], v30, s[44:47], s58 offen offset:1024 sc0 sc1\n"
      "v_mfma_f32_16x16x32_bf16 v[72:75], a[144:147], a[208:211], v[72:75]\n"
      "ds_read_b64_tr_b16 a[172:173], v1 offset:17408\n"
      "v_cvt_pk_bf16_f32 v34, v36, v37\n"
      "v_mfma_f32_16x16x32_bf16 v[72:75], a[148:151], a[212:215], v[72:75]\n"
      "v_cvt_pk_bf16_f32 v35, v38, v39\n"
      "v_cvt_pk_bf16_f32 v68, v68, v69\n"
      "v_mfma_f32_16x16x32_bf16 v[104:107], a[156:159], a[212:215], v[104:107]\n"
      "v_cvt_pk_bf16_f32 v69, v70, v71\n"
      "v_cvt_pk_bf16_f32 v70, v100, v101\n"
      "v_mfma_f32_16x16x32_bf16 v[104:107], a[152:155], a[208:211], v[104:107]\n"
      "v_cvt_pk_bf16_f32 v71, v102, v103\n"
      "v_accvgpr_read_b32 v36, a8\n"
      "v_mfma_f32_16x16x32_bf16 a[12:15], a[128:131], a[216:219], a[12:15]\n"
      "buffer_store_dwordx4 v[32:35], v30, s[44:47], s58 offen offset:2048 sc0 sc1\n"
      "v_mfma_f32_16x16x32_bf16 a[12:15], a[132:135], a[220:223], a[12:15]\n"
      "ds_read_b64_tr_b16 a[174:175], v1 offset:17920\n"
      "v_accvgpr_read_b32 v37, a9\n"
      "v_mfma_f32_16x16x32_bf16 v[44:47], a[140:143], a[220:223], v[44:47]\n"
      "v_xor_b32 v1, s56, v1\n"
      "v_accvgpr_read_b32 v38, a10\n"
      "v_mfma_f32_16x16x32_bf16 v[44:47], a[136:139], a[216:219], v[44:47]\n"
      "ds_read_b64_tr_b16 a[176:177], v2 offset:1024\n"
      "v_accvgpr_read_b32 v39, a11\n"
      "v_mfma_f32_16x16x32_bf16 v[76:79], a[144:147], a[216:219], v[76:79]\n"
      "v_cvt_pk_bf16_f32 v36, v36, v37\n"
      "v_cvt_pk_bf16_f32 v37, v38, v39\n"
      "v_mfma_f32_16x16x32_bf16 v[76:79], a[148:151], a[220:223], v[76:79]\n"
      "buffer_store_dwordx4 v[68:71], v30, s[44:47], s58 offen offset:3072 sc0 sc1\n"
      "v_mfma_f32_16x16x32_bf16 v[108:111], a[156:159], a[220:223], v[108:111]\n"
      "s_add_u32 s58, s58, 0x1000\n"
      "v_cvt_pk_bf16_f32 v38, v40, v41\n"
      "v_mfma_f32_16x16x32_bf16 v[108:111], a[152:155], a[216:219], v[108:111]\n"
      "ds_read_b64_tr_b16 a[178:179], v2 offset:1536\n"
      "v_cvt_pk_bf16_f32 v39, v42, v43\n"
      "v_mfma_f32_16x16x32_bf16 a[16:19], a[128:131], a[224:227], a[16:19]\n"
      "v_cvt_pk_bf16_f32 v72, v72, v73\n"
      "v_cvt_pk_bf16_f32 v73, v74, v75\n"
      "v_mfma_f32_16x16x32_bf16 a[16:19], a[132:135], a[228:231], a[16:19]\n"
      "v_cvt_pk_bf16_f32 v74, v104, v105\n"
      "v_cvt_pk_bf16_f32 v75, v106, v107\n"
      "v_mfma_f32_16x16x32_bf16 v[48:51], a[140:143], a[228:231], v[48:51]\n"
      "buffer_store_dwordx4 v[36:39], v30, s[44:47], s58 offen offset:0 sc0 sc1\n"
      "v_mfma_f32_16x16x32_bf16 v[48:51], a[136:139], a[224:227], v[48:51]\n"
      "ds_read_b64_tr_b16 a[180:181], v2 offset:17408\n"
      "v_accvgpr_read_b32 v40, a12\n"
      "v_mfma_f32_16x16x32_bf16 v[80:83], a[144:147], a[224:227], v[80:83]\n"
      "v_accvgpr_read_b32 v41, a13\n"
      "v_accvgpr_read_b32 v42, a14\n"
      "v_mfma_f32_16x16x32_bf16 v[80:83], a[148:151], a[228:231], v[80:83]\n"
      "ds_read_b64_tr_b16 a[182:183], v2 offset:17920\n"
      "v_accvgpr_read_b32 v43, a15\n"
      "v_mfma_f32_16x16x32_bf16 v[112:115], a[156:159], a[228:231], v[112:115]\n"
      "v_xor_b32 v2, s56, v2\n"
      "v_cvt_pk_bf16_f32 v40, v40, v41\n"
      "v_mfma_f32_16x16x32_bf16 v[112:115], a[152:155], a[224:227], v[112:115]\n"
      "buffer_store_dwordx4 v[72:75], v30, s[44:47], s58 offen offset:1024 sc0 sc1\n"
      "v_mfma_f32_16x16x32_bf16 a[20:23], a[128:131], a[232:235], a[20:23]\n"
      "ds_read_b64_tr_b16 a[184:185], v3 offset:1024\n"
      "v_cvt_pk_bf16_f32 v41, v42, v43\n"
      "v_mfma_f32_16x16x32_bf16 a[20:23], a[132:135], a[236:239], a[20:23]\n"
      "v_cvt_pk_bf16_f32 v42, v44, v45\n"
      "v_cvt_pk_bf16_f32 v43, v46, v47\n"
      "v_mfma_f32_16x16x32_bf16 v[52:55], a[140:143], a[236:239], v[52:55]\n"
      "v_cvt_pk_bf16_f32 v76, v76, v77\n"
      "v_cvt_pk_bf16_f32 v77, v78, v79\n"
      "v_mfma_f32_16x16x32_bf16 v[52:55], a[136:139], a[232:235], v[52:55]\n"
      "ds_read_b64_tr_b16 a[186:187], v3 offset:1536\n"
      "v_cvt_pk_bf16_f32 v78, v108, v109\n"
      "v_mfma_f32_16x16x32_bf16 v[84:87], a[144:147], a[232:235], v[84:87]\n"
      "buffer_store_dwordx4 v[40:43], v30, s[44:47], s58 offen offset:2048 sc0 sc1\n"
      "v_mfma_f32_16x16x32_bf16 v[84:87], a[148:151], a[236:239], v[84:87]\n"
      "v_cvt_pk_bf16_f32 v79, v110, v111\n"
      "v_accvgpr_read_b32 v44, a16\n"
      "v_mfma_f32_16x16x32_bf16 v[116:119], a[156:159], a[236:239], v[116:119]\n"
      "v_accvgpr_read_b32 v45, a17\n"
      "v_accvgpr_read_b32 v46, a18\n"
      "v_mfma_f32_16x16x32_bf16 v[116:119], a[152:155], a[232:235], v[116:119]\n"
      "ds_read_b64_tr_b16 a[188:189], v3 offset:17408\n"
      "v_accvgpr_read_b32 v47, a19\n"
      "v_mfma_f32_16x16x32_bf16 a[24:27], a[128:131], a[240:243], a[24:27]\n"
      "v_cvt_pk_bf16_f32 v44, v44, v45\n"
      "v_cvt_pk_bf16_f32 v45, v46, v47\n"
      "v_mfma_f32_16x16x32_bf16 a[24:27], a[132:135], a[244:247], a[24:27]\n"
      "buffer_store_dwordx4 v[76:79], v30, s[44:47], s58 offen offset:3072 sc0 sc1\n"
      "v_mfma_f32_16x16x32_bf16 v[56:59], a[140:143], a[244:247], v[56:59]\n"
      "s_add_u32 s58, s58, 0x1000\n"
      "v_cvt_pk_bf16_f32 v46, v48, v49\n"
      "v_mfma_f32_16x16x32_bf16 v[56:59], a[136:139], a[240:243], v[56:59]\n"
      "ds_read_b64_tr_b16 a[190:191], v3 offset:17920\n"
      "v_cvt_pk_bf16_f32 v47, v50, v51\n"
      "v_mfma_f32_16x16x32_bf16 v[88:91], a[144:147], a[240:243], v[88:91]\n"
      "v_xor_b32 v3, s56, v3\n"
      "v_cvt_pk_bf16_f32 v80, v80, v81\n"
      "v_mfma_f32_16x16x32_bf16 v[88:91], a[148:151], a[244:247], v[88:91]\n"
      "v_cvt_pk_bf16_f32 v81, v82, v83\n"
      "v_cvt_pk_bf16_f32 v82, v112, v113\n"
      "v_mfma_f32_16x16x32_bf16 v[120:123], a[156:159], a[244:247], v[120:123]\n"
      "buffer_store_dwordx4 v[44:47], v30, s[44:47], s58 offen offset:0 sc0 sc1\n"
      "v_mfma_f32_16x16x32_bf16 v[120:123], a[152:155], a[240:243], v[120:123]\n"
      "v_cvt_pk_bf16_f32 v83, v114, v115\n"
      "v_accvgpr_read_b32 v48, a20\n"
      "s_waitcnt lgkmcnt(4)\n"
      "v_mfma_f32_16x16x32_bf16 a[28:31], a[128:131], a[248:251], a[28:31]\n"
      "v_accvgpr_read_b32 v49, a21\n"
      "v_accvgpr_read_b32 v50, a22\n"
      "v_mfma_f32_16x16x32_bf16 a[28:31], a[132:135], a[252:255], a[28:31]\n"
      "v_accvgpr_read_b32 v51, a23\n"
      "v_cvt_pk_bf16_f32 v48, v48, v49\n"
      "v_mfma_f32_16x16x32_bf16 v[60:63], a[140:143], a[252:255], v[60:63]\n"
      "v_cvt_pk_bf16_f32 v49, v50, v51\n"
      "v_cvt_pk_bf16_f32 v50, v52, v53\n"
      "v_mfma_f32_16x16x32_bf16 v[60:63], a[136:139], a[248:251], v[60:63]\n"
      "v_cvt_pk_bf16_f32 v51, v54, v55\n"
      "v_cvt_pk_bf16_f32 v84, v84, v85\n"
      "v_mfma_f32_16x16x32_bf16 v[92:95], a[144:147], a[248:251], v[92:95]\n"
      "s_waitcnt vmcnt(17) lgkmcnt(0)\n"
      "s_barrier\n"
      "v_mfma_f32_16x16x32_bf16 v[92:95], a[148:151], a[252:255], v[92:95]\n"
      "v_cvt_pk_bf16_f32 v85, v86, v87\n"
      "v_cvt_pk_bf16_f32 v86, v116, v117\n"
      "v_mfma_f32_16x16x32_bf16 v[124:127], a[156:159], a[252:255], v[124:127]\n"
      "buffer_store_dwordx4 v[80:83], v30, s[44:47], s58 offen offset:1024 sc0 sc1\n"
      "v_mfma_f32_16x16x32_bf16 v[124:127], a[152:155], a[248:251], v[124:127]\n"
      "v_cvt_pk_bf16_f32 v87, v118, v119\n"
      "v_accvgpr_read_b32 v52, a24\n"
      "v_mfma_f32_16x16x32_bf16 v[128:131], a[160:163], a[192:195], v[128:131]\n"
      "buffer_store_dwordx4 v[48:51], v30, s[44:47], s58 offen offset:2048 sc0 sc1\n"
      "v_mfma_f32_16x16x32_bf16 v[128:131], a[164:167], a[196:199], v[128:131]\n"
      "v_accvgpr_read_b32 v53, a25\n"
      "v_accvgpr_read_b32 v54, a26\n"
      "v_mfma_f32_16x16x32_bf16 v[160:163], a[172:175], a[196:199], v[160:163]\n"
      "ds_read_b64_tr_b16 a[128:129], v0 offset:1024\n"
      "v_accvgpr_read_b32 v55, a27\n"
      "v_mfma_f32_16x16x32_bf16 v[160:163], a[168:171], a[192:195], v[160:163]\n"
      "v_cvt_pk_bf16_f32 v52, v52, v53\n"
      "v_cvt_pk_bf16_f32 v53, v54, v55\n"
      "v_mfma_f32_16x16x32_bf16 v[192:195], a[176:179], a[192:195], v[192:195]\n"
      "v_cvt_pk_bf16_f32 v54, v56, v57\n"
      "v_cvt_pk_bf16_f32 v55, v58, v59\n"
      "v_mfma_f32_16x16x32_bf16 v[192:195], a[180:183], a[196:199], v[192:195]\n"
      "buffer_store_dwordx4 v[84:87], v30, s[44:47], s58 offen offset:3072 sc0 sc1\n"
      "v_mfma_f32_16x16x32_bf16 v[224:227], a[188:191], a[196:199], v[224:227]\n"
      "s_add_u32 s58, s58, 0x1000\n"
      "v_cvt_pk_bf16_f32 v88, v88, v89\n"
      "v_mfma_f32_16x16x32_bf16 v[224:227], a[184:187], a[192:195], v[224:227]\n"
      "ds_read_b64_tr_b16 a[130:131], v0 offset:1536\n"
      "v_cvt_pk_bf16_f32 v89, v90, v91\n"
      "v_mfma_f32_16x16x32_bf16 v[228:231], a[184:187], a[200:203], v[228:231]\n"
      "ds_read_b64_tr_b16 a[132:133], v0 offset:17408\n"
      "v_cvt_pk_bf16_f32 v90, v120, v121\n"
      "v_mfma_f32_16x16x32_bf16 v[228:231], a[188:191], a[204:207], v[228:231]\n"
      "v_cvt_pk_bf16_f32 v91, v122, v123\n"
      "v_accvgpr_read_b32 v56, a28\n"
      "v_mfma_f32_16x16x32_bf16 v[196:199], a[180:183], a[204:207], v[196:199]\n"
      "buffer_store_dwordx4 v[52:55], v30, s[44:47], s58 offen offset:0 sc0 sc1\n"
      "v_mfma_f32_16x16x32_bf16 v[196:199], a[176:179], a[200:203], v[196:199]\n"
      "ds_read_b64_tr_b16 a[134:135], v0 offset:17920\n"
      "v_accvgpr_read_b32 v57, a29\n"
      "v_mfma_f32_16x16x32_bf16 v[164:167], a[168:171], a[200:203], v[164:167]\n"
      "v_accvgpr_read_b32 v58, a30\n"
      "v_accvgpr_read_b32 v59, a31\n"
      "v_mfma_f32_16x16x32_bf16 v[164:167], a[172:175], a[204:207], v[164:167]\n"
      "v_cvt_pk_bf16_f32 v56, v56, v57\n"
      "v_cvt_pk_bf16_f32 v57, v58, v59\n"
      "v_mfma_f32_16x16x32_bf16 v[132:135], a[164:167], a[204:207], v[132:135]\n"
      "ds_read_b64_tr_b16 a[136:137], v1 offset:1024\n"
      "v_cvt_pk_bf16_f32 v58, v60, v61\n"
      "v_mfma_f32_16x16x32_bf16 v[132:135], a[160:163], a[200:203], v[132:135]\n"
      "buffer_store_dwordx4 v[88:91], v30, s[44:47], s58 offen offset:1024 sc0 sc1\n"
      "v_mfma_f32_16x16x32_bf16 v[136:139], a[160:163], a[208:211], v[136:139]\n"
      "v_cvt_pk_bf16_f32 v59, v62, v63\n"
      "v_cvt_pk_bf16_f32 v92, v92, v93\n"
      "v_mfma_f32_16x16x32_bf16 v[136:139], a[164:167], a[212:215], v[136:139]\n"
      "ds_read_b64_tr_b16 a[138:139], v1 offset:1536\n"
      "v_cvt_pk_bf16_f32 v93, v94, v95\n"
      "v_mfma_f32_16x16x32_bf16 v[168:171], a[172:175], a[212:215], v[168:171]\n"
      "v_cvt_pk_bf16_f32 v94, v124, v125\n"
      "v_cvt_pk_bf16_f32 v95, v126, v127\n"
      "v_mfma_f32_16x16x32_bf16 v[168:171], a[168:171], a[208:211], v[168:171]\n"
      "v_mfma_f32_16x16x32_bf16 v[200:203], a[176:179], a[208:211], v[200:203]\n"
      "buffer_store_dwordx4 v[56:59], v30, s[44:47], s58 offen offset:2048 sc0 sc1\n"
      "v_mfma_f32_16x16x32_bf16 v[200:203], a[180:183], a[212:215], v[200:203]\n"
      "ds_read_b64_tr_b16 a[140:141], v1 offset:17408\n"
      "v_mfma_f32_16x16x32_bf16 v[232:235], a[188:191], a[212:215], v[232:235]\n"
      "v_mfma_f32_16x16x32_bf16 v[232:235], a[184:187], a[208:211], v[232:235]\n"
      "ds_read_b64_tr_b16 a[142:143], v1 offset:17920\n"
      "v_mfma_f32_16x16x32_bf16 v[236:239], a[184:187], a[216:219], v[236:239]\n"
      "s_add_u32 s90, s71, 2\n"
      "v_mfma_f32_16x16x32_bf16 v[236:239], a[188:191], a[220:223], v[236:239]\n"
      "buffer_store_dwordx4 v[92:95], v30, s[44:47], s58 offen offset:3072 sc0 sc1\n"
      "v_mfma_f32_16x16x32_bf16 v[204:207], a[180:183], a[220:223], v[204:207]\n"
      "s_xor_b32 s58, %48, 0x10000\n"
      "v_readlane_b32 s57, v22, s90\n"
      "v_mfma_f32_16x16x32_bf16 v[204:207], a[176:179], a[216:219], v[204:207]\n"
      "s_add_u32 s65, s57, 0\n"
      "v_mfma_f32_16x16x32_bf16 v[172:175], a[168:171], a[216:219], v[172:175]\n"
      "s_or_b32 m0, s54, %42\n"
      "v_mfma_f32_16x16x32_bf16 v[172:175], a[172:175], a[220:223], v[172:175]\n"
      "buffer_load_dwordx4 v7, %36, s65 offen offset:1024 lds\n"
      "v_mfma_f32_16x16x32_bf16 v[140:143], a[164:167], a[220:223], v[140:143]\n"
      "ds_read_b64_tr_b16 a[144:145], v2 offset:1024\n"
      "v_mfma_f32_16x16x32_bf16 v[140:143], a[160:163], a[216:219], v[140:143]\n"
      "ds_read_b64_tr_b16 a[146:147], v2 offset:1536\n"
      "v_mfma_f32_16x16x32_bf16 v[144:147], a[160:163], a[224:227], v[144:147]\n"
      "ds_read_b64_tr_b16 a[148:149], v2 offset:17408\n"
      "v_mfma_f32_16x16x32_bf16 v[144:147], a[164:167], a[228:231], v[144:147]\n"
      "buffer_load_dwordx4 v9, %36, s65 offen offset:3072 lds\n"
      "v_mfma_f32_16x16x32_bf16 v[176:179], a[172:175], a[228:231], v[176:179]\n"
      "s_bitset1_b32 m0, 13\n"
      "v_mfma_f32_16x16x32_bf16 v[176:179], a[168:171], a[224:227], v[176:179]\n"
      "ds_read_b64_tr_b16 a[150:151], v2 offset:17920\n"
      "v_mfma_f32_16x16x32_bf16 v[208:211], a[176:179], a[224:227], v[208:211]\n"
      "s_add_u32 s90, s71, 2\n"
      "v_mfma_f32_16x16x32_bf16 v[208:211], a[180:183], a[228:231], v[208:211]\n"
      "buffer_load_dwordx4 v11, %36, s65 offen offset:1024 lds\n"
      "v_mfma_f32_16x16x32_bf16 v[240:243], a[188:191], a[228:231], v[240:243]\n"
      "ds_read_b64_tr_b16 a[152:153], v3 offset:1024\n"
      "v_readlane_b32 s86, v22, s90\n"
      "v_mfma_f32_16x16x32_bf16 v[240:243], a[184:187], a[224:227], v[240:243]\n"
      "s_lshr_b32 s86, s86, 13\n"
      "v_mfma_f32_16x16x32_bf16 v[244:247], a[184:187], a[232:235], v[244:247]\n"
      "s_mul_i32 s86, s86, 0x401\n"
      "v_mfma_f32_16x16x32_bf16 v[244:247], a[188:191], a[236:239], v[244:247]\n"
      "buffer_load_dwordx4 v13, %36, s65 offen offset:3072 lds\n"
      "v_mfma_f32_16x16x32_bf16 v[212:215], a[180:183], a[236:239], v[212:215]\n"
      "ds_read_b64_tr_b16 a[154:155], v3 offset:1536\n"
      "s_and_b32 s86, s86, 0xe00780\n"
      "v_mfma_f32_16x16x32_bf16 v[212:215], a[176:179], a[232:235], v[212:215]\n"
      "s_or_b32 s64, s86, %43\n"
      "v_mfma_f32_16x16x32_bf16 v[180:183], a[168:171], a[232:235], v[180:183]\n"
      "s_or_b32 m0, s50, %41\n"
      "v_mfma_f32_16x16x32_bf16 v[180:183], a[172:175], a[236:239], v[180:183]\n"
      "buffer_load_dwordx4 v14, s[40:43], s64 offen offset:0 sc0 sc1 lds\n"
      "v_mfma_f32_16x16x32_bf16 v[148:151], a[164:167], a[236:239], v[148:151]\n"
      "ds_read_b128 a[192:195], v4 offset:0\n"
      "v_mfma_f32_16x16x32_bf16 v[148:151], a[160:163], a[232:235], v[148:151]\n"
      "ds_read_b128 a[196:199], v5 offset:0\n"
      "v_mfma_f32_16x16x32_bf16 v[152:155], a[160:163], a[240:243], v[152:155]\n"
      "ds_read_b64_tr_b16 a[156:157], v3 offset:17408\n"
      "v_mfma_f32_16x16x32_bf16 v[152:155], a[164:167], a[244:247], v[152:155]\n"
      "buffer_load_dwordx4 v15, s[40:43], s64 offen offset:1024 sc0 sc1 lds\n"
      "v_mfma_f32_16x16x32_bf16 v[184:187], a[172:175], a[244:247], v[184:187]\n"
      "ds_read_b64_tr_b16 a[158:159], v3 offset:17920\n"
      "v_mfma_f32_16x16x32_bf16 v[184:187], a[168:171], a[240:243], v[184:187]\n"
      "v_mfma_f32_16x16x32_bf16 v[216:219], a[176:179], a[240:243], v[216:219]\n"
      "ds_read_b128 a[200:203], v4 offset:2048\n"
      "v_mfma_f32_16x16x32_bf16 v[216:219], a[180:183], a[244:247], v[216:219]\n"
      "buffer_load_dwordx4 v16, s[40:43], s64 offen offset:2048 sc0 sc1 lds\n"
      "v_mfma_f32_16x16x32_bf16 v[248:251], a[188:191], a[244:247], v[248:251]\n"
      "ds_read_b128 a[204:207], v5 offset:2048\n"
      "v_mfma_f32_16x16x32_bf16 v[248:251], a[184:187], a[240:243], v[248:251]\n"
      "v_mfma_f32_16x16x32_bf16 v[252:255], a[184:187], a[248:251], v[252:255]\n"
      "v_mfma_f32_16x16x32_bf16 v[252:255], a[188:191], a[252:255], v[252:255]\n"
      "buffer_load_dwordx4 v17, s[40:43], s64 offen offset:3072 sc0 sc1 lds\n"
      "v_mfma_f32_16x16x32_bf16 v[220:223], a[180:183], a[252:255], v[220:223]\n"
      "s_bitset1_b32 m0, 12\n"
      "v_mfma_f32_16x16x32_bf16 v[220:223], a[176:179], a[248:251], v[220:223]\n"
      "v_mfma_f32_16x16x32_bf16 v[188:191], a[168:171], a[248:251], v[188:191]\n"
      "ds_read_b128 a[208:211], v4 offset:4096\n"
      "v_mfma_f32_16x16x32_bf16 v[188:191], a[172:175], a[252:255], v[188:191]\n"
      "buffer_load_dwordx4 v18, s[40:43], s64 offen offset:0 sc0 sc1 lds\n"
      "v_mfma_f32_16x16x32_bf16 v[156:159], a[164:167], a[252:255], v[156:159]\n"
      "ds_read_b128 a[212:215], v5 offset:4096\n"
      "v_mfma_f32_16x16x32_bf16 v[156:159], a[160:163], a[248:251], v[156:159]\n"
      "s_waitcnt lgkmcnt(5)\n"
      "v_mfma_f32_16x16x32_bf16 v[128:131], a[128:131], a[192:195], v[128:131]\n"
      "ds_read_b128 a[216:219], v4 offset:6144\n"
      "v_mfma_f32_16x16x32_bf16 v[128:131], a[132:135], a[196:199], v[128:131]\n"
      "buffer_load_dwordx4 v19, s[40:43], s64 offen offset:1024 sc0 sc1 lds\n"
      "v_mfma_f32_16x16x32_bf16 v[160:163], a[140:143], a[196:199], v[160:163]\n"
      "ds_read_b128 a[220:223], v5 offset:6144\n"
      "v_mfma_f32_16x16x32_bf16 v[160:163], a[136:139], a[192:195], v[160:163]\n"
      "ds_read_b128 a[224:227], v4 offset:8192\n"
      "s_xor_b32 s55, s51, s53\n"
      "v_mfma_f32_16x16x32_bf16 v[192:195], a[144:147], a[192:195], v[192:195]\n"
      "ds_read_b128 a[228:231], v5 offset:8192\n"
      "v_mfma_f32_16x16x32_bf16 v[192:195], a[148:151], a[196:199], v[192:195]\n"
      "buffer_load_dwordx4 v20, s[40:43], s64 offen offset:2048 sc0 sc1 lds\n"
      "s_waitcnt lgkmcnt(6)\n"
      "v_mfma_f32_16x16x32_bf16 v[224:227], a[156:159], a[196:199], v[224:227]\n"
      "ds_read_b128 a[232:235], v4 offset:10240\n"
      "v_mfma_f32_16x16x32_bf16 v[224:227], a[152:155], a[192:195], v[224:227]\n"
      "ds_read_b128 a[236:239], v5 offset:10240\n"
      "v_mfma_f32_16x16x32_bf16 v[228:231], a[152:155], a[200:203], v[228:231]\n"
      "ds_read_b128 a[240:243], v4 offset:12288\n"
      "s_add_u32 s90, s71, 3\n"
      "v_mfma_f32_16x16x32_bf16 v[228:231], a[156:159], a[204:207], v[228:231]\n"
      "buffer_load_dwordx4 v21, s[40:43], s64 offen offset:3072 sc0 sc1 lds\n"
      "v_mfma_f32_16x16x32_bf16 v[196:199], a[148:151], a[204:207], v[196:199]\n"
      "ds_read_b128 a[244:247], v5 offset:12288\n"
      "v_readlane_b32 s57, v22, s90\n"
      "v_mfma_f32_16x16x32_bf16 v[196:199], a[144:147], a[200:203], v[196:199]\n"
      "s_add_u32 s65, s57, 1024\n"
      "v_mfma_f32_16x16x32_bf16 v[164:167], a[136:139], a[200:203], v[164:167]\n"
      "s_or_b32 m0, s54, %42\n"
      "v_mfma_f32_16x16x32_bf16 v[164:167], a[140:143], a[204:207], v[164:167]\n"
      "buffer_load_dwordx4 v7, %36, s65 offen offset:0 lds\n"
      "v_mfma_f32_16x16x32_bf16 v[132:135], a[132:135], a[204:207], v[132:135]\n"
      "ds_read_b128 a[248:251], v4 offset:14336\n"
      "v_mfma_f32_16x16x32_bf16 v[132:135], a[128:131], a[200:203], v[132:135]\n"
      "ds_read_b128 a[252:255], v5 offset:14336\n"
      "s_waitcnt lgkmcnt(9)\n"
      "v_mfma_f32_16x16x32_bf16 v[136:139], a[128:131], a[208:211], v[136:139]\n"
      "v_xor_b32 v4, s55, v4\n"
      "v_xor_b32 v5, s55, v5\n"
      "v_mfma_f32_16x16x32_bf16 v[136:139], a[132:135], a[212:215], v[136:139]\n"
      "buffer_load_dwordx4 v9, %36, s65 offen offset:2048 lds\n"
      "v_mfma_f32_16x16x32_bf16 v[168:171], a[140:143], a[212:215], v[168:171]\n"
      "s_bitset1_b32 m0, 13\n"
      "v_mfma_f32_16x16x32_bf16 v[168:171], a[136:139], a[208:211], v[168:171]\n"
      "v_mfma_f32_16x16x32_bf16 v[200:203], a[144:147], a[208:211], v[200:203]\n"
      "s_add_u32 s90, s71, 3\n"
      "v_mfma_f32_16x16x32_bf16 v[200:203], a[148:151], a[212:215], v[200:203]\n"
      "buffer_load_dwordx4 v11, %36, s65 offen offset:0 lds\n"
      "v_mfma_f32_16x16x32_bf16 v[232:235], a[156:159], a[212:215], v[232:235]\n"
      "v_readlane_b32 s86, v22, s90\n"
      "v_mfma_f32_16x16x32_bf16 v[232:235], a[152:155], a[208:211], v[232:235]\n"
      "s_lshr_b32 s86, s86, 13\n"
      "s_waitcnt lgkmcnt(4)\n"
      "v_mfma_f32_16x16x32_bf16 v[236:239], a[152:155], a[216:219], v[236:239]\n"
      "s_mul_i32 s86, s86, 0x401\n"
      "v_mfma_f32_16x16x32_bf16 v[236:239], a[156:159], a[220:223], v[236:239]\n"
      "s_waitcnt vmcnt(31) lgkmcnt(0)\n"
      "s_barrier\n"
      "v_mfma_f32_16x16x32_bf16 v[204:207], a[148:151], a[220:223], v[204:207]\n"
      "buffer_load_dwordx4 v13, %36, s65 offen offset:2048 lds\n"
      "v_mfma_f32_16x16x32_bf16 v[204:207], a[144:147], a[216:219], v[204:207]\n"
      "ds_read_b64_tr_b16 a[160:161], v0 offset:0\n"
      "s_and_b32 s86, s86, 0xe00780\n"
      "v_mfma_f32_16x16x32_bf16 v[172:175], a[136:139], a[216:219], v[172:175]\n"
      "s_or_b32 s64, s86, %43\n"
      "v_mfma_f32_16x16x32_bf16 v[172:175], a[140:143], a[220:223], v[172:175]\n"
      "s_or_b32 m0, s51, %41\n"
      "v_mfma_f32_16x16x32_bf16 v[140:143], a[132:135], a[220:223], v[140:143]\n"
      "buffer_load_dwordx4 v14, s[40:43], s64 offen offset:0 sc0 sc1 lds\n"
      "v_mfma_f32_16x16x32_bf16 v[140:143], a[128:131], a[216:219], v[140:143]\n"
      "ds_read_b64_tr_b16 a[162:163], v0 offset:512\n"
      "v_mfma_f32_16x16x32_bf16 v[144:147], a[128:131], a[224:227], v[144:147]\n"
      "ds_read_b64_tr_b16 a[164:165], v0 offset:16384\n"
      "v_mfma_f32_16x16x32_bf16 v[144:147], a[132:135], a[228:231], v[144:147]\n"
      "ds_read_b64_tr_b16 a[166:167], v0 offset:16896\n"
      "v_mfma_f32_16x16x32_bf16 v[176:179], a[140:143], a[228:231], v[176:179]\n"
      "buffer_load_dwordx4 v15, s[40:43], s64 offen offset:1024 sc0 sc1 lds\n"
      "v_mfma_f32_16x16x32_bf16 v[176:179], a[136:139], a[224:227], v[176:179]\n"
      "ds_read_b64_tr_b16 a[168:169], v1 offset:0\n"
      "v_xor_b32 v0, s56, v0\n"
      "v_mfma_f32_16x16x32_bf16 v[208:211], a[144:147], a[224:227], v[208:211]\n"
      "ds_read_b64_tr_b16 a[170:171], v1 offset:512\n"
      "v_mfma_f32_16x16x32_bf16 v[208:211], a[148:151], a[228:231], v[208:211]\n"
      "ds_read_b64_tr_b16 a[172:173], v1 offset:16384\n"
      "v_mfma_f32_16x16x32_bf16 v[240:243], a[156:159], a[228:231], v[240:243]\n"
      "buffer_load_dwordx4 v16, s[40:43], s64 offen offset:2048 sc0 sc1 lds\n"
      "v_mfma_f32_16x16x32_bf16 v[240:243], a[152:155], a[224:227], v[240:243]\n"
      "ds_read_b64_tr_b16 a[174:175], v1 offset:16896\n"
      "v_mfma_f32_16x16x32_bf16 v[244:247], a[152:155], a[232:235], v[244:247]\n"
      "ds_read_b64_tr_b16 a[176:177], v2 offset:0\n"
      "v_xor_b32 v1, s56, v1\n"
      "v_mfma_f32_16x16x32_bf16 v[244:247], a[156:159], a[236:239], v[244:247]\n"
      "ds_read_b64_tr_b16 a[178:179], v2 offset:512\n"
      "v_mfma_f32_16x16x32_bf16 v[212:215], a[148:151], a[236:239], v[212:215]\n"
      "buffer_load_dwordx4 v17, s[40:43], s64 offen offset:3072 sc0 sc1 lds\n"
      "v_mfma_f32_16x16x32_bf16 v[212:215], a[144:147], a[232:235], v[212:215]\n"
      "s_bitset1_b32 m0, 12\n"
      "v_mfma_f32_16x16x32_bf16 v[180:183], a[136:139], a[232:235], v[180:183]\n"
      "ds_read_b128 a[192:195], v4 offset:0\n"
      "v_mfma_f32_16x16x32_bf16 v[180:183], a[140:143], a[236:239], v[180:183]\n"
      "ds_read_b128 a[196:199], v5 offset:0\n"
      "v_mfma_f32_16x16x32_bf16 v[148:151], a[132:135], a[236:239], v[148:151]\n"
      "buffer_load_dwordx4 v18, s[40:43], s64 offen offset:0 sc0 sc1 lds\n"
      "v_mfma_f32_16x16x32_bf16 v[148:151], a[128:131], a[232:235], v[148:151]\n"
      "ds_read_b64_tr_b16 a[180:181], v2 offset:16384\n"
      "v_mfma_f32_16x16x32_bf16 v[152:155], a[128:131], a[240:243], v[152:155]\n"
      "ds_read_b64_tr_b16 a[182:183], v2 offset:16896\n"
      "v_mfma_f32_16x16x32_bf16 v[152:155], a[132:135], a[244:247], v[152:155]\n"
      "ds_read_b64_tr_b16 a[184:185], v3 offset:0\n"
      "v_xor_b32 v2, s56, v2\n"
      "v_mfma_f32_16x16x32_bf16 v[184:187], a[140:143], a[244:247], v[184:187]\n"
      "buffer_load_dwordx4 v19, s[40:43], s64 offen offset:1024 sc0 sc1 lds\n"
      "v_mfma_f32_16x16x32_bf16 v[184:187], a[136:139], a[240:243], v[184:187]\n"
      "ds_read_b64_tr_b16 a[186:187], v3 offset:512\n"
      "v_mfma_f32_16x16x32_bf16 v[216:219], a[144:147], a[240:243], v[216:219]\n"
      "ds_read_b128 a[200:203], v4 offset:2048\n"
      "v_mfma_f32_16x16x32_bf16 v[216:219], a[148:151], a[244:247], v[216:219]\n"
      "ds_read_b128 a[204:207], v5 offset:2048\n"
      "v_mfma_f32_16x16x32_bf16 v[248:251], a[156:159], a[244:247], v[248:251]\n"
      "buffer_load_dwordx4 v20, s[40:43], s64 offen offset:2048 sc0 sc1 lds\n"
      "v_mfma_f32_16x16x32_bf16 v[248:251], a[152:155], a[240:243], v[248:251]\n"
      "ds_read_b64_tr_b16 a[188:189], v3 offset:16384\n"
      "v_mfma_f32_16x16x32_bf16 v[252:255], a[152:155], a[248:251], v[252:255]\n"
      "ds_read_b64_tr_b16 a[190:191], v3 offset:16896\n"
      "v_mfma_f32_16x16x32_bf16 v[252:255], a[156:159], a[252:255], v[252:255]\n"
      "v_xor_b32 v3, s56, v3\n"
      "s_add_u32 s90, s71, 4\n"
      "v_mfma_f32_16x16x32_bf16 v[220:223], a[148:151], a[252:255], v[220:223]\n"
      "buffer_load_dwordx4 v21, s[40:43], s64 offen offset:3072 sc0 sc1 lds\n"
      "v_mfma_f32_16x16x32_bf16 v[220:223], a[144:147], a[248:251], v[220:223]\n"
      "v_readlane_b32 s57, v22, s90\n"
      "v_mfma_f32_16x16x32_bf16 v[188:191], a[136:139], a[248:251], v[188:191]\n"
      "s_add_u32 s65, s57, 0\n"
      "v_mfma_f32_16x16x32_bf16 v[188:191], a[140:143], a[252:255], v[188:191]\n"
      "s_or_b32 m0, s52, %42\n"
      "v_mfma_f32_16x16x32_bf16 v[156:159], a[132:135], a[252:255], v[156:159]\n"
      "buffer_load_dwordx4 v7, %36, s65 offen offset:1024 lds\n"
      "v_mfma_f32_16x16x32_bf16 v[156:159], a[128:131], a[248:251], v[156:159]\n"
      "ds_read_b128 a[208:211], v4 offset:4096\n"
      "s_waitcnt lgkmcnt(7)\n"
      "v_mfma_f32_16x16x32_bf16 v[128:131], a[160:163], a[192:195], v[128:131]\n"
      "ds_read_b128 a[212:215], v5 offset:4096\n"
      "v_mfma_f32_16x16x32_bf16 v[128:131], a[164:167], a[196:199], v[128:131]\n"
      "ds_read_b128 a[216:219], v4 offset:6144\n"
      "v_mfma_f32_16x16x32_bf16 v[160:163], a[172:175], a[196:199], v[160:163]\n"
      "buffer_load_dwordx4 v9, %36, s65 offen offset:3072 lds\n"
      "v_mfma_f32_16x16x32_bf16 v[160:163], a[168:171], a[192:195], v[160:163]\n"
      "s_bitset1_b32 m0, 13\n"
      "s_xor_b32 s55, s53, s50\n"
      "v_mfma_f32_16x16x32_bf16 v[192:195], a[176:179], a[192:195], v[192:195]\n"
      "ds_read_b128 a[220:223], v5 offset:6144\n"
      "v_mfma_f32_16x16x32_bf16 v[192:195], a[180:183], a[196:199], v[192:195]\n"
      "ds_read_b128 a[224:227], v4 offset:8192\n"
      "s_waitcnt lgkmcnt(5)\n"
      "v_mfma_f32_16x16x32_bf16 v[224:227], a[188:191], a[196:199], v[224:227]\n"
      "buffer_load_dwordx4 v11, %36, s65 offen offset:1024 lds\n"
      "v_mfma_f32_16x16x32_bf16 v[224:227], a[184:187], a[192:195], v[224:227]\n"
      "ds_read_b128 a[228:231], v5 offset:8192\n"
      "v_mfma_f32_16x16x32_bf16 v[228:231], a[184:187], a[200:203], v[228:231]\n"
      "ds_read_b128 a[232:235], v4 offset:10240\n"
      "v_mfma_f32_16x16x32_bf16 v[228:231], a[188:191], a[204:207], v[228:231]\n"
      "ds_read_b128 a[236:239], v5 offset:10240\n"
      "v_mfma_f32_16x16x32_bf16 v[196:199], a[180:183], a[204:207], v[196:199]\n"
      "buffer_load_dwordx4 v13, %36, s65 offen offset:3072 lds\n"
      "v_mfma_f32_16x16x32_bf16 v[196:199], a[176:179], a[200:203], v[196:199]\n"
      "ds_read_b128 a[240:243], v4 offset:12288\n"
      "v_mfma_f32_16x16x32_bf16 v[164:167], a[168:171], a[200:203], v[164:167]\n"
      "ds_read_b128 a[244:247], v5 offset:12288\n"
      "v_mfma_f32_16x16x32_bf16 v[164:167], a[172:175], a[204:207], v[164:167]\n"
      "ds_read_b128 a[248:251], v4 offset:14336\n"
      "v_mfma_f32_16x16x32_bf16 v[132:135], a[164:167], a[204:207], v[132:135]\n"
      "ds_read_b128 a[252:255], v5 offset:14336\n"
      "v_mfma_f32_16x16x32_bf16 v[132:135], a[160:163], a[200:203], v[132:135]\n"
      "v_xor_b32 v4, s55, v4\n"
      "v_xor_b32 v5, s55, v5\n"
      "s_waitcnt lgkmcnt(10)\n"
      "v_mfma_f32_16x16x32_bf16 v[136:139], a[160:163], a[208:211], v[136:139]\n"
      "v_and_b32 v26, 0xe0, v30\n"
      "v_mfma_f32_16x16x32_bf16 v[136:139], a[164:167], a[212:215], v[136:139]\n"
      "v_lshlrev_b32 v26, 10, v26\n"
      "v_mfma_f32_16x16x32_bf16 v[168:171], a[172:175], a[212:215], v[168:171]\n"
      "v_lshrrev_b32 v24, 8, v30\n"
      "v_mfma_f32_16x16x32_bf16 v[168:171], a[168:171], a[208:211], v[168:171]\n"
      "s_add_u32 s90, s71, 4\n"
      "v_mfma_f32_16x16x32_bf16 v[200:203], a[176:179], a[208:211], v[200:203]\n"
      "v_readlane_b32 s86, v22, s90\n"
      "v_mfma_f32_16x16x32_bf16 v[200:203], a[180:183], a[212:215], v[200:203]\n"
      "s_lshr_b32 s86, s86, 13\n"
      "v_mfma_f32_16x16x32_bf16 v[232:235], a[188:191], a[212:215], v[232:235]\n"
      "s_mul_i32 s86, s86, 0x401\n"
      "v_mfma_f32_16x16x32_bf16 v[232:235], a[184:187], a[208:211], v[232:235]\n"
      "s_and_b32 s86, s86, 0xe00780\n"
      "s_waitcnt lgkmcnt(5)\n"
      "v_mfma_f32_16x16x32_bf16 v[236:239], a[184:187], a[216:219], v[236:239]\n"
      "s_or_b32 s64, s86, %43\n"
      "v_mfma_f32_16x16x32_bf16 v[236:239], a[188:191], a[220:223], v[236:239]\n"
      "s_waitcnt vmcnt(16) lgkmcnt(0)\n"
      "s_barrier\n"
      "v_mfma_f32_16x16x32_bf16 v[204:207], a[180:183], a[220:223], v[204:207]\n"
      "s_or_b32 m0, s53, %41\n"
      "v_mfma_f32_16x16x32_bf16 v[204:207], a[176:179], a[216:219], v[204:207]\n"
      "buffer_load_dwordx4 v14, s[40:43], s64 offen offset:0 sc0 sc1 lds\n"
      "v_mfma_f32_16x16x32_bf16 v[172:175], a[168:171], a[216:219], v[172:175]\n"
      "s_mov_b64 exec, 1\n"
      "buffer_store_dword v31, off, s[28:31], %46 sc0 sc1\n"
      "s_mov_b64 exec, -1\n"
      "v_mfma_f32_16x16x32_bf16 v[172:175], a[172:175], a[220:223], v[172:175]\n"
      "ds_read_b64_tr_b16 a[128:129], v0 offset:1024\n"
      "v_mfma_f32_16x16x32_bf16 v[140:143], a[164:167], a[220:223], v[140:143]\n"
      "ds_read_b64_tr_b16 a[130:131], v0 offset:1536\n"
      "v_mfma_f32_16x16x32_bf16 v[140:143], a[160:163], a[216:219], v[140:143]\n"
      "buffer_load_dwordx4 v15, s[40:43], s64 offen offset:1024 sc0 sc1 lds\n"
      "v_mfma_f32_16x16x32_bf16 v[144:147], a[160:163], a[224:227], v[144:147]\n"
      "ds_read_b64_tr_b16 a[132:133], v0 offset:17408\n"
      "v_mfma_f32_16x16x32_bf16 v[144:147], a[164:167], a[228:231], v[144:147]\n"
      "ds_read_b64_tr_b16 a[134:135], v0 offset:17920\n"
      "v_mfma_f32_16x16x32_bf16 v[176:179], a[172:175], a[228:231], v[176:179]\n"
      "ds_read_b64_tr_b16 a[136:137], v1 offset:1024\n"
      "v_mfma_f32_16x16x32_bf16 v[176:179], a[168:171], a[224:227], v[176:179]\n"
      "buffer_load_dwordx4 v16, s[40:43], s64 offen offset:2048 sc0 sc1 lds\n"
      "v_mfma_f32_16x16x32_bf16 v[208:211], a[176:179], a[224:227], v[208:211]\n"
      "ds_read_b64_tr_b16 a[138:139], v1 offset:1536\n"
      "v_mfma_f32_16x16x32_bf16 v[208:211], a[180:183], a[228:231], v[208:211]\n"
      "ds_read_b64_tr_b16 a[140:141], v1 offset:17408\n"
      "v_mfma_f32_16x16x32_bf16 v[240:243], a[188:191], a[228:231], v[240:243]\n"
      "ds_read_b64_tr_b16 a[142:143], v1 offset:17920\n"
      "v_mfma_f32_16x16x32_bf16 v[240:243], a[184:187], a[224:227], v[240:243]\n"
      "buffer_load_dwordx4 v17, s[40:43], s64 offen offset:3072 sc0 sc1 lds\n"
      "v_mfma_f32_16x16x32_bf16 v[244:247], a[184:187], a[232:235], v[244:247]\n"
      "s_bitset1_b32 m0, 12\n"
      "v_mfma_f32_16x16x32_bf16 v[244:247], a[188:191], a[236:239], v[244:247]\n"
      "ds_read_b64_tr_b16 a[144:145], v2 offset:1024\n"
      "v_mfma_f32_16x16x32_bf16 v[212:215], a[180:183], a[236:239], v[212:215]\n"
      "ds_read_b64_tr_b16 a[146:147], v2 offset:1536\n"
      "v_mfma_f32_16x16x32_bf16 v[212:215], a[176:179], a[232:235], v[212:215]\n"
      "buffer_load_dwordx4 v18, s[40:43], s64 offen offset:0 sc0 sc1 lds\n"
      "v_mfma_f32_16x16x32_bf16 v[180:183], a[168:171], a[232:235], v[180:183]\n"
      "ds_read_b128 a[192:195], v4 offset:0\n"
      "v_mfma_f32_16x16x32_bf16 v[180:183], a[172:175], a[236:239], v[180:183]\n"
      "ds_read_b128 a[196:199], v5 offset:0\n"
      "v_mfma_f32_16x16x32_bf16 v[148:151], a[164:167], a[236:239], v[148:151]\n"
      "ds_read_b64_tr_b16 a[148:149], v2 offset:17408\n"
      "v_mfma_f32_16x16x32_bf16 v[148:151], a[160:163], a[232:235], v[148:151]\n"
      "buffer_load_dwordx4 v19, s[40:43], s64 offen offset:1024 sc0 sc1 lds\n"
      "v_mfma_f32_16x16x32_bf16 v[152:155], a[160:163], a[240:243], v[152:155]\n"
      "ds_read_b64_tr_b16 a[150:151], v2 offset:17920\n"
      "v_mfma_f32_16x16x32_bf16 v[152:155], a[164:167], a[244:247], v[152:155]\n"
      "ds_read_b64_tr_b16 a[152:153], v3 offset:1024\n"
      "v_mfma_f32_16x16x32_bf16 v[184:187], a[172:175], a[244:247], v[184:187]\n"
      "ds_read_b64_tr_b16 a[154:155], v3 offset:1536\n"
      "v_mfma_f32_16x16x32_bf16 v[184:187], a[168:171], a[240:243], v[184:187]\n"
      "buffer_load_dwordx4 v20, s[40:43], s64 offen offset:2048 sc0 sc1 lds\n"
      "v_mfma_f32_16x16x32_bf16 v[216:219], a[176:179], a[240:243], v[216:219]\n"
      "ds_read_b128 a[200:203], v4 offset:2048\n"
      "v_mfma_f32_16x16x32_bf16 v[216:219], a[180:183], a[244:247], v[216:219]\n"
      "ds_read_b128 a[204:207], v5 offset:2048\n"
      "v_mfma_f32_16x16x32_bf16 v[248:251], a[188:191], a[244:247], v[248:251]\n"
      "ds_read_b64_tr_b16 a[156:157], v3 offset:17408\n"
      "s_add_u32 s90, s71, 5\n"
      "v_mfma_f32_16x16x32_bf16 v[248:251], a[184:187], a[240:243], v[248:251]\n"
      "buffer_load_dwordx4 v21, s[40:43], s64 offen offset:3072 sc0 sc1 lds\n"
      "v_mfma_f32_16x16x32_bf16 v[252:255], a[184:187], a[248:251], v[252:255]\n"
      "ds_read_b64_tr_b16 a[158:159], v3 offset:17920\n"
      "v_readlane_b32 s57, v22, s90\n"
      "v_mfma_f32_16x16x32_bf16 v[252:255], a[188:191], a[252:255], v[252:255]\n"
      "s_add_u32 s65, s57, 1024\n"
      "v_mfma_f32_16x16x32_bf16 v[220:223], a[180:183], a[252:255], v[220:223]\n"
      "s_or_b32 m0, s52, %42\n"
      "v_mfma_f32_16x16x32_bf16 v[220:223], a[176:179], a[248:251], v[220:223]\n"
      "buffer_load_dwordx4 v7, %36, s65 offen offset:0 lds\n"
      "v_mfma_f32_16x16x32_bf16 v[188:191], a[168:171], a[248:251], v[188:191]\n"
      "ds_read_b128 a[208:211], v4 offset:4096\n"
      "v_mfma_f32_16x16x32_bf16 v[188:191], a[172:175], a[252:255], v[188:191]\n"
      "ds_read_b128 a[212:215], v5 offset:4096\n"
      "v_mfma_f32_16x16x32_bf16 v[156:159], a[164:167], a[252:255], v[156:159]\n"
      "v_lshl_or_b32 v26, v24, 5, v26\n"
      "v_mfma_f32_16x16x32_bf16 v[156:159], a[160:163], a[248:251], v[156:159]\n"
      "buffer_load_dwordx4 v9, %36, s65 offen offset:2048 lds\n"
      "s_waitcnt lgkmcnt(8)\n"
      "v_mfma_f32_16x16x32_bf16 v[128:131], a[128:131], a[192:195], v[128:131]\n"
      "s_bitset1_b32 m0, 13\n"
      "s_xor_b32 s55, s50, s51\n"
      "v_mfma_f32_16x16x32_bf16 v[128:131], a[132:135], a[196:199], v[128:131]\n"
      "ds_read_b128 a[216:219], v4 offset:6144\n"
      "v_mfma_f32_16x16x32_bf16 v[160:163], a[140:143], a[196:199], v[160:163]\n"
      "ds_read_b128 a[220:223], v5 offset:6144\n"
      "v_mfma_f32_16x16x32_bf16 v[160:163], a[136:139], a[192:195], v[160:163]\n"
      "buffer_load_dwordx4 v11, %36, s65 offen offset:0 lds\n"
      "v_mfma_f32_16x16x32_bf16 v[192:195], a[144:147], a[192:195], v[192:195]\n"
      "ds_read_b128 a[224:227], v4 offset:8192\n"
      "v_mfma_f32_16x16x32_bf16 v[192:195], a[148:151], a[196:199], v[192:195]\n"
      "ds_read_b128 a[228:231], v5 offset:8192\n"
      "s_waitcnt lgkmcnt(6)\n"
      "v_mfma_f32_16x16x32_bf16 v[224:227], a[156:159], a[196:199], v[224:227]\n"
      "ds_read_b128 a[232:235], v4 offset:10240\n"
      "v_mfma_f32_16x16x32_bf16 v[224:227], a[152:155], a[192:195], v[224:227]\n"
      "buffer_load_dwordx4 v13, %36, s65 offen offset:2048 lds\n"
      "v_mfma_f32_16x16x32_bf16 v[228:231], a[152:155], a[200:203], v[228:231]\n"
      "ds_read_b128 a[236:239], v5 offset:10240\n"
      "v_mfma_f32_16x16x32_bf16 v[228:231], a[156:159], a[204:207], v[228:231]\n"
      "ds_read_b128 a[240:243], v4 offset:12288\n"
      "v_mfma_f32_16x16x32_bf16 v[196:199], a[148:151], a[204:207], v[196:199]\n"
      "ds_read_b128 a[244:247], v5 offset:12288\n"
      "v_mfma_f32_16x16x32_bf16 v[196:199], a[144:147], a[200:203], v[196:199]\n"
      "ds_read_b128 a[248:251], v4 offset:14336\n"
      "v_mfma_f32_16x16x32_bf16 v[164:167], a[136:139], a[200:203], v[164:167]\n"
      "ds_read_b128 a[252:255], v5 offset:14336\n"
      "v_mfma_f32_16x16x32_bf16 v[164:167], a[140:143], a[204:207], v[164:167]\n"
      "v_xor_b32 v4, s55, v4\n"
      "v_xor_b32 v5, s55, v5\n"
      "v_mfma_f32_16x16x32_bf16 v[132:135], a[132:135], a[204:207], v[132:135]\n"
      "v_and_or_b32 v26, v30, 16, v26\n"
      "v_mfma_f32_16x16x32_bf16 v[132:135], a[128:131], a[200:203], v[132:135]\n"
      "v_mov_b32 v24, 0x3f80\n"
      "s_waitcnt lgkmcnt(10)\n"
      "v_mfma_f32_16x16x32_bf16 v[136:139], a[128:131], a[208:211], v[136:139]\n"
      "v_mov_b32 v25, 0\n"
      "v_mfma_f32_16x16x32_bf16 v[136:139], a[132:135], a[212:215], v[136:139]\n"
      "v_lshlrev_b64 v[24:25], v30, v[24:25]\n"
      "v_mfma_f32_16x16x32_bf16 v[168:171], a[140:143], a[212:215], v[168:171]\n"
      "v_mfma_f32_16x16x32_bf16 v[168:171], a[136:139], a[208:211], v[168:171]\n"
      "s_add_u32 s90, s71, 5\n"
      "v_mfma_f32_16x16x32_bf16 v[200:203], a[144:147], a[208:211], v[200:203]\n"
      "v_readlane_b32 s86, v22, s90\n"
      "v_mfma_f32_16x16x32_bf16 v[200:203], a[148:151], a[212:215], v[200:203]\n"
      "s_lshr_b32 s86, s86, 13\n"
      "v_mfma_f32_16x16x32_bf16 v[232:235], a[156:159], a[212:215], v[232:235]\n"
      "s_mul_i32 s86, s86, 0x401\n"
      "v_mfma_f32_16x16x32_bf16 v[232:235], a[152:155], a[208:211], v[232:235]\n"
      "s_and_b32 s86, s86, 0xe00780\n"
      "s_waitcnt lgkmcnt(4)\n"
      "v_mfma_f32_16x16x32_bf16 v[236:239], a[152:155], a[216:219], v[236:239]\n"
      "s_or_b32 s64, s86, %43\n"
      "v_mfma_f32_16x16x32_bf16 v[236:239], a[156:159], a[220:223], v[236:239]\n"
      "s_waitcnt vmcnt(17) lgkmcnt(0)\n"
      "s_barrier\n"
      "v_mfma_f32_16x16x32_bf16 v[204:207], a[148:151], a[220:223], v[204:207]\n"
      "s_or_b32 m0, s50, %41\n"
      "v_mfma_f32_16x16x32_bf16 v[204:207], a[144:147], a[216:219], v[204:207]\n"
      "buffer_load_dwordx4 v14, s[40:43], s64 offen offset:0 sc0 sc1 lds\n"
      "v_mfma_f32_16x16x32_bf16 v[172:175], a[136:139], a[216:219], v[172:175]\n"
      "ds_read_b64_tr_b16 a[160:161], v0 offset:0\n"
      "v_mfma_f32_16x16x32_bf16 v[172:175], a[140:143], a[220:223], v[172:175]\n"
      "ds_read_b64_tr_b16 a[162:163], v0 offset:512\n"
      "v_mfma_f32_16x16x32_bf16 v[140:143], a[132:135], a[220:223], v[140:143]\n"
      "ds_read_b64_tr_b16 a[164:165], v0 offset:16384\n"
      "v_mfma_f32_16x16x32_bf16 v[140:143], a[128:131], a[216:219], v[140:143]\n"
      "buffer_load_dwordx4 v15, s[40:43], s64 offen offset:1024 sc0 sc1 lds\n"
      "v_mfma_f32_16x16x32_bf16 v[144:147], a[128:131], a[224:227], v[144:147]\n"
      "ds_read_b64_tr_b16 a[166:167], v0 offset:16896\n"
      "v_mfma_f32_16x16x32_bf16 v[144:147], a[132:135], a[228:231], v[144:147]\n"
      "ds_read_b64_tr_b16 a[168:169], v1 offset:0\n"
      "v_xor_b32 v0, s56, v0\n"
      "v_mfma_f32_16x16x32_bf16 v[176:179], a[140:143], a[228:231], v[176:179]\n"
      "ds_read_b64_tr_b16 a[170:171], v1 offset:512\n"
      "v_mfma_f32_16x16x32_bf16 v[176:179], a[136:139], a[224:227], v[176:179]\n"
      "buffer_load_dwordx4 v16, s[40:43], s64 offen offset:2048 sc0 sc1 lds\n"
      "v_mfma_f32_16x16x32_bf16 v[208:211], a[144:147], a[224:227], v[208:211]\n"
      "ds_read_b64_tr_b16 a[172:173], v1 offset:16384\n"
      "v_mfma_f32_16x16x32_bf16 v[208:211], a[148:151], a[228:231], v[208:211]\n"
      "ds_read_b64_tr_b16 a[174:175], v1 offset:16896\n"
      "v_mfma_f32_16x16x32_bf16 v[240:243], a[156:159], a[228:231], v[240:243]\n"
      "ds_read_b64_tr_b16 a[176:177], v2 offset:0\n"
      "v_xor_b32 v1, s56, v1\n"
      "v_mfma_f32_16x16x32_bf16 v[240:243], a[152:155], a[224:227], v[240:243]\n"
      "buffer_load_dwordx4 v17, s[40:43], s64 offen offset:3072 sc0 sc1 lds\n"
      "v_mfma_f32_16x16x32_bf16 v[244:247], a[152:155], a[232:235], v[244:247]\n"
      "s_bitset1_b32 m0, 12\n"
      "v_mfma_f32_16x16x32_bf16 v[244:247], a[156:159], a[236:239], v[244:247]\n"
      "ds_read_b64_tr_b16 a[178:179], v2 offset:512\n"
      "v_mfma_f32_16x16x32_bf16 v[212:215], a[148:151], a[236:239], v[212:215]\n"
      "ds_read_b64_tr_b16 a[180:181], v2 offset:16384\n"
      "v_mfma_f32_16x16x32_bf16 v[212:215], a[144:147], a[232:235], v[212:215]\n"
      "buffer_load_dwordx4 v18, s[40:43], s64 offen offset:0 sc0 sc1 lds\n"
      "v_mfma_f32_16x16x32_bf16 v[180:183], a[136:139], a[232:235], v[180:183]\n"
      "ds_read_b128 a[192:195], v4 offset:0\n"
      "v_mfma_f32_16x16x32_bf16 v[180:183], a[140:143], a[236:239], v[180:183]\n"
      "ds_read_b128 a[196:199], v5 offset:0\n"
      "v_mfma_f32_16x16x32_bf16 v[148:151], a[132:135], a[236:239], v[148:151]\n"
      "ds_read_b64_tr_b16 a[182:183], v2 offset:16896\n"
      "v_mfma_f32_16x16x32_bf16 v[148:151], a[128:131], a[232:235], v[148:151]\n"
      "buffer_load_dwordx4 v19, s[40:43], s64 offen offset:1024 sc0 sc1 lds\n"
      "v_mfma_f32_16x16x32_bf16 v[152:155], a[128:131], a[240:243], v[152:155]\n"
      "ds_read_b64_tr_b16 a[184:185], v3 offset:0\n"
      "v_xor_b32 v2, s56, v2\n"
      "v_mfma_f32_16x16x32_bf16 v[152:155], a[132:135], a[244:247], v[152:155]\n"
      "ds_read_b64_tr_b16 a[186:187], v3 offset:512\n"
      "v_mfma_f32_16x16x32_bf16 v[184:187], a[140:143], a[244:247], v[184:187]\n"
      "ds_read_b64_tr_b16 a[188:189], v3 offset:16384\n"
      "v_mfma_f32_16x16x32_bf16 v[184:187], a[136:139], a[240:243], v[184:187]\n"
      "buffer_load_dwordx4 v20, s[40:43], s64 offen offset:2048 sc0 sc1 lds\n"
      "v_mfma_f32_16x16x32_bf16 v[216:219], a[144:147], a[240:243], v[216:219]\n"
      "ds_read_b128 a[200:203], v4 offset:2048\n"
      "v_mfma_f32_16x16x32_bf16 v[216:219], a[148:151], a[244:247], v[216:219]\n"
      "ds_read_b128 a[204:207], v5 offset:2048\n"
      "v_mfma_f32_16x16x32_bf16 v[248:251], a[156:159], a[244:247], v[248:251]\n"
      "ds_read_b64_tr_b16 a[190:191], v3 offset:16896\n"
      "s_add_u32 s90, s71, 6\n"
      "v_mfma_f32_16x16x32_bf16 v[248:251], a[152:155], a[240:243], v[248:251]\n"
      "buffer_load_dwordx4 v21, s[40:43], s64 offen offset:3072 sc0 sc1 lds\n"
      "v_mfma_f32_16x16x32_bf16 v[252:255], a[152:155], a[248:251], v[252:255]\n"
      "v_xor_b32 v3, s56, v3\n"
      "v_readlane_b32 s57, v22, s90\n"
      "v_mfma_f32_16x16x32_bf16 v[252:255], a[156:159], a[252:255], v[252:255]\n"
      "s_add_u32 s65, s57, 0\n"
      "v_mfma_f32_16x16x32_bf16 v[220:223], a[148:151], a[252:255], v[220:223]\n"
      "s_or_b32 m0, s54, %42\n"
      "v_mfma_f32_16x16x32_bf16 v[220:223], a[144:147], a[248:251], v[220:223]\n"
      "buffer_load_dwordx4 v7, %36, s65 offen offset:1024 lds\n"
      "v_mfma_f32_16x16x32_bf16 v[188:191], a[136:139], a[248:251], v[188:191]\n"
      "ds_read_b128 a[208:211], v4 offset:4096\n"
      "v_mfma_f32_16x16x32_bf16 v[188:191], a[140:143], a[252:255], v[188:191]\n"
      "ds_read_b128 a[212:215], v5 offset:4096\n"
      "v_mfma_f32_16x16x32_bf16 v[156:159], a[132:135], a[252:255], v[156:159]\n"
      "v_mfma_f32_16x16x32_bf16 v[156:159], a[128:131], a[248:251], v[156:159]\n"
      "buffer_load_dwordx4 v9, %36, s65 offen offset:3072 lds\n"
      "s_waitcnt lgkmcnt(7)\n"
      "v_mfma_f32_16x16x32_bf16 v[128:131], a[160:163], a[192:195], v[128:131]\n"
      "s_bitset1_b32 m0, 13\n"
      "v_mfma_f32_16x16x32_bf16 v[128:131], a[164:167], a[196:199], v[128:131]\n"
      "ds_read_b128 a[216:219], v4 offset:6144\n"
      "v_mfma_f32_16x16x32_bf16 v[160:163], a[172:175], a[196:199], v[160:163]\n"
      "ds_read_b128 a[220:223], v5 offset:6144\n"
      "v_mfma_f32_16x16x32_bf16 v[160:163], a[168:171], a[192:195], v[160:163]\n"
      "buffer_load_dwordx4 v11, %36, s65 offen offset:1024 lds\n"
      "v_mfma_f32_16x16x32_bf16 v[192:195], a[176:179], a[192:195], v[192:195]\n"
      "ds_read_b128 a[224:227], v4 offset:8192\n"
      "s_xor_b32 s55, s51, s53\n"
      "v_mfma_f32_16x16x32_bf16 v[192:195], a[180:183], a[196:199], v[192:195]\n"
      "ds_read_b128 a[228:231], v5 offset:8192\n"
      "s_waitcnt lgkmcnt(6)\n"
      "v_mfma_f32_16x16x32_bf16 v[224:227], a[188:191], a[196:199], v[224:227]\n"
      "ds_read_b128 a[232:235], v4 offset:10240\n"
      "v_mfma_f32_16x16x32_bf16 v[224:227], a[184:187], a[192:195], v[224:227]\n"
      "buffer_load_dwordx4 v13, %36, s65 offen offset:3072 lds\n"
      "v_mfma_f32_16x16x32_bf16 v[228:231], a[184:187], a[200:203], v[228:231]\n"
      "ds_read_b128 a[236:239], v5 offset:10240\n"
      "v_mfma_f32_16x16x32_bf16 v[228:231], a[188:191], a[204:207], v[228:231]\n"
      "ds_read_b128 a[240:243], v4 offset:12288\n"
      "v_mfma_f32_16x16x32_bf16 v[196:199], a[180:183], a[204:207], v[196:199]\n"
      "ds_read_b128 a[244:247], v5 offset:12288\n"
      "v_mfma_f32_16x16x32_bf16 v[196:199], a[176:179], a[200:203], v[196:199]\n"
      "ds_read_b128 a[248:251], v4 offset:14336\n"
      "v_mfma_f32_16x16x32_bf16 v[164:167], a[168:171], a[200:203], v[164:167]\n"
      "v_mov_b32 v28, 0\n"
      "v_mfma_f32_16x16x32_bf16 v[164:167], a[172:175], a[204:207], v[164:167]\n"
      "s_mov_b32 s66, %45\n"
      "v_mfma_f32_16x16x32_bf16 v[132:135], a[164:167], a[204:207], v[132:135]\n"
      "buffer_load_dword v29, v28, s[28:31], s66 offen sc0 sc1\n"
      "v_mfma_f32_16x16x32_bf16 v[132:135], a[160:163], a[200:203], v[132:135]\n"
      "ds_read_b128 a[252:255], v5 offset:14336\n"
      "s_waitcnt lgkmcnt(10)\n"
      "v_mfma_f32_16x16x32_bf16 v[136:139], a[160:163], a[208:211], v[136:139]\n"
      "v_xor_b32 v4, s55, v4\n"
      "v_xor_b32 v5, s55, v5\n"
      "v_mfma_f32_16x16x32_bf16 v[136:139], a[164:167], a[212:215], v[136:139]\n"
      "v_mfma_f32_16x16x32_bf16 v[168:171], a[172:175], a[212:215], v[168:171]\n"
      "v_mfma_f32_16x16x32_bf16 v[168:171], a[168:171], a[208:211], v[168:171]\n"
      "s_add_u32 s90, s71, 6\n"
      "v_mfma_f32_16x16x32_bf16 v[200:203], a[176:179], a[208:211], v[200:203]\n"
      "v_readlane_b32 s86, v22, s90\n"
      "v_mfma_f32_16x16x32_bf16 v[200:203], a[180:183], a[212:215], v[200:203]\n"
      "s_lshr_b32 s86, s86, 13\n"
      "v_mfma_f32_16x16x32_bf16 v[232:235], a[188:191], a[212:215], v[232:235]\n"
      "s_mul_i32 s86, s86, 0x401\n"
      "v_mfma_f32_16x16x32_bf16 v[232:235], a[184:187], a[208:211], v[232:235]\n"
      "s_and_b32 s86, s86, 0xe00780\n"
      "s_waitcnt lgkmcnt(4)\n"
      "v_mfma_f32_16x16x32_bf16 v[236:239], a[184:187], a[216:219], v[236:239]\n"
      "s_or_b32 s64, s86, %43\n"
      "v_mfma_f32_16x16x32_bf16 v[236:239], a[188:191], a[220:223], v[236:239]\n"
      "s_waitcnt vmcnt(17) lgkmcnt(0)\n"
      "s_barrier\n"
      "v_mfma_f32_16x16x32_bf16 v[204:207], a[180:183], a[220:223], v[204:207]\n"
      "s_or_b32 m0, s51, %41\n"
      "v_mfma_f32_16x16x32_bf16 v[204:207], a[176:179], a[216:219], v[204:207]\n"
      "buffer_load_dwordx4 v14, s[40:43], s64 offen offset:0 sc0 sc1 lds\n"
      "v_mfma_f32_16x16x32_bf16 v[172:175], a[168:171], a[216:219], v[172:175]\n"
      "ds_read_b64_tr_b16 a[128:129], v0 offset:1024\n"
      "v_mfma_f32_16x16x32_bf16 v[172:175], a[172:175], a[220:223], v[172:175]\n"
      "ds_read_b64_tr_b16 a[130:131], v0 offset:1536\n"
      "v_mfma_f32_16x16x32_bf16 v[140:143], a[164:167], a[220:223], v[140:143]\n"
      "ds_read_b64_tr_b16 a[132:133], v0 offset:17408\n"
      "v_mfma_f32_16x16x32_bf16 v[140:143], a[160:163], a[216:219], v[140:143]\n"
      "buffer_load_dwordx4 v15, s[40:43], s64 offen offset:1024 sc0 sc1 lds\n"
      "v_mfma_f32_16x16x32_bf16 v[144:147], a[160:163], a[224:227], v[144:147]\n"
      "ds_read_b64_tr_b16 a[134:135], v0 offset:17920\n"
      "v_mfma_f32_16x16x32_bf16 v[144:147], a[164:167], a[228:231], v[144:147]\n"
      "ds_read_b64_tr_b16 a[136:137], v1 offset:1024\n"
      "v_mfma_f32_16x16x32_bf16 v[176:179], a[172:175], a[228:231], v[176:179]\n"
      "ds_read_b64_tr_b16 a[138:139], v1 offset:1536\n"
      "v_mfma_f32_16x16x32_bf16 v[176:179], a[168:171], a[224:227], v[176:179]\n"
      "buffer_load_dwordx4 v16, s[40:43], s64 offen offset:2048 sc0 sc1 lds\n"
      "v_mfma_f32_16x16x32_bf16 v[208:211], a[176:179], a[224:227], v[208:211]\n"
      "ds_read_b64_tr_b16 a[140:141], v1 offset:17408\n"
      "v_mfma_f32_16x16x32_bf16 v[208:211], a[180:183], a[228:231], v[208:211]\n"
      "ds_read_b64_tr_b16 a[142:143], v1 offset:17920\n"
      "v_mfma_f32_16x16x32_bf16 v[240:243], a[188:191], a[228:231], v[240:243]\n"
      "ds_read_b64_tr_b16 a[144:145], v2 offset:1024\n"
      "v_mfma_f32_16x16x32_bf16 v[240:243], a[184:187], a[224:227], v[240:243]\n"
      "buffer_load_dwordx4 v17, s[40:43], s64 offen offset:3072 sc0 sc1 lds\n"
      "v_mfma_f32_16x16x32_bf16 v[244:247], a[184:187], a[232:235], v[244:247]\n"
      "s_bitset1_b32 m0, 12\n"
      "v_mfma_f32_16x16x32_bf16 v[244:247], a[188:191], a[236:239], v[244:247]\n"
      "ds_read_b64_tr_b16 a[146:147], v2 offset:1536\n"
      "v_mfma_f32_16x16x32_bf16 v[212:215], a[180:183], a[236:239], v[212:215]\n"
      "ds_read_b64_tr_b16 a[148:149], v2 offset:17408\n"
      "v_mfma_f32_16x16x32_bf16 v[212:215], a[176:179], a[232:235], v[212:215]\n"
      "buffer_load_dwordx4 v18, s[40:43], s64 offen offset:0 sc0 sc1 lds\n"
      "v_mfma_f32_16x16x32_bf16 v[180:183], a[168:171], a[232:235], v[180:183]\n"
      "ds_read_b128 a[192:195], v4 offset:0\n"
      "v_mfma_f32_16x16x32_bf16 v[180:183], a[172:175], a[236:239], v[180:183]\n"
      "ds_read_b128 a[196:199], v5 offset:0\n"
      "v_mfma_f32_16x16x32_bf16 v[148:151], a[164:167], a[236:239], v[148:151]\n"
      "ds_read_b64_tr_b16 a[150:151], v2 offset:17920\n"
      "v_mfma_f32_16x16x32_bf16 v[148:151], a[160:163], a[232:235], v[148:151]\n"
      "buffer_load_dwordx4 v19, s[40:43], s64 offen offset:1024 sc0 sc1 lds\n"
      "v_mfma_f32_16x16x32_bf16 v[152:155], a[160:163], a[240:243], v[152:155]\n"
      "ds_read_b64_tr_b16 a[152:153], v3 offset:1024\n"
      "v_mfma_f32_16x16x32_bf16 v[152:155], a[164:167], a[244:247], v[152:155]\n"
      "ds_read_b64_tr_b16 a[154:155], v3 offset:1536\n"
      "v_mfma_f32_16x16x32_bf16 v[184:187], a[172:175], a[244:247], v[184:187]\n"
      "ds_read_b64_tr_b16 a[156:157], v3 offset:17408\n"
      "v_mfma_f32_16x16x32_bf16 v[184:187], a[168:171], a[240:243], v[184:187]\n"
      "buffer_load_dwordx4 v20, s[40:43], s64 offen offset:2048 sc0 sc1 lds\n"
      "v_mfma_f32_16x16x32_bf16 v[216:219], a[176:179], a[240:243], v[216:219]\n"
      "ds_read_b128 a[200:203], v4 offset:2048\n"
      "v_mfma_f32_16x16x32_bf16 v[216:219], a[180:183], a[244:247], v[216:219]\n"
      "ds_read_b128 a[204:207], v5 offset:2048\n"
      "v_mfma_f32_16x16x32_bf16 v[248:251], a[188:191], a[244:247], v[248:251]\n"
      "ds_read_b64_tr_b16 a[158:159], v3 offset:17920\n"
      "s_add_u32 s90, s71, 7\n"
      "v_mfma_f32_16x16x32_bf16 v[248:251], a[184:187], a[240:243], v[248:251]\n"
      "buffer_load_dwordx4 v21, s[40:43], s64 offen offset:3072 sc0 sc1 lds\n"
      "v_mfma_f32_16x16x32_bf16 v[252:255], a[184:187], a[248:251], v[252:255]\n"
      "v_readlane_b32 s57, v22, s90\n"
      "v_mfma_f32_16x16x32_bf16 v[252:255], a[188:191], a[252:255], v[252:255]\n"
      "s_add_u32 s65, s57, 1024\n"
      "v_mfma_f32_16x16x32_bf16 v[220:223], a[180:183], a[252:255], v[220:223]\n"
      "s_or_b32 m0, s54, %42\n"
      "v_mfma_f32_16x16x32_bf16 v[220:223], a[176:179], a[248:251], v[220:223]\n"
      "buffer_load_dwordx4 v7, %36, s65 offen offset:0 lds\n"
      "v_mfma_f32_16x16x32_bf16 v[188:191], a[168:171], a[248:251], v[188:191]\n"
      "ds_read_b128 a[208:211], v4 offset:4096\n"
      "v_mfma_f32_16x16x32_bf16 v[188:191], a[172:175], a[252:255], v[188:191]\n"
      "ds_read_b128 a[212:215], v5 offset:4096\n"
      "v_mfma_f32_16x16x32_bf16 v[156:159], a[164:167], a[252:255], v[156:159]\n"
      "v_mfma_f32_16x16x32_bf16 v[156:159], a[160:163], a[248:251], v[156:159]\n"
      "buffer_load_dwordx4 v9, %36, s65 offen offset:2048 lds\n"
      "s_waitcnt lgkmcnt(7)\n"
      "v_mfma_f32_16x16x32_bf16 v[128:131], a[128:131], a[192:195], v[128:131]\n"
      "s_bitset1_b32 m0, 13\n"
      "v_mfma_f32_16x16x32_bf16 v[128:131], a[132:135], a[196:199], v[128:131]\n"
      "ds_read_b128 a[216:219], v4 offset:6144\n"
      "v_mfma_f32_16x16x32_bf16 v[160:163], a[140:143], a[196:199], v[160:163]\n"
      "ds_read_b128 a[220:223], v5 offset:6144\n"
      "v_mfma_f32_16x16x32_bf16 v[160:163], a[136:139], a[192:195], v[160:163]\n"
      "buffer_load_dwordx4 v11, %36, s65 offen offset:0 lds\n"
      "v_mfma_f32_16x16x32_bf16 v[192:195], a[144:147], a[192:195], v[192:195]\n"
      "ds_read_b128 a[224:227], v4 offset:8192\n"
      "v_mfma_f32_16x16x32_bf16 v[192:195], a[148:151], a[196:199], v[192:195]\n"
      "ds_read_b128 a[228:231], v5 offset:8192\n"
      "s_xor_b32 s55, s53, s50\n"
      "s_waitcnt lgkmcnt(6)\n"
      "v_mfma_f32_16x16x32_bf16 v[224:227], a[156:159], a[196:199], v[224:227]\n"
      "ds_read_b128 a[232:235], v4 offset:10240\n"
      "v_mfma_f32_16x16x32_bf16 v[224:227], a[152:155], a[192:195], v[224:227]\n"
      "buffer_load_dwordx4 v13, %36, s65 offen offset:2048 lds\n"
      "v_mfma_f32_16x16x32_bf16 v[228:231], a[152:155], a[200:203], v[228:231]\n"
      "ds_read_b128 a[236:239], v5 offset:10240\n"
      "v_mfma_f32_16x16x32_bf16 v[228:231], a[156:159], a[204:207], v[228:231]\n"
      "ds_read_b128 a[240:243], v4 offset:12288\n"
      "v_mfma_f32_16x16x32_bf16 v[196:199], a[148:151], a[204:207], v[196:199]\n"
      "s_waitcnt vmcnt(12)\n"
      "v_cmp_gt_u32 vcc, s75, v29\n"
      "v_mfma_f32_16x16x32_bf16 v[196:199], a[144:147], a[200:203], v[196:199]\n"
      "ds_read_b128 a[244:247], v5 offset:12288\n"
      "v_mfma_f32_16x16x32_bf16 v[164:167], a[136:139], a[200:203], v[164:167]\n"
      "s_cbranch_vccnz 146f\n"
      "46:\n"
      "v_mfma_f32_16x16x32_bf16 v[164:167], a[140:143], a[204:207], v[164:167]\n"
      "buffer_load_dwordx4 a[32:35], v30, s[44:47], s58 offen offset:0 sc0 sc1\n"
      "v_mfma_f32_16x16x32_bf16 v[132:135], a[132:135], a[204:207], v[132:135]\n"
      "v_mov_b32 v27, 0x40c0000\n"
      "v_mfma_f32_16x16x32_bf16 v[132:135], a[128:131], a[200:203], v[132:135]\n"
      "global_load_dword v29, v27, s[48:49] sc1\n"
      "s_waitcnt lgkmcnt(8)\n"
      "v_mfma_f32_16x16x32_bf16 v[136:139], a[128:131], a[208:211], v[136:139]\n"
      "ds_read_b128 a[248:251], v4 offset:14336\n"
      "v_mfma_f32_16x16x32_bf16 v[136:139], a[132:135], a[212:215], v[136:139]\n"
      "ds_read_b128 a[252:255], v5 offset:14336\n"
      "s_add_u32 s90, s71, 7\n"
      "v_mfma_f32_16x16x32_bf16 v[168:171], a[140:143], a[212:215], v[168:171]\n"
      "v_xor_b32 v4, s55, v4\n"
      "v_xor_b32 v5, s55, v5\n"
      "v_mfma_f32_16x16x32_bf16 v[168:171], a[136:139], a[208:211], v[168:171]\n"
      "buffer_load_dwordx4 a[36:39], v30, s[44:47], s58 offen offset:1024 sc0 sc1\n"
      "v_mfma_f32_16x16x32_bf16 v[200:203], a[144:147], a[208:211], v[200:203]\n"
      "v_readlane_b32 s86, v22, s90\n"
      "v_mfma_f32_16x16x32_bf16 v[200:203], a[148:151], a[212:215], v[200:203]\n"
      "s_lshr_b32 s86, s86, 13\n"
      "v_mfma_f32_16x16x32_bf16 v[232:235], a[156:159], a[212:215], v[232:235]\n"
      "s_mul_i32 s86, s86, 0x401\n"
      "v_mfma_f32_16x16x32_bf16 v[232:235], a[152:155], a[208:211], v[232:235]\n"
      "s_and_b32 s86, s86, 0xe00780\n"
      "s_waitcnt lgkmcnt(4)\n"
      "v_mfma_f32_16x16x32_bf16 v[236:239], a[152:155], a[216:219], v[236:239]\n"
      "s_or_b32 s64, s86, %43\n"
      "v_mfma_f32_16x16x32_bf16 v[236:239], a[156:159], a[220:223], v[236:239]\n"
      "s_waitcnt lgkmcnt(0)\n"
      "s_barrier\n"
      "v_mfma_f32_16x16x32_bf16 v[204:207], a[148:151], a[220:223], v[204:207]\n"
      "s_or_b32 m0, s53, %41\n"
      "v_mfma_f32_16x16x32_bf16 v[204:207], a[144:147], a[216:219], v[204:207]\n"
      "buffer_load_dwordx4 v14, s[40:43], s64 offen offset:0 sc0 sc1 lds\n"
      "v_mfma_f32_16x16x32_bf16 v[172:175], a[136:139], a[216:219], v[172:175]\n"
      "ds_read_b64_tr_b16 a[160:161], v0 offset:0\n"
      "v_mfma_f32_16x16x32_bf16 v[172:175], a[140:143], a[220:223], v[172:175]\n"
      "buffer_load_dwordx4 a[40:43], v30, s[44:47], s58 offen offset:2048 sc0 sc1\n"
      "v_mfma_f32_16x16x32_bf16 v[140:143], a[132:135], a[220:223], v[140:143]\n"
      "ds_read_b64_tr_b16 a[162:163], v0 offset:512\n"
      "v_mfma_f32_16x16x32_bf16 v[140:143], a[128:131], a[216:219], v[140:143]\n"
      "buffer_load_dwordx4 v15, s[40:43], s64 offen offset:1024 sc0 sc1 lds\n"
      "v_mfma_f32_16x16x32_bf16 v[144:147], a[128:131], a[224:227], v[144:147]\n"
      "ds_read_b64_tr_b16 a[164:165], v0 offset:16384\n"
      "v_mfma_f32_16x16x32_bf16 v[144:147], a[132:135], a[228:231], v[144:147]\n"
      "ds_read_b64_tr_b16 a[166:167], v0 offset:16896\n"
      "v_mfma_f32_16x16x32_bf16 v[176:179], a[140:143], a[228:231], v[176:179]\n"
      "ds_read_b64_tr_b16 a[168:169], v1 offset:0\n"
      "v_xor_b32 v0, s56, v0\n"
      "v_mfma_f32_16x16x32_bf16 v[176:179], a[136:139], a[224:227], v[176:179]\n"
      "buffer_load_dwordx4 v16, s[40:43], s64 offen offset:2048 sc0 sc1 lds\n"
      "v_mfma_f32_16x16x32_bf16 v[208:211], a[144:147], a[224:227], v[208:211]\n"
      "ds_read_b64_tr_b16 a[170:171], v1 offset:512\n"
      "v_mfma_f32_16x16x32_bf16 v[208:211], a[148:151], a[228:231], v[208:211]\n"
      "buffer_load_dwordx4 a[44:47], v30, s[44:47], s58 offen offset:3072 sc0 sc1\n"
      "v_mfma_f32_16x16x32_bf16 v[240:243], a[156:159], a[228:231], v[240:243]\n"
      "s_add_u32 s58, s58, 0x1000\n"
      "v_mfma_f32_16x16x32_bf16 v[240:243], a[152:155], a[224:227], v[240:243]\n"
      "buffer_load_dwordx4 v17, s[40:43], s64 offen offset:3072 sc0 sc1 lds\n"
      "v_mfma_f32_16x16x32_bf16 v[244:247], a[152:155], a[232:235], v[244:247]\n"
      "s_bitset1_b32 m0, 12\n"
      "v_mfma_f32_16x16x32_bf16 v[244:247], a[156:159], a[236:239], v[244:247]\n"
      "ds_read_b64_tr_b16 a[172:173], v1 offset:16384\n"
      "v_mfma_f32_16x16x32_bf16 v[212:215], a[148:151], a[236:239], v[212:215]\n"
      "ds_read_b64_tr_b16 a[174:175], v1 offset:16896\n"
      "v_mfma_f32_16x16x32_bf16 v[212:215], a[144:147], a[232:235], v[212:215]\n"
      "buffer_load_dwordx4 v18, s[40:43], s64 offen offset:0 sc0 sc1 lds\n"
      "v_mfma_f32_16x16x32_bf16 v[180:183], a[136:139], a[232:235], v[180:183]\n"
      "ds_read_b128 a[192:195], v4 offset:0\n"
      "v_xor_b32 v1, s56, v1\n"
      "v_mfma_f32_16x16x32_bf16 v[180:183], a[140:143], a[236:239], v[180:183]\n"
      "buffer_load_dwordx4 a[48:51], v30, s[44:47], s58 offen offset:0 sc0 sc1\n"
      "v_mfma_f32_16x16x32_bf16 v[148:151], a[132:135], a[236:239], v[148:151]\n"
      "ds_read_b128 a[196:199], v5 offset:0\n"
      "v_mfma_f32_16x16x32_bf16 v[148:151], a[128:131], a[232:235], v[148:151]\n"
      "buffer_load_dwordx4 v19, s[40:43], s64 offen offset:1024 sc0 sc1 lds\n"
      "v_mfma_f32_16x16x32_bf16 v[152:155], a[128:131], a[240:243], v[152:155]\n"
      "ds_read_b64_tr_b16 a[176:177], v2 offset:0\n"
      "ds_read_b64_tr_b16 a[188:189], v3 offset:16384\n"
      "v_mfma_f32_16x16x32_bf16 v[152:155], a[132:135], a[244:247], v[152:155]\n"
      "ds_read_b64_tr_b16 a[178:179], v2 offset:512\n"
      "ds_read_b64_tr_b16 a[190:191], v3 offset:16896\n"
      "v_mfma_f32_16x16x32_bf16 v[184:187], a[140:143], a[244:247], v[184:187]\n"
      "ds_read_b64_tr_b16 a[180:181], v2 offset:16384\n"
      "v_mfma_f32_16x16x32_bf16 v[184:187], a[136:139], a[240:243], v[184:187]\n"
      "buffer_load_dwordx4 v20, s[40:43], s64 offen offset:2048 sc0 sc1 lds\n"
      "v_mfma_f32_16x16x32_bf16 v[216:219], a[144:147], a[240:243], v[216:219]\n"
      "ds_read_b128 a[200:203], v4 offset:2048\n"
      "v_mfma_f32_16x16x32_bf16 v[216:219], a[148:151], a[244:247], v[216:219]\n"
      "buffer_load_dwordx4 a[52:55], v30, s[44:47], s58 offen offset:1024 sc0 sc1\n"
      "v_mfma_f32_16x16x32_bf16 v[248:251], a[156:159], a[244:247], v[248:251]\n"
      "ds_read_b128 a[204:207], v5 offset:2048\n"
      "v_mfma_f32_16x16x32_bf16 v[248:251], a[152:155], a[240:243], v[248:251]\n"
      "buffer_load_dwordx4 v21, s[40:43], s64 offen offset:3072 sc0 sc1 lds\n"
      "v_mfma_f32_16x16x32_bf16 v[252:255], a[152:155], a[248:251], v[252:255]\n"
      "ds_read_b64_tr_b16 a[182:183], v2 offset:16896\n"
      "v_mfma_f32_16x16x32_bf16 v[252:255], a[156:159], a[252:255], v[252:255]\n"
      "ds_read_b64_tr_b16 a[184:185], v3 offset:0\n"
      "v_xor_b32 v2, s56, v2\n"
      "v_mfma_f32_16x16x32_bf16 v[220:223], a[148:151], a[252:255], v[220:223]\n"
      "ds_read_b64_tr_b16 a[186:187], v3 offset:512\n"
      "v_mfma_f32_16x16x32_bf16 v[220:223], a[144:147], a[248:251], v[220:223]\n"
      "buffer_load_dwordx4 a[56:59], v30, s[44:47], s58 offen offset:2048 sc0 sc1\n"
      "v_mfma_f32_16x16x32_bf16 v[188:191], a[136:139], a[248:251], v[188:191]\n"
      "ds_read_b128 a[208:211], v4 offset:4096\n"
      "v_xor_b32 v3, s56, v3\n"
      "v_mfma_f32_16x16x32_bf16 v[188:191], a[140:143], a[252:255], v[188:191]\n"
      "ds_read_b128 a[212:215], v5 offset:4096\n"
      "v_mfma_f32_16x16x32_bf16 v[156:159], a[132:135], a[252:255], v[156:159]\n"
      "v_mfma_f32_16x16x32_bf16 v[156:159], a[128:131], a[248:251], v[156:159]\n"
      "s_waitcnt lgkmcnt(10)\n"
      "v_mfma_f32_16x16x32_bf16 v[128:131], a[160:163], a[192:195], v[128:131]\n"
      "ds_read_b128 a[216:219], v4 offset:6144\n"
      "s_xor_b32 s55, s50, s51\n"
      "v_mfma_f32_16x16x32_bf16 v[128:131], a[164:167], a[196:199], v[128:131]\n"
      "buffer_load_dwordx4 a[60:63], v30, s[44:47], s58 offen offset:3072 sc0 sc1\n"
      "v_mfma_f32_16x16x32_bf16 v[160:163], a[172:175], a[196:199], v[160:163]\n"
      "s_add_u32 s58, s58, 0x1000\n"
      "v_mfma_f32_16x16x32_bf16 v[160:163], a[168:171], a[192:195], v[160:163]\n"
      "ds_read_b128 a[220:223], v5 offset:6144\n"
      "s_waitcnt lgkmcnt(6)\n"
      "v_mfma_f32_16x16x32_bf16 v[192:195], a[176:179], a[192:195], v[192:195]\n"
      "ds_read_b128 a[224:227], v4 offset:8192\n"
      "v_mfma_f32_16x16x32_bf16 v[192:195], a[180:183], a[196:199], v[192:195]\n"
      "ds_read_b128 a[228:231], v5 offset:8192\n"
      "s_waitcnt lgkmcnt(6)\n"
      "v_mfma_f32_16x16x32_bf16 v[224:227], a[188:191], a[196:199], v[224:227]\n"
      "ds_read_b128 a[232:235], v4 offset:10240\n"
      "v_mfma_f32_16x16x32_bf16 v[224:227], a[184:187], a[192:195], v[224:227]\n"
      "buffer_load_dwordx4 a[64:67], v30, s[44:47], s58 offen offset:0 sc0 sc1\n"
      "v_mfma_f32_16x16x32_bf16 v[228:231], a[184:187], a[200:203], v[228:231]\n"
      "ds_read_b128 a[236:239], v5 offset:10240\n"
      "v_mfma_f32_16x16x32_bf16 v[228:231], a[188:191], a[204:207], v[228:231]\n"
      "ds_read_b128 a[240:243], v4 offset:12288\n"
      "v_mfma_f32_16x16x32_bf16 v[196:199], a[180:183], a[204:207], v[196:199]\n"
      "ds_read_b128 a[244:247], v5 offset:12288\n"
      "v_mfma_f32_16x16x32_bf16 v[196:199], a[176:179], a[200:203], v[196:199]\n"
      "ds_read_b128 a[248:251], v4 offset:14336\n"
      "v_mfma_f32_16x16x32_bf16 v[164:167], a[168:171], a[200:203], v[164:167]\n"
      "ds_read_b128 a[252:255], v5 offset:14336\n"
      "v_mfma_f32_16x16x32_bf16 v[164:167], a[172:175], a[204:207], v[164:167]\n"
      "buffer_load_dwordx4 a[68:71], v30, s[44:47], s58 offen offset:1024 sc0 sc1\n"
      "v_mfma_f32_16x16x32_bf16 v[132:135], a[164:167], a[204:207], v[132:135]\n"
      "v_xor_b32 v4, s55, v4\n"
      "v_xor_b32 v5, s55, v5\n"
      "v_mfma_f32_16x16x32_bf16 v[132:135], a[160:163], a[200:203], v[132:135]\n"
      "s_waitcnt lgkmcnt(9)\n"
      "v_mfma_f32_16x16x32_bf16 v[136:139], a[160:163], a[208:211], v[136:139]\n"
      "v_mfma_f32_16x16x32_bf16 v[136:139], a[164:167], a[212:215], v[136:139]\n"
      "v_mfma_f32_16x16x32_bf16 v[168:171], a[172:175], a[212:215], v[168:171]\n"
      "v_mfma_f32_16x16x32_bf16 v[168:171], a[168:171], a[208:211], v[168:171]\n"
      "buffer_load_dwordx4 a[72:75], v30, s[44:47], s58 offen offset:2048 sc0 sc1\n"
      "v_mfma_f32_16x16x32_bf16 v[200:203], a[176:179], a[208:211], v[200:203]\n"
      "v_mfma_f32_16x16x32_bf16 v[200:203], a[180:183], a[212:215], v[200:203]\n"
      "v_mfma_f32_16x16x32_bf16 v[232:235], a[188:191], a[212:215], v[232:235]\n"
      "v_mfma_f32_16x16x32_bf16 v[232:235], a[184:187], a[208:211], v[232:235]\n"
      "s_waitcnt lgkmcnt(4)\n"
      "v_mfma_f32_16x16x32_bf16 v[236:239], a[184:187], a[216:219], v[236:239]\n"
      "v_mfma_f32_16x16x32_bf16 v[236:239], a[188:191], a[220:223], v[236:239]\n"
      "s_waitcnt vmcnt(20)\n"
      "s_barrier\n"
      "v_mfma_f32_16x16x32_bf16 v[204:207], a[180:183], a[220:223], v[204:207]\n"
      "ds_read_b128 a[192:195], v4 offset:0\n"
      "v_mfma_f32_16x16x32_bf16 v[204:207], a[176:179], a[216:219], v[204:207]\n"
      "buffer_load_dwordx4 a[76:79], v30, s[44:47], s58 offen offset:3072 sc0 sc1\n"
      "v_mfma_f32_16x16x32_bf16 v[172:175], a[168:171], a[216:219], v[172:175]\n"
      "s_add_u32 s58, s58, 0x1000\n"
      "v_mfma_f32_16x16x32_bf16 v[172:175], a[172:175], a[220:223], v[172:175]\n"
      "ds_read_b128 a[196:199], v5 offset:0\n"
      "v_mfma_f32_16x16x32_bf16 v[140:143], a[164:167], a[220:223], v[140:143]\n"
      "ds_read_b128 a[200:203], v4 offset:2048\n"
      "v_mfma_f32_16x16x32_bf16 v[140:143], a[160:163], a[216:219], v[140:143]\n"
      "ds_read_b128 a[204:207], v5 offset:2048\n"
      "v_mfma_f32_16x16x32_bf16 v[144:147], a[160:163], a[224:227], v[144:147]\n"
      "ds_read_b128 a[208:211], v4 offset:4096\n"
      "v_mfma_f32_16x16x32_bf16 v[144:147], a[164:167], a[228:231], v[144:147]\n"
      "buffer_load_dwordx4 a[80:83], v30, s[44:47], s58 offen offset:0 sc0 sc1\n"
      "v_mfma_f32_16x16x32_bf16 v[176:179], a[172:175], a[228:231], v[176:179]\n"
      "ds_read_b128 a[212:215], v5 offset:4096\n"
      "v_mfma_f32_16x16x32_bf16 v[176:179], a[168:171], a[224:227], v[176:179]\n"
      "ds_read_b128 a[216:219], v4 offset:6144\n"
      "v_mfma_f32_16x16x32_bf16 v[208:211], a[176:179], a[224:227], v[208:211]\n"
      "ds_read_b128 a[220:223], v5 offset:6144\n"
      "v_mfma_f32_16x16x32_bf16 v[208:211], a[180:183], a[228:231], v[208:211]\n"
      "ds_read_b64_tr_b16 a[128:129], v0 offset:1024\n"
      "v_mfma_f32_16x16x32_bf16 v[240:243], a[188:191], a[228:231], v[240:243]\n"
      "ds_read_b64_tr_b16 a[130:131], v0 offset:1536\n"
      "v_mfma_f32_16x16x32_bf16 v[240:243], a[184:187], a[224:227], v[240:243]\n"
      "buffer_load_dwordx4 a[84:87], v30, s[44:47], s58 offen offset:1024 sc0 sc1\n"
      "v_mfma_f32_16x16x32_bf16 v[244:247], a[184:187], a[232:235], v[244:247]\n"
      "ds_read_b64_tr_b16 a[132:133], v0 offset:17408\n"
      "v_mfma_f32_16x16x32_bf16 v[244:247], a[188:191], a[236:239], v[244:247]\n"
      "ds_read_b64_tr_b16 a[134:135], v0 offset:17920\n"
      "v_mfma_f32_16x16x32_bf16 v[212:215], a[180:183], a[236:239], v[212:215]\n"
      "ds_read_b64_tr_b16 a[136:137], v1 offset:1024\n"
      "v_mfma_f32_16x16x32_bf16 v[212:215], a[176:179], a[232:235], v[212:215]\n"
      "ds_read_b64_tr_b16 a[138:139], v1 offset:1536\n"
      "v_mfma_f32_16x16x32_bf16 v[180:183], a[168:171], a[232:235], v[180:183]\n"
      "ds_read_b64_tr_b16 a[140:141], v1 offset:17408\n"
      "v_mfma_f32_16x16x32_bf16 v[180:183], a[172:175], a[236:239], v[180:183]\n"
      "buffer_load_dwordx4 a[88:91], v30, s[44:47], s58 offen offset:2048 sc0 sc1\n"
      "v_mfma_f32_16x16x32_bf16 v[148:151], a[164:167], a[236:239], v[148:151]\n"
      "ds_read_b64_tr_b16 a[142:143], v1 offset:17920\n"
      "v_mfma_f32_16x16x32_bf16 v[148:151], a[160:163], a[232:235], v[148:151]\n"
      "ds_read_b64_tr_b16 a[144:145], v2 offset:1024\n"
      "s_waitcnt lgkmcnt(12)\n"
      "v_mfma_f32_16x16x32_bf16 v[152:155], a[160:163], a[240:243], v[152:155]\n"
      "ds_read_b64_tr_b16 a[146:147], v2 offset:1536\n"
      "ds_read_b64_tr_b16 a[158:159], v3 offset:17920\n"
      "v_mfma_f32_16x16x32_bf16 v[152:155], a[164:167], a[244:247], v[152:155]\n"
      "ds_read_b64_tr_b16 a[148:149], v2 offset:17408\n"
      "v_mfma_f32_16x16x32_bf16 v[184:187], a[172:175], a[244:247], v[184:187]\n"
      "ds_read_b64_tr_b16 a[150:151], v2 offset:17920\n"
      "s_xor_b32 s55, s51, s53\n"
      "v_mfma_f32_16x16x32_bf16 v[184:187], a[168:171], a[240:243], v[184:187]\n"
      "buffer_load_dwordx4 a[92:95], v30, s[44:47], s58 offen offset:3072 sc0 sc1\n"
      "v_mfma_f32_16x16x32_bf16 v[216:219], a[176:179], a[240:243], v[216:219]\n"
      "ds_read_b128 a[0:3], v4 offset:8192\n"
      "v_mfma_f32_16x16x32_bf16 v[216:219], a[180:183], a[244:247], v[216:219]\n"
      "ds_read_b128 a[4:7], v5 offset:8192\n"
      "v_mfma_f32_16x16x32_bf16 v[248:251], a[188:191], a[244:247], v[248:251]\n"
      "ds_read_b64_tr_b16 a[152:153], v3 offset:1024\n"
      "v_mfma_f32_16x16x32_bf16 v[248:251], a[184:187], a[240:243], v[248:251]\n"
      "ds_read_b128 a[8:11], v4 offset:10240\n"
      "v_mfma_f32_16x16x32_bf16 v[252:255], a[184:187], a[248:251], v[252:255]\n"
      "ds_read_b128 a[12:15], v5 offset:10240\n"
      "v_mfma_f32_16x16x32_bf16 v[252:255], a[188:191], a[252:255], v[252:255]\n"
      "ds_read_b64_tr_b16 a[154:155], v3 offset:1536\n"
      "v_mfma_f32_16x16x32_bf16 v[220:223], a[180:183], a[252:255], v[220:223]\n"
      "ds_read_b128 a[16:19], v4 offset:12288\n"
      "v_mfma_f32_16x16x32_bf16 v[220:223], a[176:179], a[248:251], v[220:223]\n"
      "ds_read_b128 a[20:23], v5 offset:12288\n"
      "v_mfma_f32_16x16x32_bf16 v[188:191], a[168:171], a[248:251], v[188:191]\n"
      "ds_read_b64_tr_b16 a[156:157], v3 offset:17408\n"
      "v_mfma_f32_16x16x32_bf16 v[188:191], a[172:175], a[252:255], v[188:191]\n"
      "ds_read_b128 a[24:27], v4 offset:14336\n"
      "v_mfma_f32_16x16x32_bf16 v[156:159], a[164:167], a[252:255], v[156:159]\n"
      "ds_read_b128 a[28:31], v5 offset:14336\n"
      "v_mfma_f32_16x16x32_bf16 v[156:159], a[160:163], a[248:251], v[156:159]\n"
      "v_xor_b32 v4, s55, v4\n"
      "v_xor_b32 v5, s55, v5\n"
      "s_waitcnt lgkmcnt(13)\n"
      "v_mfma_f32_16x16x32_bf16 v[128:131], a[128:131], a[192:195], v[128:131]\n"
      "v_mfma_f32_16x16x32_bf16 v[128:131], a[132:135], a[196:199], v[128:131]\n"
      "ds_read_b64_tr_b16 a[160:161], v0 offset:0\n"
      "v_mfma_f32_16x16x32_bf16 v[160:163], a[140:143], a[196:199], v[160:163]\n"
      "ds_read_b64_tr_b16 a[162:163], v0 offset:512\n"
      "v_mfma_f32_16x16x32_bf16 v[160:163], a[136:139], a[192:195], v[160:163]\n"
      "ds_read_b64_tr_b16 a[164:165], v0 offset:16384\n"
      "s_waitcnt lgkmcnt(13)\n"
      "v_mfma_f32_16x16x32_bf16 v[192:195], a[144:147], a[192:195], v[192:195]\n"
      "ds_read_b64_tr_b16 a[166:167], v0 offset:16896\n"
      "v_mfma_f32_16x16x32_bf16 v[192:195], a[148:151], a[196:199], v[192:195]\n"
      "ds_read_b64_tr_b16 a[168:169], v1 offset:0\n"
      "s_waitcnt lgkmcnt(7)\n"
      "v_mfma_f32_16x16x32_bf16 v[224:227], a[156:159], a[196:199], v[224:227]\n"
      "ds_read_b64_tr_b16 a[170:171], v1 offset:512\n"
      "v_mfma_f32_16x16x32_bf16 v[224:227], a[152:155], a[192:195], v[224:227]\n"
      "ds_read_b64_tr_b16 a[172:173], v1 offset:16384\n"
      "v_mfma_f32_16x16x32_bf16 v[228:231], a[152:155], a[200:203], v[228:231]\n"
      "ds_read_b64_tr_b16 a[174:175], v1 offset:16896\n"
      "v_mfma_f32_16x16x32_bf16 v[228:231], a[156:159], a[204:207], v[228:231]\n"
      "s_waitcnt vmcnt(24)\n"
      "v_mfma_f32_4x4x4_16b_bf16 v[128:131], v[24:25], a[32:33], v[128:131]\n"
      "v_mfma_f32_16x16x32_bf16 v[196:199], a[148:151], a[204:207], v[196:199]\n"
      "ds_read_b64_tr_b16 a[176:177], v2 offset:0\n"
      "v_mfma_f32_16x16x32_bf16 v[196:199], a[144:147], a[200:203], v[196:199]\n"
      "ds_read_b64_tr_b16 a[178:179], v2 offset:512\n"
      "v_mfma_f32_16x16x32_bf16 v[164:167], a[136:139], a[200:203], v[164:167]\n"
      "ds_read_b64_tr_b16 a[180:181], v2 offset:16384\n"
      "v_mfma_f32_16x16x32_bf16 v[164:167], a[140:143], a[204:207], v[164:167]\n"
      "ds_read_b64_tr_b16 a[182:183], v2 offset:16896\n"
      "v_mfma_f32_16x16x32_bf16 v[132:135], a[132:135], a[204:207], v[132:135]\n"
      "ds_read_b64_tr_b16 a[184:185], v3 offset:0\n"
      "v_mfma_f32_16x16x32_bf16 v[132:135], a[128:131], a[200:203], v[132:135]\n"
      "ds_read_b64_tr_b16 a[186:187], v3 offset:512\n"
      "v_mfma_f32_16x16x32_bf16 v[136:139], a[128:131], a[208:211], v[136:139]\n"
      "ds_read_b64_tr_b16 a[188:189], v3 offset:16384\n"
      "v_mfma_f32_16x16x32_bf16 v[136:139], a[132:135], a[212:215], v[136:139]\n"
      "s_waitcnt vmcnt(10)\n"
      "s_barrier\n"
      "v_mfma_f32_16x16x32_bf16 v[168:171], a[140:143], a[212:215], v[168:171]\n"
      "ds_read_b128 a[224:227], v4 offset:0\n"
      "v_mfma_f32_16x16x32_bf16 v[168:171], a[136:139], a[208:211], v[168:171]\n"
      "ds_read_b128 a[228:231], v5 offset:0\n"
      "v_mfma_f32_16x16x32_bf16 v[200:203], a[144:147], a[208:211], v[200:203]\n"
      "ds_read_b64_tr_b16 a[190:191], v3 offset:16896\n"
      "v_cmp_eq_u32 vcc, %31, v29\n"
      "v_mfma_f32_16x16x32_bf16 v[200:203], a[148:151], a[212:215], v[200:203]\n"
      "v_mfma_f32_4x4x4_16b_bf16 v[160:163], v[24:25], a[34:35], v[160:163]\n"
      "v_mfma_f32_16x16x32_bf16 v[232:235], a[156:159], a[212:215], v[232:235]\n"
      "ds_read_b128 a[232:235], v4 offset:2048\n"
      "s_cmp_lg_u64 vcc, 0\n"
      "s_cselect_b32 s59, 0x7fc07fc0, 0\n"
      "v_mfma_f32_16x16x32_bf16 v[232:235], a[152:155], a[208:211], v[232:235]\n"
      "ds_read_b128 a[236:239], v5 offset:2048\n"
      "s_mov_b32 vcc_lo, 0xaaaaaaaa\n"
      "v_mfma_f32_16x16x32_bf16 v[236:239], a[152:155], a[216:219], v[236:239]\n"
      "v_mfma_f32_4x4x4_16b_bf16 v[192:195], v[24:25], a[36:37], v[192:195]\n"
      "s_mov_b32 vcc_hi, vcc_lo\n"
      "v_mfma_f32_16x16x32_bf16 v[236:239], a[156:159], a[220:223], v[236:239]\n"
      "v_mfma_f32_4x4x4_16b_bf16 v[224:227], v[24:25], a[38:39], v[224:227]\n"
      "v_mfma_f32_16x16x32_bf16 v[204:207], a[148:151], a[220:223], v[204:207]\n"
      "ds_read_b128 a[240:243], v4 offset:4096\n"
      "v_mfma_f32_16x16x32_bf16 v[204:207], a[144:147], a[216:219], v[204:207]\n"
      "ds_read_b128 a[244:247], v5 offset:4096\n"
      "v_mfma_f32_16x16x32_bf16 v[172:175], a[136:139], a[216:219], v[172:175]\n"
      "v_mfma_f32_4x4x4_16b_bf16 v[132:135], v[24:25], a[40:41], v[132:135]\n"
      "v_mfma_f32_16x16x32_bf16 v[172:175], a[140:143], a[220:223], v[172:175]\n"
      "v_mfma_f32_4x4x4_16b_bf16 v[164:167], v[24:25], a[42:43], v[164:167]\n"
      "v_mfma_f32_16x16x32_bf16 v[140:143], a[132:135], a[220:223], v[140:143]\n"
      "ds_read_b128 a[248:251], v4 offset:6144\n"
      "v_mfma_f32_16x16x32_bf16 v[140:143], a[128:131], a[216:219], v[140:143]\n"
      "ds_read_b128 a[252:255], v5 offset:6144\n"
      "s_waitcnt lgkmcnt(7)\n"
      "v_mfma_f32_16x16x32_bf16 v[128:131], a[160:163], a[224:227], v[128:131]\n"
      "v_mfma_f32_4x4x4_16b_bf16 v[196:199], v[24:25], a[44:45], v[196:199]\n"
      "v_mfma_f32_16x16x32_bf16 v[128:131], a[164:167], a[228:231], v[128:131]\n"
      "v_mfma_f32_4x4x4_16b_bf16 v[228:231], v[24:25], a[46:47], v[228:231]\n"
      "v_mfma_f32_16x16x32_bf16 v[160:163], a[172:175], a[228:231], v[160:163]\n"
      "v_mfma_f32_4x4x4_16b_bf16 v[136:139], v[24:25], a[48:49], v[136:139]\n"
      "v_mfma_f32_16x16x32_bf16 v[160:163], a[168:171], a[224:227], v[160:163]\n"
      "v_mfma_f32_4x4x4_16b_bf16 v[168:171], v[24:25], a[50:51], v[168:171]\n"
      "v_mfma_f32_16x16x32_bf16 v[192:195], a[176:179], a[224:227], v[192:195]\n"
      "v_mfma_f32_4x4x4_16b_bf16 v[200:203], v[24:25], a[52:53], v[200:203]\n"
      "v_mfma_f32_16x16x32_bf16 v[192:195], a[180:183], a[228:231], v[192:195]\n"
      "v_mfma_f32_4x4x4_16b_bf16 v[232:235], v[24:25], a[54:55], v[232:235]\n"
      "s_waitcnt lgkmcnt(5)\n"
      "v_mfma_f32_16x16x32_bf16 v[224:227], a[188:191], a[228:231], v[224:227]\n"
      "s_waitcnt vmcnt(9)\n"
      "v_mfma_f32_4x4x4_16b_bf16 v[140:143], v[24:25], a[56:57], v[140:143]\n"
      "v_cvt_pk_bf16_f32 v32, v128, v129\n"
      "v_cvt_pk_bf16_f32 v33, v130, v131\n"
      "v_mfma_f32_16x16x32_bf16 v[224:227], a[184:187], a[224:227], v[224:227]\n"
      "v_mfma_f32_4x4x4_16b_bf16 v[172:175], v[24:25], a[58:59], v[172:175]\n"
      "v_pk_max_i16 v32, v32, s59\n"
      "v_pk_max_i16 v33, v33, s59\n"
      "s_waitcnt lgkmcnt(4)\n"
      "v_mfma_f32_16x16x32_bf16 v[228:231], a[184:187], a[232:235], v[228:231]\n"
      "s_waitcnt vmcnt(8)\n"
      "v_mfma_f32_4x4x4_16b_bf16 v[204:207], v[24:25], a[60:61], v[204:207]\n"
      "v_cvt_pk_bf16_f32 v34, v160, v161\n"
      "v_cvt_pk_bf16_f32 v35, v162, v163\n"
      "v_mfma_f32_16x16x32_bf16 v[228:231], a[188:191], a[236:239], v[228:231]\n"
      "v_mfma_f32_4x4x4_16b_bf16 v[236:239], v[24:25], a[62:63], v[236:239]\n"
      "v_pk_max_i16 v34, v34, s59\n"
      "v_pk_max_i16 v35, v35, s59\n"
      "v_mfma_f32_16x16x32_bf16 v[196:199], a[180:183], a[236:239], v[196:199]\n"
      "s_waitcnt vmcnt(7)\n"
      "v_mfma_f32_4x4x4_16b_bf16 v[144:147], v[24:25], a[64:65], v[144:147]\n"
      "v_cvt_pk_bf16_f32 v64, v192, v193\n"
      "v_cvt_pk_bf16_f32 v65, v194, v195\n"
      "v_mfma_f32_16x16x32_bf16 v[196:199], a[176:179], a[232:235], v[196:199]\n"
      "v_mfma_f32_4x4x4_16b_bf16 v[176:179], v[24:25], a[66:67], v[176:179]\n"
      "v_permlane16_swap_b32 v32, v34\n"
      "v_permlane16_swap_b32 v33, v35\n"
      "v_mfma_f32_16x16x32_bf16 v[164:167], a[168:171], a[232:235], v[164:167]\n"
      "ds_read_b128 a[192:195], v4 offset:8192\n"
      "v_pk_max_i16 v64, v64, s59\n"
      "v_pk_max_i16 v65, v65, s59\n"
      "v_mfma_f32_16x16x32_bf16 v[164:167], a[172:175], a[236:239], v[164:167]\n"
      "ds_read_b128 a[196:199], v5 offset:8192\n"
      "v_cvt_pk_bf16_f32 v66, v224, v225\n"
      "v_cvt_pk_bf16_f32 v67, v226, v227\n"
      "v_mfma_f32_16x16x32_bf16 v[132:135], a[164:167], a[236:239], v[132:135]\n"
      "s_waitcnt vmcnt(6)\n"
      "v_mfma_f32_4x4x4_16b_bf16 v[208:211], v[24:25], a[68:69], v[208:211]\n"
      "v_pk_max_i16 v66, v66, s59\n"
      "v_pk_max_i16 v67, v67, s59\n"
      "v_mfma_f32_16x16x32_bf16 v[132:135], a[160:163], a[232:235], v[132:135]\n"
      "v_mfma_f32_4x4x4_16b_bf16 v[240:243], v[24:25], a[70:71], v[240:243]\n"
      "s_mov_b32 s90, 0x4000\n"
      "s_mov_b32 s91, 0x0\n"
      "s_waitcnt lgkmcnt(4)\n"
      "v_mfma_f32_16x16x32_bf16 v[136:139], a[160:163], a[240:243], v[136:139]\n"
      "s_waitcnt vmcnt(5)\n"
      "v_mfma_f32_4x4x4_16b_bf16 v[148:151], v[24:25], a[72:73], v[148:151]\n"
      "v_permlane16_swap_b32 v64, v66\n"
      "v_permlane16_swap_b32 v65, v67\n"
      "v_mfma_f32_16x16x32_bf16 v[136:139], a[164:167], a[244:247], v[136:139]\n"
      "v_mfma_f32_4x4x4_16b_bf16 v[180:183], v[24:25], a[74:75], v[180:183]\n"
      "v_cndmask_b32_dpp v96, v32, v64, vcc quad_perm:[1,0,3,2] row_mask:0xf bank_mask:0xf\n"
      "v_cndmask_b32_dpp v98, v34, v66, vcc quad_perm:[1,0,3,2] row_mask:0xf bank_mask:0xf\n"
      "v_mfma_f32_16x16x32_bf16 v[168:171], a[172:175], a[244:247], v[168:171]\n"
      "v_cndmask_b32_dpp v97, v33, v65, vcc quad_perm:[1,0,3,2] row_mask:0xf bank_mask:0xf\n"
      "v_cndmask_b32_dpp v99, v35, v67, vcc quad_perm:[1,0,3,2] row_mask:0xf bank_mask:0xf\n"
      "s_not_b64 vcc, vcc\n"
      "v_mfma_f32_16x16x32_bf16 v[168:171], a[168:171], a[240:243], v[168:171]\n"
      "v_cndmask_b32_dpp v32, v64, v32, vcc quad_perm:[1,0,3,2] row_mask:0xf bank_mask:0xf\n"
      "v_cndmask_b32_dpp v34, v66, v34, vcc quad_perm:[1,0,3,2] row_mask:0xf bank_mask:0xf\n"
      "buffer_store_dwordx4 v[96:99], v26, s[80:83], s90 offen sc0 sc1 nt\n"
      "v_mfma_f32_16x16x32_bf16 v[200:203], a[176:179], a[240:243], v[200:203]\n"
      "v_cndmask_b32_dpp v33, v65, v33, vcc quad_perm:[1,0,3,2] row_mask:0xf bank_mask:0xf\n"
      "v_cndmask_b32_dpp v35, v67, v35, vcc quad_perm:[1,0,3,2] row_mask:0xf bank_mask:0xf\n"
      "buffer_store_dwordx4 v[32:35], v26, s[80:83], s91 offen sc0 sc1 nt\n"
      "v_mfma_f32_16x16x32_bf16 v[200:203], a[180:183], a[244:247], v[200:203]\n"
      "s_waitcnt vmcnt(6)\n"
      "v_mfma_f32_4x4x4_16b_bf16 v[212:215], v[24:25], a[76:77], v[212:215]\n"
      "v_cvt_pk_bf16_f32 v70, v228, v229\n"
      "v_cvt_pk_bf16_f32 v71, v230, v231\n"
      "v_mfma_f32_16x16x32_bf16 v[232:235], a[188:191], a[244:247], v[232:235]\n"
      "v_mfma_f32_4x4x4_16b_bf16 v[244:247], v[24:25], a[78:79], v[244:247]\n"
      "v_pk_max_i16 v70, v70, s59\n"
      "v_pk_max_i16 v71, v71, s59\n"
      "v_mfma_f32_16x16x32_bf16 v[232:235], a[184:187], a[240:243], v[232:235]\n"
      "v_cvt_pk_bf16_f32 v68, v196, v197\n"
      "v_cvt_pk_bf16_f32 v69, v198, v199\n"
      "v_pk_max_i16 v68, v68, s59\n"
      "s_waitcnt lgkmcnt(2)\n"
      "v_mfma_f32_16x16x32_bf16 v[236:239], a[184:187], a[248:251], v[236:239]\n"
      "v_pk_max_i16 v69, v69, s59\n"
      "v_cvt_pk_bf16_f32 v38, v164, v165\n"
      "v_cvt_pk_bf16_f32 v39, v166, v167\n"
      "v_mfma_f32_16x16x32_bf16 v[236:239], a[188:191], a[252:255], v[236:239]\n"
      "v_permlane16_swap_b32 v68, v70\n"
      "v_permlane16_swap_b32 v69, v71\n"
      "v_pk_max_i16 v38, v38, s59\n"
      "v_mfma_f32_16x16x32_bf16 v[204:207], a[180:183], a[252:255], v[204:207]\n"
      "v_pk_max_i16 v39, v39, s59\n"
      "v_cvt_pk_bf16_f32 v36, v132, v133\n"
      "v_cvt_pk_bf16_f32 v37, v134, v135\n"
      "v_mfma_f32_16x16x32_bf16 v[204:207], a[176:179], a[248:251], v[204:207]\n"
      "s_waitcnt vmcnt(5)\n"
      "v_mfma_f32_4x4x4_16b_bf16 v[152:155], v[24:25], a[80:81], v[152:155]\n"
      "v_pk_max_i16 v36, v36, s59\n"
      "v_pk_max_i16 v37, v37, s59\n"
      "v_mfma_f32_16x16x32_bf16 v[172:175], a[168:171], a[248:251], v[172:175]\n"
      "ds_read_b128 a[200:203], v4 offset:10240\n"
      "s_mov_b32 s90, 0x40000\n"
      "s_mov_b32 s91, 0x44000\n"
      "v_mfma_f32_16x16x32_bf16 v[172:175], a[172:175], a[252:255], v[172:175]\n"
      "ds_read_b128 a[204:207], v5 offset:10240\n"
      "v_permlane16_swap_b32 v36, v38\n"
      "v_permlane16_swap_b32 v37, v39\n"
      "v_mfma_f32_16x16x32_bf16 v[140:143], a[164:167], a[252:255], v[140:143]\n"
      "v_mfma_f32_4x4x4_16b_bf16 v[184:187], v[24:25], a[82:83], v[184:187]\n"
      "v_cndmask_b32_dpp v100, v68, v36, vcc quad_perm:[1,0,3,2] row_mask:0xf bank_mask:0xf\n"
      "v_cndmask_b32_dpp v102, v70, v38, vcc quad_perm:[1,0,3,2] row_mask:0xf bank_mask:0xf\n"
      "v_mfma_f32_16x16x32_bf16 v[140:143], a[160:163], a[248:251], v[140:143]\n"
      "v_cndmask_b32_dpp v101, v69, v37, vcc quad_perm:[1,0,3,2] row_mask:0xf bank_mask:0xf\n"
      "v_cndmask_b32_dpp v103, v71, v39, vcc quad_perm:[1,0,3,2] row_mask:0xf bank_mask:0xf\n"
      "s_not_b64 vcc, vcc\n"
      "v_mfma_f32_16x16x32_bf16 v[144:147], a[128:131], a[0:3], v[144:147]\n"
      "v_cndmask_b32_dpp v68, v36, v68, vcc quad_perm:[1,0,3,2] row_mask:0xf bank_mask:0xf\n"
      "v_cndmask_b32_dpp v70, v38, v70, vcc quad_perm:[1,0,3,2] row_mask:0xf bank_mask:0xf\n"
      "buffer_store_dwordx4 v[100:103], v26, s[80:83], s90 offen sc0 sc1 nt\n"
      "v_mfma_f32_16x16x32_bf16 v[144:147], a[132:135], a[4:7], v[144:147]\n"
      "s_waitcnt vmcnt(5)\n"
      "v_mfma_f32_4x4x4_16b_bf16 v[216:219], v[24:25], a[84:85], v[216:219]\n"
      "v_cndmask_b32_dpp v69, v37, v69, vcc quad_perm:[1,0,3,2] row_mask:0xf bank_mask:0xf\n"
      "v_cndmask_b32_dpp v71, v39, v71, vcc quad_perm:[1,0,3,2] row_mask:0xf bank_mask:0xf\n"
      "v_mfma_f32_16x16x32_bf16 v[176:179], a[140:143], a[4:7], v[176:179]\n"
      "v_mfma_f32_4x4x4_16b_bf16 v[248:251], v[24:25], a[86:87], v[248:251]\n"
      "buffer_store_dwordx4 v[68:71], v26, s[80:83], s91 offen sc0 sc1 nt\n"
      "v_cvt_pk_bf16_f32 v40, v136, v137\n"
      "v_mfma_f32_16x16x32_bf16 v[176:179], a[136:139], a[0:3], v[176:179]\n"
      "v_cvt_pk_bf16_f32 v41, v138, v139\n"
      "v_pk_max_i16 v40, v40, s59\n"
      "v_pk_max_i16 v41, v41, s59\n"
      "v_mfma_f32_16x16x32_bf16 v[208:211], a[144:147], a[0:3], v[208:211]\n"
      "v_cvt_pk_bf16_f32 v42, v168, v169\n"
      "v_cvt_pk_bf16_f32 v43, v170, v171\n"
      "v_pk_max_i16 v42, v42, s59\n"
      "v_mfma_f32_16x16x32_bf16 v[208:211], a[148:151], a[4:7], v[208:211]\n"
      "v_pk_max_i16 v43, v43, s59\n"
      "v_cvt_pk_bf16_f32 v72, v200, v201\n"
      "v_cvt_pk_bf16_f32 v73, v202, v203\n"
      "v_mfma_f32_16x16x32_bf16 v[240:243], a[156:159], a[4:7], v[240:243]\n"
      "v_permlane16_swap_b32 v40, v42\n"
      "v_permlane16_swap_b32 v41, v43\n"
      "v_pk_max_i16 v72, v72, s59\n"
      "v_mfma_f32_16x16x32_bf16 v[240:243], a[152:155], a[0:3], v[240:243]\n"
      "s_waitcnt vmcnt(5)\n"
      "v_mfma_f32_4x4x4_16b_bf16 v[156:159], v[24:25], a[88:89], v[156:159]\n"
      "v_pk_max_i16 v73, v73, s59\n"
      "v_cvt_pk_bf16_f32 v74, v232, v233\n"
      "s_waitcnt lgkmcnt(2)\n"
      "v_mfma_f32_16x16x32_bf16 v[144:147], a[160:163], a[192:195], v[144:147]\n"
      "v_mfma_f32_4x4x4_16b_bf16 v[188:191], v[24:25], a[90:91], v[188:191]\n"
      "v_cvt_pk_bf16_f32 v75, v234, v235\n"
      "v_pk_max_i16 v74, v74, s59\n"
      "v_mfma_f32_16x16x32_bf16 v[144:147], a[164:167], a[196:199], v[144:147]\n"
      "v_pk_max_i16 v75, v75, s59\n"
      "s_mov_b32 s90, 0x84000\n"
      "s_mov_b32 s91, 0x80000\n"
      "v_mfma_f32_16x16x32_bf16 v[176:179], a[172:175], a[196:199], v[176:179]\n"
      "v_permlane16_swap_b32 v72, v74\n"
      "v_permlane16_swap_b32 v73, v75\n"
      "v_cndmask_b32_dpp v104, v40, v72, vcc quad_perm:[1,0,3,2] row_mask:0xf bank_mask:0xf\n"
      "v_mfma_f32_16x16x32_bf16 v[176:179], a[168:171], a[192:195], v[176:179]\n"
      "v_cndmask_b32_dpp v106, v42, v74, vcc quad_perm:[1,0,3,2] row_mask:0xf bank_mask:0xf\n"
      "v_cndmask_b32_dpp v105, v41, v73, vcc quad_perm:[1,0,3,2] row_mask:0xf bank_mask:0xf\n"
      "v_cndmask_b32_dpp v107, v43, v75, vcc quad_perm:[1,0,3,2] row_mask:0xf bank_mask:0xf\n"
      "v_mfma_f32_16x16x32_bf16 v[208:211], a[176:179], a[192:195], v[208:211]\n"
      "ds_read_b128 a[208:211], v4 offset:12288\n"
      "s_not_b64 vcc, vcc\n"
      "v_cndmask_b32_dpp v40, v72, v40, vcc quad_perm:[1,0,3,2] row_mask:0xf bank_mask:0xf\n"
      "v_mfma_f32_16x16x32_bf16 v[208:211], a[180:183], a[196:199], v[208:211]\n"
      "ds_read_b128 a[212:215], v5 offset:12288\n"
      "v_cndmask_b32_dpp v42, v74, v42, vcc quad_perm:[1,0,3,2] row_mask:0xf bank_mask:0xf\n"
      "buffer_store_dwordx4 v[104:107], v26, s[80:83], s90 offen sc0 sc1 nt\n"
      "v_mfma_f32_16x16x32_bf16 v[240:243], a[188:191], a[196:199], v[240:243]\n"
      "s_waitcnt vmcnt(5)\n"
      "v_mfma_f32_4x4x4_16b_bf16 v[220:223], v[24:25], a[92:93], v[220:223]\n"
      "v_cndmask_b32_dpp v41, v73, v41, vcc quad_perm:[1,0,3,2] row_mask:0xf bank_mask:0xf\n"
      "v_cndmask_b32_dpp v43, v75, v43, vcc quad_perm:[1,0,3,2] row_mask:0xf bank_mask:0xf\n"
      "v_mfma_f32_16x16x32_bf16 v[240:243], a[184:187], a[192:195], v[240:243]\n"
      "v_mfma_f32_4x4x4_16b_bf16 v[252:255], v[24:25], a[94:95], v[252:255]\n"
      "buffer_store_dwordx4 v[40:43], v26, s[80:83], s91 offen sc0 sc1 nt\n"
      "v_cvt_pk_bf16_f32 v78, v236, v237\n"
      "v_mfma_f32_16x16x32_bf16 v[244:247], a[152:155], a[8:11], v[244:247]\n"
      "v_cvt_pk_bf16_f32 v79, v238, v239\n"
      "v_pk_max_i16 v78, v78, s59\n"
      "v_pk_max_i16 v79, v79, s59\n"
      "v_mfma_f32_16x16x32_bf16 v[244:247], a[156:159], a[12:15], v[244:247]\n"
      "v_cvt_pk_bf16_f32 v76, v204, v205\n"
      "v_cvt_pk_bf16_f32 v77, v206, v207\n"
      "v_pk_max_i16 v76, v76, s59\n"
      "v_mfma_f32_16x16x32_bf16 v[212:215], a[148:151], a[12:15], v[212:215]\n"
      "v_pk_max_i16 v77, v77, s59\n"
      "v_cvt_pk_bf16_f32 v46, v172, v173\n"
      "v_cvt_pk_bf16_f32 v47, v174, v175\n"
      "v_mfma_f32_16x16x32_bf16 v[212:215], a[144:147], a[8:11], v[212:215]\n"
      "v_permlane16_swap_b32 v76, v78\n"
      "v_permlane16_swap_b32 v77, v79\n"
      "v_pk_max_i16 v46, v46, s59\n"
      "v_mfma_f32_16x16x32_bf16 v[180:183], a[136:139], a[8:11], v[180:183]\n"
      "v_pk_max_i16 v47, v47, s59\n"
      "v_cvt_pk_bf16_f32 v44, v140, v141\n"
      "v_cvt_pk_bf16_f32 v45, v142, v143\n"
      "v_mfma_f32_16x16x32_bf16 v[180:183], a[140:143], a[12:15], v[180:183]\n"
      "v_pk_max_i16 v44, v44, s59\n"
      "v_pk_max_i16 v45, v45, s59\n"
      "s_mov_b32 s90, 0xc0000\n"
      "v_mfma_f32_16x16x32_bf16 v[148:151], a[132:135], a[12:15], v[148:151]\n"
      "s_mov_b32 s91, 0xc4000\n"
      "v_permlane16_swap_b32 v44, v46\n"
      "v_permlane16_swap_b32 v45, v47\n"
      "v_mfma_f32_16x16x32_bf16 v[148:151], a[128:131], a[8:11], v[148:151]\n"
      "v_cndmask_b32_dpp v108, v76, v44, vcc quad_perm:[1,0,3,2] row_mask:0xf bank_mask:0xf\n"
      "v_cndmask_b32_dpp v110, v78, v46, vcc quad_perm:[1,0,3,2] row_mask:0xf bank_mask:0xf\n"
      "v_cndmask_b32_dpp v109, v77, v45, vcc quad_perm:[1,0,3,2] row_mask:0xf bank_mask:0xf\n"
      "s_waitcnt lgkmcnt(2)\n"
      "v_mfma_f32_16x16x32_bf16 v[244:247], a[184:187], a[200:203], v[244:247]\n"
      "v_cndmask_b32_dpp v111, v79, v47, vcc quad_perm:[1,0,3,2] row_mask:0xf bank_mask:0xf\n"
      "s_not_b64 vcc, vcc\n"
      "v_cndmask_b32_dpp v76, v44, v76, vcc quad_perm:[1,0,3,2] row_mask:0xf bank_mask:0xf\n"
      "v_mfma_f32_16x16x32_bf16 v[244:247], a[188:191], a[204:207], v[244:247]\n"
      "v_cndmask_b32_dpp v78, v46, v78, vcc quad_perm:[1,0,3,2] row_mask:0xf bank_mask:0xf\n"
      "buffer_store_dwordx4 v[108:111], v26, s[80:83], s90 offen sc0 sc1 nt\n"
      "v_cndmask_b32_dpp v77, v45, v77, vcc quad_perm:[1,0,3,2] row_mask:0xf bank_mask:0xf\n"
      "v_mfma_f32_16x16x32_bf16 v[212:215], a[180:183], a[204:207], v[212:215]\n"
      "v_cndmask_b32_dpp v79, v47, v79, vcc quad_perm:[1,0,3,2] row_mask:0xf bank_mask:0xf\n"
      "buffer_store_dwordx4 v[76:79], v26, s[80:83], s91 offen sc0 sc1 nt\n"
      "v_cvt_pk_bf16_f32 v48, v144, v145\n"
      "v_mfma_f32_16x16x32_bf16 v[212:215], a[176:179], a[200:203], v[212:215]\n"
      "v_cvt_pk_bf16_f32 v49, v146, v147\n"
      "v_pk_max_i16 v48, v48, s59\n"
      "v_pk_max_i16 v49, v49, s59\n"
      "v_mfma_f32_16x16x32_bf16 v[180:183], a[168:171], a[200:203], v[180:183]\n"
      "ds_read_b128 a[216:219], v4 offset:14336\n"
      "v_cvt_pk_bf16_f32 v50, v176, v177\n"
      "v_cvt_pk_bf16_f32 v51, v178, v179\n"
      "v_mfma_f32_16x16x32_bf16 v[180:183], a[172:175], a[204:207], v[180:183]\n"
      "ds_read_b128 a[220:223], v5 offset:14336\n"
      "v_pk_max_i16 v50, v50, s59\n"
      "v_pk_max_i16 v51, v51, s59\n"
      "v_mfma_f32_16x16x32_bf16 v[148:151], a[164:167], a[204:207], v[148:151]\n"
      "v_cvt_pk_bf16_f32 v80, v208, v209\n"
      "v_cvt_pk_bf16_f32 v81, v210, v211\n"
      "v_permlane16_swap_b32 v48, v50\n"
      "v_mfma_f32_16x16x32_bf16 v[148:151], a[160:163], a[200:203], v[148:151]\n"
      "v_permlane16_swap_b32 v49, v51\n"
      "v_pk_max_i16 v80, v80, s59\n"
      "v_pk_max_i16 v81, v81, s59\n"
      "v_mfma_f32_16x16x32_bf16 v[152:155], a[128:131], a[16:19], v[152:155]\n"
      "v_cvt_pk_bf16_f32 v82, v240, v241\n"
      "v_cvt_pk_bf16_f32 v83, v242, v243\n"
      "v_pk_max_i16 v82, v82, s59\n"
      "v_mfma_f32_16x16x32_bf16 v[152:155], a[132:135], a[20:23], v[152:155]\n"
      "v_pk_max_i16 v83, v83, s59\n"
      "s_mov_b32 s90, 0x104000\n"
      "s_mov_b32 s91, 0x100000\n"
      "v_mfma_f32_16x16x32_bf16 v[184:187], a[140:143], a[20:23], v[184:187]\n"
      "v_permlane16_swap_b32 v80, v82\n"
      "v_permlane16_swap_b32 v81, v83\n"
      "v_cndmask_b32_dpp v112, v48, v80, vcc quad_perm:[1,0,3,2] row_mask:0xf bank_mask:0xf\n"
      "v_mfma_f32_16x16x32_bf16 v[184:187], a[136:139], a[16:19], v[184:187]\n"
      "v_cndmask_b32_dpp v114, v50, v82, vcc quad_perm:[1,0,3,2] row_mask:0xf bank_mask:0xf\n"
      "v_cndmask_b32_dpp v113, v49, v81, vcc quad_perm:[1,0,3,2] row_mask:0xf bank_mask:0xf\n"
      "v_cndmask_b32_dpp v115, v51, v83, vcc quad_perm:[1,0,3,2] row_mask:0xf bank_mask:0xf\n"
      "v_mfma_f32_16x16x32_bf16 v[216:219], a[144:147], a[16:19], v[216:219]\n"
      "s_not_b64 vcc, vcc\n"
      "v_cndmask_b32_dpp v48, v80, v48, vcc quad_perm:[1,0,3,2] row_mask:0xf bank_mask:0xf\n"
      "v_cndmask_b32_dpp v50, v82, v50, vcc quad_perm:[1,0,3,2] row_mask:0xf bank_mask:0xf\n"
      "v_mfma_f32_16x16x32_bf16 v[216:219], a[148:151], a[20:23], v[216:219]\n"
      "buffer_store_dwordx4 v[112:115], v26, s[80:83], s90 offen sc0 sc1 nt\n"
      "v_cndmask_b32_dpp v49, v81, v49, vcc quad_perm:[1,0,3,2] row_mask:0xf bank_mask:0xf\n"
      "v_cndmask_b32_dpp v51, v83, v51, vcc quad_perm:[1,0,3,2] row_mask:0xf bank_mask:0xf\n"
      "v_mfma_f32_16x16x32_bf16 v[248:251], a[156:159], a[20:23], v[248:251]\n"
      "buffer_store_dwordx4 v[48:51], v26, s[80:83], s91 offen sc0 sc1 nt\n"
      "v_cvt_pk_bf16_f32 v86, v244, v245\n"
      "v_cvt_pk_bf16_f32 v87, v246, v247\n"
      "v_mfma_f32_16x16x32_bf16 v[248:251], a[152:155], a[16:19], v[248:251]\n"
      "v_pk_max_i16 v86, v86, s59\n"
      "v_pk_max_i16 v87, v87, s59\n"
      "v_cvt_pk_bf16_f32 v84, v212, v213\n"
      "s_waitcnt lgkmcnt(2)\n"
      "v_mfma_f32_16x16x32_bf16 v[152:155], a[160:163], a[208:211], v[152:155]\n"
      "v_cvt_pk_bf16_f32 v85, v214, v215\n"
      "v_pk_max_i16 v84, v84, s59\n"
      "v_pk_max_i16 v85, v85, s59\n"
      "v_mfma_f32_16x16x32_bf16 v[152:155], a[164:167], a[212:215], v[152:155]\n"
      "v_cvt_pk_bf16_f32 v54, v180, v181\n"
      "v_cvt_pk_bf16_f32 v55, v182, v183\n"
      "v_permlane16_swap_b32 v84, v86\n"
      "v_mfma_f32_16x16x32_bf16 v[184:187], a[172:175], a[212:215], v[184:187]\n"
      "v_permlane16_swap_b32 v85, v87\n"
      "v_pk_max_i16 v54, v54, s59\n"
      "v_pk_max_i16 v55, v55, s59\n"
      "v_mfma_f32_16x16x32_bf16 v[184:187], a[168:171], a[208:211], v[184:187]\n"
      "v_cvt_pk_bf16_f32 v52, v148, v149\n"
      "v_cvt_pk_bf16_f32 v53, v150, v151\n"
      "v_pk_max_i16 v52, v52, s59\n"
      "v_mfma_f32_16x16x32_bf16 v[216:219], a[176:179], a[208:211], v[216:219]\n"
      "v_pk_max_i16 v53, v53, s59\n"
      "s_mov_b32 s90, 0x140000\n"
      "s_mov_b32 s91, 0x144000\n"
      "v_mfma_f32_16x16x32_bf16 v[216:219], a[180:183], a[212:215], v[216:219]\n"
      "v_permlane16_swap_b32 v52, v54\n"
      "v_permlane16_swap_b32 v53, v55\n"
      "v_cndmask_b32_dpp v116, v84, v52, vcc quad_perm:[1,0,3,2] row_mask:0xf bank_mask:0xf\n"
      "v_mfma_f32_16x16x32_bf16 v[248:251], a[188:191], a[212:215], v[248:251]\n"
      "v_cndmask_b32_dpp v118, v86, v54, vcc quad_perm:[1,0,3,2] row_mask:0xf bank_mask:0xf\n"
      "v_cndmask_b32_dpp v117, v85, v53, vcc quad_perm:[1,0,3,2] row_mask:0xf bank_mask:0xf\n"
      "v_cndmask_b32_dpp v119, v87, v55, vcc quad_perm:[1,0,3,2] row_mask:0xf bank_mask:0xf\n"
      "v_mfma_f32_16x16x32_bf16 v[248:251], a[184:187], a[208:211], v[248:251]\n"
      "s_not_b64 vcc, vcc\n"
      "v_cndmask_b32_dpp v84, v52, v84, vcc quad_perm:[1,0,3,2] row_mask:0xf bank_mask:0xf\n"
      "v_cndmask_b32_dpp v86, v54, v86, vcc quad_perm:[1,0,3,2] row_mask:0xf bank_mask:0xf\n"
      "v_mfma_f32_16x16x32_bf16 v[252:255], a[152:155], a[24:27], v[252:255]\n"
      "buffer_store_dwordx4 v[116:119], v26, s[80:83], s90 offen sc0 sc1 nt\n"
      "v_cndmask_b32_dpp v85, v53, v85, vcc quad_perm:[1,0,3,2] row_mask:0xf bank_mask:0xf\n"
      "v_cndmask_b32_dpp v87, v55, v87, vcc quad_perm:[1,0,3,2] row_mask:0xf bank_mask:0xf\n"
      "v_mfma_f32_16x16x32_bf16 v[252:255], a[156:159], a[28:31], v[252:255]\n"
      "buffer_store_dwordx4 v[84:87], v26, s[80:83], s91 offen sc0 sc1 nt\n"
      "v_cvt_pk_bf16_f32 v56, v152, v153\n"
      "v_cvt_pk_bf16_f32 v57, v154, v155\n"
      "v_mfma_f32_16x16x32_bf16 v[220:223], a[148:151], a[28:31], v[220:223]\n"
      "v_pk_max_i16 v56, v56, s59\n"
      "v_pk_max_i16 v57, v57, s59\n"
      "v_cvt_pk_bf16_f32 v58, v184, v185\n"
      "v_mfma_f32_16x16x32_bf16 v[220:223], a[144:147], a[24:27], v[220:223]\n"
      "v_cvt_pk_bf16_f32 v59, v186, v187\n"
      "v_pk_max_i16 v58, v58, s59\n"
      "v_pk_max_i16 v59, v59, s59\n"
      "v_mfma_f32_16x16x32_bf16 v[188:191], a[136:139], a[24:27], v[188:191]\n"
      "v_cvt_pk_bf16_f32 v88, v216, v217\n"
      "v_cvt_pk_bf16_f32 v89, v218, v219\n"
      "v_permlane16_swap_b32 v56, v58\n"
      "v_mfma_f32_16x16x32_bf16 v[188:191], a[140:143], a[28:31], v[188:191]\n"
      "v_permlane16_swap_b32 v57, v59\n"
      "v_pk_max_i16 v88, v88, s59\n"
      "v_pk_max_i16 v89, v89, s59\n"
      "v_mfma_f32_16x16x32_bf16 v[156:159], a[132:135], a[28:31], v[156:159]\n"
      "v_cvt_pk_bf16_f32 v90, v248, v249\n"
      "v_cvt_pk_bf16_f32 v91, v250, v251\n"
      "v_pk_max_i16 v90, v90, s59\n"
      "v_mfma_f32_16x16x32_bf16 v[156:159], a[128:131], a[24:27], v[156:159]\n"
      "v_pk_max_i16 v91, v91, s59\n"
      "s_mov_b32 s90, 0x184000\n"
      "s_mov_b32 s91, 0x180000\n"
      "s_waitcnt lgkmcnt(0)\n"
      "v_mfma_f32_16x16x32_bf16 v[252:255], a[184:187], a[216:219], v[252:255]\n"
      "v_permlane16_swap_b32 v88, v90\n"
      "v_permlane16_swap_b32 v89, v91\n"
      "v_cndmask_b32_dpp v120, v56, v88, vcc quad_perm:[1,0,3,2] row_mask:0xf bank_mask:0xf\n"
      "v_mfma_f32_16x16x32_bf16 v[252:255], a[188:191], a[220:223], v[252:255]\n"
      "v_cndmask_b32_dpp v122, v58, v90, vcc quad_perm:[1,0,3,2] row_mask:0xf bank_mask:0xf\n"
      "v_cndmask_b32_dpp v121, v57, v89, vcc quad_perm:[1,0,3,2] row_mask:0xf bank_mask:0xf\n"
      "v_cndmask_b32_dpp v123, v59, v91, vcc quad_perm:[1,0,3,2] row_mask:0xf bank_mask:0xf\n"
      "v_mfma_f32_16x16x32_bf16 v[220:223], a[180:183], a[220:223], v[220:223]\n"
      "s_not_b64 vcc, vcc\n"
      "v_cndmask_b32_dpp v56, v88, v56, vcc quad_perm:[1,0,3,2] row_mask:0xf bank_mask:0xf\n"
      "v_cndmask_b32_dpp v58, v90, v58, vcc quad_perm:[1,0,3,2] row_mask:0xf bank_mask:0xf\n"
      "v_mfma_f32_16x16x32_bf16 v[220:223], a[176:179], a[216:219], v[220:223]\n"
      "buffer_store_dwordx4 v[120:123], v26, s[80:83], s90 offen sc0 sc1 nt\n"
      "v_cndmask_b32_dpp v57, v89, v57, vcc quad_perm:[1,0,3,2] row_mask:0xf bank_mask:0xf\n"
      "v_cndmask_b32_dpp v59, v91, v59, vcc quad_perm:[1,0,3,2] row_mask:0xf bank_mask:0xf\n"
      "v_mfma_f32_16x16x32_bf16 v[188:191], a[168:171], a[216:219], v[188:191]\n"
      "buffer_store_dwordx4 v[56:59], v26, s[80:83], s91 offen sc0 sc1 nt\n"
      "v_mfma_f32_16x16x32_bf16 v[188:191], a[172:175], a[220:223], v[188:191]\n"
      "v_mfma_f32_16x16x32_bf16 v[156:159], a[164:167], a[220:223], v[156:159]\n"
      "v_cvt_pk_bf16_f32 v94, v252, v253\n"
      "v_cvt_pk_bf16_f32 v95, v254, v255\n"
      "v_pk_max_i16 v94, v94, s59\n"
      "v_mfma_f32_16x16x32_bf16 v[156:159], a[160:163], a[216:219], v[156:159]\n"
      "v_pk_max_i16 v95, v95, s59\n"
      "v_cvt_pk_bf16_f32 v92, v220, v221\n"
      "v_cvt_pk_bf16_f32 v93, v222, v223\n"
      "v_pk_max_i16 v92, v92, s59\n"
      "v_pk_max_i16 v93, v93, s59\n"
      "v_cvt_pk_bf16_f32 v62, v188, v189\n"
      "v_cvt_pk_bf16_f32 v63, v190, v191\n"
      "v_permlane16_swap_b32 v92, v94\n"
      "v_permlane16_swap_b32 v93, v95\n"
      "v_pk_max_i16 v62, v62, s59\n"
      "v_pk_max_i16 v63, v63, s59\n"
      "v_cvt_pk_bf16_f32 v60, v156, v157\n"
      "v_cvt_pk_bf16_f32 v61, v158, v159\n"
      "v_pk_max_i16 v60, v60, s59\n"
      "v_pk_max_i16 v61, v61, s59\n"
      "s_mov_b32 s90, 0x1c0000\n"
      "s_mov_b32 s91, 0x1c4000\n"
      "v_permlane16_swap_b32 v60, v62\n"
      "v_permlane16_swap_b32 v61, v63\n"
      "v_cndmask_b32_dpp v124, v92, v60, vcc quad_perm:[1,0,3,2] row_mask:0xf bank_mask:0xf\n"
      "v_cndmask_b32_dpp v126, v94, v62, vcc quad_perm:[1,0,3,2] row_mask:0xf bank_mask:0xf\n"
      "v_cndmask_b32_dpp v125, v93, v61, vcc quad_perm:[1,0,3,2] row_mask:0xf bank_mask:0xf\n"
      "v_cndmask_b32_dpp v127, v95, v63, vcc quad_perm:[1,0,3,2] row_mask:0xf bank_mask:0xf\n"
      "s_not_b64 vcc, vcc\n"
      "v_cndmask_b32_dpp v92, v60, v92, vcc quad_perm:[1,0,3,2] row_mask:0xf bank_mask:0xf\n"
      "v_cndmask_b32_dpp v94, v62, v94, vcc quad_perm:[1,0,3,2] row_mask:0xf bank_mask:0xf\n"
      "buffer_store_dwordx4 v[124:127], v26, s[80:83], s90 offen sc0 sc1 nt\n"
      "v_cndmask_b32_dpp v93, v61, v93, vcc quad_perm:[1,0,3,2] row_mask:0xf bank_mask:0xf\n"
      "v_cndmask_b32_dpp v95, v63, v95, vcc quad_perm:[1,0,3,2] row_mask:0xf bank_mask:0xf\n"
      "buffer_store_dwordx4 v[92:95], v26, s[80:83], s91 offen sc0 sc1 nt\n"
      "s_branch 59f\n"
      "58:\n"
      "v_mfma_f32_16x16x32_bf16 v[128:131], a[160:163], a[192:195], v[128:131]\n"
      "v_cmp_gt_u32 vcc, s75, v29\n"
      "v_mfma_f32_16x16x32_bf16 v[128:131], a[164:167], a[196:199], v[128:131]\n"
      "v_mfma_f32_16x16x32_bf16 v[160:163], a[172:175], a[196:199], v[160:163]\n"
      "s_cbranch_vccnz 147f\n"
      "47:\n"
      "v_mfma_f32_16x16x32_bf16 v[160:163], a[168:171], a[192:195], v[160:163]\n"
      "buffer_load_dwordx4 v14, s[40:43], s64 offen offset:0 sc0 sc1 lds\n"
      "v_mfma_f32_16x16x32_bf16 v[192:195], a[176:179], a[192:195], v[192:195]\n"
      "v_accvgpr_read_b32 v22, a104\n"
      "v_mfma_f32_16x16x32_bf16 v[192:195], a[180:183], a[196:199], v[192:195]\n"
      "v_lshrrev_b32 v28, 12, v22\n"
      "v_mfma_f32_16x16x32_bf16 v[224:227], a[188:191], a[196:199], v[224:227]\n"
      "s_xor_b32 s69, s69, s70\n"
      "v_mfma_f32_16x16x32_bf16 v[224:227], a[184:187], a[192:195], v[224:227]\n"
      "s_xor_b32 s70, s70, s69\n"
      "v_mfma_f32_16x16x32_bf16 v[228:231], a[184:187], a[200:203], v[228:231]\n"
      "buffer_load_dwordx4 v15, s[40:43], s64 offen offset:1024 sc0 sc1 lds\n"
      "v_mfma_f32_16x16x32_bf16 v[228:231], a[188:191], a[204:207], v[228:231]\n"
      "s_and_b32 s52, s68, 0x38000\n"
      "v_mfma_f32_16x16x32_bf16 v[196:199], a[180:183], a[204:207], v[196:199]\n"
      "s_xor_b32 s53, s52, s70\n"
      "v_mfma_f32_16x16x32_bf16 v[196:199], a[176:179], a[200:203], v[196:199]\n"
      "s_xor_b32 s54, s53, s69\n"
      "v_mfma_f32_16x16x32_bf16 v[164:167], a[168:171], a[200:203], v[164:167]\n"
      "s_and_b32 s50, s67, 0x8000\n"
      "v_mfma_f32_16x16x32_bf16 v[164:167], a[172:175], a[204:207], v[164:167]\n"
      "buffer_load_dwordx4 v16, s[40:43], s64 offen offset:2048 sc0 sc1 lds\n"
      "v_mfma_f32_16x16x32_bf16 v[132:135], a[164:167], a[204:207], v[132:135]\n"
      "s_xor_b32 s51, s50, 0x8000\n"
      "v_mfma_f32_16x16x32_bf16 v[132:135], a[160:163], a[200:203], v[132:135]\n"
      "s_mov_b32 s66, 0\n"
      "v_mfma_f32_16x16x32_bf16 v[136:139], a[160:163], a[208:211], v[136:139]\n"
      "v_lshrrev_b32 v30, 10, v30\n"
      "v_mfma_f32_16x16x32_bf16 v[136:139], a[164:167], a[212:215], v[136:139]\n"
      "ds_read_b128 a[192:195], v4 offset:0\n"
      "v_mfma_f32_16x16x32_bf16 v[168:171], a[172:175], a[212:215], v[168:171]\n"
      "buffer_load_dwordx4 v17, s[40:43], s64 offen offset:3072 sc0 sc1 lds\n"
      "v_mfma_f32_16x16x32_bf16 v[168:171], a[168:171], a[208:211], v[168:171]\n"
      "s_bitset1_b32 m0, 12\n"
      "v_mfma_f32_16x16x32_bf16 v[200:203], a[176:179], a[208:211], v[200:203]\n"
      "ds_read_b128 a[196:199], v5 offset:0\n"
      "v_mfma_f32_16x16x32_bf16 v[200:203], a[180:183], a[212:215], v[200:203]\n"
      "ds_read_b128 a[200:203], v4 offset:2048\n"
      "v_mfma_f32_16x16x32_bf16 v[232:235], a[188:191], a[212:215], v[232:235]\n"
      "ds_read_b128 a[204:207], v5 offset:2048\n"
      "v_mfma_f32_16x16x32_bf16 v[232:235], a[184:187], a[208:211], v[232:235]\n"
      "buffer_load_dwordx4 v18, s[40:43], s64 offen offset:0 sc0 sc1 lds\n"
      "v_mfma_f32_16x16x32_bf16 v[236:239], a[184:187], a[216:219], v[236:239]\n"
      "ds_read_b128 a[208:211], v4 offset:4096\n"
      "v_mfma_f32_16x16x32_bf16 v[236:239], a[188:191], a[220:223], v[236:239]\n"
      "ds_read_b128 a[212:215], v5 offset:4096\n"
      "v_mfma_f32_16x16x32_bf16 v[204:207], a[180:183], a[220:223], v[204:207]\n"
      "ds_read_b64_tr_b16 a[128:129], v0 offset:0\n"
      "v_mfma_f32_16x16x32_bf16 v[204:207], a[176:179], a[216:219], v[204:207]\n"
      "ds_read_b64_tr_b16 a[130:131], v0 offset:512\n"
      "v_mfma_f32_16x16x32_bf16 v[172:175], a[168:171], a[216:219], v[172:175]\n"
      "buffer_load_dwordx4 v19, s[40:43], s64 offen offset:1024 sc0 sc1 lds\n"
      "v_mfma_f32_16x16x32_bf16 v[172:175], a[172:175], a[220:223], v[172:175]\n"
      "ds_read_b64_tr_b16 a[132:133], v0 offset:16384\n"
      "v_mfma_f32_16x16x32_bf16 v[140:143], a[164:167], a[220:223], v[140:143]\n"
      "ds_read_b64_tr_b16 a[134:135], v0 offset:16896\n"
      "v_mfma_f32_16x16x32_bf16 v[140:143], a[160:163], a[216:219], v[140:143]\n"
      "ds_read_b64_tr_b16 a[136:137], v1 offset:0\n"
      "v_mfma_f32_16x16x32_bf16 v[144:147], a[160:163], a[224:227], v[144:147]\n"
      "ds_read_b128 a[216:219], v4 offset:6144\n"
      "v_mfma_f32_16x16x32_bf16 v[144:147], a[164:167], a[228:231], v[144:147]\n"
      "buffer_load_dwordx4 v20, s[40:43], s64 offen offset:2048 sc0 sc1 lds\n"
      "v_mfma_f32_16x16x32_bf16 v[176:179], a[172:175], a[228:231], v[176:179]\n"
      "ds_read_b128 a[220:223], v5 offset:6144\n"
      "v_mfma_f32_16x16x32_bf16 v[176:179], a[168:171], a[224:227], v[176:179]\n"
      "ds_read_b64_tr_b16 a[138:139], v1 offset:512\n"
      "v_mfma_f32_16x16x32_bf16 v[208:211], a[176:179], a[224:227], v[208:211]\n"
      "buffer_load_dwordx4 v[24:27], v28, s[28:31], 0 offen sc0 sc1\n"
      "v_mfma_f32_16x16x32_bf16 v[208:211], a[180:183], a[228:231], v[208:211]\n"
      "ds_read_b64_tr_b16 a[140:141], v1 offset:16384\n"
      "v_mfma_f32_16x16x32_bf16 v[240:243], a[188:191], a[228:231], v[240:243]\n"
      "buffer_load_dwordx4 v21, s[40:43], s64 offen offset:3072 sc0 sc1 lds\n"
      "v_mfma_f32_16x16x32_bf16 v[240:243], a[184:187], a[224:227], v[240:243]\n"
      "ds_read_b64_tr_b16 a[142:143], v1 offset:16896\n"
      "v_mfma_f32_16x16x32_bf16 v[244:247], a[184:187], a[232:235], v[244:247]\n"
      "ds_read_b128 a[224:227], v4 offset:8192\n"
      "v_mfma_f32_16x16x32_bf16 v[244:247], a[188:191], a[236:239], v[244:247]\n"
      "ds_read_b128 a[228:231], v5 offset:8192\n"
      "v_mfma_f32_16x16x32_bf16 v[212:215], a[180:183], a[236:239], v[212:215]\n"
      "ds_read_b64_tr_b16 a[144:145], v2 offset:0\n"
      "v_mfma_f32_16x16x32_bf16 v[212:215], a[176:179], a[232:235], v[212:215]\n"
      "ds_read_b64_tr_b16 a[146:147], v2 offset:512\n"
      "v_mfma_f32_16x16x32_bf16 v[180:183], a[168:171], a[232:235], v[180:183]\n"
      "ds_read_b64_tr_b16 a[148:149], v2 offset:16384\n"
      "v_mfma_f32_16x16x32_bf16 v[180:183], a[172:175], a[236:239], v[180:183]\n"
      "s_or_b32 m0, s52, %42\n"
      "v_mfma_f32_16x16x32_bf16 v[148:151], a[164:167], a[236:239], v[148:151]\n"
      "buffer_load_dwordx4 v7, %36, s65 offen offset:1024 lds\n"
      "v_mfma_f32_16x16x32_bf16 v[148:151], a[160:163], a[232:235], v[148:151]\n"
      "ds_read_b64_tr_b16 a[150:151], v2 offset:16896\n"
      "v_mfma_f32_16x16x32_bf16 v[152:155], a[160:163], a[240:243], v[152:155]\n"
      "ds_read_b128 a[232:235], v4 offset:10240\n"
      "v_mfma_f32_16x16x32_bf16 v[152:155], a[164:167], a[244:247], v[152:155]\n"
      "ds_read_b128 a[236:239], v5 offset:10240\n"
      "v_mfma_f32_16x16x32_bf16 v[184:187], a[172:175], a[244:247], v[184:187]\n"
      "ds_read_b64_tr_b16 a[152:153], v3 offset:0\n"
      "v_mfma_f32_16x16x32_bf16 v[184:187], a[168:171], a[240:243], v[184:187]\n"
      "s_xor_b32 s55, s50, s51\n"
      "v_mfma_f32_16x16x32_bf16 v[216:219], a[176:179], a[240:243], v[216:219]\n"
      "ds_read_b64_tr_b16 a[154:155], v3 offset:512\n"
      "v_mfma_f32_16x16x32_bf16 v[216:219], a[180:183], a[244:247], v[216:219]\n"
      "s_xor_b32 s56, s54, s53\n"
      "v_mfma_f32_16x16x32_bf16 v[248:251], a[188:191], a[244:247], v[248:251]\n"
      "buffer_load_dwordx4 v9, %36, s65 offen offset:3072 lds\n"
      "v_mfma_f32_16x16x32_bf16 v[248:251], a[184:187], a[240:243], v[248:251]\n"
      "s_bitset1_b32 m0, 13\n"
      "v_mfma_f32_16x16x32_bf16 v[252:255], a[184:187], a[248:251], v[252:255]\n"
      "ds_read_b128 a[240:243], v4 offset:12288\n"
      "v_mfma_f32_16x16x32_bf16 v[252:255], a[188:191], a[252:255], v[252:255]\n"
      "ds_read_b128 a[244:247], v5 offset:12288\n"
      "v_mfma_f32_16x16x32_bf16 v[220:223], a[180:183], a[252:255], v[220:223]\n"
      "ds_read_b64_tr_b16 a[156:157], v3 offset:16384\n"
      "v_mfma_f32_16x16x32_bf16 v[220:223], a[176:179], a[248:251], v[220:223]\n"
      "ds_read_b64_tr_b16 a[158:159], v3 offset:16896\n"
      "v_mfma_f32_16x16x32_bf16 v[188:191], a[168:171], a[248:251], v[188:191]\n"
      "v_mfma_f32_16x16x32_bf16 v[188:191], a[172:175], a[252:255], v[188:191]\n"
      "v_mfma_f32_16x16x32_bf16 v[156:159], a[164:167], a[252:255], v[156:159]\n"
      "buffer_load_dwordx4 v11, %36, s65 offen offset:1024 lds\n"
      "v_mfma_f32_16x16x32_bf16 v[156:159], a[160:163], a[248:251], v[156:159]\n"
      "s_waitcnt lgkmcnt(7)\n"
      "v_mfma_f32_16x16x32_bf16 a[0:3], a[128:131], a[192:195], a[0:3]\n"
      "ds_read_b128 a[248:251], v4 offset:14336\n"
      "v_mfma_f32_16x16x32_bf16 a[0:3], a[132:135], a[196:199], a[0:3]\n"
      "ds_read_b128 a[252:255], v5 offset:14336\n"
      "v_mfma_f32_16x16x32_bf16 a[4:7], a[132:135], a[204:207], a[4:7]\n"
      "ds_read_b64_tr_b16 a[160:161], v0 offset:1024\n"
      "v_xor_b32 v4, s55, v4\n"
      "v_mfma_f32_16x16x32_bf16 a[4:7], a[128:131], a[200:203], a[4:7]\n"
      "v_xor_b32 v5, s55, v5\n"
      "v_mfma_f32_16x16x32_bf16 a[8:11], a[128:131], a[208:211], a[8:11]\n"
      "ds_read_b64_tr_b16 a[162:163], v0 offset:1536\n"
      "v_mfma_f32_16x16x32_bf16 a[8:11], a[132:135], a[212:215], a[8:11]\n"
      "v_mfma_f32_16x16x32_bf16 a[12:15], a[132:135], a[220:223], a[12:15]\n"
      "buffer_load_dwordx4 v13, %36, s65 offen offset:3072 lds\n"
      "v_mfma_f32_16x16x32_bf16 a[12:15], a[128:131], a[216:219], a[12:15]\n"
      "ds_read_b64_tr_b16 a[164:165], v0 offset:17408\n"
      "v_mfma_f32_16x16x32_bf16 a[16:19], a[128:131], a[224:227], a[16:19]\n"
      "s_add_u32 s80, s26, %39\n"
      "s_addc_u32 s81, s27, 0\n"
      "v_mfma_f32_16x16x32_bf16 a[16:19], a[132:135], a[228:231], a[16:19]\n"
      "s_mov_b32 s82, 0x200000\n"
      "s_waitcnt lgkmcnt(6)\n"
      "v_mfma_f32_16x16x32_bf16 a[20:23], a[132:135], a[236:239], a[20:23]\n"
      "ds_read_b64_tr_b16 a[166:167], v0 offset:17920\n"
      "s_add_u32 s90, s71, 1\n"
      "v_mfma_f32_16x16x32_bf16 a[20:23], a[128:131], a[232:235], a[20:23]\n"
      "v_xor_b32 v0, s56, v0\n"
      "v_readlane_b32 s57, v22, s90\n"
      "v_mfma_f32_16x16x32_bf16 a[24:27], a[128:131], a[240:243], a[24:27]\n"
      "s_add_u32 s65, s57, 1024\n"
      "v_mfma_f32_16x16x32_bf16 a[24:27], a[132:135], a[244:247], a[24:27]\n"
      "s_or_b32 m0, s52, %42\n"
      "s_waitcnt lgkmcnt(4)\n"
      "v_mfma_f32_16x16x32_bf16 a[28:31], a[132:135], a[252:255], a[28:31]\n"
      "buffer_load_dwordx4 v7, %36, s65 offen offset:0 lds\n"
      "v_mfma_f32_16x16x32_bf16 a[28:31], a[128:131], a[248:251], a[28:31]\n"
      "ds_read_b64_tr_b16 a[168:169], v1 offset:1024\n"
      "v_mfma_f32_16x16x32_bf16 v[60:63], a[136:139], a[248:251], v[60:63]\n"
      "s_mov_b32 s83, 0x20000\n"
      "v_mfma_f32_16x16x32_bf16 v[60:63], a[140:143], a[252:255], v[60:63]\n"
      "ds_read_b64_tr_b16 a[170:171], v1 offset:1536\n"
      "v_mfma_f32_16x16x32_bf16 v[56:59], a[140:143], a[244:247], v[56:59]\n"
      "v_mfma_f32_16x16x32_bf16 v[56:59], a[136:139], a[240:243], v[56:59]\n"
      "v_mfma_f32_16x16x32_bf16 v[52:55], a[136:139], a[232:235], v[52:55]\n"
      "ds_read_b64_tr_b16 a[172:173], v1 offset:17408\n"
      "v_mfma_f32_16x16x32_bf16 v[52:55], a[140:143], a[236:239], v[52:55]\n"
      "v_mfma_f32_16x16x32_bf16 v[48:51], a[140:143], a[228:231], v[48:51]\n"
      "buffer_load_dwordx4 v9, %36, s65 offen offset:2048 lds\n"
      "v_mfma_f32_16x16x32_bf16 v[48:51], a[136:139], a[224:227], v[48:51]\n"
      "s_bitset1_b32 m0, 13\n"
      "v_mfma_f32_16x16x32_bf16 v[44:47], a[136:139], a[216:219], v[44:47]\n"
      "ds_read_b64_tr_b16 a[174:175], v1 offset:17920\n"
      "v_mfma_f32_16x16x32_bf16 v[44:47], a[140:143], a[220:223], v[44:47]\n"
      "v_xor_b32 v1, s56, v1\n"
      "v_mfma_f32_16x16x32_bf16 v[40:43], a[140:143], a[212:215], v[40:43]\n"
      "v_mfma_f32_16x16x32_bf16 v[40:43], a[136:139], a[208:211], v[40:43]\n"
      "ds_read_b64_tr_b16 a[176:177], v2 offset:1024\n"
      "v_mfma_f32_16x16x32_bf16 v[36:39], a[136:139], a[200:203], v[36:39]\n"
      "v_mfma_f32_16x16x32_bf16 v[36:39], a[140:143], a[204:207], v[36:39]\n"
      "v_mfma_f32_16x16x32_bf16 v[32:35], a[140:143], a[196:199], v[32:35]\n"
      "buffer_load_dwordx4 v11, %36, s65 offen offset:0 lds\n"
      "v_mfma_f32_16x16x32_bf16 v[32:35], a[136:139], a[192:195], v[32:35]\n"
      "ds_read_b64_tr_b16 a[178:179], v2 offset:1536\n"
      "v_mfma_f32_16x16x32_bf16 v[64:67], a[144:147], a[192:195], v[64:67]\n"
      "v_mfma_f32_16x16x32_bf16 v[64:67], a[148:151], a[196:199], v[64:67]\n"
      "v_mfma_f32_16x16x32_bf16 v[68:71], a[148:151], a[204:207], v[68:71]\n"
      "ds_read_b64_tr_b16 a[180:181], v2 offset:17408\n"
      "v_mfma_f32_16x16x32_bf16 v[68:71], a[144:147], a[200:203], v[68:71]\n"
      "v_mfma_f32_16x16x32_bf16 v[72:75], a[144:147], a[208:211], v[72:75]\n"
      "v_mfma_f32_16x16x32_bf16 v[72:75], a[148:151], a[212:215], v[72:75]\n"
      "ds_read_b64_tr_b16 a[182:183], v2 offset:17920\n"
      "v_mfma_f32_16x16x32_bf16 v[76:79], a[148:151], a[220:223], v[76:79]\n"
      "buffer_load_dwordx4 v13, %36, s65 offen offset:2048 lds\n"
      "v_mfma_f32_16x16x32_bf16 v[76:79], a[144:147], a[216:219], v[76:79]\n"
      "v_xor_b32 v2, s56, v2\n"
      "v_mfma_f32_16x16x32_bf16 v[80:83], a[144:147], a[224:227], v[80:83]\n"
      "ds_read_b64_tr_b16 a[184:185], v3 offset:1024\n"
      "v_mfma_f32_16x16x32_bf16 v[80:83], a[148:151], a[228:231], v[80:83]\n"
      "v_mfma_f32_16x16x32_bf16 v[84:87], a[148:151], a[236:239], v[84:87]\n"
      "v_mfma_f32_16x16x32_bf16 v[84:87], a[144:147], a[232:235], v[84:87]\n"
      "ds_read_b64_tr_b16 a[186:187], v3 offset:1536\n"
      "v_mfma_f32_16x16x32_bf16 v[88:91], a[144:147], a[240:243], v[88:91]\n"
      "v_mfma_f32_16x16x32_bf16 v[88:91], a[148:151], a[244:247], v[88:91]\n"
      "v_mfma_f32_16x16x32_bf16 v[92:95], a[148:151], a[252:255], v[92:95]\n"
      "v_mfma_f32_16x16x32_bf16 v[92:95], a[144:147], a[248:251], v[92:95]\n"
      "ds_read_b64_tr_b16 a[188:189], v3 offset:17408\n"
      "v_mfma_f32_16x16x32_bf16 v[124:127], a[152:155], a[248:251], v[124:127]\n"
      "v_mfma_f32_16x16x32_bf16 v[124:127], a[156:159], a[252:255], v[124:127]\n"
      "v_mfma_f32_16x16x32_bf16 v[120:123], a[156:159], a[244:247], v[120:123]\n"
      "ds_read_b64_tr_b16 a[190:191], v3 offset:17920\n"
      "v_mfma_f32_16x16x32_bf16 v[120:123], a[152:155], a[240:243], v[120:123]\n"
      "v_xor_b32 v3, s56, v3\n"
      "v_mfma_f32_16x16x32_bf16 v[116:119], a[152:155], a[232:235], v[116:119]\n"
      "v_mfma_f32_16x16x32_bf16 v[116:119], a[156:159], a[236:239], v[116:119]\n"
      "v_mfma_f32_16x16x32_bf16 v[112:115], a[156:159], a[228:231], v[112:115]\n"
      "v_mfma_f32_16x16x32_bf16 v[112:115], a[152:155], a[224:227], v[112:115]\n"
      "v_mfma_f32_16x16x32_bf16 v[108:111], a[152:155], a[216:219], v[108:111]\n"
      "v_mfma_f32_16x16x32_bf16 v[108:111], a[156:159], a[220:223], v[108:111]\n"
      "s_add_u32 s90, s71, 0\n"
      "v_mfma_f32_16x16x32_bf16 v[104:107], a[156:159], a[212:215], v[104:107]\n"
      "v_readlane_b32 s86, v22, s90\n"
      "v_mfma_f32_16x16x32_bf16 v[104:107], a[152:155], a[208:211], v[104:107]\n"
      "s_lshr_b32 s86, s86, 13\n"
      "v_mfma_f32_16x16x32_bf16 v[100:103], a[152:155], a[200:203], v[100:103]\n"
      "s_waitcnt vmcnt(8) lgkmcnt(0)\n"
      "s_barrier\n"
      "v_mfma_f32_16x16x32_bf16 v[100:103], a[156:159], a[204:207], v[100:103]\n"
      "s_mul_i32 s86, s86, 0x401\n"
      "v_mfma_f32_16x16x32_bf16 v[96:99], a[156:159], a[196:199], v[96:99]\n"
      "s_and_b32 s86, s86, 0xe00780\n"
      "v_mfma_f32_16x16x32_bf16 v[96:99], a[152:155], a[192:195], v[96:99]\n"
      "s_or_b32 s64, s86, %43\n"
      "v_mfma_f32_16x16x32_bf16 v[128:131], a[160:163], a[192:195], v[128:131]\n"
      "v_min_u32 v29, v24, v25\n"
      "v_min3_u32 v29, v29, v26, v27\n"
      "v_cmp_gt_u32 vcc, s75, v29\n"
      "v_mfma_f32_16x16x32_bf16 v[128:131], a[164:167], a[196:199], v[128:131]\n"
      "s_or_b32 m0, s51, %41\n"
      "v_mfma_f32_16x16x32_bf16 v[160:163], a[172:175], a[196:199], v[160:163]\n"
      "s_cbranch_vccnz 148f\n"
      "48:\n"
      "v_mfma_f32_16x16x32_bf16 v[160:163], a[168:171], a[192:195], v[160:163]\n"
      "buffer_load_dwordx4 v14, s[40:43], s64 offen offset:0 sc0 sc1 lds\n"
      "v_mfma_f32_16x16x32_bf16 v[192:195], a[176:179], a[192:195], v[192:195]\n"
      "ds_read_b64_tr_b16 a[128:129], v0 offset:0\n"
      "v_mfma_f32_16x16x32_bf16 v[192:195], a[180:183], a[196:199], v[192:195]\n"
      "s_add_u32 s44, s48, %47\n"
      "s_addc_u32 s45, s49, 0\n"
      "v_mfma_f32_16x16x32_bf16 v[224:227], a[188:191], a[196:199], v[224:227]\n"
      "s_mov_b32 s58, %48\n"
      "v_mfma_f32_16x16x32_bf16 v[224:227], a[184:187], a[192:195], v[224:227]\n"
      "buffer_load_dwordx4 v15, s[40:43], s64 offen offset:1024 sc0 sc1 lds\n"
      "v_mfma_f32_16x16x32_bf16 v[228:231], a[184:187], a[200:203], v[228:231]\n"
      "ds_read_b128 a[192:195], v4 offset:0\n"
      "v_mfma_f32_16x16x32_bf16 v[228:231], a[188:191], a[204:207], v[228:231]\n"
      "ds_read_b128 a[196:199], v5 offset:0\n"
      "v_mfma_f32_16x16x32_bf16 v[196:199], a[180:183], a[204:207], v[196:199]\n"
      "ds_read_b64_tr_b16 a[130:131], v0 offset:512\n"
      "v_mfma_f32_16x16x32_bf16 v[196:199], a[176:179], a[200:203], v[196:199]\n"
      "buffer_load_dwordx4 v16, s[40:43], s64 offen offset:2048 sc0 sc1 lds\n"
      "v_mfma_f32_16x16x32_bf16 v[164:167], a[168:171], a[200:203], v[164:167]\n"
      "ds_read_b64_tr_b16 a[132:133], v0 offset:16384\n"
      "v_mfma_f32_16x16x32_bf16 v[164:167], a[172:175], a[204:207], v[164:167]\n"
      "s_min_u32 s46, s42, 0x20000\n"
      "v_mfma_f32_16x16x32_bf16 v[132:135], a[164:167], a[204:207], v[132:135]\n"
      "s_mov_b32 s47, 0x20000\n"
      "v_mfma_f32_16x16x32_bf16 v[132:135], a[160:163], a[200:203], v[132:135]\n"
      "buffer_load_dwordx4 v17, s[40:43], s64 offen offset:3072 sc0 sc1 lds\n"
      "v_mfma_f32_16x16x32_bf16 v[136:139], a[160:163], a[208:211], v[136:139]\n"
      "s_bitset1_b32 m0, 12\n"
      "v_mfma_f32_16x16x32_bf16 v[136:139], a[164:167], a[212:215], v[136:139]\n"
      "ds_read_b128 a[200:203], v4 offset:2048\n"
      "v_mfma_f32_16x16x32_bf16 v[168:171], a[172:175], a[212:215], v[168:171]\n"
      "ds_read_b128 a[204:207], v5 offset:2048\n"
      "v_mfma_f32_16x16x32_bf16 v[168:171], a[168:171], a[208:211], v[168:171]\n"
      "buffer_load_dwordx4 v18, s[40:43], s64 offen offset:0 sc0 sc1 lds\n"
      "v_mfma_f32_16x16x32_bf16 v[200:203], a[176:179], a[208:211], v[200:203]\n"
      "ds_read_b64_tr_b16 a[134:135], v0 offset:16896\n"
      "v_mfma_f32_16x16x32_bf16 v[200:203], a[180:183], a[212:215], v[200:203]\n"
      "ds_read_b64_tr_b16 a[136:137], v1 offset:0\n"
      "v_mfma_f32_16x16x32_bf16 v[232:235], a[188:191], a[212:215], v[232:235]\n"
      "ds_read_b64_tr_b16 a[138:139], v1 offset:512\n"
      "v_mfma_f32_16x16x32_bf16 v[232:235], a[184:187], a[208:211], v[232:235]\n"
      "buffer_load_dwordx4 v19, s[40:43], s64 offen offset:1024 sc0 sc1 lds\n"
      "v_mfma_f32_16x16x32_bf16 v[236:239], a[184:187], a[216:219], v[236:239]\n"
      "ds_read_b128 a[208:211], v4 offset:4096\n"
      "v_mfma_f32_16x16x32_bf16 v[236:239], a[188:191], a[220:223], v[236:239]\n"
      "ds_read_b128 a[212:215], v5 offset:4096\n"
      "v_mfma_f32_16x16x32_bf16 v[204:207], a[180:183], a[220:223], v[204:207]\n"
      "ds_read_b64_tr_b16 a[140:141], v1 offset:16384\n"
      "s_add_u32 s90, s71, 1\n"
      "v_mfma_f32_16x16x32_bf16 v[204:207], a[176:179], a[216:219], v[204:207]\n"
      "buffer_load_dwordx4 v20, s[40:43], s64 offen offset:2048 sc0 sc1 lds\n"
      "v_mfma_f32_16x16x32_bf16 v[172:175], a[168:171], a[216:219], v[172:175]\n"
      "v_readlane_b32 s86, v22, s90\n"
      "v_mfma_f32_16x16x32_bf16 v[172:175], a[172:175], a[220:223], v[172:175]\n"
      "ds_read_b64_tr_b16 a[142:143], v1 offset:16896\n"
      "s_lshr_b32 s86, s86, 13\n"
      "v_mfma_f32_16x16x32_bf16 v[140:143], a[164:167], a[220:223], v[140:143]\n"
      "s_mul_i32 s86, s86, 0x401\n"
      "v_mfma_f32_16x16x32_bf16 v[140:143], a[160:163], a[216:219], v[140:143]\n"
      "buffer_load_dwordx4 v21, s[40:43], s64 offen offset:3072 sc0 sc1 lds\n"
      "v_mfma_f32_16x16x32_bf16 v[144:147], a[160:163], a[224:227], v[144:147]\n"
      "ds_read_b128 a[216:219], v4 offset:6144\n"
      "s_and_b32 s86, s86, 0xe00780\n"
      "v_mfma_f32_16x16x32_bf16 v[144:147], a[164:167], a[228:231], v[144:147]\n"
      "s_or_b32 s64, s86, %43\n"
      "v_mfma_f32_16x16x32_bf16 v[176:179], a[172:175], a[228:231], v[176:179]\n"
      "s_or_b32 m0, s53, %41\n"
      "v_mfma_f32_16x16x32_bf16 v[176:179], a[168:171], a[224:227], v[176:179]\n"
      "buffer_load_dwordx4 v14, s[40:43], s64 offen offset:0 sc0 sc1 lds\n"
      "v_mfma_f32_16x16x32_bf16 v[208:211], a[176:179], a[224:227], v[208:211]\n"
      "ds_read_b128 a[220:223], v5 offset:6144\n"
      "v_mfma_f32_16x16x32_bf16 v[208:211], a[180:183], a[228:231], v[208:211]\n"
      "ds_read_b64_tr_b16 a[144:145], v2 offset:0\n"
      "v_mfma_f32_16x16x32_bf16 v[240:243], a[188:191], a[228:231], v[240:243]\n"
      "ds_read_b64_tr_b16 a[146:147], v2 offset:512\n"
      "v_mfma_f32_16x16x32_bf16 v[240:243], a[184:187], a[224:227], v[240:243]\n"
      "buffer_load_dwordx4 v15, s[40:43], s64 offen offset:1024 sc0 sc1 lds\n"
      "v_mfma_f32_16x16x32_bf16 v[244:247], a[184:187], a[232:235], v[244:247]\n"
      "ds_read_b128 a[224:227], v4 offset:8192\n"
      "v_mfma_f32_16x16x32_bf16 v[244:247], a[188:191], a[236:239], v[244:247]\n"
      "ds_read_b128 a[228:231], v5 offset:8192\n"
      "v_mfma_f32_16x16x32_bf16 v[212:215], a[180:183], a[236:239], v[212:215]\n"
      "ds_read_b64_tr_b16 a[148:149], v2 offset:16384\n"
      "v_mfma_f32_16x16x32_bf16 v[212:215], a[176:179], a[232:235], v[212:215]\n"
      "buffer_load_dwordx4 v16, s[40:43], s64 offen offset:2048 sc0 sc1 lds\n"
      "v_mfma_f32_16x16x32_bf16 v[180:183], a[168:171], a[232:235], v[180:183]\n"
      "ds_read_b64_tr_b16 a[150:151], v2 offset:16896\n"
      "v_mfma_f32_16x16x32_bf16 v[180:183], a[172:175], a[236:239], v[180:183]\n"
      "s_xor_b32 s55, s50, s51\n"
      "v_mfma_f32_16x16x32_bf16 v[148:151], a[164:167], a[236:239], v[148:151]\n"
      "s_xor_b32 s56, s54, s52\n"
      "v_mfma_f32_16x16x32_bf16 v[148:151], a[160:163], a[232:235], v[148:151]\n"
      "buffer_load_dwordx4 v17, s[40:43], s64 offen offset:3072 sc0 sc1 lds\n"
      "v_mfma_f32_16x16x32_bf16 v[152:155], a[160:163], a[240:243], v[152:155]\n"
      "s_bitset1_b32 m0, 12\n"
      "v_mfma_f32_16x16x32_bf16 v[152:155], a[164:167], a[244:247], v[152:155]\n"
      "ds_read_b128 a[232:235], v4 offset:10240\n"
      "v_mfma_f32_16x16x32_bf16 v[184:187], a[172:175], a[244:247], v[184:187]\n"
      "ds_read_b128 a[236:239], v5 offset:10240\n"
      "v_mfma_f32_16x16x32_bf16 v[184:187], a[168:171], a[240:243], v[184:187]\n"
      "buffer_load_dwordx4 v18, s[40:43], s64 offen offset:0 sc0 sc1 lds\n"
      "v_mfma_f32_16x16x32_bf16 v[216:219], a[176:179], a[240:243], v[216:219]\n"
      "ds_read_b64_tr_b16 a[152:153], v3 offset:0\n"
      "v_mfma_f32_16x16x32_bf16 v[216:219], a[180:183], a[244:247], v[216:219]\n"
      "ds_read_b64_tr_b16 a[154:155], v3 offset:512\n"
      "v_mfma_f32_16x16x32_bf16 v[248:251], a[188:191], a[244:247], v[248:251]\n"
      "v_mfma_f32_16x16x32_bf16 v[248:251], a[184:187], a[240:243], v[248:251]\n"
      "buffer_load_dwordx4 v19, s[40:43], s64 offen offset:1024 sc0 sc1 lds\n"
      "v_mfma_f32_16x16x32_bf16 v[252:255], a[184:187], a[248:251], v[252:255]\n"
      "ds_read_b128 a[240:243], v4 offset:12288\n"
      "v_mfma_f32_16x16x32_bf16 v[252:255], a[188:191], a[252:255], v[252:255]\n"
      "ds_read_b128 a[244:247], v5 offset:12288\n"
      "v_mfma_f32_16x16x32_bf16 v[220:223], a[180:183], a[252:255], v[220:223]\n"
      "ds_read_b64_tr_b16 a[156:157], v3 offset:16384\n"
      "v_mfma_f32_16x16x32_bf16 v[220:223], a[176:179], a[248:251], v[220:223]\n"
      "buffer_load_dwordx4 v20, s[40:43], s64 offen offset:2048 sc0 sc1 lds\n"
      "v_mfma_f32_16x16x32_bf16 v[188:191], a[168:171], a[248:251], v[188:191]\n"
      "ds_read_b64_tr_b16 a[158:159], v3 offset:16896\n"
      "v_mfma_f32_16x16x32_bf16 v[188:191], a[172:175], a[252:255], v[188:191]\n"
      "v_mfma_f32_16x16x32_bf16 v[156:159], a[164:167], a[252:255], v[156:159]\n"
      "v_mfma_f32_16x16x32_bf16 v[156:159], a[160:163], a[248:251], v[156:159]\n"
      "buffer_load_dwordx4 v21, s[40:43], s64 offen offset:3072 sc0 sc1 lds\n"
      "s_waitcnt lgkmcnt(8)\n"
      "v_mfma_f32_16x16x32_bf16 a[0:3], a[128:131], a[192:195], a[0:3]\n"
      "ds_read_b128 a[248:251], v4 offset:14336\n"
      "v_mfma_f32_16x16x32_bf16 a[0:3], a[132:135], a[196:199], a[0:3]\n"
      "ds_read_b128 a[252:255], v5 offset:14336\n"
      "v_mfma_f32_16x16x32_bf16 v[32:35], a[140:143], a[196:199], v[32:35]\n"
      "ds_read_b64_tr_b16 a[160:161], v0 offset:1024\n"
      "v_xor_b32 v4, s55, v4\n"
      "v_mfma_f32_16x16x32_bf16 v[32:35], a[136:139], a[192:195], v[32:35]\n"
      "v_xor_b32 v5, s55, v5\n"
      "v_mfma_f32_16x16x32_bf16 v[64:67], a[144:147], a[192:195], v[64:67]\n"
      "ds_read_b64_tr_b16 a[162:163], v0 offset:1536\n"
      "v_mfma_f32_16x16x32_bf16 v[64:67], a[148:151], a[196:199], v[64:67]\n"
      "s_waitcnt lgkmcnt(4)\n"
      "v_mfma_f32_16x16x32_bf16 v[96:99], a[156:159], a[196:199], v[96:99]\n"
      "v_accvgpr_read_b32 v24, a0\n"
      "v_accvgpr_read_b32 v25, a1\n"
      "v_mfma_f32_16x16x32_bf16 v[96:99], a[152:155], a[192:195], v[96:99]\n"
      "ds_read_b64_tr_b16 a[164:165], v0 offset:17408\n"
      "v_accvgpr_read_b32 v26, a2\n"
      "v_mfma_f32_16x16x32_bf16 a[4:7], a[128:131], a[200:203], a[4:7]\n"
      "v_accvgpr_read_b32 v27, a3\n"
      "v_cvt_pk_bf16_f32 v24, v24, v25\n"
      "v_mfma_f32_16x16x32_bf16 a[4:7], a[132:135], a[204:207], a[4:7]\n"
      "v_cvt_pk_bf16_f32 v25, v26, v27\n"
      "v_cvt_pk_bf16_f32 v26, v32, v33\n"
      "v_mfma_f32_16x16x32_bf16 v[36:39], a[140:143], a[204:207], v[36:39]\n"
      "ds_read_b64_tr_b16 a[166:167], v0 offset:17920\n"
      "v_cvt_pk_bf16_f32 v27, v34, v35\n"
      "v_mfma_f32_16x16x32_bf16 v[36:39], a[136:139], a[200:203], v[36:39]\n"
      "v_xor_b32 v0, s56, v0\n"
      "v_cvt_pk_bf16_f32 v64, v64, v65\n"
      "v_mfma_f32_16x16x32_bf16 v[68:71], a[144:147], a[200:203], v[68:71]\n"
      "v_cvt_pk_bf16_f32 v65, v66, v67\n"
      "v_cvt_pk_bf16_f32 v66, v96, v97\n"
      "v_mfma_f32_16x16x32_bf16 v[68:71], a[148:151], a[204:207], v[68:71]\n"
      "v_cvt_pk_bf16_f32 v67, v98, v99\n"
      "v_mfma_f32_16x16x32_bf16 v[100:103], a[156:159], a[204:207], v[100:103]\n"
      "buffer_store_dwordx4 v[24:27], v30, s[44:47], s58 offen offset:0 sc0 sc1\n"
      "v_mfma_f32_16x16x32_bf16 v[100:103], a[152:155], a[200:203], v[100:103]\n"
      "ds_read_b64_tr_b16 a[168:169], v1 offset:1024\n"
      "v_accvgpr_read_b32 v32, a4\n"
      "v_mfma_f32_16x16x32_bf16 a[8:11], a[128:131], a[208:211], a[8:11]\n"
      "v_accvgpr_read_b32 v33, a5\n"
      "v_accvgpr_read_b32 v34, a6\n"
      "v_mfma_f32_16x16x32_bf16 a[8:11], a[132:135], a[212:215], a[8:11]\n"
      "ds_read_b64_tr_b16 a[170:171], v1 offset:1536\n"
      "v_accvgpr_read_b32 v35, a7\n"
      "v_mfma_f32_16x16x32_bf16 v[40:43], a[140:143], a[212:215], v[40:43]\n"
      "v_cvt_pk_bf16_f32 v32, v32, v33\n"
      "v_cvt_pk_bf16_f32 v33, v34, v35\n"
      "v_mfma_f32_16x16x32_bf16 v[40:43], a[136:139], a[208:211], v[40:43]\n"
      "buffer_store_dwordx4 v[64:67], v30, s[44:47], s58 offen offset:1024 sc0 sc1\n"
      "v_mfma_f32_16x16x32_bf16 v[72:75], a[144:147], a[208:211], v[72:75]\n"
      "ds_read_b64_tr_b16 a[172:173], v1 offset:17408\n"
      "v_cvt_pk_bf16_f32 v34, v36, v37\n"
      "v_mfma_f32_16x16x32_bf16 v[72:75], a[148:151], a[212:215], v[72:75]\n"
      "v_cvt_pk_bf16_f32 v35, v38, v39\n"
      "v_cvt_pk_bf16_f32 v68, v68, v69\n"
      "v_mfma_f32_16x16x32_bf16 v[104:107], a[156:159], a[212:215], v[104:107]\n"
      "v_cvt_pk_bf16_f32 v69, v70, v71\n"
      "v_cvt_pk_bf16_f32 v70, v100, v101\n"
      "v_mfma_f32_16x16x32_bf16 v[104:107], a[152:155], a[208:211], v[104:107]\n"
      "v_cvt_pk_bf16_f32 v71, v102, v103\n"
      "v_accvgpr_read_b32 v36, a8\n"
      "v_mfma_f32_16x16x32_bf16 a[12:15], a[128:131], a[216:219], a[12:15]\n"
      "buffer_store_dwordx4 v[32:35], v30, s[44:47], s58 offen offset:2048 sc0 sc1\n"
      "v_mfma_f32_16x16x32_bf16 a[12:15], a[132:135], a[220:223], a[12:15]\n"
      "ds_read_b64_tr_b16 a[174:175], v1 offset:17920\n"
      "v_accvgpr_read_b32 v37, a9\n"
      "v_mfma_f32_16x16x32_bf16 v[44:47], a[140:143], a[220:223], v[44:47]\n"
      "v_xor_b32 v1, s56, v1\n"
      "v_accvgpr_read_b32 v38, a10\n"
      "v_mfma_f32_16x16x32_bf16 v[44:47], a[136:139], a[216:219], v[44:47]\n"
      "ds_read_b64_tr_b16 a[176:177], v2 offset:1024\n"
      "v_accvgpr_read_b32 v39, a11\n"
      "v_mfma_f32_16x16x32_bf16 v[76:79], a[144:147], a[216:219], v[76:79]\n"
      "v_cvt_pk_bf16_f32 v36, v36, v37\n"
      "v_cvt_pk_bf16_f32 v37, v38, v39\n"
      "v_mfma_f32_16x16x32_bf16 v[76:79], a[148:151], a[220:223], v[76:79]\n"
      "buffer_store_dwordx4 v[68:71], v30, s[44:47], s58 offen offset:3072 sc0 sc1\n"
      "v_mfma_f32_16x16x32_bf16 v[108:111], a[156:159], a[220:223], v[108:111]\n"
      "s_add_u32 s58, s58, 0x1000\n"
      "v_cvt_pk_bf16_f32 v38, v40, v41\n"
      "v_mfma_f32_16x16x32_bf16 v[108:111], a[152:155], a[216:219], v[108:111]\n"
      "ds_read_b64_tr_b16 a[178:179], v2 offset:1536\n"
      "v_cvt_pk_bf16_f32 v39, v42, v43\n"
      "v_mfma_f32_16x16x32_bf16 a[16:19], a[128:131], a[224:227], a[16:19]\n"
      "v_cvt_pk_bf16_f32 v72, v72, v73\n"
      "v_cvt_pk_bf16_f32 v73, v74, v75\n"
      "v_mfma_f32_16x16x32_bf16 a[16:19], a[132:135], a[228:231], a[16:19]\n"
      "v_cvt_pk_bf16_f32 v74, v104, v105\n"
      "v_cvt_pk_bf16_f32 v75, v106, v107\n"
      "v_mfma_f32_16x16x32_bf16 v[48:51], a[140:143], a[228:231], v[48:51]\n"
      "buffer_store_dwordx4 v[36:39], v30, s[44:47], s58 offen offset:0 sc0 sc1\n"
      "v_mfma_f32_16x16x32_bf16 v[48:51], a[136:139], a[224:227], v[48:51]\n"
      "ds_read_b64_tr_b16 a[180:181], v2 offset:17408\n"
      "v_accvgpr_read_b32 v40, a12\n"
      "v_mfma_f32_16x16x32_bf16 v[80:83], a[144:147], a[224:227], v[80:83]\n"
      "v_accvgpr_read_b32 v41, a13\n"
      "v_accvgpr_read_b32 v42, a14\n"
      "v_mfma_f32_16x16x32_bf16 v[80:83], a[148:151], a[228:231], v[80:83]\n"
      "ds_read_b64_tr_b16 a[182:183], v2 offset:17920\n"
      "v_accvgpr_read_b32 v43, a15\n"
      "v_mfma_f32_16x16x32_bf16 v[112:115], a[156:159], a[228:231], v[112:115]\n"
      "v_xor_b32 v2, s56, v2\n"
      "v_cvt_pk_bf16_f32 v40, v40, v41\n"
      "v_mfma_f32_16x16x32_bf16 v[112:115], a[152:155], a[224:227], v[112:115]\n"
      "buffer_store_dwordx4 v[72:75], v30, s[44:47], s58 offen offset:1024 sc0 sc1\n"
      "v_mfma_f32_16x16x32_bf16 a[20:23], a[128:131], a[232:235], a[20:23]\n"
      "ds_read_b64_tr_b16 a[184:185], v3 offset:1024\n"
      "v_cvt_pk_bf16_f32 v41, v42, v43\n"
      "v_mfma_f32_16x16x32_bf16 a[20:23], a[132:135], a[236:239], a[20:23]\n"
      "v_cvt_pk_bf16_f32 v42, v44, v45\n"
      "v_cvt_pk_bf16_f32 v43, v46, v47\n"
      "v_mfma_f32_16x16x32_bf16 v[52:55], a[140:143], a[236:239], v[52:55]\n"
      "v_cvt_pk_bf16_f32 v76, v76, v77\n"
      "v_cvt_pk_bf16_f32 v77, v78, v79\n"
      "v_mfma_f32_16x16x32_bf16 v[52:55], a[136:139], a[232:235], v[52:55]\n"
      "ds_read_b64_tr_b16 a[186:187], v3 offset:1536\n"
      "v_cvt_pk_bf16_f32 v78, v108, v109\n"
      "v_mfma_f32_16x16x32_bf16 v[84:87], a[144:147], a[232:235], v[84:87]\n"
      "buffer_store_dwordx4 v[40:43], v30, s[44:47], s58 offen offset:2048 sc0 sc1\n"
      "v_mfma_f32_16x16x32_bf16 v[84:87], a[148:151], a[236:239], v[84:87]\n"
      "v_cvt_pk_bf16_f32 v79, v110, v111\n"
      "v_accvgpr_read_b32 v44, a16\n"
      "v_mfma_f32_16x16x32_bf16 v[116:119], a[156:159], a[236:239], v[116:119]\n"
      "v_accvgpr_read_b32 v45, a17\n"
      "v_accvgpr_read_b32 v46, a18\n"
      "v_mfma_f32_16x16x32_bf16 v[116:119], a[152:155], a[232:235], v[116:119]\n"
      "ds_read_b64_tr_b16 a[188:189], v3 offset:17408\n"
      "v_accvgpr_read_b32 v47, a19\n"
      "v_mfma_f32_16x16x32_bf16 a[24:27], a[128:131], a[240:243], a[24:27]\n"
      "v_cvt_pk_bf16_f32 v44, v44, v45\n"
      "v_cvt_pk_bf16_f32 v45, v46, v47\n"
      "v_mfma_f32_16x16x32_bf16 a[24:27], a[132:135], a[244:247], a[24:27]\n"
      "buffer_store_dwordx4 v[76:79], v30, s[44:47], s58 offen offset:3072 sc0 sc1\n"
      "v_mfma_f32_16x16x32_bf16 v[56:59], a[140:143], a[244:247], v[56:59]\n"
      "s_add_u32 s58, s58, 0x1000\n"
      "v_cvt_pk_bf16_f32 v46, v48, v49\n"
      "v_mfma_f32_16x16x32_bf16 v[56:59], a[136:139], a[240:243], v[56:59]\n"
      "ds_read_b64_tr_b16 a[190:191], v3 offset:17920\n"
      "v_cvt_pk_bf16_f32 v47, v50, v51\n"
      "v_mfma_f32_16x16x32_bf16 v[88:91], a[144:147], a[240:243], v[88:91]\n"
      "v_xor_b32 v3, s56, v3\n"
      "v_cvt_pk_bf16_f32 v80, v80, v81\n"
      "v_mfma_f32_16x16x32_bf16 v[88:91], a[148:151], a[244:247], v[88:91]\n"
      "v_cvt_pk_bf16_f32 v81, v82, v83\n"
      "v_cvt_pk_bf16_f32 v82, v112, v113\n"
      "v_mfma_f32_16x16x32_bf16 v[120:123], a[156:159], a[244:247], v[120:123]\n"
      "buffer_store_dwordx4 v[44:47], v30, s[44:47], s58 offen offset:0 sc0 sc1\n"
      "v_mfma_f32_16x16x32_bf16 v[120:123], a[152:155], a[240:243], v[120:123]\n"
      "v_cvt_pk_bf16_f32 v83, v114, v115\n"
      "v_accvgpr_read_b32 v48, a20\n"
      "s_waitcnt lgkmcnt(4)\n"
      "v_mfma_f32_16x16x32_bf16 a[28:31], a[128:131], a[248:251], a[28:31]\n"
      "v_accvgpr_read_b32 v49, a21\n"
      "v_accvgpr_read_b32 v50, a22\n"
      "v_mfma_f32_16x16x32_bf16 a[28:31], a[132:135], a[252:255], a[28:31]\n"
      "v_accvgpr_read_b32 v51, a23\n"
      "v_cvt_pk_bf16_f32 v48, v48, v49\n"
      "v_mfma_f32_16x16x32_bf16 v[60:63], a[140:143], a[252:255], v[60:63]\n"
      "v_cvt_pk_bf16_f32 v49, v50, v51\n"
      "v_cvt_pk_bf16_f32 v50, v52, v53\n"
      "v_mfma_f32_16x16x32_bf16 v[60:63], a[136:139], a[248:251], v[60:63]\n"
      "v_cvt_pk_bf16_f32 v51, v54, v55\n"
      "v_cvt_pk_bf16_f32 v84, v84, v85\n"
      "v_mfma_f32_16x16x32_bf16 v[92:95], a[144:147], a[248:251], v[92:95]\n"
      "s_waitcnt vmcnt(17) lgkmcnt(0)\n"
      "s_barrier\n"
      "v_mfma_f32_16x16x32_bf16 v[92:95], a[148:151], a[252:255], v[92:95]\n"
      "v_cvt_pk_bf16_f32 v85, v86, v87\n"
      "v_cvt_pk_bf16_f32 v86, v116, v117\n"
      "v_mfma_f32_16x16x32_bf16 v[124:127], a[156:159], a[252:255], v[124:127]\n"
      "buffer_store_dwordx4 v[80:83], v30, s[44:47], s58 offen offset:1024 sc0 sc1\n"
      "v_mfma_f32_16x16x32_bf16 v[124:127], a[152:155], a[248:251], v[124:127]\n"
      "v_cvt_pk_bf16_f32 v87, v118, v119\n"
      "v_accvgpr_read_b32 v52, a24\n"
      "v_mfma_f32_16x16x32_bf16 v[128:131], a[160:163], a[192:195], v[128:131]\n"
      "buffer_store_dwordx4 v[48:51], v30, s[44:47], s58 offen offset:2048 sc0 sc1\n"
      "v_mfma_f32_16x16x32_bf16 v[128:131], a[164:167], a[196:199], v[128:131]\n"
      "v_accvgpr_read_b32 v53, a25\n"
      "v_accvgpr_read_b32 v54, a26\n"
      "v_mfma_f32_16x16x32_bf16 v[160:163], a[172:175], a[196:199], v[160:163]\n"
      "ds_read_b64_tr_b16 a[128:129], v0 offset:1024\n"
      "v_accvgpr_read_b32 v55, a27\n"
      "v_mfma_f32_16x16x32_bf16 v[160:163], a[168:171], a[192:195], v[160:163]\n"
      "v_cvt_pk_bf16_f32 v52, v52, v53\n"
      "v_cvt_pk_bf16_f32 v53, v54, v55\n"
      "v_mfma_f32_16x16x32_bf16 v[192:195], a[176:179], a[192:195], v[192:195]\n"
      "v_cvt_pk_bf16_f32 v54, v56, v57\n"
      "v_cvt_pk_bf16_f32 v55, v58, v59\n"
      "v_mfma_f32_16x16x32_bf16 v[192:195], a[180:183], a[196:199], v[192:195]\n"
      "buffer_store_dwordx4 v[84:87], v30, s[44:47], s58 offen offset:3072 sc0 sc1\n"
      "v_mfma_f32_16x16x32_bf16 v[224:227], a[188:191], a[196:199], v[224:227]\n"
      "s_add_u32 s58, s58, 0x1000\n"
      "v_cvt_pk_bf16_f32 v88, v88, v89\n"
      "v_mfma_f32_16x16x32_bf16 v[224:227], a[184:187], a[192:195], v[224:227]\n"
      "ds_read_b64_tr_b16 a[130:131], v0 offset:1536\n"
      "v_cvt_pk_bf16_f32 v89, v90, v91\n"
      "v_mfma_f32_16x16x32_bf16 v[228:231], a[184:187], a[200:203], v[228:231]\n"
      "ds_read_b64_tr_b16 a[132:133], v0 offset:17408\n"
      "v_cvt_pk_bf16_f32 v90, v120, v121\n"
      "v_mfma_f32_16x16x32_bf16 v[228:231], a[188:191], a[204:207], v[228:231]\n"
      "v_cvt_pk_bf16_f32 v91, v122, v123\n"
      "v_accvgpr_read_b32 v56, a28\n"
      "v_mfma_f32_16x16x32_bf16 v[196:199], a[180:183], a[204:207], v[196:199]\n"
      "buffer_store_dwordx4 v[52:55], v30, s[44:47], s58 offen offset:0 sc0 sc1\n"
      "v_mfma_f32_16x16x32_bf16 v[196:199], a[176:179], a[200:203], v[196:199]\n"
      "ds_read_b64_tr_b16 a[134:135], v0 offset:17920\n"
      "v_accvgpr_read_b32 v57, a29\n"
      "v_mfma_f32_16x16x32_bf16 v[164:167], a[168:171], a[200:203], v[164:167]\n"
      "v_accvgpr_read_b32 v58, a30\n"
      "v_accvgpr_read_b32 v59, a31\n"
      "v_mfma_f32_16x16x32_bf16 v[164:167], a[172:175], a[204:207], v[164:167]\n"
      "v_cvt_pk_bf16_f32 v56, v56, v57\n"
      "v_cvt_pk_bf16_f32 v57, v58, v59\n"
      "v_mfma_f32_16x16x32_bf16 v[132:135], a[164:167], a[204:207], v[132:135]\n"
      "ds_read_b64_tr_b16 a[136:137], v1 offset:1024\n"
      "v_cvt_pk_bf16_f32 v58, v60, v61\n"
      "v_mfma_f32_16x16x32_bf16 v[132:135], a[160:163], a[200:203], v[132:135]\n"
      "buffer_store_dwordx4 v[88:91], v30, s[44:47], s58 offen offset:1024 sc0 sc1\n"
      "v_mfma_f32_16x16x32_bf16 v[136:139], a[160:163], a[208:211], v[136:139]\n"
      "v_cvt_pk_bf16_f32 v59, v62, v63\n"
      "v_cvt_pk_bf16_f32 v92, v92, v93\n"
      "v_mfma_f32_16x16x32_bf16 v[136:139], a[164:167], a[212:215], v[136:139]\n"
      "ds_read_b64_tr_b16 a[138:139], v1 offset:1536\n"
      "v_cvt_pk_bf16_f32 v93, v94, v95\n"
      "v_mfma_f32_16x16x32_bf16 v[168:171], a[172:175], a[212:215], v[168:171]\n"
      "v_cvt_pk_bf16_f32 v94, v124, v125\n"
      "v_cvt_pk_bf16_f32 v95, v126, v127\n"
      "v_mfma_f32_16x16x32_bf16 v[168:171], a[168:171], a[208:211], v[168:171]\n"
      "v_mfma_f32_16x16x32_bf16 v[200:203], a[176:179], a[208:211], v[200:203]\n"
      "buffer_store_dwordx4 v[56:59], v30, s[44:47], s58 offen offset:2048 sc0 sc1\n"
      "v_mfma_f32_16x16x32_bf16 v[200:203], a[180:183], a[212:215], v[200:203]\n"
      "ds_read_b64_tr_b16 a[140:141], v1 offset:17408\n"
      "v_mfma_f32_16x16x32_bf16 v[232:235], a[188:191], a[212:215], v[232:235]\n"
      "v_mfma_f32_16x16x32_bf16 v[232:235], a[184:187], a[208:211], v[232:235]\n"
      "ds_read_b64_tr_b16 a[142:143], v1 offset:17920\n"
      "v_mfma_f32_16x16x32_bf16 v[236:239], a[184:187], a[216:219], v[236:239]\n"
      "s_add_u32 s90, s71, 2\n"
      "v_mfma_f32_16x16x32_bf16 v[236:239], a[188:191], a[220:223], v[236:239]\n"
      "buffer_store_dwordx4 v[92:95], v30, s[44:47], s58 offen offset:3072 sc0 sc1\n"
      "v_mfma_f32_16x16x32_bf16 v[204:207], a[180:183], a[220:223], v[204:207]\n"
      "s_xor_b32 s58, %48, 0x10000\n"
      "v_readlane_b32 s57, v22, s90\n"
      "v_mfma_f32_16x16x32_bf16 v[204:207], a[176:179], a[216:219], v[204:207]\n"
      "s_add_u32 s65, s57, 0\n"
      "v_mfma_f32_16x16x32_bf16 v[172:175], a[168:171], a[216:219], v[172:175]\n"
      "s_or_b32 m0, s54, %42\n"
      "v_mfma_f32_16x16x32_bf16 v[172:175], a[172:175], a[220:223], v[172:175]\n"
      "buffer_load_dwordx4 v7, %36, s65 offen offset:1024 lds\n"
      "v_mfma_f32_16x16x32_bf16 v[140:143], a[164:167], a[220:223], v[140:143]\n"
      "ds_read_b64_tr_b16 a[144:145], v2 offset:1024\n"
      "v_mfma_f32_16x16x32_bf16 v[140:143], a[160:163], a[216:219], v[140:143]\n"
      "ds_read_b64_tr_b16 a[146:147], v2 offset:1536\n"
      "v_mfma_f32_16x16x32_bf16 v[144:147], a[160:163], a[224:227], v[144:147]\n"
      "ds_read_b64_tr_b16 a[148:149], v2 offset:17408\n"
      "v_mfma_f32_16x16x32_bf16 v[144:147], a[164:167], a[228:231], v[144:147]\n"
      "buffer_load_dwordx4 v9, %36, s65 offen offset:3072 lds\n"
      "v_mfma_f32_16x16x32_bf16 v[176:179], a[172:175], a[228:231], v[176:179]\n"
      "s_bitset1_b32 m0, 13\n"
      "v_mfma_f32_16x16x32_bf16 v[176:179], a[168:171], a[224:227], v[176:179]\n"
      "ds_read_b64_tr_b16 a[150:151], v2 offset:17920\n"
      "v_mfma_f32_16x16x32_bf16 v[208:211], a[176:179], a[224:227], v[208:211]\n"
      "s_add_u32 s90, s71, 2\n"
      "v_mfma_f32_16x16x32_bf16 v[208:211], a[180:183], a[228:231], v[208:211]\n"
      "buffer_load_dwordx4 v11, %36, s65 offen offset:1024 lds\n"
      "v_mfma_f32_16x16x32_bf16 v[240:243], a[188:191], a[228:231], v[240:243]\n"
      "ds_read_b64_tr_b16 a[152:153], v3 offset:1024\n"
      "v_readlane_b32 s86, v22, s90\n"
      "v_mfma_f32_16x16x32_bf16 v[240:243], a[184:187], a[224:227], v[240:243]\n"
      "s_lshr_b32 s86, s86, 13\n"
      "v_mfma_f32_16x16x32_bf16 v[244:247], a[184:187], a[232:235], v[244:247]\n"
      "s_mul_i32 s86, s86, 0x401\n"
      "v_mfma_f32_16x16x32_bf16 v[244:247], a[188:191], a[236:239], v[244:247]\n"
      "buffer_load_dwordx4 v13, %36, s65 offen offset:3072 lds\n"
      "v_mfma_f32_16x16x32_bf16 v[212:215], a[180:183], a[236:239], v[212:215]\n"
      "ds_read_b64_tr_b16 a[154:155], v3 offset:1536\n"
      "s_and_b32 s86, s86, 0xe00780\n"
      "v_mfma_f32_16x16x32_bf16 v[212:215], a[176:179], a[232:235], v[212:215]\n"
      "s_or_b32 s64, s86, %43\n"
      "v_mfma_f32_16x16x32_bf16 v[180:183], a[168:171], a[232:235], v[180:183]\n"
      "s_or_b32 m0, s50, %41\n"
      "v_mfma_f32_16x16x32_bf16 v[180:183], a[172:175], a[236:239], v[180:183]\n"
      "buffer_load_dwordx4 v14, s[40:43], s64 offen offset:0 sc0 sc1 lds\n"
      "v_mfma_f32_16x16x32_bf16 v[148:151], a[164:167], a[236:239], v[148:151]\n"
      "ds_read_b128 a[192:195], v4 offset:0\n"
      "v_mfma_f32_16x16x32_bf16 v[148:151], a[160:163], a[232:235], v[148:151]\n"
      "ds_read_b128 a[196:199], v5 offset:0\n"
      "v_mfma_f32_16x16x32_bf16 v[152:155], a[160:163], a[240:243], v[152:155]\n"
      "ds_read_b64_tr_b16 a[156:157], v3 offset:17408\n"
      "v_mfma_f32_16x16x32_bf16 v[152:155], a[164:167], a[244:247], v[152:155]\n"
      "buffer_load_dwordx4 v15, s[40:43], s64 offen offset:1024 sc0 sc1 lds\n"
      "v_mfma_f32_16x16x32_bf16 v[184:187], a[172:175], a[244:247], v[184:187]\n"
      "ds_read_b64_tr_b16 a[158:159], v3 offset:17920\n"
      "v_mfma_f32_16x16x32_bf16 v[184:187], a[168:171], a[240:243], v[184:187]\n"
      "v_mfma_f32_16x16x32_bf16 v[216:219], a[176:179], a[240:243], v[216:219]\n"
      "ds_read_b128 a[200:203], v4 offset:2048\n"
      "v_mfma_f32_16x16x32_bf16 v[216:219], a[180:183], a[244:247], v[216:219]\n"
      "buffer_load_dwordx4 v16, s[40:43], s64 offen offset:2048 sc0 sc1 lds\n"
      "v_mfma_f32_16x16x32_bf16 v[248:251], a[188:191], a[244:247], v[248:251]\n"
      "ds_read_b128 a[204:207], v5 offset:2048\n"
      "v_mfma_f32_16x16x32_bf16 v[248:251], a[184:187], a[240:243], v[248:251]\n"
      "v_mfma_f32_16x16x32_bf16 v[252:255], a[184:187], a[248:251], v[252:255]\n"
      "v_mfma_f32_16x16x32_bf16 v[252:255], a[188:191], a[252:255], v[252:255]\n"
      "buffer_load_dwordx4 v17, s[40:43], s64 offen offset:3072 sc0 sc1 lds\n"
      "v_mfma_f32_16x16x32_bf16 v[220:223], a[180:183], a[252:255], v[220:223]\n"
      "s_bitset1_b32 m0, 12\n"
      "v_mfma_f32_16x16x32_bf16 v[220:223], a[176:179], a[248:251], v[220:223]\n"
      "v_mfma_f32_16x16x32_bf16 v[188:191], a[168:171], a[248:251], v[188:191]\n"
      "ds_read_b128 a[208:211], v4 offset:4096\n"
      "v_mfma_f32_16x16x32_bf16 v[188:191], a[172:175], a[252:255], v[188:191]\n"
      "buffer_load_dwordx4 v18, s[40:43], s64 offen offset:0 sc0 sc1 lds\n"
      "v_mfma_f32_16x16x32_bf16 v[156:159], a[164:167], a[252:255], v[156:159]\n"
      "ds_read_b128 a[212:215], v5 offset:4096\n"
      "v_mfma_f32_16x16x32_bf16 v[156:159], a[160:163], a[248:251], v[156:159]\n"
      "s_waitcnt lgkmcnt(5)\n"
      "v_mfma_f32_16x16x32_bf16 v[128:131], a[128:131], a[192:195], v[128:131]\n"
      "ds_read_b128 a[216:219], v4 offset:6144\n"
      "v_mfma_f32_16x16x32_bf16 v[128:131], a[132:135], a[196:199], v[128:131]\n"
      "buffer_load_dwordx4 v19, s[40:43], s64 offen offset:1024 sc0 sc1 lds\n"
      "v_mfma_f32_16x16x32_bf16 v[160:163], a[140:143], a[196:199], v[160:163]\n"
      "ds_read_b128 a[220:223], v5 offset:6144\n"
      "v_mfma_f32_16x16x32_bf16 v[160:163], a[136:139], a[192:195], v[160:163]\n"
      "ds_read_b128 a[224:227], v4 offset:8192\n"
      "s_xor_b32 s55, s51, s53\n"
      "v_mfma_f32_16x16x32_bf16 v[192:195], a[144:147], a[192:195], v[192:195]\n"
      "ds_read_b128 a[228:231], v5 offset:8192\n"
      "v_mfma_f32_16x16x32_bf16 v[192:195], a[148:151], a[196:199], v[192:195]\n"
      "buffer_load_dwordx4 v20, s[40:43], s64 offen offset:2048 sc0 sc1 lds\n"
      "s_waitcnt lgkmcnt(6)\n"
      "v_mfma_f32_16x16x32_bf16 v[224:227], a[156:159], a[196:199], v[224:227]\n"
      "ds_read_b128 a[232:235], v4 offset:10240\n"
      "v_mfma_f32_16x16x32_bf16 v[224:227], a[152:155], a[192:195], v[224:227]\n"
      "ds_read_b128 a[236:239], v5 offset:10240\n"
      "v_mfma_f32_16x16x32_bf16 v[228:231], a[152:155], a[200:203], v[228:231]\n"
      "ds_read_b128 a[240:243], v4 offset:12288\n"
      "s_add_u32 s90, s71, 3\n"
      "v_mfma_f32_16x16x32_bf16 v[228:231], a[156:159], a[204:207], v[228:231]\n"
      "buffer_load_dwordx4 v21, s[40:43], s64 offen offset:3072 sc0 sc1 lds\n"
      "v_mfma_f32_16x16x32_bf16 v[196:199], a[148:151], a[204:207], v[196:199]\n"
      "ds_read_b128 a[244:247], v5 offset:12288\n"
      "v_readlane_b32 s57, v22, s90\n"
      "v_mfma_f32_16x16x32_bf16 v[196:199], a[144:147], a[200:203], v[196:199]\n"
      "s_add_u32 s65, s57, 1024\n"
      "v_mfma_f32_16x16x32_bf16 v[164:167], a[136:139], a[200:203], v[164:167]\n"
      "s_or_b32 m0, s54, %42\n"
      "v_mfma_f32_16x16x32_bf16 v[164:167], a[140:143], a[204:207], v[164:167]\n"
      "buffer_load_dwordx4 v7, %36, s65 offen offset:0 lds\n"
      "v_mfma_f32_16x16x32_bf16 v[132:135], a[132:135], a[204:207], v[132:135]\n"
      "ds_read_b128 a[248:251], v4 offset:14336\n"
      "v_mfma_f32_16x16x32_bf16 v[132:135], a[128:131], a[200:203], v[132:135]\n"
      "ds_read_b128 a[252:255], v5 offset:14336\n"
      "s_waitcnt lgkmcnt(9)\n"
      "v_mfma_f32_16x16x32_bf16 v[136:139], a[128:131], a[208:211], v[136:139]\n"
      "v_xor_b32 v4, s55, v4\n"
      "v_xor_b32 v5, s55, v5\n"
      "v_mfma_f32_16x16x32_bf16 v[136:139], a[132:135], a[212:215], v[136:139]\n"
      "buffer_load_dwordx4 v9, %36, s65 offen offset:2048 lds\n"
      "v_mfma_f32_16x16x32_bf16 v[168:171], a[140:143], a[212:215], v[168:171]\n"
      "s_bitset1_b32 m0, 13\n"
      "v_mfma_f32_16x16x32_bf16 v[168:171], a[136:139], a[208:211], v[168:171]\n"
      "v_mfma_f32_16x16x32_bf16 v[200:203], a[144:147], a[208:211], v[200:203]\n"
      "s_add_u32 s90, s71, 3\n"
      "v_mfma_f32_16x16x32_bf16 v[200:203], a[148:151], a[212:215], v[200:203]\n"
      "buffer_load_dwordx4 v11, %36, s65 offen offset:0 lds\n"
      "v_mfma_f32_16x16x32_bf16 v[232:235], a[156:159], a[212:215], v[232:235]\n"
      "v_readlane_b32 s86, v22, s90\n"
      "v_mfma_f32_16x16x32_bf16 v[232:235], a[152:155], a[208:211], v[232:235]\n"
      "s_lshr_b32 s86, s86, 13\n"
      "s_waitcnt lgkmcnt(4)\n"
      "v_mfma_f32_16x16x32_bf16 v[236:239], a[152:155], a[216:219], v[236:239]\n"
      "s_mul_i32 s86, s86, 0x401\n"
      "v_mfma_f32_16x16x32_bf16 v[236:239], a[156:159], a[220:223], v[236:239]\n"
      "s_waitcnt vmcnt(31) lgkmcnt(0)\n"
      "s_barrier\n"
      "v_mfma_f32_16x16x32_bf16 v[204:207], a[148:151], a[220:223], v[204:207]\n"
      "buffer_load_dwordx4 v13, %36, s65 offen offset:2048 lds\n"
      "v_mfma_f32_16x16x32_bf16 v[204:207], a[144:147], a[216:219], v[204:207]\n"
      "ds_read_b64_tr_b16 a[160:161], v0 offset:0\n"
      "s_and_b32 s86, s86, 0xe00780\n"
      "v_mfma_f32_16x16x32_bf16 v[172:175], a[136:139], a[216:219], v[172:175]\n"
      "s_or_b32 s64, s86, %43\n"
      "v_mfma_f32_16x16x32_bf16 v[172:175], a[140:143], a[220:223], v[172:175]\n"
      "s_or_b32 m0, s51, %41\n"
      "v_mfma_f32_16x16x32_bf16 v[140:143], a[132:135], a[220:223], v[140:143]\n"
      "buffer_load_dwordx4 v14, s[40:43], s64 offen offset:0 sc0 sc1 lds\n"
      "v_mfma_f32_16x16x32_bf16 v[140:143], a[128:131], a[216:219], v[140:143]\n"
      "ds_read_b64_tr_b16 a[162:163], v0 offset:512\n"
      "v_mfma_f32_16x16x32_bf16 v[144:147], a[128:131], a[224:227], v[144:147]\n"
      "ds_read_b64_tr_b16 a[164:165], v0 offset:16384\n"
      "v_mfma_f32_16x16x32_bf16 v[144:147], a[132:135], a[228:231], v[144:147]\n"
      "ds_read_b64_tr_b16 a[166:167], v0 offset:16896\n"
      "v_mfma_f32_16x16x32_bf16 v[176:179], a[140:143], a[228:231], v[176:179]\n"
      "buffer_load_dwordx4 v15, s[40:43], s64 offen offset:1024 sc0 sc1 lds\n"
      "v_mfma_f32_16x16x32_bf16 v[176:179], a[136:139], a[224:227], v[176:179]\n"
      "ds_read_b64_tr_b16 a[168:169], v1 offset:0\n"
      "v_xor_b32 v0, s56, v0\n"
      "v_mfma_f32_16x16x32_bf16 v[208:211], a[144:147], a[224:227], v[208:211]\n"
      "ds_read_b64_tr_b16 a[170:171], v1 offset:512\n"
      "v_mfma_f32_16x16x32_bf16 v[208:211], a[148:151], a[228:231], v[208:211]\n"
      "ds_read_b64_tr_b16 a[172:173], v1 offset:16384\n"
      "v_mfma_f32_16x16x32_bf16 v[240:243], a[156:159], a[228:231], v[240:243]\n"
      "buffer_load_dwordx4 v16, s[40:43], s64 offen offset:2048 sc0 sc1 lds\n"
      "v_mfma_f32_16x16x32_bf16 v[240:243], a[152:155], a[224:227], v[240:243]\n"
      "ds_read_b64_tr_b16 a[174:175], v1 offset:16896\n"
      "v_mfma_f32_16x16x32_bf16 v[244:247], a[152:155], a[232:235], v[244:247]\n"
      "ds_read_b64_tr_b16 a[176:177], v2 offset:0\n"
      "v_xor_b32 v1, s56, v1\n"
      "v_mfma_f32_16x16x32_bf16 v[244:247], a[156:159], a[236:239], v[244:247]\n"
      "ds_read_b64_tr_b16 a[178:179], v2 offset:512\n"
      "v_mfma_f32_16x16x32_bf16 v[212:215], a[148:151], a[236:239], v[212:215]\n"
      "buffer_load_dwordx4 v17, s[40:43], s64 offen offset:3072 sc0 sc1 lds\n"
      "v_mfma_f32_16x16x32_bf16 v[212:215], a[144:147], a[232:235], v[212:215]\n"
      "s_bitset1_b32 m0, 12\n"
      "v_mfma_f32_16x16x32_bf16 v[180:183], a[136:139], a[232:235], v[180:183]\n"
      "ds_read_b128 a[192:195], v4 offset:0\n"
      "v_mfma_f32_16x16x32_bf16 v[180:183], a[140:143], a[236:239], v[180:183]\n"
      "ds_read_b128 a[196:199], v5 offset:0\n"
      "v_mfma_f32_16x16x32_bf16 v[148:151], a[132:135], a[236:239], v[148:151]\n"
      "buffer_load_dwordx4 v18, s[40:43], s64 offen offset:0 sc0 sc1 lds\n"
      "v_mfma_f32_16x16x32_bf16 v[148:151], a[128:131], a[232:235], v[148:151]\n"
      "ds_read_b64_tr_b16 a[180:181], v2 offset:16384\n"
      "v_mfma_f32_16x16x32_bf16 v[152:155], a[128:131], a[240:243], v[152:155]\n"
      "ds_read_b64_tr_b16 a[182:183], v2 offset:16896\n"
      "v_mfma_f32_16x16x32_bf16 v[152:155], a[132:135], a[244:247], v[152:155]\n"
      "ds_read_b64_tr_b16 a[184:185], v3 offset:0\n"
      "v_xor_b32 v2, s56, v2\n"
      "v_mfma_f32_16x16x32_bf16 v[184:187], a[140:143], a[244:247], v[184:187]\n"
      "buffer_load_dwordx4 v19, s[40:43], s64 offen offset:1024 sc0 sc1 lds\n"
      "v_mfma_f32_16x16x32_bf16 v[184:187], a[136:139], a[240:243], v[184:187]\n"
      "ds_read_b64_tr_b16 a[186:187], v3 offset:512\n"
      "v_mfma_f32_16x16x32_bf16 v[216:219], a[144:147], a[240:243], v[216:219]\n"
      "ds_read_b128 a[200:203], v4 offset:2048\n"
      "v_mfma_f32_16x16x32_bf16 v[216:219], a[148:151], a[244:247], v[216:219]\n"
      "ds_read_b128 a[204:207], v5 offset:2048\n"
      "v_mfma_f32_16x16x32_bf16 v[248:251], a[156:159], a[244:247], v[248:251]\n"
      "buffer_load_dwordx4 v20, s[40:43], s64 offen offset:2048 sc0 sc1 lds\n"
      "v_mfma_f32_16x16x32_bf16 v[248:251], a[152:155], a[240:243], v[248:251]\n"
      "ds_read_b64_tr_b16 a[188:189], v3 offset:16384\n"
      "v_mfma_f32_16x16x32_bf16 v[252:255], a[152:155], a[248:251], v[252:255]\n"
      "ds_read_b64_tr_b16 a[190:191], v3 offset:16896\n"
      "v_mfma_f32_16x16x32_bf16 v[252:255], a[156:159], a[252:255], v[252:255]\n"
      "v_xor_b32 v3, s56, v3\n"
      "s_add_u32 s90, s71, 4\n"
      "v_mfma_f32_16x16x32_bf16 v[220:223], a[148:151], a[252:255], v[220:223]\n"
      "buffer_load_dwordx4 v21, s[40:43], s64 offen offset:3072 sc0 sc1 lds\n"
      "v_mfma_f32_16x16x32_bf16 v[220:223], a[144:147], a[248:251], v[220:223]\n"
      "v_readlane_b32 s57, v22, s90\n"
      "v_mfma_f32_16x16x32_bf16 v[188:191], a[136:139], a[248:251], v[188:191]\n"
      "s_add_u32 s65, s57, 0\n"
      "v_mfma_f32_16x16x32_bf16 v[188:191], a[140:143], a[252:255], v[188:191]\n"
      "s_or_b32 m0, s52, %42\n"
      "v_mfma_f32_16x16x32_bf16 v[156:159], a[132:135], a[252:255], v[156:159]\n"
      "buffer_load_dwordx4 v7, %36, s65 offen offset:1024 lds\n"
      "v_mfma_f32_16x16x32_bf16 v[156:159], a[128:131], a[248:251], v[156:159]\n"
      "ds_read_b128 a[208:211], v4 offset:4096\n"
      "s_waitcnt lgkmcnt(7)\n"
      "v_mfma_f32_16x16x32_bf16 v[128:131], a[160:163], a[192:195], v[128:131]\n"
      "ds_read_b128 a[212:215], v5 offset:4096\n"
      "v_mfma_f32_16x16x32_bf16 v[128:131], a[164:167], a[196:199], v[128:131]\n"
      "ds_read_b128 a[216:219], v4 offset:6144\n"
      "v_mfma_f32_16x16x32_bf16 v[160:163], a[172:175], a[196:199], v[160:163]\n"
      "buffer_load_dwordx4 v9, %36, s65 offen offset:3072 lds\n"
      "v_mfma_f32_16x16x32_bf16 v[160:163], a[168:171], a[192:195], v[160:163]\n"
      "s_bitset1_b32 m0, 13\n"
      "s_xor_b32 s55, s53, s50\n"
      "v_mfma_f32_16x16x32_bf16 v[192:195], a[176:179], a[192:195], v[192:195]\n"
      "ds_read_b128 a[220:223], v5 offset:6144\n"
      "v_mfma_f32_16x16x32_bf16 v[192:195], a[180:183], a[196:199], v[192:195]\n"
      "ds_read_b128 a[224:227], v4 offset:8192\n"
      "s_waitcnt lgkmcnt(5)\n"
      "v_mfma_f32_16x16x32_bf16 v[224:227], a[188:191], a[196:199], v[224:227]\n"
      "buffer_load_dwordx4 v11, %36, s65 offen offset:1024 lds\n"
      "v_mfma_f32_16x16x32_bf16 v[224:227], a[184:187], a[192:195], v[224:227]\n"
      "ds_read_b128 a[228:231], v5 offset:8192\n"
      "v_mfma_f32_16x16x32_bf16 v[228:231], a[184:187], a[200:203], v[228:231]\n"
      "ds_read_b128 a[232:235], v4 offset:10240\n"
      "v_mfma_f32_16x16x32_bf16 v[228:231], a[188:191], a[204:207], v[228:231]\n"
      "ds_read_b128 a[236:239], v5 offset:10240\n"
      "v_mfma_f32_16x16x32_bf16 v[196:199], a[180:183], a[204:207], v[196:199]\n"
      "buffer_load_dwordx4 v13, %36, s65 offen offset:3072 lds\n"
      "v_mfma_f32_16x16x32_bf16 v[196:199], a[176:179], a[200:203], v[196:199]\n"
      "ds_read_b128 a[240:243], v4 offset:12288\n"
      "v_mfma_f32_16x16x32_bf16 v[164:167], a[168:171], a[200:203], v[164:167]\n"
      "ds_read_b128 a[244:247], v5 offset:12288\n"
      "v_mfma_f32_16x16x32_bf16 v[164:167], a[172:175], a[204:207], v[164:167]\n"
      "ds_read_b128 a[248:251], v4 offset:14336\n"
      "v_mfma_f32_16x16x32_bf16 v[132:135], a[164:167], a[204:207], v[132:135]\n"
      "ds_read_b128 a[252:255], v5 offset:14336\n"
      "v_mfma_f32_16x16x32_bf16 v[132:135], a[160:163], a[200:203], v[132:135]\n"
      "v_xor_b32 v4, s55, v4\n"
      "v_xor_b32 v5, s55, v5\n"
      "s_waitcnt lgkmcnt(10)\n"
      "v_mfma_f32_16x16x32_bf16 v[136:139], a[160:163], a[208:211], v[136:139]\n"
      "v_and_b32 v26, 0xe0, v30\n"
      "v_mfma_f32_16x16x32_bf16 v[136:139], a[164:167], a[212:215], v[136:139]\n"
      "v_lshlrev_b32 v26, 10, v26\n"
      "v_mfma_f32_16x16x32_bf16 v[168:171], a[172:175], a[212:215], v[168:171]\n"
      "v_lshrrev_b32 v24, 8, v30\n"
      "v_mfma_f32_16x16x32_bf16 v[168:171], a[168:171], a[208:211], v[168:171]\n"
      "s_add_u32 s90, s71, 4\n"
      "v_mfma_f32_16x16x32_bf16 v[200:203], a[176:179], a[208:211], v[200:203]\n"
      "v_readlane_b32 s86, v22, s90\n"
      "v_mfma_f32_16x16x32_bf16 v[200:203], a[180:183], a[212:215], v[200:203]\n"
      "s_lshr_b32 s86, s86, 13\n"
      "v_mfma_f32_16x16x32_bf16 v[232:235], a[188:191], a[212:215], v[232:235]\n"
      "s_mul_i32 s86, s86, 0x401\n"
      "v_mfma_f32_16x16x32_bf16 v[232:235], a[184:187], a[208:211], v[232:235]\n"
      "s_and_b32 s86, s86, 0xe00780\n"
      "s_waitcnt lgkmcnt(5)\n"
      "v_mfma_f32_16x16x32_bf16 v[236:239], a[184:187], a[216:219], v[236:239]\n"
      "s_or_b32 s64, s86, %43\n"
      "v_mfma_f32_16x16x32_bf16 v[236:239], a[188:191], a[220:223], v[236:239]\n"
      "s_waitcnt vmcnt(16) lgkmcnt(0)\n"
      "s_barrier\n"
      "v_mfma_f32_16x16x32_bf16 v[204:207], a[180:183], a[220:223], v[204:207]\n"
      "s_or_b32 m0, s53, %41\n"
      "v_mfma_f32_16x16x32_bf16 v[204:207], a[176:179], a[216:219], v[204:207]\n"
      "buffer_load_dwordx4 v14, s[40:43], s64 offen offset:0 sc0 sc1 lds\n"
      "v_mfma_f32_16x16x32_bf16 v[172:175], a[168:171], a[216:219], v[172:175]\n"
      "s_mov_b64 exec, 1\n"
      "buffer_store_dword v31, off, s[28:31], %46 sc0 sc1\n"
      "s_mov_b64 exec, -1\n"
      "v_mfma_f32_16x16x32_bf16 v[172:175], a[172:175], a[220:223], v[172:175]\n"
      "ds_read_b64_tr_b16 a[128:129], v0 offset:1024\n"
      "v_mfma_f32_16x16x32_bf16 v[140:143], a[164:167], a[220:223], v[140:143]\n"
      "ds_read_b64_tr_b16 a[130:131], v0 offset:1536\n"
      "v_mfma_f32_16x16x32_bf16 v[140:143], a[160:163], a[216:219], v[140:143]\n"
      "buffer_load_dwordx4 v15, s[40:43], s64 offen offset:1024 sc0 sc1 lds\n"
      "v_mfma_f32_16x16x32_bf16 v[144:147], a[160:163], a[224:227], v[144:147]\n"
      "ds_read_b64_tr_b16 a[132:133], v0 offset:17408\n"
      "v_mfma_f32_16x16x32_bf16 v[144:147], a[164:167], a[228:231], v[144:147]\n"
      "ds_read_b64_tr_b16 a[134:135], v0 offset:17920\n"
      "v_mfma_f32_16x16x32_bf16 v[176:179], a[172:175], a[228:231], v[176:179]\n"
      "ds_read_b64_tr_b16 a[136:137], v1 offset:1024\n"
      "v_mfma_f32_16x16x32_bf16 v[176:179], a[168:171], a[224:227], v[176:179]\n"
      "buffer_load_dwordx4 v16, s[40:43], s64 offen offset:2048 sc0 sc1 lds\n"
      "v_mfma_f32_16x16x32_bf16 v[208:211], a[176:179], a[224:227], v[208:211]\n"
      "ds_read_b64_tr_b16 a[138:139], v1 offset:1536\n"
      "v_mfma_f32_16x16x32_bf16 v[208:211], a[180:183], a[228:231], v[208:211]\n"
      "ds_read_b64_tr_b16 a[140:141], v1 offset:17408\n"
      "v_mfma_f32_16x16x32_bf16 v[240:243], a[188:191], a[228:231], v[240:243]\n"
      "ds_read_b64_tr_b16 a[142:143], v1 offset:17920\n"
      "v_mfma_f32_16x16x32_bf16 v[240:243], a[184:187], a[224:227], v[240:243]\n"
      "buffer_load_dwordx4 v17, s[40:43], s64 offen offset:3072 sc0 sc1 lds\n"
      "v_mfma_f32_16x16x32_bf16 v[244:247], a[184:187], a[232:235], v[244:247]\n"
      "s_bitset1_b32 m0, 12\n"
      "v_mfma_f32_16x16x32_bf16 v[244:247], a[188:191], a[236:239], v[244:247]\n"
      "ds_read_b64_tr_b16 a[144:145], v2 offset:1024\n"
      "v_mfma_f32_16x16x32_bf16 v[212:215], a[180:183], a[236:239], v[212:215]\n"
      "ds_read_b64_tr_b16 a[146:147], v2 offset:1536\n"
      "v_mfma_f32_16x16x32_bf16 v[212:215], a[176:179], a[232:235], v[212:215]\n"
      "buffer_load_dwordx4 v18, s[40:43], s64 offen offset:0 sc0 sc1 lds\n"
      "v_mfma_f32_16x16x32_bf16 v[180:183], a[168:171], a[232:235], v[180:183]\n"
      "ds_read_b128 a[192:195], v4 offset:0\n"
      "v_mfma_f32_16x16x32_bf16 v[180:183], a[172:175], a[236:239], v[180:183]\n"
      "ds_read_b128 a[196:199], v5 offset:0\n"
      "v_mfma_f32_16x16x32_bf16 v[148:151], a[164:167], a[236:239], v[148:151]\n"
      "ds_read_b64_tr_b16 a[148:149], v2 offset:17408\n"
      "v_mfma_f32_16x16x32_bf16 v[148:151], a[160:163], a[232:235], v[148:151]\n"
      "buffer_load_dwordx4 v19, s[40:43], s64 offen offset:1024 sc0 sc1 lds\n"
      "v_mfma_f32_16x16x32_bf16 v[152:155], a[160:163], a[240:243], v[152:155]\n"
      "ds_read_b64_tr_b16 a[150:151], v2 offset:17920\n"
      "v_mfma_f32_16x16x32_bf16 v[152:155], a[164:167], a[244:247], v[152:155]\n"
      "ds_read_b64_tr_b16 a[152:153], v3 offset:1024\n"
      "v_mfma_f32_16x16x32_bf16 v[184:187], a[172:175], a[244:247], v[184:187]\n"
      "ds_read_b64_tr_b16 a[154:155], v3 offset:1536\n"
      "v_mfma_f32_16x16x32_bf16 v[184:187], a[168:171], a[240:243], v[184:187]\n"
      "buffer_load_dwordx4 v20, s[40:43], s64 offen offset:2048 sc0 sc1 lds\n"
      "v_mfma_f32_16x16x32_bf16 v[216:219], a[176:179], a[240:243], v[216:219]\n"
      "ds_read_b128 a[200:203], v4 offset:2048\n"
      "v_mfma_f32_16x16x32_bf16 v[216:219], a[180:183], a[244:247], v[216:219]\n"
      "ds_read_b128 a[204:207], v5 offset:2048\n"
      "v_mfma_f32_16x16x32_bf16 v[248:251], a[188:191], a[244:247], v[248:251]\n"
      "ds_read_b64_tr_b16 a[156:157], v3 offset:17408\n"
      "s_add_u32 s90, s71, 5\n"
      "v_mfma_f32_16x16x32_bf16 v[248:251], a[184:187], a[240:243], v[248:251]\n"
      "buffer_load_dwordx4 v21, s[40:43], s64 offen offset:3072 sc0 sc1 lds\n"
      "v_mfma_f32_16x16x32_bf16 v[252:255], a[184:187], a[248:251], v[252:255]\n"
      "ds_read_b64_tr_b16 a[158:159], v3 offset:17920\n"
      "v_readlane_b32 s57, v22, s90\n"
      "v_mfma_f32_16x16x32_bf16 v[252:255], a[188:191], a[252:255], v[252:255]\n"
      "s_add_u32 s65, s57, 1024\n"
      "v_mfma_f32_16x16x32_bf16 v[220:223], a[180:183], a[252:255], v[220:223]\n"
      "s_or_b32 m0, s52, %42\n"
      "v_mfma_f32_16x16x32_bf16 v[220:223], a[176:179], a[248:251], v[220:223]\n"
      "buffer_load_dwordx4 v7, %36, s65 offen offset:0 lds\n"
      "v_mfma_f32_16x16x32_bf16 v[188:191], a[168:171], a[248:251], v[188:191]\n"
      "ds_read_b128 a[208:211], v4 offset:4096\n"
      "v_mfma_f32_16x16x32_bf16 v[188:191], a[172:175], a[252:255], v[188:191]\n"
      "ds_read_b128 a[212:215], v5 offset:4096\n"
      "v_mfma_f32_16x16x32_bf16 v[156:159], a[164:167], a[252:255], v[156:159]\n"
      "v_lshl_or_b32 v26, v24, 5, v26\n"
      "v_mfma_f32_16x16x32_bf16 v[156:159], a[160:163], a[248:251], v[156:159]\n"
      "buffer_load_dwordx4 v9, %36, s65 offen offset:2048 lds\n"
      "s_waitcnt lgkmcnt(8)\n"
      "v_mfma_f32_16x16x32_bf16 v[128:131], a[128:131], a[192:195], v[128:131]\n"
      "s_bitset1_b32 m0, 13\n"
      "s_xor_b32 s55, s50, s51\n"
      "v_mfma_f32_16x16x32_bf16 v[128:131], a[132:135], a[196:199], v[128:131]\n"
      "ds_read_b128 a[216:219], v4 offset:6144\n"
      "v_mfma_f32_16x16x32_bf16 v[160:163], a[140:143], a[196:199], v[160:163]\n"
      "ds_read_b128 a[220:223], v5 offset:6144\n"
      "v_mfma_f32_16x16x32_bf16 v[160:163], a[136:139], a[192:195], v[160:163]\n"
      "buffer_load_dwordx4 v11, %36, s65 offen offset:0 lds\n"
      "v_mfma_f32_16x16x32_bf16 v[192:195], a[144:147], a[192:195], v[192:195]\n"
      "ds_read_b128 a[224:227], v4 offset:8192\n"
      "v_mfma_f32_16x16x32_bf16 v[192:195], a[148:151], a[196:199], v[192:195]\n"
      "ds_read_b128 a[228:231], v5 offset:8192\n"
      "s_waitcnt lgkmcnt(6)\n"
      "v_mfma_f32_16x16x32_bf16 v[224:227], a[156:159], a[196:199], v[224:227]\n"
      "ds_read_b128 a[232:235], v4 offset:10240\n"
      "v_mfma_f32_16x16x32_bf16 v[224:227], a[152:155], a[192:195], v[224:227]\n"
      "buffer_load_dwordx4 v13, %36, s65 offen offset:2048 lds\n"
      "v_mfma_f32_16x16x32_bf16 v[228:231], a[152:155], a[200:203], v[228:231]\n"
      "ds_read_b128 a[236:239], v5 offset:10240\n"
      "v_mfma_f32_16x16x32_bf16 v[228:231], a[156:159], a[204:207], v[228:231]\n"
      "ds_read_b128 a[240:243], v4 offset:12288\n"
      "v_mfma_f32_16x16x32_bf16 v[196:199], a[148:151], a[204:207], v[196:199]\n"
      "ds_read_b128 a[244:247], v5 offset:12288\n"
      "v_mfma_f32_16x16x32_bf16 v[196:199], a[144:147], a[200:203], v[196:199]\n"
      "ds_read_b128 a[248:251], v4 offset:14336\n"
      "v_mfma_f32_16x16x32_bf16 v[164:167], a[136:139], a[200:203], v[164:167]\n"
      "ds_read_b128 a[252:255], v5 offset:14336\n"
      "v_mfma_f32_16x16x32_bf16 v[164:167], a[140:143], a[204:207], v[164:167]\n"
      "v_xor_b32 v4, s55, v4\n"
      "v_xor_b32 v5, s55, v5\n"
      "v_mfma_f32_16x16x32_bf16 v[132:135], a[132:135], a[204:207], v[132:135]\n"
      "v_and_or_b32 v26, v30, 16, v26\n"
      "v_mfma_f32_16x16x32_bf16 v[132:135], a[128:131], a[200:203], v[132:135]\n"
      "v_mov_b32 v24, 0x3f80\n"
      "s_waitcnt lgkmcnt(10)\n"
      "v_mfma_f32_16x16x32_bf16 v[136:139], a[128:131], a[208:211], v[136:139]\n"
      "v_mov_b32 v25, 0\n"
      "v_mfma_f32_16x16x32_bf16 v[136:139], a[132:135], a[212:215], v[136:139]\n"
      "v_lshlrev_b64 v[24:25], v30, v[24:25]\n"
      "v_mfma_f32_16x16x32_bf16 v[168:171], a[140:143], a[212:215], v[168:171]\n"
      "v_mfma_f32_16x16x32_bf16 v[168:171], a[136:139], a[208:211], v[168:171]\n"
      "s_add_u32 s90, s71, 5\n"
      "v_mfma_f32_16x16x32_bf16 v[200:203], a[144:147], a[208:211], v[200:203]\n"
      "v_readlane_b32 s86, v22, s90\n"
      "v_mfma_f32_16x16x32_bf16 v[200:203], a[148:151], a[212:215], v[200:203]\n"
      "s_lshr_b32 s86, s86, 13\n"
      "v_mfma_f32_16x16x32_bf16 v[232:235], a[156:159], a[212:215], v[232:235]\n"
      "s_mul_i32 s86, s86, 0x401\n"
      "v_mfma_f32_16x16x32_bf16 v[232:235], a[152:155], a[208:211], v[232:235]\n"
      "s_and_b32 s86, s86, 0xe00780\n"
      "s_waitcnt lgkmcnt(4)\n"
      "v_mfma_f32_16x16x32_bf16 v[236:239], a[152:155], a[216:219], v[236:239]\n"
      "s_or_b32 s64, s86, %43\n"
      "v_mfma_f32_16x16x32_bf16 v[236:239], a[156:159], a[220:223], v[236:239]\n"
      "s_waitcnt vmcnt(17) lgkmcnt(0)\n"
      "s_barrier\n"
      "v_mfma_f32_16x16x32_bf16 v[204:207], a[148:151], a[220:223], v[204:207]\n"
      "s_or_b32 m0, s50, %41\n"
      "v_mfma_f32_16x16x32_bf16 v[204:207], a[144:147], a[216:219], v[204:207]\n"
      "buffer_load_dwordx4 v14, s[40:43], s64 offen offset:0 sc0 sc1 lds\n"
      "v_mfma_f32_16x16x32_bf16 v[172:175], a[136:139], a[216:219], v[172:175]\n"
      "ds_read_b64_tr_b16 a[160:161], v0 offset:0\n"
      "v_mfma_f32_16x16x32_bf16 v[172:175], a[140:143], a[220:223], v[172:175]\n"
      "ds_read_b64_tr_b16 a[162:163], v0 offset:512\n"
      "v_mfma_f32_16x16x32_bf16 v[140:143], a[132:135], a[220:223], v[140:143]\n"
      "ds_read_b64_tr_b16 a[164:165], v0 offset:16384\n"
      "v_mfma_f32_16x16x32_bf16 v[140:143], a[128:131], a[216:219], v[140:143]\n"
      "buffer_load_dwordx4 v15, s[40:43], s64 offen offset:1024 sc0 sc1 lds\n"
      "v_mfma_f32_16x16x32_bf16 v[144:147], a[128:131], a[224:227], v[144:147]\n"
      "ds_read_b64_tr_b16 a[166:167], v0 offset:16896\n"
      "v_mfma_f32_16x16x32_bf16 v[144:147], a[132:135], a[228:231], v[144:147]\n"
      "ds_read_b64_tr_b16 a[168:169], v1 offset:0\n"
      "v_xor_b32 v0, s56, v0\n"
      "v_mfma_f32_16x16x32_bf16 v[176:179], a[140:143], a[228:231], v[176:179]\n"
      "ds_read_b64_tr_b16 a[170:171], v1 offset:512\n"
      "v_mfma_f32_16x16x32_bf16 v[176:179], a[136:139], a[224:227], v[176:179]\n"
      "buffer_load_dwordx4 v16, s[40:43], s64 offen offset:2048 sc0 sc1 lds\n"
      "v_mfma_f32_16x16x32_bf16 v[208:211], a[144:147], a[224:227], v[208:211]\n"
      "ds_read_b64_tr_b16 a[172:173], v1 offset:16384\n"
      "v_mfma_f32_16x16x32_bf16 v[208:211], a[148:151], a[228:231], v[208:211]\n"
      "ds_read_b64_tr_b16 a[174:175], v1 offset:16896\n"
      "v_mfma_f32_16x16x32_bf16 v[240:243], a[156:159], a[228:231], v[240:243]\n"
      "ds_read_b64_tr_b16 a[176:177], v2 offset:0\n"
      "v_xor_b32 v1, s56, v1\n"
      "v_mfma_f32_16x16x32_bf16 v[240:243], a[152:155], a[224:227], v[240:243]\n"
      "buffer_load_dwordx4 v17, s[40:43], s64 offen offset:3072 sc0 sc1 lds\n"
      "v_mfma_f32_16x16x32_bf16 v[244:247], a[152:155], a[232:235], v[244:247]\n"
      "s_bitset1_b32 m0, 12\n"
      "v_mfma_f32_16x16x32_bf16 v[244:247], a[156:159], a[236:239], v[244:247]\n"
      "ds_read_b64_tr_b16 a[178:179], v2 offset:512\n"
      "v_mfma_f32_16x16x32_bf16 v[212:215], a[148:151], a[236:239], v[212:215]\n"
      "ds_read_b64_tr_b16 a[180:181], v2 offset:16384\n"
      "v_mfma_f32_16x16x32_bf16 v[212:215], a[144:147], a[232:235], v[212:215]\n"
      "buffer_load_dwordx4 v18, s[40:43], s64 offen offset:0 sc0 sc1 lds\n"
      "v_mfma_f32_16x16x32_bf16 v[180:183], a[136:139], a[232:235], v[180:183]\n"
      "ds_read_b128 a[192:195], v4 offset:0\n"
      "v_mfma_f32_16x16x32_bf16 v[180:183], a[140:143], a[236:239], v[180:183]\n"
      "ds_read_b128 a[196:199], v5 offset:0\n"
      "v_mfma_f32_16x16x32_bf16 v[148:151], a[132:135], a[236:239], v[148:151]\n"
      "ds_read_b64_tr_b16 a[182:183], v2 offset:16896\n"
      "v_mfma_f32_16x16x32_bf16 v[148:151], a[128:131], a[232:235], v[148:151]\n"
      "buffer_load_dwordx4 v19, s[40:43], s64 offen offset:1024 sc0 sc1 lds\n"
      "v_mfma_f32_16x16x32_bf16 v[152:155], a[128:131], a[240:243], v[152:155]\n"
      "ds_read_b64_tr_b16 a[184:185], v3 offset:0\n"
      "v_xor_b32 v2, s56, v2\n"
      "v_mfma_f32_16x16x32_bf16 v[152:155], a[132:135], a[244:247], v[152:155]\n"
      "ds_read_b64_tr_b16 a[186:187], v3 offset:512\n"
      "v_mfma_f32_16x16x32_bf16 v[184:187], a[140:143], a[244:247], v[184:187]\n"
      "ds_read_b64_tr_b16 a[188:189], v3 offset:16384\n"
      "v_mfma_f32_16x16x32_bf16 v[184:187], a[136:139], a[240:243], v[184:187]\n"
      "buffer_load_dwordx4 v20, s[40:43], s64 offen offset:2048 sc0 sc1 lds\n"
      "v_mfma_f32_16x16x32_bf16 v[216:219], a[144:147], a[240:243], v[216:219]\n"
      "ds_read_b128 a[200:203], v4 offset:2048\n"
      "v_mfma_f32_16x16x32_bf16 v[216:219], a[148:151], a[244:247], v[216:219]\n"
      "ds_read_b128 a[204:207], v5 offset:2048\n"
      "v_mfma_f32_16x16x32_bf16 v[248:251], a[156:159], a[244:247], v[248:251]\n"
      "ds_read_b64_tr_b16 a[190:191], v3 offset:16896\n"
      "s_add_u32 s90, s71, 6\n"
      "v_mfma_f32_16x16x32_bf16 v[248:251], a[152:155], a[240:243], v[248:251]\n"
      "buffer_load_dwordx4 v21, s[40:43], s64 offen offset:3072 sc0 sc1 lds\n"
      "v_mfma_f32_16x16x32_bf16 v[252:255], a[152:155], a[248:251], v[252:255]\n"
      "v_xor_b32 v3, s56, v3\n"
      "v_readlane_b32 s57, v22, s90\n"
      "v_mfma_f32_16x16x32_bf16 v[252:255], a[156:159], a[252:255], v[252:255]\n"
      "s_add_u32 s65, s57, 0\n"
      "v_mfma_f32_16x16x32_bf16 v[220:223], a[148:151], a[252:255], v[220:223]\n"
      "s_or_b32 m0, s54, %42\n"
      "v_mfma_f32_16x16x32_bf16 v[220:223], a[144:147], a[248:251], v[220:223]\n"
      "buffer_load_dwordx4 v7, %36, s65 offen offset:1024 lds\n"
      "v_mfma_f32_16x16x32_bf16 v[188:191], a[136:139], a[248:251], v[188:191]\n"
      "ds_read_b128 a[208:211], v4 offset:4096\n"
      "v_mfma_f32_16x16x32_bf16 v[188:191], a[140:143], a[252:255], v[188:191]\n"
      "ds_read_b128 a[212:215], v5 offset:4096\n"
      "v_mfma_f32_16x16x32_bf16 v[156:159], a[132:135], a[252:255], v[156:159]\n"
      "v_mfma_f32_16x16x32_bf16 v[156:159], a[128:131], a[248:251], v[156:159]\n"
      "buffer_load_dwordx4 v9, %36, s65 offen offset:3072 lds\n"
      "s_waitcnt lgkmcnt(7)\n"
      "v_mfma_f32_16x16x32_bf16 v[128:131], a[160:163], a[192:195], v[128:131]\n"
      "s_bitset1_b32 m0, 13\n"
      "s_xor_b32 s55, s51, s53\n"
      "v_mfma_f32_16x16x32_bf16 v[128:131], a[164:167], a[196:199], v[128:131]\n"
      "ds_read_b128 a[216:219], v4 offset:6144\n"
      "v_mfma_f32_16x16x32_bf16 v[160:163], a[172:175], a[196:199], v[160:163]\n"
      "ds_read_b128 a[220:223], v5 offset:6144\n"
      "v_mfma_f32_16x16x32_bf16 v[160:163], a[168:171], a[192:195], v[160:163]\n"
      "buffer_load_dwordx4 v11, %36, s65 offen offset:1024 lds\n"
      "v_mfma_f32_16x16x32_bf16 v[192:195], a[176:179], a[192:195], v[192:195]\n"
      "ds_read_b128 a[224:227], v4 offset:8192\n"
      "v_mfma_f32_16x16x32_bf16 v[192:195], a[180:183], a[196:199], v[192:195]\n"
      "ds_read_b128 a[228:231], v5 offset:8192\n"
      "s_waitcnt lgkmcnt(6)\n"
      "v_mfma_f32_16x16x32_bf16 v[224:227], a[188:191], a[196:199], v[224:227]\n"
      "ds_read_b128 a[232:235], v4 offset:10240\n"
      "v_mfma_f32_16x16x32_bf16 v[224:227], a[184:187], a[192:195], v[224:227]\n"
      "buffer_load_dwordx4 v13, %36, s65 offen offset:3072 lds\n"
      "v_mfma_f32_16x16x32_bf16 v[228:231], a[184:187], a[200:203], v[228:231]\n"
      "ds_read_b128 a[236:239], v5 offset:10240\n"
      "v_mfma_f32_16x16x32_bf16 v[228:231], a[188:191], a[204:207], v[228:231]\n"
      "ds_read_b128 a[240:243], v4 offset:12288\n"
      "v_mfma_f32_16x16x32_bf16 v[196:199], a[180:183], a[204:207], v[196:199]\n"
      "ds_read_b128 a[244:247], v5 offset:12288\n"
      "v_mfma_f32_16x16x32_bf16 v[196:199], a[176:179], a[200:203], v[196:199]\n"
      "ds_read_b128 a[248:251], v4 offset:14336\n"
      "v_mfma_f32_16x16x32_bf16 v[164:167], a[168:171], a[200:203], v[164:167]\n"
      "ds_read_b128 a[252:255], v5 offset:14336\n"
      "v_mfma_f32_16x16x32_bf16 v[164:167], a[172:175], a[204:207], v[164:167]\n"
      "v_xor_b32 v4, s55, v4\n"
      "v_xor_b32 v5, s55, v5\n"
      "v_mfma_f32_16x16x32_bf16 v[132:135], a[164:167], a[204:207], v[132:135]\n"
      "v_mfma_f32_16x16x32_bf16 v[132:135], a[160:163], a[200:203], v[132:135]\n"
      "s_waitcnt lgkmcnt(10)\n"
      "v_mfma_f32_16x16x32_bf16 v[136:139], a[160:163], a[208:211], v[136:139]\n"
      "v_mfma_f32_16x16x32_bf16 v[136:139], a[164:167], a[212:215], v[136:139]\n"
      "v_mfma_f32_16x16x32_bf16 v[168:171], a[172:175], a[212:215], v[168:171]\n"
      "v_mfma_f32_16x16x32_bf16 v[168:171], a[168:171], a[208:211], v[168:171]\n"
      "s_add_u32 s90, s71, 6\n"
      "v_mfma_f32_16x16x32_bf16 v[200:203], a[176:179], a[208:211], v[200:203]\n"
      "v_readlane_b32 s86, v22, s90\n"
      "v_mfma_f32_16x16x32_bf16 v[200:203], a[180:183], a[212:215], v[200:203]\n"
      "s_lshr_b32 s86, s86, 13\n"
      "v_mfma_f32_16x16x32_bf16 v[232:235], a[188:191], a[212:215], v[232:235]\n"
      "s_mul_i32 s86, s86, 0x401\n"
      "v_mfma_f32_16x16x32_bf16 v[232:235], a[184:187], a[208:211], v[232:235]\n"
      "s_and_b32 s86, s86, 0xe00780\n"
      "s_waitcnt lgkmcnt(4)\n"
      "v_mfma_f32_16x16x32_bf16 v[236:239], a[184:187], a[216:219], v[236:239]\n"
      "s_or_b32 s64, s86, %43\n"
      "v_mfma_f32_16x16x32_bf16 v[236:239], a[188:191], a[220:223], v[236:239]\n"
      "s_waitcnt vmcnt(16) lgkmcnt(0)\n"
      "s_barrier\n"
      "v_mfma_f32_16x16x32_bf16 v[204:207], a[180:183], a[220:223], v[204:207]\n"
      "s_or_b32 m0, s51, %41\n"
      "v_mfma_f32_16x16x32_bf16 v[204:207], a[176:179], a[216:219], v[204:207]\n"
      "buffer_load_dwordx4 v14, s[40:43], s64 offen offset:0 sc0 sc1 lds\n"
      "v_mfma_f32_16x16x32_bf16 v[172:175], a[168:171], a[216:219], v[172:175]\n"
      "ds_read_b64_tr_b16 a[128:129], v0 offset:1024\n"
      "v_mfma_f32_16x16x32_bf16 v[172:175], a[172:175], a[220:223], v[172:175]\n"
      "ds_read_b64_tr_b16 a[130:131], v0 offset:1536\n"
      "v_mfma_f32_16x16x32_bf16 v[140:143], a[164:167], a[220:223], v[140:143]\n"
      "ds_read_b64_tr_b16 a[132:133], v0 offset:17408\n"
      "v_mfma_f32_16x16x32_bf16 v[140:143], a[160:163], a[216:219], v[140:143]\n"
      "buffer_load_dwordx4 v15, s[40:43], s64 offen offset:1024 sc0 sc1 lds\n"
      "v_mfma_f32_16x16x32_bf16 v[144:147], a[160:163], a[224:227], v[144:147]\n"
      "ds_read_b64_tr_b16 a[134:135], v0 offset:17920\n"
      "v_mfma_f32_16x16x32_bf16 v[144:147], a[164:167], a[228:231], v[144:147]\n"
      "ds_read_b64_tr_b16 a[136:137], v1 offset:1024\n"
      "v_mfma_f32_16x16x32_bf16 v[176:179], a[172:175], a[228:231], v[176:179]\n"
      "ds_read_b64_tr_b16 a[138:139], v1 offset:1536\n"
      "v_mfma_f32_16x16x32_bf16 v[176:179], a[168:171], a[224:227], v[176:179]\n"
      "buffer_load_dwordx4 v16, s[40:43], s64 offen offset:2048 sc0 sc1 lds\n"
      "v_mfma_f32_16x16x32_bf16 v[208:211], a[176:179], a[224:227], v[208:211]\n"
      "ds_read_b64_tr_b16 a[140:141], v1 offset:17408\n"
      "v_mfma_f32_16x16x32_bf16 v[208:211], a[180:183], a[228:231], v[208:211]\n"
      "ds_read_b64_tr_b16 a[142:143], v1 offset:17920\n"
      "v_mfma_f32_16x16x32_bf16 v[240:243], a[188:191], a[228:231], v[240:243]\n"
      "ds_read_b64_tr_b16 a[144:145], v2 offset:1024\n"
      "v_mfma_f32_16x16x32_bf16 v[240:243], a[184:187], a[224:227], v[240:243]\n"
      "buffer_load_dwordx4 v17, s[40:43], s64 offen offset:3072 sc0 sc1 lds\n"
      "v_mfma_f32_16x16x32_bf16 v[244:247], a[184:187], a[232:235], v[244:247]\n"
      "s_bitset1_b32 m0, 12\n"
      "v_mfma_f32_16x16x32_bf16 v[244:247], a[188:191], a[236:239], v[244:247]\n"
      "ds_read_b64_tr_b16 a[146:147], v2 offset:1536\n"
      "v_mfma_f32_16x16x32_bf16 v[212:215], a[180:183], a[236:239], v[212:215]\n"
      "ds_read_b64_tr_b16 a[148:149], v2 offset:17408\n"
      "v_mfma_f32_16x16x32_bf16 v[212:215], a[176:179], a[232:235], v[212:215]\n"
      "buffer_load_dwordx4 v18, s[40:43], s64 offen offset:0 sc0 sc1 lds\n"
      "v_mfma_f32_16x16x32_bf16 v[180:183], a[168:171], a[232:235], v[180:183]\n"
      "ds_read_b128 a[192:195], v4 offset:0\n"
      "v_mfma_f32_16x16x32_bf16 v[180:183], a[172:175], a[236:239], v[180:183]\n"
      "ds_read_b128 a[196:199], v5 offset:0\n"
      "v_mfma_f32_16x16x32_bf16 v[148:151], a[164:167], a[236:239], v[148:151]\n"
      "ds_read_b64_tr_b16 a[150:151], v2 offset:17920\n"
      "v_mfma_f32_16x16x32_bf16 v[148:151], a[160:163], a[232:235], v[148:151]\n"
      "buffer_load_dwordx4 v19, s[40:43], s64 offen offset:1024 sc0 sc1 lds\n"
      "v_mfma_f32_16x16x32_bf16 v[152:155], a[160:163], a[240:243], v[152:155]\n"
      "ds_read_b64_tr_b16 a[152:153], v3 offset:1024\n"
      "v_mfma_f32_16x16x32_bf16 v[152:155], a[164:167], a[244:247], v[152:155]\n"
      "ds_read_b64_tr_b16 a[154:155], v3 offset:1536\n"
      "v_mfma_f32_16x16x32_bf16 v[184:187], a[172:175], a[244:247], v[184:187]\n"
      "ds_read_b64_tr_b16 a[156:157], v3 offset:17408\n"
      "v_mfma_f32_16x16x32_bf16 v[184:187], a[168:171], a[240:243], v[184:187]\n"
      "buffer_load_dwordx4 v20, s[40:43], s64 offen offset:2048 sc0 sc1 lds\n"
      "v_mfma_f32_16x16x32_bf16 v[216:219], a[176:179], a[240:243], v[216:219]\n"
      "ds_read_b128 a[200:203], v4 offset:2048\n"
      "v_mfma_f32_16x16x32_bf16 v[216:219], a[180:183], a[244:247], v[216:219]\n"
      "ds_read_b128 a[204:207], v5 offset:2048\n"
      "v_mfma_f32_16x16x32_bf16 v[248:251], a[188:191], a[244:247], v[248:251]\n"
      "ds_read_b64_tr_b16 a[158:159], v3 offset:17920\n"
      "s_add_u32 s90, s71, 7\n"
      "v_mfma_f32_16x16x32_bf16 v[248:251], a[184:187], a[240:243], v[248:251]\n"
      "buffer_load_dwordx4 v21, s[40:43], s64 offen offset:3072 sc0 sc1 lds\n"
      "v_mfma_f32_16x16x32_bf16 v[252:255], a[184:187], a[248:251], v[252:255]\n"
      "v_readlane_b32 s57, v22, s90\n"
      "v_mfma_f32_16x16x32_bf16 v[252:255], a[188:191], a[252:255], v[252:255]\n"
      "s_add_u32 s65, s57, 1024\n"
      "v_mfma_f32_16x16x32_bf16 v[220:223], a[180:183], a[252:255], v[220:223]\n"
      "s_or_b32 m0, s54, %42\n"
      "v_mfma_f32_16x16x32_bf16 v[220:223], a[176:179], a[248:251], v[220:223]\n"
      "buffer_load_dwordx4 v7, %36, s65 offen offset:0 lds\n"
      "v_mfma_f32_16x16x32_bf16 v[188:191], a[168:171], a[248:251], v[188:191]\n"
      "ds_read_b128 a[208:211], v4 offset:4096\n"
      "v_mfma_f32_16x16x32_bf16 v[188:191], a[172:175], a[252:255], v[188:191]\n"
      "ds_read_b128 a[212:215], v5 offset:4096\n"
      "v_mfma_f32_16x16x32_bf16 v[156:159], a[164:167], a[252:255], v[156:159]\n"
      "v_mfma_f32_16x16x32_bf16 v[156:159], a[160:163], a[248:251], v[156:159]\n"
      "buffer_load_dwordx4 v9, %36, s65 offen offset:2048 lds\n"
      "s_waitcnt lgkmcnt(7)\n"
      "v_mfma_f32_16x16x32_bf16 v[128:131], a[128:131], a[192:195], v[128:131]\n"
      "s_bitset1_b32 m0, 13\n"
      "v_mfma_f32_16x16x32_bf16 v[128:131], a[132:135], a[196:199], v[128:131]\n"
      "ds_read_b128 a[216:219], v4 offset:6144\n"
      "v_mfma_f32_16x16x32_bf16 v[160:163], a[140:143], a[196:199], v[160:163]\n"
      "ds_read_b128 a[220:223], v5 offset:6144\n"
      "v_mfma_f32_16x16x32_bf16 v[160:163], a[136:139], a[192:195], v[160:163]\n"
      "buffer_load_dwordx4 v11, %36, s65 offen offset:0 lds\n"
      "v_mfma_f32_16x16x32_bf16 v[192:195], a[144:147], a[192:195], v[192:195]\n"
      "ds_read_b128 a[224:227], v4 offset:8192\n"
      "s_xor_b32 s55, s53, s50\n"
      "v_mfma_f32_16x16x32_bf16 v[192:195], a[148:151], a[196:199], v[192:195]\n"
      "ds_read_b128 a[228:231], v5 offset:8192\n"
      "s_waitcnt lgkmcnt(6)\n"
      "v_mfma_f32_16x16x32_bf16 v[224:227], a[156:159], a[196:199], v[224:227]\n"
      "ds_read_b128 a[232:235], v4 offset:10240\n"
      "v_mfma_f32_16x16x32_bf16 v[224:227], a[152:155], a[192:195], v[224:227]\n"
      "buffer_load_dwordx4 v13, %36, s65 offen offset:2048 lds\n"
      "v_mfma_f32_16x16x32_bf16 v[228:231], a[152:155], a[200:203], v[228:231]\n"
      "ds_read_b128 a[236:239], v5 offset:10240\n"
      "v_mfma_f32_16x16x32_bf16 v[228:231], a[156:159], a[204:207], v[228:231]\n"
      "ds_read_b128 a[240:243], v4 offset:12288\n"
      "v_mfma_f32_16x16x32_bf16 v[196:199], a[148:151], a[204:207], v[196:199]\n"
      "ds_read_b128 a[244:247], v5 offset:12288\n"
      "v_mfma_f32_16x16x32_bf16 v[196:199], a[144:147], a[200:203], v[196:199]\n"
      "ds_read_b128 a[248:251], v4 offset:14336\n"
      "v_mfma_f32_16x16x32_bf16 v[164:167], a[136:139], a[200:203], v[164:167]\n"
      "v_mov_b32 v28, 0\n"
      "v_mfma_f32_16x16x32_bf16 v[164:167], a[140:143], a[204:207], v[164:167]\n"
      "s_mov_b32 s66, %45\n"
      "v_mfma_f32_16x16x32_bf16 v[132:135], a[132:135], a[204:207], v[132:135]\n"
      "buffer_load_dword v29, v28, s[28:31], s66 offen sc0 sc1\n"
      "v_mfma_f32_16x16x32_bf16 v[132:135], a[128:131], a[200:203], v[132:135]\n"
      "ds_read_b128 a[252:255], v5 offset:14336\n"
      "s_waitcnt lgkmcnt(10)\n"
      "v_mfma_f32_16x16x32_bf16 v[136:139], a[128:131], a[208:211], v[136:139]\n"
      "v_xor_b32 v4, s55, v4\n"
      "v_xor_b32 v5, s55, v5\n"
      "v_mfma_f32_16x16x32_bf16 v[136:139], a[132:135], a[212:215], v[136:139]\n"
      "v_mfma_f32_16x16x32_bf16 v[168:171], a[140:143], a[212:215], v[168:171]\n"
      "v_mfma_f32_16x16x32_bf16 v[168:171], a[136:139], a[208:211], v[168:171]\n"
      "s_add_u32 s90, s71, 7\n"
      "v_mfma_f32_16x16x32_bf16 v[200:203], a[144:147], a[208:211], v[200:203]\n"
      "v_readlane_b32 s86, v22, s90\n"
      "v_mfma_f32_16x16x32_bf16 v[200:203], a[148:151], a[212:215], v[200:203]\n"
      "s_lshr_b32 s86, s86, 13\n"
      "v_mfma_f32_16x16x32_bf16 v[232:235], a[156:159], a[212:215], v[232:235]\n"
      "s_mul_i32 s86, s86, 0x401\n"
      "v_mfma_f32_16x16x32_bf16 v[232:235], a[152:155], a[208:211], v[232:235]\n"
      "s_and_b32 s86, s86, 0xe00780\n"
      "s_waitcnt lgkmcnt(4)\n"
      "v_mfma_f32_16x16x32_bf16 v[236:239], a[152:155], a[216:219], v[236:239]\n"
      "s_or_b32 s64, s86, %43\n"
      "v_mfma_f32_16x16x32_bf16 v[236:239], a[156:159], a[220:223], v[236:239]\n"
      "s_waitcnt vmcnt(17) lgkmcnt(0)\n"
      "s_barrier\n"
      "v_mfma_f32_16x16x32_bf16 v[204:207], a[148:151], a[220:223], v[204:207]\n"
      "s_or_b32 m0, s53, %41\n"
      "v_mfma_f32_16x16x32_bf16 v[204:207], a[144:147], a[216:219], v[204:207]\n"
      "buffer_load_dwordx4 v14, s[40:43], s64 offen offset:0 sc0 sc1 lds\n"
      "v_mfma_f32_16x16x32_bf16 v[172:175], a[136:139], a[216:219], v[172:175]\n"
      "ds_read_b64_tr_b16 a[160:161], v0 offset:0\n"
      "v_mfma_f32_16x16x32_bf16 v[172:175], a[140:143], a[220:223], v[172:175]\n"
      "ds_read_b64_tr_b16 a[162:163], v0 offset:512\n"
      "v_mfma_f32_16x16x32_bf16 v[140:143], a[132:135], a[220:223], v[140:143]\n"
      "ds_read_b64_tr_b16 a[164:165], v0 offset:16384\n"
      "v_mfma_f32_16x16x32_bf16 v[140:143], a[128:131], a[216:219], v[140:143]\n"
      "buffer_load_dwordx4 v15, s[40:43], s64 offen offset:1024 sc0 sc1 lds\n"
      "v_mfma_f32_16x16x32_bf16 v[144:147], a[128:131], a[224:227], v[144:147]\n"
      "ds_read_b64_tr_b16 a[166:167], v0 offset:16896\n"
      "v_mfma_f32_16x16x32_bf16 v[144:147], a[132:135], a[228:231], v[144:147]\n"
      "ds_read_b64_tr_b16 a[168:169], v1 offset:0\n"
      "v_xor_b32 v0, s56, v0\n"
      "v_mfma_f32_16x16x32_bf16 v[176:179], a[140:143], a[228:231], v[176:179]\n"
      "ds_read_b64_tr_b16 a[170:171], v1 offset:512\n"
      "v_mfma_f32_16x16x32_bf16 v[176:179], a[136:139], a[224:227], v[176:179]\n"
      "buffer_load_dwordx4 v16, s[40:43], s64 offen offset:2048 sc0 sc1 lds\n"
      "v_mfma_f32_16x16x32_bf16 v[208:211], a[144:147], a[224:227], v[208:211]\n"
      "ds_read_b64_tr_b16 a[172:173], v1 offset:16384\n"
      "v_mfma_f32_16x16x32_bf16 v[208:211], a[148:151], a[228:231], v[208:211]\n"
      "ds_read_b64_tr_b16 a[174:175], v1 offset:16896\n"
      "v_mfma_f32_16x16x32_bf16 v[240:243], a[156:159], a[228:231], v[240:243]\n"
      "ds_read_b64_tr_b16 a[176:177], v2 offset:0\n"
      "v_xor_b32 v1, s56, v1\n"
      "v_mfma_f32_16x16x32_bf16 v[240:243], a[152:155], a[224:227], v[240:243]\n"
      "buffer_load_dwordx4 v17, s[40:43], s64 offen offset:3072 sc0 sc1 lds\n"
      "v_mfma_f32_16x16x32_bf16 v[244:247], a[152:155], a[232:235], v[244:247]\n"
      "s_bitset1_b32 m0, 12\n"
      "v_mfma_f32_16x16x32_bf16 v[244:247], a[156:159], a[236:239], v[244:247]\n"
      "ds_read_b64_tr_b16 a[178:179], v2 offset:512\n"
      "v_mfma_f32_16x16x32_bf16 v[212:215], a[148:151], a[236:239], v[212:215]\n"
      "ds_read_b64_tr_b16 a[180:181], v2 offset:16384\n"
      "v_mfma_f32_16x16x32_bf16 v[212:215], a[144:147], a[232:235], v[212:215]\n"
      "buffer_load_dwordx4 v18, s[40:43], s64 offen offset:0 sc0 sc1 lds\n"
      "v_mfma_f32_16x16x32_bf16 v[180:183], a[136:139], a[232:235], v[180:183]\n"
      "ds_read_b128 a[192:195], v4 offset:0\n"
      "v_mfma_f32_16x16x32_bf16 v[180:183], a[140:143], a[236:239], v[180:183]\n"
      "ds_read_b128 a[196:199], v5 offset:0\n"
      "v_mfma_f32_16x16x32_bf16 v[148:151], a[132:135], a[236:239], v[148:151]\n"
      "ds_read_b64_tr_b16 a[182:183], v2 offset:16896\n"
      "v_mfma_f32_16x16x32_bf16 v[148:151], a[128:131], a[232:235], v[148:151]\n"
      "buffer_load_dwordx4 v19, s[40:43], s64 offen offset:1024 sc0 sc1 lds\n"
      "v_mfma_f32_16x16x32_bf16 v[152:155], a[128:131], a[240:243], v[152:155]\n"
      "ds_read_b64_tr_b16 a[184:185], v3 offset:0\n"
      "v_xor_b32 v2, s56, v2\n"
      "v_mfma_f32_16x16x32_bf16 v[152:155], a[132:135], a[244:247], v[152:155]\n"
      "ds_read_b64_tr_b16 a[186:187], v3 offset:512\n"
      "v_mfma_f32_16x16x32_bf16 v[184:187], a[140:143], a[244:247], v[184:187]\n"
      "ds_read_b64_tr_b16 a[188:189], v3 offset:16384\n"
      "v_mfma_f32_16x16x32_bf16 v[184:187], a[136:139], a[240:243], v[184:187]\n"
      "buffer_load_dwordx4 v20, s[40:43], s64 offen offset:2048 sc0 sc1 lds\n"
      "v_mfma_f32_16x16x32_bf16 v[216:219], a[144:147], a[240:243], v[216:219]\n"
      "ds_read_b128 a[200:203], v4 offset:2048\n"
      "v_mfma_f32_16x16x32_bf16 v[216:219], a[148:151], a[244:247], v[216:219]\n"
      "ds_read_b128 a[204:207], v5 offset:2048\n"
      "v_mfma_f32_16x16x32_bf16 v[248:251], a[156:159], a[244:247], v[248:251]\n"
      "ds_read_b64_tr_b16 a[190:191], v3 offset:16896\n"
      "s_add_u32 s90, s71, 8\n"
      "v_mfma_f32_16x16x32_bf16 v[248:251], a[152:155], a[240:243], v[248:251]\n"
      "buffer_load_dwordx4 v21, s[40:43], s64 offen offset:3072 sc0 sc1 lds\n"
      "v_mfma_f32_16x16x32_bf16 v[252:255], a[152:155], a[248:251], v[252:255]\n"
      "v_xor_b32 v3, s56, v3\n"
      "v_readlane_b32 s57, v22, s90\n"
      "v_mfma_f32_16x16x32_bf16 v[252:255], a[156:159], a[252:255], v[252:255]\n"
      "s_add_u32 s65, s57, 0\n"
      "v_mfma_f32_16x16x32_bf16 v[220:223], a[148:151], a[252:255], v[220:223]\n"
      "s_or_b32 m0, s52, %42\n"
      "v_mfma_f32_16x16x32_bf16 v[220:223], a[144:147], a[248:251], v[220:223]\n"
      "buffer_load_dwordx4 v7, %36, s65 offen offset:1024 lds\n"
      "v_mfma_f32_16x16x32_bf16 v[188:191], a[136:139], a[248:251], v[188:191]\n"
      "ds_read_b128 a[208:211], v4 offset:4096\n"
      "v_mfma_f32_16x16x32_bf16 v[188:191], a[140:143], a[252:255], v[188:191]\n"
      "ds_read_b128 a[212:215], v5 offset:4096\n"
      "v_mfma_f32_16x16x32_bf16 v[156:159], a[132:135], a[252:255], v[156:159]\n"
      "v_mfma_f32_16x16x32_bf16 v[156:159], a[128:131], a[248:251], v[156:159]\n"
      "buffer_load_dwordx4 v9, %36, s65 offen offset:3072 lds\n"
      "s_waitcnt lgkmcnt(7)\n"
      "v_mfma_f32_16x16x32_bf16 v[128:131], a[160:163], a[192:195], v[128:131]\n"
      "s_bitset1_b32 m0, 13\n"
      "v_mfma_f32_16x16x32_bf16 v[128:131], a[164:167], a[196:199], v[128:131]\n"
      "ds_read_b128 a[216:219], v4 offset:6144\n"
      "v_mfma_f32_16x16x32_bf16 v[160:163], a[172:175], a[196:199], v[160:163]\n"
      "ds_read_b128 a[220:223], v5 offset:6144\n"
      "v_mfma_f32_16x16x32_bf16 v[160:163], a[168:171], a[192:195], v[160:163]\n"
      "buffer_load_dwordx4 v11, %36, s65 offen offset:1024 lds\n"
      "v_mfma_f32_16x16x32_bf16 v[192:195], a[176:179], a[192:195], v[192:195]\n"
      "ds_read_b128 a[224:227], v4 offset:8192\n"
      "v_mfma_f32_16x16x32_bf16 v[192:195], a[180:183], a[196:199], v[192:195]\n"
      "ds_read_b128 a[228:231], v5 offset:8192\n"
      "s_xor_b32 s55, s50, s51\n"
      "s_waitcnt lgkmcnt(6)\n"
      "v_mfma_f32_16x16x32_bf16 v[224:227], a[188:191], a[196:199], v[224:227]\n"
      "ds_read_b128 a[232:235], v4 offset:10240\n"
      "v_mfma_f32_16x16x32_bf16 v[224:227], a[184:187], a[192:195], v[224:227]\n"
      "buffer_load_dwordx4 v13, %36, s65 offen offset:3072 lds\n"
      "v_mfma_f32_16x16x32_bf16 v[228:231], a[184:187], a[200:203], v[228:231]\n"
      "ds_read_b128 a[236:239], v5 offset:10240\n"
      "v_mfma_f32_16x16x32_bf16 v[228:231], a[188:191], a[204:207], v[228:231]\n"
      "ds_read_b128 a[240:243], v4 offset:12288\n"
      "v_mfma_f32_16x16x32_bf16 v[196:199], a[180:183], a[204:207], v[196:199]\n"
      "s_waitcnt vmcnt(12)\n"
      "v_cmp_gt_u32 vcc, s75, v29\n"
      "v_mfma_f32_16x16x32_bf16 v[196:199], a[176:179], a[200:203], v[196:199]\n"
      "ds_read_b128 a[244:247], v5 offset:12288\n"
      "v_mfma_f32_16x16x32_bf16 v[164:167], a[168:171], a[200:203], v[164:167]\n"
      "s_cbranch_vccnz 149f\n"
      "49:\n"
      "v_mfma_f32_16x16x32_bf16 v[164:167], a[172:175], a[204:207], v[164:167]\n"
      "buffer_load_dwordx4 a[32:35], v30, s[44:47], s58 offen offset:0 sc0 sc1\n"
      "v_mfma_f32_16x16x32_bf16 v[132:135], a[164:167], a[204:207], v[132:135]\n"
      "v_mov_b32 v27, 0x40c0000\n"
      "v_mfma_f32_16x16x32_bf16 v[132:135], a[160:163], a[200:203], v[132:135]\n"
      "global_load_dword v29, v27, s[48:49] sc1\n"
      "s_waitcnt lgkmcnt(8)\n"
      "v_mfma_f32_16x16x32_bf16 v[136:139], a[160:163], a[208:211], v[136:139]\n"
      "ds_read_b128 a[248:251], v4 offset:14336\n"
      "v_mfma_f32_16x16x32_bf16 v[136:139], a[164:167], a[212:215], v[136:139]\n"
      "ds_read_b128 a[252:255], v5 offset:14336\n"
      "s_add_u32 s90, s71, 8\n"
      "v_mfma_f32_16x16x32_bf16 v[168:171], a[172:175], a[212:215], v[168:171]\n"
      "v_xor_b32 v4, s55, v4\n"
      "v_xor_b32 v5, s55, v5\n"
      "v_mfma_f32_16x16x32_bf16 v[168:171], a[168:171], a[208:211], v[168:171]\n"
      "buffer_load_dwordx4 a[36:39], v30, s[44:47], s58 offen offset:1024 sc0 sc1\n"
      "v_mfma_f32_16x16x32_bf16 v[200:203], a[176:179], a[208:211], v[200:203]\n"
      "v_readlane_b32 s86, v22, s90\n"
      "v_mfma_f32_16x16x32_bf16 v[200:203], a[180:183], a[212:215], v[200:203]\n"
      "s_lshr_b32 s86, s86, 13\n"
      "v_mfma_f32_16x16x32_bf16 v[232:235], a[188:191], a[212:215], v[232:235]\n"
      "s_mul_i32 s86, s86, 0x401\n"
      "v_mfma_f32_16x16x32_bf16 v[232:235], a[184:187], a[208:211], v[232:235]\n"
      "s_and_b32 s86, s86, 0xe00780\n"
      "s_waitcnt lgkmcnt(4)\n"
      "v_mfma_f32_16x16x32_bf16 v[236:239], a[184:187], a[216:219], v[236:239]\n"
      "s_or_b32 s64, s86, %43\n"
      "v_mfma_f32_16x16x32_bf16 v[236:239], a[188:191], a[220:223], v[236:239]\n"
      "s_waitcnt lgkmcnt(0)\n"
      "s_barrier\n"
      "v_mfma_f32_16x16x32_bf16 v[204:207], a[180:183], a[220:223], v[204:207]\n"
      "s_or_b32 m0, s50, %41\n"
      "v_mfma_f32_16x16x32_bf16 v[204:207], a[176:179], a[216:219], v[204:207]\n"
      "buffer_load_dwordx4 v14, s[40:43], s64 offen offset:0 sc0 sc1 lds\n"
      "v_mfma_f32_16x16x32_bf16 v[172:175], a[168:171], a[216:219], v[172:175]\n"
      "ds_read_b64_tr_b16 a[128:129], v0 offset:1024\n"
      "v_mfma_f32_16x16x32_bf16 v[172:175], a[172:175], a[220:223], v[172:175]\n"
      "buffer_load_dwordx4 a[40:43], v30, s[44:47], s58 offen offset:2048 sc0 sc1\n"
      "v_mfma_f32_16x16x32_bf16 v[140:143], a[164:167], a[220:223], v[140:143]\n"
      "ds_read_b64_tr_b16 a[130:131], v0 offset:1536\n"
      "v_mfma_f32_16x16x32_bf16 v[140:143], a[160:163], a[216:219], v[140:143]\n"
      "buffer_load_dwordx4 v15, s[40:43], s64 offen offset:1024 sc0 sc1 lds\n"
      "v_mfma_f32_16x16x32_bf16 v[144:147], a[160:163], a[224:227], v[144:147]\n"
      "ds_read_b64_tr_b16 a[132:133], v0 offset:17408\n"
      "v_mfma_f32_16x16x32_bf16 v[144:147], a[164:167], a[228:231], v[144:147]\n"
      "ds_read_b64_tr_b16 a[134:135], v0 offset:17920\n"
      "v_mfma_f32_16x16x32_bf16 v[176:179], a[172:175], a[228:231], v[176:179]\n"
      "ds_read_b64_tr_b16 a[136:137], v1 offset:1024\n"
      "v_mfma_f32_16x16x32_bf16 v[176:179], a[168:171], a[224:227], v[176:179]\n"
      "buffer_load_dwordx4 v16, s[40:43], s64 offen offset:2048 sc0 sc1 lds\n"
      "v_mfma_f32_16x16x32_bf16 v[208:211], a[176:179], a[224:227], v[208:211]\n"
      "ds_read_b64_tr_b16 a[138:139], v1 offset:1536\n"
      "v_mfma_f32_16x16x32_bf16 v[208:211], a[180:183], a[228:231], v[208:211]\n"
      "buffer_load_dwordx4 a[44:47], v30, s[44:47], s58 offen offset:3072 sc0 sc1\n"
      "v_mfma_f32_16x16x32_bf16 v[240:243], a[188:191], a[228:231], v[240:243]\n"
      "s_add_u32 s58, s58, 0x1000\n"
      "v_mfma_f32_16x16x32_bf16 v[240:243], a[184:187], a[224:227], v[240:243]\n"
      "buffer_load_dwordx4 v17, s[40:43], s64 offen offset:3072 sc0 sc1 lds\n"
      "v_mfma_f32_16x16x32_bf16 v[244:247], a[184:187], a[232:235], v[244:247]\n"
      "s_bitset1_b32 m0, 12\n"
      "v_mfma_f32_16x16x32_bf16 v[244:247], a[188:191], a[236:239], v[244:247]\n"
      "ds_read_b64_tr_b16 a[140:141], v1 offset:17408\n"
      "v_mfma_f32_16x16x32_bf16 v[212:215], a[180:183], a[236:239], v[212:215]\n"
      "ds_read_b64_tr_b16 a[142:143], v1 offset:17920\n"
      "v_mfma_f32_16x16x32_bf16 v[212:215], a[176:179], a[232:235], v[212:215]\n"
      "buffer_load_dwordx4 v18, s[40:43], s64 offen offset:0 sc0 sc1 lds\n"
      "v_mfma_f32_16x16x32_bf16 v[180:183], a[168:171], a[232:235], v[180:183]\n"
      "ds_read_b128 a[192:195], v4 offset:0\n"
      "v_mfma_f32_16x16x32_bf16 v[180:183], a[172:175], a[236:239], v[180:183]\n"
      "buffer_load_dwordx4 a[48:51], v30, s[44:47], s58 offen offset:0 sc0 sc1\n"
      "v_mfma_f32_16x16x32_bf16 v[148:151], a[164:167], a[236:239], v[148:151]\n"
      "ds_read_b128 a[196:199], v5 offset:0\n"
      "v_mfma_f32_16x16x32_bf16 v[148:151], a[160:163], a[232:235], v[148:151]\n"
      "buffer_load_dwordx4 v19, s[40:43], s64 offen offset:1024 sc0 sc1 lds\n"
      "v_mfma_f32_16x16x32_bf16 v[152:155], a[160:163], a[240:243], v[152:155]\n"
      "ds_read_b64_tr_b16 a[144:145], v2 offset:1024\n"
      "ds_read_b64_tr_b16 a[156:157], v3 offset:17408\n"
      "v_mfma_f32_16x16x32_bf16 v[152:155], a[164:167], a[244:247], v[152:155]\n"
      "ds_read_b64_tr_b16 a[146:147], v2 offset:1536\n"
      "ds_read_b64_tr_b16 a[158:159], v3 offset:17920\n"
      "v_mfma_f32_16x16x32_bf16 v[184:187], a[172:175], a[244:247], v[184:187]\n"
      "ds_read_b64_tr_b16 a[148:149], v2 offset:17408\n"
      "v_mfma_f32_16x16x32_bf16 v[184:187], a[168:171], a[240:243], v[184:187]\n"
      "buffer_load_dwordx4 v20, s[40:43], s64 offen offset:2048 sc0 sc1 lds\n"
      "v_mfma_f32_16x16x32_bf16 v[216:219], a[176:179], a[240:243], v[216:219]\n"
      "ds_read_b128 a[200:203], v4 offset:2048\n"
      "v_mfma_f32_16x16x32_bf16 v[216:219], a[180:183], a[244:247], v[216:219]\n"
      "buffer_load_dwordx4 a[52:55], v30, s[44:47], s58 offen offset:1024 sc0 sc1\n"
      "v_mfma_f32_16x16x32_bf16 v[248:251], a[188:191], a[244:247], v[248:251]\n"
      "ds_read_b128 a[204:207], v5 offset:2048\n"
      "v_mfma_f32_16x16x32_bf16 v[248:251], a[184:187], a[240:243], v[248:251]\n"
      "buffer_load_dwordx4 v21, s[40:43], s64 offen offset:3072 sc0 sc1 lds\n"
      "v_mfma_f32_16x16x32_bf16 v[252:255], a[184:187], a[248:251], v[252:255]\n"
      "ds_read_b64_tr_b16 a[150:151], v2 offset:17920\n"
      "v_mfma_f32_16x16x32_bf16 v[252:255], a[188:191], a[252:255], v[252:255]\n"
      "ds_read_b64_tr_b16 a[152:153], v3 offset:1024\n"
      "v_mfma_f32_16x16x32_bf16 v[220:223], a[180:183], a[252:255], v[220:223]\n"
      "ds_read_b64_tr_b16 a[154:155], v3 offset:1536\n"
      "v_mfma_f32_16x16x32_bf16 v[220:223], a[176:179], a[248:251], v[220:223]\n"
      "buffer_load_dwordx4 a[56:59], v30, s[44:47], s58 offen offset:2048 sc0 sc1\n"
      "v_mfma_f32_16x16x32_bf16 v[188:191], a[168:171], a[248:251], v[188:191]\n"
      "ds_read_b128 a[208:211], v4 offset:4096\n"
      "v_mfma_f32_16x16x32_bf16 v[188:191], a[172:175], a[252:255], v[188:191]\n"
      "ds_read_b128 a[212:215], v5 offset:4096\n"
      "v_mfma_f32_16x16x32_bf16 v[156:159], a[164:167], a[252:255], v[156:159]\n"
      "v_mfma_f32_16x16x32_bf16 v[156:159], a[160:163], a[248:251], v[156:159]\n"
      "s_waitcnt lgkmcnt(10)\n"
      "v_mfma_f32_16x16x32_bf16 v[128:131], a[128:131], a[192:195], v[128:131]\n"
      "ds_read_b128 a[216:219], v4 offset:6144\n"
      "s_xor_b32 s55, s51, s53\n"
      "v_mfma_f32_16x16x32_bf16 v[128:131], a[132:135], a[196:199], v[128:131]\n"
      "buffer_load_dwordx4 a[60:63], v30, s[44:47], s58 offen offset:3072 sc0 sc1\n"
      "v_mfma_f32_16x16x32_bf16 v[160:163], a[140:143], a[196:199], v[160:163]\n"
      "s_add_u32 s58, s58, 0x1000\n"
      "v_mfma_f32_16x16x32_bf16 v[160:163], a[136:139], a[192:195], v[160:163]\n"
      "ds_read_b128 a[220:223], v5 offset:6144\n"
      "s_waitcnt lgkmcnt(6)\n"
      "v_mfma_f32_16x16x32_bf16 v[192:195], a[144:147], a[192:195], v[192:195]\n"
      "ds_read_b128 a[224:227], v4 offset:8192\n"
      "v_mfma_f32_16x16x32_bf16 v[192:195], a[148:151], a[196:199], v[192:195]\n"
      "ds_read_b128 a[228:231], v5 offset:8192\n"
      "s_waitcnt lgkmcnt(6)\n"
      "v_mfma_f32_16x16x32_bf16 v[224:227], a[156:159], a[196:199], v[224:227]\n"
      "ds_read_b128 a[232:235], v4 offset:10240\n"
      "v_mfma_f32_16x16x32_bf16 v[224:227], a[152:155], a[192:195], v[224:227]\n"
      "buffer_load_dwordx4 a[64:67], v30, s[44:47], s58 offen offset:0 sc0 sc1\n"
      "v_mfma_f32_16x16x32_bf16 v[228:231], a[152:155], a[200:203], v[228:231]\n"
      "ds_read_b128 a[236:239], v5 offset:10240\n"
      "v_mfma_f32_16x16x32_bf16 v[228:231], a[156:159], a[204:207], v[228:231]\n"
      "ds_read_b128 a[240:243], v4 offset:12288\n"
      "v_mfma_f32_16x16x32_bf16 v[196:199], a[148:151], a[204:207], v[196:199]\n"
      "ds_read_b128 a[244:247], v5 offset:12288\n"
      "v_mfma_f32_16x16x32_bf16 v[196:199], a[144:147], a[200:203], v[196:199]\n"
      "ds_read_b128 a[248:251], v4 offset:14336\n"
      "v_mfma_f32_16x16x32_bf16 v[164:167], a[136:139], a[200:203], v[164:167]\n"
      "ds_read_b128 a[252:255], v5 offset:14336\n"
      "v_mfma_f32_16x16x32_bf16 v[164:167], a[140:143], a[204:207], v[164:167]\n"
      "buffer_load_dwordx4 a[68:71], v30, s[44:47], s58 offen offset:1024 sc0 sc1\n"
      "v_mfma_f32_16x16x32_bf16 v[132:135], a[132:135], a[204:207], v[132:135]\n"
      "v_xor_b32 v4, s55, v4\n"
      "v_xor_b32 v5, s55, v5\n"
      "v_mfma_f32_16x16x32_bf16 v[132:135], a[128:131], a[200:203], v[132:135]\n"
      "s_waitcnt lgkmcnt(9)\n"
      "v_mfma_f32_16x16x32_bf16 v[136:139], a[128:131], a[208:211], v[136:139]\n"
      "v_mfma_f32_16x16x32_bf16 v[136:139], a[132:135], a[212:215], v[136:139]\n"
      "v_mfma_f32_16x16x32_bf16 v[168:171], a[140:143], a[212:215], v[168:171]\n"
      "v_mfma_f32_16x16x32_bf16 v[168:171], a[136:139], a[208:211], v[168:171]\n"
      "buffer_load_dwordx4 a[72:75], v30, s[44:47], s58 offen offset:2048 sc0 sc1\n"
      "v_mfma_f32_16x16x32_bf16 v[200:203], a[144:147], a[208:211], v[200:203]\n"
      "v_mfma_f32_16x16x32_bf16 v[200:203], a[148:151], a[212:215], v[200:203]\n"
      "v_mfma_f32_16x16x32_bf16 v[232:235], a[156:159], a[212:215], v[232:235]\n"
      "v_mfma_f32_16x16x32_bf16 v[232:235], a[152:155], a[208:211], v[232:235]\n"
      "s_waitcnt lgkmcnt(4)\n"
      "v_mfma_f32_16x16x32_bf16 v[236:239], a[152:155], a[216:219], v[236:239]\n"
      "v_mfma_f32_16x16x32_bf16 v[236:239], a[156:159], a[220:223], v[236:239]\n"
      "s_waitcnt vmcnt(20)\n"
      "s_barrier\n"
      "v_mfma_f32_16x16x32_bf16 v[204:207], a[148:151], a[220:223], v[204:207]\n"
      "ds_read_b128 a[192:195], v4 offset:0\n"
      "v_mfma_f32_16x16x32_bf16 v[204:207], a[144:147], a[216:219], v[204:207]\n"
      "buffer_load_dwordx4 a[76:79], v30, s[44:47], s58 offen offset:3072 sc0 sc1\n"
      "v_mfma_f32_16x16x32_bf16 v[172:175], a[136:139], a[216:219], v[172:175]\n"
      "s_add_u32 s58, s58, 0x1000\n"
      "v_mfma_f32_16x16x32_bf16 v[172:175], a[140:143], a[220:223], v[172:175]\n"
      "ds_read_b128 a[196:199], v5 offset:0\n"
      "v_mfma_f32_16x16x32_bf16 v[140:143], a[132:135], a[220:223], v[140:143]\n"
      "ds_read_b128 a[200:203], v4 offset:2048\n"
      "v_mfma_f32_16x16x32_bf16 v[140:143], a[128:131], a[216:219], v[140:143]\n"
      "ds_read_b128 a[204:207], v5 offset:2048\n"
      "v_mfma_f32_16x16x32_bf16 v[144:147], a[128:131], a[224:227], v[144:147]\n"
      "ds_read_b128 a[208:211], v4 offset:4096\n"
      "v_mfma_f32_16x16x32_bf16 v[144:147], a[132:135], a[228:231], v[144:147]\n"
      "buffer_load_dwordx4 a[80:83], v30, s[44:47], s58 offen offset:0 sc0 sc1\n"
      "v_mfma_f32_16x16x32_bf16 v[176:179], a[140:143], a[228:231], v[176:179]\n"
      "ds_read_b128 a[212:215], v5 offset:4096\n"
      "v_mfma_f32_16x16x32_bf16 v[176:179], a[136:139], a[224:227], v[176:179]\n"
      "ds_read_b128 a[216:219], v4 offset:6144\n"
      "v_mfma_f32_16x16x32_bf16 v[208:211], a[144:147], a[224:227], v[208:211]\n"
      "ds_read_b128 a[220:223], v5 offset:6144\n"
      "v_mfma_f32_16x16x32_bf16 v[208:211], a[148:151], a[228:231], v[208:211]\n"
      "ds_read_b64_tr_b16 a[160:161], v0 offset:0\n"
      "v_mfma_f32_16x16x32_bf16 v[240:243], a[156:159], a[228:231], v[240:243]\n"
      "ds_read_b64_tr_b16 a[162:163], v0 offset:512\n"
      "v_mfma_f32_16x16x32_bf16 v[240:243], a[152:155], a[224:227], v[240:243]\n"
      "buffer_load_dwordx4 a[84:87], v30, s[44:47], s58 offen offset:1024 sc0 sc1\n"
      "v_mfma_f32_16x16x32_bf16 v[244:247], a[152:155], a[232:235], v[244:247]\n"
      "ds_read_b64_tr_b16 a[164:165], v0 offset:16384\n"
      "v_mfma_f32_16x16x32_bf16 v[244:247], a[156:159], a[236:239], v[244:247]\n"
      "ds_read_b64_tr_b16 a[166:167], v0 offset:16896\n"
      "v_mfma_f32_16x16x32_bf16 v[212:215], a[148:151], a[236:239], v[212:215]\n"
      "ds_read_b64_tr_b16 a[168:169], v1 offset:0\n"
      "v_xor_b32 v0, s56, v0\n"
      "v_mfma_f32_16x16x32_bf16 v[212:215], a[144:147], a[232:235], v[212:215]\n"
      "ds_read_b64_tr_b16 a[170:171], v1 offset:512\n"
      "v_mfma_f32_16x16x32_bf16 v[180:183], a[136:139], a[232:235], v[180:183]\n"
      "ds_read_b64_tr_b16 a[172:173], v1 offset:16384\n"
      "v_mfma_f32_16x16x32_bf16 v[180:183], a[140:143], a[236:239], v[180:183]\n"
      "buffer_load_dwordx4 a[88:91], v30, s[44:47], s58 offen offset:2048 sc0 sc1\n"
      "v_mfma_f32_16x16x32_bf16 v[148:151], a[132:135], a[236:239], v[148:151]\n"
      "ds_read_b64_tr_b16 a[174:175], v1 offset:16896\n"
      "v_mfma_f32_16x16x32_bf16 v[148:151], a[128:131], a[232:235], v[148:151]\n"
      "ds_read_b64_tr_b16 a[176:177], v2 offset:0\n"
      "v_xor_b32 v1, s56, v1\n"
      "s_waitcnt lgkmcnt(12)\n"
      "v_mfma_f32_16x16x32_bf16 v[152:155], a[128:131], a[240:243], v[152:155]\n"
      "ds_read_b64_tr_b16 a[178:179], v2 offset:512\n"
      "ds_read_b64_tr_b16 a[190:191], v3 offset:16896\n"
      "v_mfma_f32_16x16x32_bf16 v[152:155], a[132:135], a[244:247], v[152:155]\n"
      "ds_read_b64_tr_b16 a[180:181], v2 offset:16384\n"
      "v_mfma_f32_16x16x32_bf16 v[184:187], a[140:143], a[244:247], v[184:187]\n"
      "ds_read_b64_tr_b16 a[182:183], v2 offset:16896\n"
      "s_xor_b32 s55, s53, s50\n"
      "v_mfma_f32_16x16x32_bf16 v[184:187], a[136:139], a[240:243], v[184:187]\n"
      "buffer_load_dwordx4 a[92:95], v30, s[44:47], s58 offen offset:3072 sc0 sc1\n"
      "v_mfma_f32_16x16x32_bf16 v[216:219], a[144:147], a[240:243], v[216:219]\n"
      "ds_read_b128 a[0:3], v4 offset:8192\n"
      "v_xor_b32 v2, s56, v2\n"
      "v_mfma_f32_16x16x32_bf16 v[216:219], a[148:151], a[244:247], v[216:219]\n"
      "ds_read_b128 a[4:7], v5 offset:8192\n"
      "v_mfma_f32_16x16x32_bf16 v[248:251], a[156:159], a[244:247], v[248:251]\n"
      "ds_read_b64_tr_b16 a[184:185], v3 offset:0\n"
      "v_mfma_f32_16x16x32_bf16 v[248:251], a[152:155], a[240:243], v[248:251]\n"
      "ds_read_b128 a[8:11], v4 offset:10240\n"
      "v_mfma_f32_16x16x32_bf16 v[252:255], a[152:155], a[248:251], v[252:255]\n"
      "ds_read_b128 a[12:15], v5 offset:10240\n"
      "v_mfma_f32_16x16x32_bf16 v[252:255], a[156:159], a[252:255], v[252:255]\n"
      "ds_read_b64_tr_b16 a[186:187], v3 offset:512\n"
      "v_mfma_f32_16x16x32_bf16 v[220:223], a[148:151], a[252:255], v[220:223]\n"
      "ds_read_b128 a[16:19], v4 offset:12288\n"
      "v_mfma_f32_16x16x32_bf16 v[220:223], a[144:147], a[248:251], v[220:223]\n"
      "ds_read_b128 a[20:23], v5 offset:12288\n"
      "v_mfma_f32_16x16x32_bf16 v[188:191], a[136:139], a[248:251], v[188:191]\n"
      "ds_read_b64_tr_b16 a[188:189], v3 offset:16384\n"
      "v_mfma_f32_16x16x32_bf16 v[188:191], a[140:143], a[252:255], v[188:191]\n"
      "ds_read_b128 a[24:27], v4 offset:14336\n"
      "v_xor_b32 v3, s56, v3\n"
      "v_mfma_f32_16x16x32_bf16 v[156:159], a[132:135], a[252:255], v[156:159]\n"
      "ds_read_b128 a[28:31], v5 offset:14336\n"
      "v_mfma_f32_16x16x32_bf16 v[156:159], a[128:131], a[248:251], v[156:159]\n"
      "v_xor_b32 v4, s55, v4\n"
      "v_xor_b32 v5, s55, v5\n"
      "s_waitcnt lgkmcnt(13)\n"
      "v_mfma_f32_16x16x32_bf16 v[128:131], a[160:163], a[192:195], v[128:131]\n"
      "v_mfma_f32_16x16x32_bf16 v[128:131], a[164:167], a[196:199], v[128:131]\n"
      "ds_read_b64_tr_b16 a[128:129], v0 offset:1024\n"
      "v_mfma_f32_16x16x32_bf16 v[160:163], a[172:175], a[196:199], v[160:163]\n"
      "ds_read_b64_tr_b16 a[130:131], v0 offset:1536\n"
      "v_mfma_f32_16x16x32_bf16 v[160:163], a[168:171], a[192:195], v[160:163]\n"
      "ds_read_b64_tr_b16 a[132:133], v0 offset:17408\n"
      "s_waitcnt lgkmcnt(13)\n"
      "v_mfma_f32_16x16x32_bf16 v[192:195], a[176:179], a[192:195], v[192:195]\n"
      "ds_read_b64_tr_b16 a[134:135], v0 offset:17920\n"
      "v_mfma_f32_16x16x32_bf16 v[192:195], a[180:183], a[196:199], v[192:195]\n"
      "ds_read_b64_tr_b16 a[136:137], v1 offset:1024\n"
      "s_waitcnt lgkmcnt(7)\n"
      "v_mfma_f32_16x16x32_bf16 v[224:227], a[188:191], a[196:199], v[224:227]\n"
      "ds_read_b64_tr_b16 a[138:139], v1 offset:1536\n"
      "v_mfma_f32_16x16x32_bf16 v[224:227], a[184:187], a[192:195], v[224:227]\n"
      "ds_read_b64_tr_b16 a[140:141], v1 offset:17408\n"
      "v_mfma_f32_16x16x32_bf16 v[228:231], a[184:187], a[200:203], v[228:231]\n"
      "ds_read_b64_tr_b16 a[142:143], v1 offset:17920\n"
      "v_mfma_f32_16x16x32_bf16 v[228:231], a[188:191], a[204:207], v[228:231]\n"
      "s_waitcnt vmcnt(24)\n"
      "v_mfma_f32_4x4x4_16b_bf16 v[128:131], v[24:25], a[32:33], v[128:131]\n"
      "v_mfma_f32_16x16x32_bf16 v[196:199], a[180:183], a[204:207], v[196:199]\n"
      "ds_read_b64_tr_b16 a[144:145], v2 offset:1024\n"
      "v_mfma_f32_16x16x32_bf16 v[196:199], a[176:179], a[200:203], v[196:199]\n"
      "ds_read_b64_tr_b16 a[146:147], v2 offset:1536\n"
      "v_mfma_f32_16x16x32_bf16 v[164:167], a[168:171], a[200:203], v[164:167]\n"
      "ds_read_b64_tr_b16 a[148:149], v2 offset:17408\n"
      "v_mfma_f32_16x16x32_bf16 v[164:167], a[172:175], a[204:207], v[164:167]\n"
      "ds_read_b64_tr_b16 a[150:151], v2 offset:17920\n"
      "v_mfma_f32_16x16x32_bf16 v[132:135], a[164:167], a[204:207], v[132:135]\n"
      "ds_read_b64_tr_b16 a[152:153], v3 offset:1024\n"
      "v_mfma_f32_16x16x32_bf16 v[132:135], a[160:163], a[200:203], v[132:135]\n"
      "ds_read_b64_tr_b16 a[154:155], v3 offset:1536\n"
      "v_mfma_f32_16x16x32_bf16 v[136:139], a[160:163], a[208:211], v[136:139]\n"
      "ds_read_b64_tr_b16 a[156:157], v3 offset:17408\n"
      "v_mfma_f32_16x16x32_bf16 v[136:139], a[164:167], a[212:215], v[136:139]\n"
      "s_waitcnt vmcnt(10)\n"
      "s_barrier\n"
      "v_mfma_f32_16x16x32_bf16 v[168:171], a[172:175], a[212:215], v[168:171]\n"
      "ds_read_b128 a[224:227], v4 offset:0\n"
      "v_mfma_f32_16x16x32_bf16 v[168:171], a[168:171], a[208:211], v[168:171]\n"
      "ds_read_b128 a[228:231], v5 offset:0\n"
      "v_mfma_f32_16x16x32_bf16 v[200:203], a[176:179], a[208:211], v[200:203]\n"
      "ds_read_b64_tr_b16 a[158:159], v3 offset:17920\n"
      "v_cmp_eq_u32 vcc, %31, v29\n"
      "v_mfma_f32_16x16x32_bf16 v[200:203], a[180:183], a[212:215], v[200:203]\n"
      "v_mfma_f32_4x4x4_16b_bf16 v[160:163], v[24:25], a[34:35], v[160:163]\n"
      "v_mfma_f32_16x16x32_bf16 v[232:235], a[188:191], a[212:215], v[232:235]\n"
      "ds_read_b128 a[232:235], v4 offset:2048\n"
      "s_cmp_lg_u64 vcc, 0\n"
      "s_cselect_b32 s59, 0x7fc07fc0, 0\n"
      "v_mfma_f32_16x16x32_bf16 v[232:235], a[184:187], a[208:211], v[232:235]\n"
      "ds_read_b128 a[236:239], v5 offset:2048\n"
      "s_mov_b32 vcc_lo, 0xaaaaaaaa\n"
      "v_mfma_f32_16x16x32_bf16 v[236:239], a[184:187], a[216:219], v[236:239]\n"
      "v_mfma_f32_4x4x4_16b_bf16 v[192:195], v[24:25], a[36:37], v[192:195]\n"
      "s_mov_b32 vcc_hi, vcc_lo\n"
      "v_mfma_f32_16x16x32_bf16 v[236:239], a[188:191], a[220:223], v[236:239]\n"
      "v_mfma_f32_4x4x4_16b_bf16 v[224:227], v[24:25], a[38:39], v[224:227]\n"
      "v_mfma_f32_16x16x32_bf16 v[204:207], a[180:183], a[220:223], v[204:207]\n"
      "ds_read_b128 a[240:243], v4 offset:4096\n"
      "v_mfma_f32_16x16x32_bf16 v[204:207], a[176:179], a[216:219], v[204:207]\n"
      "ds_read_b128 a[244:247], v5 offset:4096\n"
      "v_mfma_f32_16x16x32_bf16 v[172:175], a[168:171], a[216:219], v[172:175]\n"
      "v_mfma_f32_4x4x4_16b_bf16 v[132:135], v[24:25], a[40:41], v[132:135]\n"
      "v_mfma_f32_16x16x32_bf16 v[172:175], a[172:175], a[220:223], v[172:175]\n"
      "v_mfma_f32_4x4x4_16b_bf16 v[164:167], v[24:25], a[42:43], v[164:167]\n"
      "v_mfma_f32_16x16x32_bf16 v[140:143], a[164:167], a[220:223], v[140:143]\n"
      "ds_read_b128 a[248:251], v4 offset:6144\n"
      "v_mfma_f32_16x16x32_bf16 v[140:143], a[160:163], a[216:219], v[140:143]\n"
      "ds_read_b128 a[252:255], v5 offset:6144\n"
      "s_waitcnt lgkmcnt(7)\n"
      "v_mfma_f32_16x16x32_bf16 v[128:131], a[128:131], a[224:227], v[128:131]\n"
      "v_mfma_f32_4x4x4_16b_bf16 v[196:199], v[24:25], a[44:45], v[196:199]\n"
      "v_mfma_f32_16x16x32_bf16 v[128:131], a[132:135], a[228:231], v[128:131]\n"
      "v_mfma_f32_4x4x4_16b_bf16 v[228:231], v[24:25], a[46:47], v[228:231]\n"
      "v_mfma_f32_16x16x32_bf16 v[160:163], a[140:143], a[228:231], v[160:163]\n"
      "v_mfma_f32_4x4x4_16b_bf16 v[136:139], v[24:25], a[48:49], v[136:139]\n"
      "v_mfma_f32_16x16x32_bf16 v[160:163], a[136:139], a[224:227], v[160:163]\n"
      "v_mfma_f32_4x4x4_16b_bf16 v[168:171], v[24:25], a[50:51], v[168:171]\n"
      "v_mfma_f32_16x16x32_bf16 v[192:195], a[144:147], a[224:227], v[192:195]\n"
      "v_mfma_f32_4x4x4_16b_bf16 v[200:203], v[24:25], a[52:53], v[200:203]\n"
      "v_mfma_f32_16x16x32_bf16 v[192:195], a[148:151], a[228:231], v[192:195]\n"
      "v_mfma_f32_4x4x4_16b_bf16 v[232:235], v[24:25], a[54:55], v[232:235]\n"
      "s_waitcnt lgkmcnt(5)\n"
      "v_mfma_f32_16x16x32_bf16 v[224:227], a[156:159], a[228:231], v[224:227]\n"
      "s_waitcnt vmcnt(9)\n"
      "v_mfma_f32_4x4x4_16b_bf16 v[140:143], v[24:25], a[56:57], v[140:143]\n"
      "v_cvt_pk_bf16_f32 v32, v128, v129\n"
      "v_cvt_pk_bf16_f32 v33, v130, v131\n"
      "v_mfma_f32_16x16x32_bf16 v[224:227], a[152:155], a[224:227], v[224:227]\n"
      "v_mfma_f32_4x4x4_16b_bf16 v[172:175], v[24:25], a[58:59], v[172:175]\n"
      "v_pk_max_i16 v32, v32, s59\n"
      "v_pk_max_i16 v33, v33, s59\n"
      "s_waitcnt lgkmcnt(4)\n"
      "v_mfma_f32_16x16x32_bf16 v[228:231], a[152:155], a[232:235], v[228:231]\n"
      "s_waitcnt vmcnt(8)\n"
      "v_mfma_f32_4x4x4_16b_bf16 v[204:207], v[24:25], a[60:61], v[204:207]\n"
      "v_cvt_pk_bf16_f32 v34, v160, v161\n"
      "v_cvt_pk_bf16_f32 v35, v162, v163\n"
      "v_mfma_f32_16x16x32_bf16 v[228:231], a[156:159], a[236:239], v[228:231]\n"
      "v_mfma_f32_4x4x4_16b_bf16 v[236:239], v[24:25], a[62:63], v[236:239]\n"
      "v_pk_max_i16 v34, v34, s59\n"
      "v_pk_max_i16 v35, v35, s59\n"
      "v_mfma_f32_16x16x32_bf16 v[196:199], a[148:151], a[236:239], v[196:199]\n"
      "s_waitcnt vmcnt(7)\n"
      "v_mfma_f32_4x4x4_16b_bf16 v[144:147], v[24:25], a[64:65], v[144:147]\n"
      "v_cvt_pk_bf16_f32 v64, v192, v193\n"
      "v_cvt_pk_bf16_f32 v65, v194, v195\n"
      "v_mfma_f32_16x16x32_bf16 v[196:199], a[144:147], a[232:235], v[196:199]\n"
      "v_mfma_f32_4x4x4_16b_bf16 v[176:179], v[24:25], a[66:67], v[176:179]\n"
      "v_permlane16_swap_b32 v32, v34\n"
      "v_permlane16_swap_b32 v33, v35\n"
      "v_mfma_f32_16x16x32_bf16 v[164:167], a[136:139], a[232:235], v[164:167]\n"
      "ds_read_b128 a[192:195], v4 offset:8192\n"
      "v_pk_max_i16 v64, v64, s59\n"
      "v_pk_max_i16 v65, v65, s59\n"
      "v_mfma_f32_16x16x32_bf16 v[164:167], a[140:143], a[236:239], v[164:167]\n"
      "ds_read_b128 a[196:199], v5 offset:8192\n"
      "v_cvt_pk_bf16_f32 v66, v224, v225\n"
      "v_cvt_pk_bf16_f32 v67, v226, v227\n"
      "v_mfma_f32_16x16x32_bf16 v[132:135], a[132:135], a[236:239], v[132:135]\n"
      "s_waitcnt vmcnt(6)\n"
      "v_mfma_f32_4x4x4_16b_bf16 v[208:211], v[24:25], a[68:69], v[208:211]\n"
      "v_pk_max_i16 v66, v66, s59\n"
      "v_pk_max_i16 v67, v67, s59\n"
      "v_mfma_f32_16x16x32_bf16 v[132:135], a[128:131], a[232:235], v[132:135]\n"
      "v_mfma_f32_4x4x4_16b_bf16 v[240:243], v[24:25], a[70:71], v[240:243]\n"
      "s_mov_b32 s90, 0x4000\n"
      "s_mov_b32 s91, 0x0\n"
      "s_waitcnt lgkmcnt(4)\n"
      "v_mfma_f32_16x16x32_bf16 v[136:139], a[128:131], a[240:243], v[136:139]\n"
      "s_waitcnt vmcnt(5)\n"
      "v_mfma_f32_4x4x4_16b_bf16 v[148:151], v[24:25], a[72:73], v[148:151]\n"
      "v_permlane16_swap_b32 v64, v66\n"
      "v_permlane16_swap_b32 v65, v67\n"
      "v_mfma_f32_16x16x32_bf16 v[136:139], a[132:135], a[244:247], v[136:139]\n"
      "v_mfma_f32_4x4x4_16b_bf16 v[180:183], v[24:25], a[74:75], v[180:183]\n"
      "v_cndmask_b32_dpp v96, v32, v64, vcc quad_perm:[1,0,3,2] row_mask:0xf bank_mask:0xf\n"
      "v_cndmask_b32_dpp v98, v34, v66, vcc quad_perm:[1,0,3,2] row_mask:0xf bank_mask:0xf\n"
      "v_mfma_f32_16x16x32_bf16 v[168:171], a[140:143], a[244:247], v[168:171]\n"
      "v_cndmask_b32_dpp v97, v33, v65, vcc quad_perm:[1,0,3,2] row_mask:0xf bank_mask:0xf\n"
      "v_cndmask_b32_dpp v99, v35, v67, vcc quad_perm:[1,0,3,2] row_mask:0xf bank_mask:0xf\n"
      "s_not_b64 vcc, vcc\n"
      "v_mfma_f32_16x16x32_bf16 v[168:171], a[136:139], a[240:243], v[168:171]\n"
      "v_cndmask_b32_dpp v32, v64, v32, vcc quad_perm:[1,0,3,2] row_mask:0xf bank_mask:0xf\n"
      "v_cndmask_b32_dpp v34, v66, v34, vcc quad_perm:[1,0,3,2] row_mask:0xf bank_mask:0xf\n"
      "buffer_store_dwordx4 v[96:99], v26, s[80:83], s90 offen sc0 sc1 nt\n"
      "v_mfma_f32_16x16x32_bf16 v[200:203], a[144:147], a[240:243], v[200:203]\n"
      "v_cndmask_b32_dpp v33, v65, v33, vcc quad_perm:[1,0,3,2] row_mask:0xf bank_mask:0xf\n"
      "v_cndmask_b32_dpp v35, v67, v35, vcc quad_perm:[1,0,3,2] row_mask:0xf bank_mask:0xf\n"
      "buffer_store_dwordx4 v[32:35], v26, s[80:83], s91 offen sc0 sc1 nt\n"
      "v_mfma_f32_16x16x32_bf16 v[200:203], a[148:151], a[244:247], v[200:203]\n"
      "s_waitcnt vmcnt(6)\n"
      "v_mfma_f32_4x4x4_16b_bf16 v[212:215], v[24:25], a[76:77], v[212:215]\n"
      "v_cvt_pk_bf16_f32 v70, v228, v229\n"
      "v_cvt_pk_bf16_f32 v71, v230, v231\n"
      "v_mfma_f32_16x16x32_bf16 v[232:235], a[156:159], a[244:247], v[232:235]\n"
      "v_mfma_f32_4x4x4_16b_bf16 v[244:247], v[24:25], a[78:79], v[244:247]\n"
      "v_pk_max_i16 v70, v70, s59\n"
      "v_pk_max_i16 v71, v71, s59\n"
      "v_mfma_f32_16x16x32_bf16 v[232:235], a[152:155], a[240:243], v[232:235]\n"
      "v_cvt_pk_bf16_f32 v68, v196, v197\n"
      "v_cvt_pk_bf16_f32 v69, v198, v199\n"
      "v_pk_max_i16 v68, v68, s59\n"
      "s_waitcnt lgkmcnt(2)\n"
      "v_mfma_f32_16x16x32_bf16 v[236:239], a[152:155], a[248:251], v[236:239]\n"
      "v_pk_max_i16 v69, v69, s59\n"
      "v_cvt_pk_bf16_f32 v38, v164, v165\n"
      "v_cvt_pk_bf16_f32 v39, v166, v167\n"
      "v_mfma_f32_16x16x32_bf16 v[236:239], a[156:159], a[252:255], v[236:239]\n"
      "v_permlane16_swap_b32 v68, v70\n"
      "v_permlane16_swap_b32 v69, v71\n"
      "v_pk_max_i16 v38, v38, s59\n"
      "v_mfma_f32_16x16x32_bf16 v[204:207], a[148:151], a[252:255], v[204:207]\n"
      "v_pk_max_i16 v39, v39, s59\n"
      "v_cvt_pk_bf16_f32 v36, v132, v133\n"
      "v_cvt_pk_bf16_f32 v37, v134, v135\n"
      "v_mfma_f32_16x16x32_bf16 v[204:207], a[144:147], a[248:251], v[204:207]\n"
      "s_waitcnt vmcnt(5)\n"
      "v_mfma_f32_4x4x4_16b_bf16 v[152:155], v[24:25], a[80:81], v[152:155]\n"
      "v_pk_max_i16 v36, v36, s59\n"
      "v_pk_max_i16 v37, v37, s59\n"
      "v_mfma_f32_16x16x32_bf16 v[172:175], a[136:139], a[248:251], v[172:175]\n"
      "ds_read_b128 a[200:203], v4 offset:10240\n"
      "s_mov_b32 s90, 0x40000\n"
      "s_mov_b32 s91, 0x44000\n"
      "v_mfma_f32_16x16x32_bf16 v[172:175], a[140:143], a[252:255], v[172:175]\n"
      "ds_read_b128 a[204:207], v5 offset:10240\n"
      "v_permlane16_swap_b32 v36, v38\n"
      "v_permlane16_swap_b32 v37, v39\n"
      "v_mfma_f32_16x16x32_bf16 v[140:143], a[132:135], a[252:255], v[140:143]\n"
      "v_mfma_f32_4x4x4_16b_bf16 v[184:187], v[24:25], a[82:83], v[184:187]\n"
      "v_cndmask_b32_dpp v100, v68, v36, vcc quad_perm:[1,0,3,2] row_mask:0xf bank_mask:0xf\n"
      "v_cndmask_b32_dpp v102, v70, v38, vcc quad_perm:[1,0,3,2] row_mask:0xf bank_mask:0xf\n"
      "v_mfma_f32_16x16x32_bf16 v[140:143], a[128:131], a[248:251], v[140:143]\n"
      "v_cndmask_b32_dpp v101, v69, v37, vcc quad_perm:[1,0,3,2] row_mask:0xf bank_mask:0xf\n"
      "v_cndmask_b32_dpp v103, v71, v39, vcc quad_perm:[1,0,3,2] row_mask:0xf bank_mask:0xf\n"
      "s_not_b64 vcc, vcc\n"
      "v_mfma_f32_16x16x32_bf16 v[144:147], a[160:163], a[0:3], v[144:147]\n"
      "v_cndmask_b32_dpp v68, v36, v68, vcc quad_perm:[1,0,3,2] row_mask:0xf bank_mask:0xf\n"
      "v_cndmask_b32_dpp v70, v38, v70, vcc quad_perm:[1,0,3,2] row_mask:0xf bank_mask:0xf\n"
      "buffer_store_dwordx4 v[100:103], v26, s[80:83], s90 offen sc0 sc1 nt\n"
      "v_mfma_f32_16x16x32_bf16 v[144:147], a[164:167], a[4:7], v[144:147]\n"
      "s_waitcnt vmcnt(5)\n"
      "v_mfma_f32_4x4x4_16b_bf16 v[216:219], v[24:25], a[84:85], v[216:219]\n"
      "v_cndmask_b32_dpp v69, v37, v69, vcc quad_perm:[1,0,3,2] row_mask:0xf bank_mask:0xf\n"
      "v_cndmask_b32_dpp v71, v39, v71, vcc quad_perm:[1,0,3,2] row_mask:0xf bank_mask:0xf\n"
      "v_mfma_f32_16x16x32_bf16 v[176:179], a[172:175], a[4:7], v[176:179]\n"
      "v_mfma_f32_4x4x4_16b_bf16 v[248:251], v[24:25], a[86:87], v[248:251]\n"
      "buffer_store_dwordx4 v[68:71], v26, s[80:83], s91 offen sc0 sc1 nt\n"
      "v_cvt_pk_bf16_f32 v40, v136, v137\n"
      "v_mfma_f32_16x16x32_bf16 v[176:179], a[168:171], a[0:3], v[176:179]\n"
      "v_cvt_pk_bf16_f32 v41, v138, v139\n"
      "v_pk_max_i16 v40, v40, s59\n"
      "v_pk_max_i16 v41, v41, s59\n"
      "v_mfma_f32_16x16x32_bf16 v[208:211], a[176:179], a[0:3], v[208:211]\n"
      "v_cvt_pk_bf16_f32 v42, v168, v169\n"
      "v_cvt_pk_bf16_f32 v43, v170, v171\n"
      "v_pk_max_i16 v42, v42, s59\n"
      "v_mfma_f32_16x16x32_bf16 v[208:211], a[180:183], a[4:7], v[208:211]\n"
      "v_pk_max_i16 v43, v43, s59\n"
      "v_cvt_pk_bf16_f32 v72, v200, v201\n"
      "v_cvt_pk_bf16_f32 v73, v202, v203\n"
      "v_mfma_f32_16x16x32_bf16 v[240:243], a[188:191], a[4:7], v[240:243]\n"
      "v_permlane16_swap_b32 v40, v42\n"
      "v_permlane16_swap_b32 v41, v43\n"
      "v_pk_max_i16 v72, v72, s59\n"
      "v_mfma_f32_16x16x32_bf16 v[240:243], a[184:187], a[0:3], v[240:243]\n"
      "s_waitcnt vmcnt(5)\n"
      "v_mfma_f32_4x4x4_16b_bf16 v[156:159], v[24:25], a[88:89], v[156:159]\n"
      "v_pk_max_i16 v73, v73, s59\n"
      "v_cvt_pk_bf16_f32 v74, v232, v233\n"
      "s_waitcnt lgkmcnt(2)\n"
      "v_mfma_f32_16x16x32_bf16 v[144:147], a[128:131], a[192:195], v[144:147]\n"
      "v_mfma_f32_4x4x4_16b_bf16 v[188:191], v[24:25], a[90:91], v[188:191]\n"
      "v_cvt_pk_bf16_f32 v75, v234, v235\n"
      "v_pk_max_i16 v74, v74, s59\n"
      "v_mfma_f32_16x16x32_bf16 v[144:147], a[132:135], a[196:199], v[144:147]\n"
      "v_pk_max_i16 v75, v75, s59\n"
      "s_mov_b32 s90, 0x84000\n"
      "s_mov_b32 s91, 0x80000\n"
      "v_mfma_f32_16x16x32_bf16 v[176:179], a[140:143], a[196:199], v[176:179]\n"
      "v_permlane16_swap_b32 v72, v74\n"
      "v_permlane16_swap_b32 v73, v75\n"
      "v_cndmask_b32_dpp v104, v40, v72, vcc quad_perm:[1,0,3,2] row_mask:0xf bank_mask:0xf\n"
      "v_mfma_f32_16x16x32_bf16 v[176:179], a[136:139], a[192:195], v[176:179]\n"
      "v_cndmask_b32_dpp v106, v42, v74, vcc quad_perm:[1,0,3,2] row_mask:0xf bank_mask:0xf\n"
      "v_cndmask_b32_dpp v105, v41, v73, vcc quad_perm:[1,0,3,2] row_mask:0xf bank_mask:0xf\n"
      "v_cndmask_b32_dpp v107, v43, v75, vcc quad_perm:[1,0,3,2] row_mask:0xf bank_mask:0xf\n"
      "v_mfma_f32_16x16x32_bf16 v[208:211], a[144:147], a[192:195], v[208:211]\n"
      "ds_read_b128 a[208:211], v4 offset:12288\n"
      "s_not_b64 vcc, vcc\n"
      "v_cndmask_b32_dpp v40, v72, v40, vcc quad_perm:[1,0,3,2] row_mask:0xf bank_mask:0xf\n"
      "v_mfma_f32_16x16x32_bf16 v[208:211], a[148:151], a[196:199], v[208:211]\n"
      "ds_read_b128 a[212:215], v5 offset:12288\n"
      "v_cndmask_b32_dpp v42, v74, v42, vcc quad_perm:[1,0,3,2] row_mask:0xf bank_mask:0xf\n"
      "buffer_store_dwordx4 v[104:107], v26, s[80:83], s90 offen sc0 sc1 nt\n"
      "v_mfma_f32_16x16x32_bf16 v[240:243], a[156:159], a[196:199], v[240:243]\n"
      "s_waitcnt vmcnt(5)\n"
      "v_mfma_f32_4x4x4_16b_bf16 v[220:223], v[24:25], a[92:93], v[220:223]\n"
      "v_cndmask_b32_dpp v41, v73, v41, vcc quad_perm:[1,0,3,2] row_mask:0xf bank_mask:0xf\n"
      "v_cndmask_b32_dpp v43, v75, v43, vcc quad_perm:[1,0,3,2] row_mask:0xf bank_mask:0xf\n"
      "v_mfma_f32_16x16x32_bf16 v[240:243], a[152:155], a[192:195], v[240:243]\n"
      "v_mfma_f32_4x4x4_16b_bf16 v[252:255], v[24:25], a[94:95], v[252:255]\n"
      "buffer_store_dwordx4 v[40:43], v26, s[80:83], s91 offen sc0 sc1 nt\n"
      "v_cvt_pk_bf16_f32 v78, v236, v237\n"
      "v_mfma_f32_16x16x32_bf16 v[244:247], a[184:187], a[8:11], v[244:247]\n"
      "v_cvt_pk_bf16_f32 v79, v238, v239\n"
      "v_pk_max_i16 v78, v78, s59\n"
      "v_pk_max_i16 v79, v79, s59\n"
      "v_mfma_f32_16x16x32_bf16 v[244:247], a[188:191], a[12:15], v[244:247]\n"
      "v_cvt_pk_bf16_f32 v76, v204, v205\n"
      "v_cvt_pk_bf16_f32 v77, v206, v207\n"
      "v_pk_max_i16 v76, v76, s59\n"
      "v_mfma_f32_16x16x32_bf16 v[212:215], a[180:183], a[12:15], v[212:215]\n"
      "v_pk_max_i16 v77, v77, s59\n"
      "v_cvt_pk_bf16_f32 v46, v172, v173\n"
      "v_cvt_pk_bf16_f32 v47, v174, v175\n"
      "v_mfma_f32_16x16x32_bf16 v[212:215], a[176:179], a[8:11], v[212:215]\n"
      "v_permlane16_swap_b32 v76, v78\n"
      "v_permlane16_swap_b32 v77, v79\n"
      "v_pk_max_i16 v46, v46, s59\n"
      "v_mfma_f32_16x16x32_bf16 v[180:183], a[168:171], a[8:11], v[180:183]\n"
      "v_pk_max_i16 v47, v47, s59\n"
      "v_cvt_pk_bf16_f32 v44, v140, v141\n"
      "v_cvt_pk_bf16_f32 v45, v142, v143\n"
      "v_mfma_f32_16x16x32_bf16 v[180:183], a[172:175], a[12:15], v[180:183]\n"
      "v_pk_max_i16 v44, v44, s59\n"
      "v_pk_max_i16 v45, v45, s59\n"
      "s_mov_b32 s90, 0xc0000\n"
      "v_mfma_f32_16x16x32_bf16 v[148:151], a[164:167], a[12:15], v[148:151]\n"
      "s_mov_b32 s91, 0xc4000\n"
      "v_permlane16_swap_b32 v44, v46\n"
      "v_permlane16_swap_b32 v45, v47\n"
      "v_mfma_f32_16x16x32_bf16 v[148:151], a[160:163], a[8:11], v[148:151]\n"
      "v_cndmask_b32_dpp v108, v76, v44, vcc quad_perm:[1,0,3,2] row_mask:0xf bank_mask:0xf\n"
      "v_cndmask_b32_dpp v110, v78, v46, vcc quad_perm:[1,0,3,2] row_mask:0xf bank_mask:0xf\n"
      "v_cndmask_b32_dpp v109, v77, v45, vcc quad_perm:[1,0,3,2] row_mask:0xf bank_mask:0xf\n"
      "s_waitcnt lgkmcnt(2)\n"
      "v_mfma_f32_16x16x32_bf16 v[244:247], a[152:155], a[200:203], v[244:247]\n"
      "v_cndmask_b32_dpp v111, v79, v47, vcc quad_perm:[1,0,3,2] row_mask:0xf bank_mask:0xf\n"
      "s_not_b64 vcc, vcc\n"
      "v_cndmask_b32_dpp v76, v44, v76, vcc quad_perm:[1,0,3,2] row_mask:0xf bank_mask:0xf\n"
      "v_mfma_f32_16x16x32_bf16 v[244:247], a[156:159], a[204:207], v[244:247]\n"
      "v_cndmask_b32_dpp v78, v46, v78, vcc quad_perm:[1,0,3,2] row_mask:0xf bank_mask:0xf\n"
      "buffer_store_dwordx4 v[108:111], v26, s[80:83], s90 offen sc0 sc1 nt\n"
      "v_cndmask_b32_dpp v77, v45, v77, vcc quad_perm:[1,0,3,2] row_mask:0xf bank_mask:0xf\n"
      "v_mfma_f32_16x16x32_bf16 v[212:215], a[148:151], a[204:207], v[212:215]\n"
      "v_cndmask_b32_dpp v79, v47, v79, vcc quad_perm:[1,0,3,2] row_mask:0xf bank_mask:0xf\n"
      "buffer_store_dwordx4 v[76:79], v26, s[80:83], s91 offen sc0 sc1 nt\n"
      "v_cvt_pk_bf16_f32 v48, v144, v145\n"
      "v_mfma_f32_16x16x32_bf16 v[212:215], a[144:147], a[200:203], v[212:215]\n"
      "v_cvt_pk_bf16_f32 v49, v146, v147\n"
      "v_pk_max_i16 v48, v48, s59\n"
      "v_pk_max_i16 v49, v49, s59\n"
      "v_mfma_f32_16x16x32_bf16 v[180:183], a[136:139], a[200:203], v[180:183]\n"
      "ds_read_b128 a[216:219], v4 offset:14336\n"
      "v_cvt_pk_bf16_f32 v50, v176, v177\n"
      "v_cvt_pk_bf16_f32 v51, v178, v179\n"
      "v_mfma_f32_16x16x32_bf16 v[180:183], a[140:143], a[204:207], v[180:183]\n"
      "ds_read_b128 a[220:223], v5 offset:14336\n"
      "v_pk_max_i16 v50, v50, s59\n"
      "v_pk_max_i16 v51, v51, s59\n"
      "v_mfma_f32_16x16x32_bf16 v[148:151], a[132:135], a[204:207], v[148:151]\n"
      "v_cvt_pk_bf16_f32 v80, v208, v209\n"
      "v_cvt_pk_bf16_f32 v81, v210, v211\n"
      "v_permlane16_swap_b32 v48, v50\n"
      "v_mfma_f32_16x16x32_bf16 v[148:151], a[128:131], a[200:203], v[148:151]\n"
      "v_permlane16_swap_b32 v49, v51\n"
      "v_pk_max_i16 v80, v80, s59\n"
      "v_pk_max_i16 v81, v81, s59\n"
      "v_mfma_f32_16x16x32_bf16 v[152:155], a[160:163], a[16:19], v[152:155]\n"
      "v_cvt_pk_bf16_f32 v82, v240, v241\n"
      "v_cvt_pk_bf16_f32 v83, v242, v243\n"
      "v_pk_max_i16 v82, v82, s59\n"
      "v_mfma_f32_16x16x32_bf16 v[152:155], a[164:167], a[20:23], v[152:155]\n"
      "v_pk_max_i16 v83, v83, s59\n"
      "s_mov_b32 s90, 0x104000\n"
      "s_mov_b32 s91, 0x100000\n"
      "v_mfma_f32_16x16x32_bf16 v[184:187], a[172:175], a[20:23], v[184:187]\n"
      "v_permlane16_swap_b32 v80, v82\n"
      "v_permlane16_swap_b32 v81, v83\n"
      "v_cndmask_b32_dpp v112, v48, v80, vcc quad_perm:[1,0,3,2] row_mask:0xf bank_mask:0xf\n"
      "v_mfma_f32_16x16x32_bf16 v[184:187], a[168:171], a[16:19], v[184:187]\n"
      "v_cndmask_b32_dpp v114, v50, v82, vcc quad_perm:[1,0,3,2] row_mask:0xf bank_mask:0xf\n"
      "v_cndmask_b32_dpp v113, v49, v81, vcc quad_perm:[1,0,3,2] row_mask:0xf bank_mask:0xf\n"
      "v_cndmask_b32_dpp v115, v51, v83, vcc quad_perm:[1,0,3,2] row_mask:0xf bank_mask:0xf\n"
      "v_mfma_f32_16x16x32_bf16 v[216:219], a[176:179], a[16:19], v[216:219]\n"
      "s_not_b64 vcc, vcc\n"
      "v_cndmask_b32_dpp v48, v80, v48, vcc quad_perm:[1,0,3,2] row_mask:0xf bank_mask:0xf\n"
      "v_cndmask_b32_dpp v50, v82, v50, vcc quad_perm:[1,0,3,2] row_mask:0xf bank_mask:0xf\n"
      "v_mfma_f32_16x16x32_bf16 v[216:219], a[180:183], a[20:23], v[216:219]\n"
      "buffer_store_dwordx4 v[112:115], v26, s[80:83], s90 offen sc0 sc1 nt\n"
      "v_cndmask_b32_dpp v49, v81, v49, vcc quad_perm:[1,0,3,2] row_mask:0xf bank_mask:0xf\n"
      "v_cndmask_b32_dpp v51, v83, v51, vcc quad_perm:[1,0,3,2] row_mask:0xf bank_mask:0xf\n"
      "v_mfma_f32_16x16x32_bf16 v[248:251], a[188:191], a[20:23], v[248:251]\n"
      "buffer_store_dwordx4 v[48:51], v26, s[80:83], s91 offen sc0 sc1 nt\n"
      "v_cvt_pk_bf16_f32 v86, v244, v245\n"
      "v_cvt_pk_bf16_f32 v87, v246, v247\n"
      "v_mfma_f32_16x16x32_bf16 v[248:251], a[184:187], a[16:19], v[248:251]\n"
      "v_pk_max_i16 v86, v86, s59\n"
      "v_pk_max_i16 v87, v87, s59\n"
      "v_cvt_pk_bf16_f32 v84, v212, v213\n"
      "s_waitcnt lgkmcnt(2)\n"
      "v_mfma_f32_16x16x32_bf16 v[152:155], a[128:131], a[208:211], v[152:155]\n"
      "v_cvt_pk_bf16_f32 v85, v214, v215\n"
      "v_pk_max_i16 v84, v84, s59\n"
      "v_pk_max_i16 v85, v85, s59\n"
      "v_mfma_f32_16x16x32_bf16 v[152:155], a[132:135], a[212:215], v[152:155]\n"
      "v_cvt_pk_bf16_f32 v54, v180, v181\n"
      "v_cvt_pk_bf16_f32 v55, v182, v183\n"
      "v_permlane16_swap_b32 v84, v86\n"
      "v_mfma_f32_16x16x32_bf16 v[184:187], a[140:143], a[212:215], v[184:187]\n"
      "v_permlane16_swap_b32 v85, v87\n"
      "v_pk_max_i16 v54, v54, s59\n"
      "v_pk_max_i16 v55, v55, s59\n"
      "v_mfma_f32_16x16x32_bf16 v[184:187], a[136:139], a[208:211], v[184:187]\n"
      "v_cvt_pk_bf16_f32 v52, v148, v149\n"
      "v_cvt_pk_bf16_f32 v53, v150, v151\n"
      "v_pk_max_i16 v52, v52, s59\n"
      "v_mfma_f32_16x16x32_bf16 v[216:219], a[144:147], a[208:211], v[216:219]\n"
      "v_pk_max_i16 v53, v53, s59\n"
      "s_mov_b32 s90, 0x140000\n"
      "s_mov_b32 s91, 0x144000\n"
      "v_mfma_f32_16x16x32_bf16 v[216:219], a[148:151], a[212:215], v[216:219]\n"
      "v_permlane16_swap_b32 v52, v54\n"
      "v_permlane16_swap_b32 v53, v55\n"
      "v_cndmask_b32_dpp v116, v84, v52, vcc quad_perm:[1,0,3,2] row_mask:0xf bank_mask:0xf\n"
      "v_mfma_f32_16x16x32_bf16 v[248:251], a[156:159], a[212:215], v[248:251]\n"
      "v_cndmask_b32_dpp v118, v86, v54, vcc quad_perm:[1,0,3,2] row_mask:0xf bank_mask:0xf\n"
      "v_cndmask_b32_dpp v117, v85, v53, vcc quad_perm:[1,0,3,2] row_mask:0xf bank_mask:0xf\n"
      "v_cndmask_b32_dpp v119, v87, v55, vcc quad_perm:[1,0,3,2] row_mask:0xf bank_mask:0xf\n"
      "v_mfma_f32_16x16x32_bf16 v[248:251], a[152:155], a[208:211], v[248:251]\n"
      "s_not_b64 vcc, vcc\n"
      "v_cndmask_b32_dpp v84, v52, v84, vcc quad_perm:[1,0,3,2] row_mask:0xf bank_mask:0xf\n"
      "v_cndmask_b32_dpp v86, v54, v86, vcc quad_perm:[1,0,3,2] row_mask:0xf bank_mask:0xf\n"
      "v_mfma_f32_16x16x32_bf16 v[252:255], a[184:187], a[24:27], v[252:255]\n"
      "buffer_store_dwordx4 v[116:119], v26, s[80:83], s90 offen sc0 sc1 nt\n"
      "v_cndmask_b32_dpp v85, v53, v85, vcc quad_perm:[1,0,3,2] row_mask:0xf bank_mask:0xf\n"
      "v_cndmask_b32_dpp v87, v55, v87, vcc quad_perm:[1,0,3,2] row_mask:0xf bank_mask:0xf\n"
      "v_mfma_f32_16x16x32_bf16 v[252:255], a[188:191], a[28:31], v[252:255]\n"
      "buffer_store_dwordx4 v[84:87], v26, s[80:83], s91 offen sc0 sc1 nt\n"
      "v_cvt_pk_bf16_f32 v56, v152, v153\n"
      "v_cvt_pk_bf16_f32 v57, v154, v155\n"
      "v_mfma_f32_16x16x32_bf16 v[220:223], a[180:183], a[28:31], v[220:223]\n"
      "v_pk_max_i16 v56, v56, s59\n"
      "v_pk_max_i16 v57, v57, s59\n"
      "v_cvt_pk_bf16_f32 v58, v184, v185\n"
      "v_mfma_f32_16x16x32_bf16 v[220:223], a[176:179], a[24:27], v[220:223]\n"
      "v_cvt_pk_bf16_f32 v59, v186, v187\n"
      "v_pk_max_i16 v58, v58, s59\n"
      "v_pk_max_i16 v59, v59, s59\n"
      "v_mfma_f32_16x16x32_bf16 v[188:191], a[168:171], a[24:27], v[188:191]\n"
      "v_cvt_pk_bf16_f32 v88, v216, v217\n"
      "v_cvt_pk_bf16_f32 v89, v218, v219\n"
      "v_permlane16_swap_b32 v56, v58\n"
      "v_mfma_f32_16x16x32_bf16 v[188:191], a[172:175], a[28:31], v[188:191]\n"
      "v_permlane16_swap_b32 v57, v59\n"
      "v_pk_max_i16 v88, v88, s59\n"
      "v_pk_max_i16 v89, v89, s59\n"
      "v_mfma_f32_16x16x32_bf16 v[156:159], a[164:167], a[28:31], v[156:159]\n"
      "v_cvt_pk_bf16_f32 v90, v248, v249\n"
      "v_cvt_pk_bf16_f32 v91, v250, v251\n"
      "v_pk_max_i16 v90, v90, s59\n"
      "v_mfma_f32_16x16x32_bf16 v[156:159], a[160:163], a[24:27], v[156:159]\n"
      "v_pk_max_i16 v91, v91, s59\n"
      "s_mov_b32 s90, 0x184000\n"
      "s_mov_b32 s91, 0x180000\n"
      "s_waitcnt lgkmcnt(0)\n"
      "v_mfma_f32_16x16x32_bf16 v[252:255], a[152:155], a[216:219], v[252:255]\n"
      "v_permlane16_swap_b32 v88, v90\n"
      "v_permlane16_swap_b32 v89, v91\n"
      "v_cndmask_b32_dpp v120, v56, v88, vcc quad_perm:[1,0,3,2] row_mask:0xf bank_mask:0xf\n"
      "v_mfma_f32_16x16x32_bf16 v[252:255], a[156:159], a[220:223], v[252:255]\n"
      "v_cndmask_b32_dpp v122, v58, v90, vcc quad_perm:[1,0,3,2] row_mask:0xf bank_mask:0xf\n"
      "v_cndmask_b32_dpp v121, v57, v89, vcc quad_perm:[1,0,3,2] row_mask:0xf bank_mask:0xf\n"
      "v_cndmask_b32_dpp v123, v59, v91, vcc quad_perm:[1,0,3,2] row_mask:0xf bank_mask:0xf\n"
      "v_mfma_f32_16x16x32_bf16 v[220:223], a[148:151], a[220:223], v[220:223]\n"
      "s_not_b64 vcc, vcc\n"
      "v_cndmask_b32_dpp v56, v88, v56, vcc quad_perm:[1,0,3,2] row_mask:0xf bank_mask:0xf\n"
      "v_cndmask_b32_dpp v58, v90, v58, vcc quad_perm:[1,0,3,2] row_mask:0xf bank_mask:0xf\n"
      "v_mfma_f32_16x16x32_bf16 v[220:223], a[144:147], a[216:219], v[220:223]\n"
      "buffer_store_dwordx4 v[120:123], v26, s[80:83], s90 offen sc0 sc1 nt\n"
      "v_cndmask_b32_dpp v57, v89, v57, vcc quad_perm:[1,0,3,2] row_mask:0xf bank_mask:0xf\n"
      "v_cndmask_b32_dpp v59, v91, v59, vcc quad_perm:[1,0,3,2] row_mask:0xf bank_mask:0xf\n"
      "v_mfma_f32_16x16x32_bf16 v[188:191], a[136:139], a[216:219], v[188:191]\n"
      "buffer_store_dwordx4 v[56:59], v26, s[80:83], s91 offen sc0 sc1 nt\n"
      "v_mfma_f32_16x16x32_bf16 v[188:191], a[140:143], a[220:223], v[188:191]\n"
      "v_mfma_f32_16x16x32_bf16 v[156:159], a[132:135], a[220:223], v[156:159]\n"
      "v_cvt_pk_bf16_f32 v94, v252, v253\n"
      "v_cvt_pk_bf16_f32 v95, v254, v255\n"
      "v_pk_max_i16 v94, v94, s59\n"
      "v_mfma_f32_16x16x32_bf16 v[156:159], a[128:131], a[216:219], v[156:159]\n"
      "v_pk_max_i16 v95, v95, s59\n"
      "v_cvt_pk_bf16_f32 v92, v220, v221\n"
      "v_cvt_pk_bf16_f32 v93, v222, v223\n"
      "v_pk_max_i16 v92, v92, s59\n"
      "v_pk_max_i16 v93, v93, s59\n"
      "v_cvt_pk_bf16_f32 v62, v188, v189\n"
      "v_cvt_pk_bf16_f32 v63, v190, v191\n"
      "v_permlane16_swap_b32 v92, v94\n"
      "v_permlane16_swap_b32 v93, v95\n"
      "v_pk_max_i16 v62, v62, s59\n"
      "v_pk_max_i16 v63, v63, s59\n"
      "v_cvt_pk_bf16_f32 v60, v156, v157\n"
      "v_cvt_pk_bf16_f32 v61, v158, v159\n"
      "v_pk_max_i16 v60, v60, s59\n"
      "v_pk_max_i16 v61, v61, s59\n"
      "s_mov_b32 s90, 0x1c0000\n"
      "s_mov_b32 s91, 0x1c4000\n"
      "v_permlane16_swap_b32 v60, v62\n"
      "v_permlane16_swap_b32 v61, v63\n"
      "v_cndmask_b32_dpp v124, v92, v60, vcc quad_perm:[1,0,3,2] row_mask:0xf bank_mask:0xf\n"
      "v_cndmask_b32_dpp v126, v94, v62, vcc quad_perm:[1,0,3,2] row_mask:0xf bank_mask:0xf\n"
      "v_cndmask_b32_dpp v125, v93, v61, vcc quad_perm:[1,0,3,2] row_mask:0xf bank_mask:0xf\n"
      "v_cndmask_b32_dpp v127, v95, v63, vcc quad_perm:[1,0,3,2] row_mask:0xf bank_mask:0xf\n"
      "s_not_b64 vcc, vcc\n"
      "v_cndmask_b32_dpp v92, v60, v92, vcc quad_perm:[1,0,3,2] row_mask:0xf bank_mask:0xf\n"
      "v_cndmask_b32_dpp v94, v62, v94, vcc quad_perm:[1,0,3,2] row_mask:0xf bank_mask:0xf\n"
      "buffer_store_dwordx4 v[124:127], v26, s[80:83], s90 offen sc0 sc1 nt\n"
      "v_cndmask_b32_dpp v93, v61, v93, vcc quad_perm:[1,0,3,2] row_mask:0xf bank_mask:0xf\n"
      "v_cndmask_b32_dpp v95, v63, v95, vcc quad_perm:[1,0,3,2] row_mask:0xf bank_mask:0xf\n"
      "buffer_store_dwordx4 v[92:95], v26, s[80:83], s91 offen sc0 sc1 nt\n"
      "59:\n"
      "s_branch 80f\n"
      "31:\n"
      "s_mov_b32 s90, s66\n"
      "s_add_u32 s66, %49, 4\n"
      "v_lshrrev_b32 v28, 11, v30\n"
      "s_getpc_b64 s[78:79]\n"
      "s_branch 73f\n"
      "s_mov_b32 s66, s90\n"
      "v_bfe_u32 v28, v30, 12, 4\n"
      "v_lshrrev_b32 v29, 11, v30\n"
      "buffer_load_dwordx2 a[106:107], v29, s[28:31], %49 offen sc0 sc1\n"
      "s_waitcnt vmcnt(0)\n"
      "v_accvgpr_read_b32 v29, a106\n"
      "s_nop 1\n"
      "s_branch 28b\n"
      "160:\n"
      "s_cmp_eq_u32 s19, 0\n"
      "s_cselect_b32 s3, s46, s47\n"
      "s_addc_u32 s92, s92, 0\n"
      "s_sub_u32 s3, s3, s1\n"
      "s_cmp_gt_i32 s3, 170\n"
      "s_cmov_b32 s92, 64\n"
      "s_cmp_le_u32 s92, 2\n"
      "s_cbranch_scc1 18f\n"
      "s_sleep 8\n"
      "s_cmp_eq_u32 s19, 0\n"
      "s_cmov_b32 s19, 5\n"
      "s_cmp_lt_i32 s1, s47\n"
      "s_cmov_b32 s19, 5\n"
      "s_branch 19f\n"
      "18:\n"
      "s_cmp_eq_u32 s19, 0\n"
      "s_cmov_b32 s19, 3\n"
      "19:\n"
      "s_mov_b32 s47, s1\n"
      "s_sub_u32 s19, s19, 1\n"
      "s_cmp_lg_u32 s19, 0\n"
      "s_cselect_b32 s3, 0x100000, 0\n"
      "s_add_u32 s1, s1, s3\n"
      "s_branch 60b\n"
      "161:\n"
      "s_cmp_eq_u32 s19, 0\n"
      "s_cselect_b32 s3, s46, s47\n"
      "s_addc_u32 s92, s92, 0\n"
      "s_sub_u32 s3, s3, s1\n"
      "s_cmp_gt_i32 s3, 170\n"
      "s_cmov_b32 s92, 64\n"
      "s_cmp_le_u32 s92, 2\n"
      "s_cbranch_scc1 18f\n"
      "s_sleep 8\n"
      "s_cmp_eq_u32 s19, 0\n"
      "s_cmov_b32 s19, 5\n"
      "s_cmp_lt_i32 s1, s47\n"
      "s_cmov_b32 s19, 5\n"
      "s_branch 19f\n"
      "18:\n"
      "s_cmp_eq_u32 s19, 0\n"
      "s_cmov_b32 s19, 3\n"
      "19:\n"
      "s_mov_b32 s47, s1\n"
      "s_sub_u32 s19, s19, 1\n"
      "s_cmp_lg_u32 s19, 0\n"
      "s_cselect_b32 s3, 0x100000, 0\n"
      "s_add_u32 s1, s1, s3\n"
      "s_branch 61b\n"
      "162:\n"
      "s_cmp_eq_u32 s19, 0\n"
      "s_cselect_b32 s3, s46, s47\n"
      "s_addc_u32 s92, s92, 0\n"
      "s_sub_u32 s3, s3, s1\n"
      "s_cmp_gt_i32 s3, 170\n"
      "s_cmov_b32 s92, 64\n"
      "s_cmp_le_u32 s92, 2\n"
      "s_cbranch_scc1 18f\n"
      "s_sleep 8\n"
      "s_cmp_eq_u32 s19, 0\n"
      "s_cmov_b32 s19, 5\n"
      "s_cmp_lt_i32 s1, s47\n"
      "s_cmov_b32 s19, 5\n"
      "s_branch 19f\n"
      "18:\n"
      "s_cmp_eq_u32 s19, 0\n"
      "s_cmov_b32 s19, 3\n"
      "19:\n"
      "s_mov_b32 s47, s1\n"
      "s_sub_u32 s19, s19, 1\n"
      "s_cmp_lg_u32 s19, 0\n"
      "s_cselect_b32 s3, 0x100000, 0\n"
      "s_add_u32 s1, s1, s3\n"
      "s_branch 62b\n"
      "163:\n"
      "s_cmp_eq_u32 s19, 0\n"
      "s_cselect_b32 s3, s46, s47\n"
      "s_addc_u32 s92, s92, 0\n"
      "s_sub_u32 s3, s3, s1\n"
      "s_cmp_gt_i32 s3, 170\n"
      "s_cmov_b32 s92, 64\n"
      "s_cmp_le_u32 s92, 2\n"
      "s_cbranch_scc1 18f\n"
      "s_sleep 8\n"
      "s_cmp_eq_u32 s19, 0\n"
      "s_cmov_b32 s19, 5\n"
      "s_cmp_lt_i32 s1, s47\n"
      "s_cmov_b32 s19, 5\n"
      "s_branch 19f\n"
      "18:\n"
      "s_cmp_eq_u32 s19, 0\n"
      "s_cmov_b32 s19, 3\n"
      "19:\n"
      "s_mov_b32 s47, s1\n"
      "s_sub_u32 s19, s19, 1\n"
      "s_cmp_lg_u32 s19, 0\n"
      "s_cselect_b32 s3, 0x100000, 0\n"
      "s_add_u32 s1, s1, s3\n"
      "s_branch 63b\n"
      "164:\n"
      "s_cmp_eq_u32 s19, 0\n"
      "s_cselect_b32 s3, s46, s47\n"
      "s_addc_u32 s92, s92, 0\n"
      "s_sub_u32 s3, s3, s1\n"
      "s_cmp_gt_i32 s3, 170\n"
      "s_cmov_b32 s92, 64\n"
      "s_cmp_le_u32 s92, 2\n"
      "s_cbranch_scc1 18f\n"
      "s_sleep 8\n"
      "s_cmp_eq_u32 s19, 0\n"
      "s_cmov_b32 s19, 5\n"
      "s_cmp_lt_i32 s1, s47\n"
      "s_cmov_b32 s19, 5\n"
      "s_branch 19f\n"
      "18:\n"
      "s_cmp_eq_u32 s19, 0\n"
      "s_cmov_b32 s19, 3\n"
      "19:\n"
      "s_mov_b32 s47, s1\n"
      "s_sub_u32 s19, s19, 1\n"
      "s_cmp_lg_u32 s19, 0\n"
      "s_cselect_b32 s3, 0x100000, 0\n"
      "s_add_u32 s1, s1, s3\n"
      "s_branch 64b\n"
      "165:\n"
      "s_cmp_eq_u32 s19, 0\n"
      "s_cselect_b32 s3, s46, s47\n"
      "s_addc_u32 s92, s92, 0\n"
      "s_sub_u32 s3, s3, s1\n"
      "s_cmp_gt_i32 s3, 170\n"
      "s_cmov_b32 s92, 64\n"
      "s_cmp_le_u32 s92, 2\n"
      "s_cbranch_scc1 18f\n"
      "s_sleep 8\n"
      "s_cmp_eq_u32 s19, 0\n"
      "s_cmov_b32 s19, 5\n"
      "s_cmp_lt_i32 s1, s47\n"
      "s_cmov_b32 s19, 5\n"
      "s_branch 19f\n"
      "18:\n"
      "s_cmp_eq_u32 s19, 0\n"
      "s_cmov_b32 s19, 3\n"
      "19:\n"
      "s_mov_b32 s47, s1\n"
      "s_sub_u32 s19, s19, 1\n"
      "s_cmp_lg_u32 s19, 0\n"
      "s_cselect_b32 s3, 0x100000, 0\n"
      "s_add_u32 s1, s1, s3\n"
      "s_branch 65b\n"
      "140:\n"
      "s_getpc_b64 s[78:79]\n"
      "s_branch 73f\n"
      "s_branch 40b\n"
      "141:\n"
      "s_getpc_b64 s[78:79]\n"
      "s_branch 73f\n"
      "s_branch 41b\n"
      "142:\n"
      "s_getpc_b64 s[78:79]\n"
      "s_branch 73f\n"
      "s_branch 42b\n"
      "143:\n"
      "s_getpc_b64 s[78:79]\n"
      "s_branch 73f\n"
      "s_branch 43b\n"
      "144:\n"
      "s_getpc_b64 s[78:79]\n"
      "s_branch 73f\n"
      "s_branch 44b\n"
      "145:\n"
      "s_getpc_b64 s[78:79]\n"
      "s_branch 73f\n"
      "s_add_u32 s66, s66, 4\n"
      "s_getpc_b64 s[78:79]\n"
      "s_branch 73f\n"
      "s_add_u32 s66, s66, 4\n"
      "s_getpc_b64 s[78:79]\n"
      "s_branch 73f\n"
      "s_add_u32 s66, s66, 4\n"
      "s_getpc_b64 s[78:79]\n"
      "s_branch 73f\n"
      "s_branch 45b\n"
      "146:\n"
      "s_getpc_b64 s[78:79]\n"
      "s_branch 73f\n"
      "s_branch 46b\n"
      "147:\n"
      "s_getpc_b64 s[78:79]\n"
      "s_branch 73f\n"
      "s_branch 47b\n"
      "148:\n"
      "s_getpc_b64 s[78:79]\n"
      "s_branch 73f\n"
      "s_add_u32 s66, s66, 4\n"
      "s_getpc_b64 s[78:79]\n"
      "s_branch 73f\n"
      "s_add_u32 s66, s66, 4\n"
      "s_getpc_b64 s[78:79]\n"
      "s_branch 73f\n"
      "s_add_u32 s66, s66, 4\n"
      "s_getpc_b64 s[78:79]\n"
      "s_branch 73f\n"
      "s_branch 48b\n"
      "149:\n"
      "s_getpc_b64 s[78:79]\n"
      "s_branch 73f\n"
      "s_branch 49b\n"
      "73:\n"
      "s_mov_b32 s76, s86\n"
      "s_memrealtime s[86:87]\n"
      "s_add_u32 s78, s78, 4\n"
      "s_addc_u32 s79, s79, 0\n"
      "76:\n"
      "buffer_load_dword v29, v28, s[28:31], s66 offen sc0 sc1\n"
      "s_memrealtime s[88:89]\n"
      "s_waitcnt vmcnt(0)\n"
      "v_cmp_gt_u32 vcc, s75, v29\n"
      "s_cbranch_vccz 77f\n"
      "s_waitcnt lgkmcnt(0)\n"
      "s_sub_u32 s88, s88, s86\n"
      "s_sleep 1\n"
      "s_cmp_lt_u32 s88, 20000\n"
      "s_cbranch_scc1 76b\n"
      "v_mov_b32 v29, 0x40c0000\n"
      "global_load_dword v29, v29, s[48:49] sc1\n"
      "s_waitcnt vmcnt(0)\n"
      "v_cmp_eq_u32 vcc, %31, v29\n"
      "s_cbranch_vccnz 78f\n"
      "s_sleep 8\n"
      "s_cmp_gt_u32 s88, 40000000\n"
      "s_cbranch_scc0 76b\n"
      "78:\n"
      "v_mov_b32 v29, 0x40c0000\n"
      "global_store_dword v29, v31, s[48:49] sc1\n"
      "s_mov_b32 s75, 0\n"
      "s_mov_b32 s42, 0\n"
      "s_mov_b32 s46, 0\n"
      "s_waitcnt vmcnt(0)\n"
      "77:\n"
      "s_waitcnt lgkmcnt(0)\n"
      "s_mov_b32 s86, s76\n"
      "s_setpc_b64 s[78:79]\n"
      "80:\n"
      : "+{v0}"(b0), "+{v1}"(b1), "+{v2}"(b2), "+{v3}"(b3), "+{v4}"(a0), "+{v5}"(a1), "+{v6}"(ws0), "+{v7}"(ws1), "+{v8}"(ws2), "+{v9}"(ws3), "+{v10}"(ws4), "+{v11}"(ws5), "+{v12}"(ws6), "+{v13}"(ws7), "+{v14}"(av0), "+{v15}"(av1), "+{v16}"(av2), "+{v17}"(av3), "+{v18}"(av4), "+{v19}"(av5), "+{v20}"(av6), "+{v21}"(av7), "+{v22}"(tabW), "+{v23}"(tabA), "+{v24}"(tabS), "+{v25}"(tabD), "+{v26}"(tabF), "+{v27}"(tabE), "+{v28}"(foff), "+{v29}"(tabT), "+{v30}"(lane), "+{s20}"(call), "+{s28}"(fo), "+{s65}"(wof), "+{s64}"(aof), "+{s21}"(bas)
      : "{s[4:7]}"(rW), "{s[36:39]}"(rX), "{s[26:27]}"(out), "{s8}"(oo), "{s9}"(w), "{s10}"(wa), "{s11}"(ww), "{s12}"(rowo), "{s13}"(cnto), "{s14}"(xfo), "{s15}"(xfs), "{s16}"(xo), "{s17}"(xs), "{s18}"(bal), "{s[48:63]}"(pt)
      : "v31", "v32", "v33", "v34", "v35", "v36", "v37", "v38", "v39", "v40", "v41", "v42", "v43", "v44", "v45", "v46", "v47", "v48", "v49", "v50", "v51", "v52", "v53", "v54", "v55", "v56", "v57", "v58", "v59", "v60", "v61", "v62", "v63", "v64", "v65", "v66", "v67", "v68", "v69", "v70", "v71", "v72", "v73", "v74", "v75", "v76", "v77", "v78", "v79", "v80", "v81", "v82", "v83", "v84", "v85", "v86", "v87", "v88", "v89", "v90", "v91", "v92", "v93", "v94", "v95", "v96", "v97", "v98", "v99", "v100", "v101", "v102", "v103", "v104", "v105", "v106", "v107", "v108", "v109", "v110", "v111", "v112", "v113", "v114", "v115", "v116", "v117", "v118", "v119", "v120", "v121", "v122", "v123", "v124", "v125", "v126", "v127", "v128", "v129", "v130", "v131", "v132", "v133", "v134", "v135", "v136", "v137", "v138", "v139", "v140", "v141", "v142", "v143", "v144", "v145", "v146", "v147", "v148", "v149", "v150", "v151", "v152", "v153", "v154", "v155", "v156", "v157", "v158", "v159", "v160", "v161", "v162", "v163", "v164", "v165", "v166", "v167", "v168", "v169", "v170", "v171", "v172", "v173", "v174", "v175", "v176", "v177", "v178", "v179", "v180", "v181", "v182", "v183", "v184", "v185", "v186", "v187", "v188", "v189", "v190", "v191", "v192", "v193", "v194", "v195", "v196", "v197", "v198", "v199", "v200", "v201", "v202", "v203", "v204", "v205", "v206", "v207", "v208", "v209", "v210", "v211", "v212", "v213", "v214", "v215", "v216", "v217", "v218", "v219", "v220", "v221", "v222", "v223", "v224", "v225", "v226", "v227", "v228", "v229", "v230", "v231", "v232", "v233", "v234", "v235", "v236", "v237", "v238", "v239", "v240", "v241", "v242", "v243", "v244", "v245", "v246", "v247", "v248", "v249", "v250", "v251", "v252", "v253", "v254", "v255", "a0", "a1", "a2", "a3", "a4", "a5", "a6", "a7", "a8", "a9", "a10", "a11", "a12", "a13", "a14", "a15", "a16", "a17", "a18", "a19", "a20", "a21", "a22", "a23", "a24", "a25", "a26", "a27", "a28", "a29", "a30", "a31", "a32", "a33", "a34", "a35", "a36", "a37", "a38", "a39", "a40", "a41", "a42", "a43", "a44", "a45", "a46", "a47", "a48", "a49", "a50", "a51", "a52", "a53", "a54", "a55", "a56", "a57", "a58", "a59", "a60", "a61", "a62", "a63", "a64", "a65", "a66", "a67", "a68", "a69", "a70", "a71", "a72", "a73", "a74", "a75", "a76", "a77", "a78", "a79", "a80", "a81", "a82", "a83", "a84", "a85", "a86", "a87", "a88", "a89", "a90", "a91", "a92", "a93", "a94", "a95", "a96", "a97", "a98", "a99", "a100", "a101", "a102", "a103", "a104", "a105", "a106", "a107", "a108", "a109", "a110", "a111", "a112", "a113", "a114", "a115", "a116", "a117", "a118", "a119", "a120", "a121", "a122", "a123", "a124", "a125", "a126", "a127", "a128", "a129", "a130", "a131", "a132", "a133", "a134", "a135", "a136", "a137", "a138", "a139", "a140", "a141", "a142", "a143", "a144", "a145", "a146", "a147", "a148", "a149", "a150", "a151", "a152", "a153", "a154", "a155", "a156", "a157", "a158", "a159", "a160", "a161", "a162", "a163", "a164", "a165", "a166", "a167", "a168", "a169", "a170", "a171", "a172", "a173", "a174", "a175", "a176", "a177", "a178", "a179", "a180", "a181", "a182", "a183", "a184", "a185", "a186", "a187", "a188", "a189", "a190", "a191", "a192", "a193", "a194", "a195", "a196", "a197", "a198", "a199", "a200", "a201", "a202", "a203", "a204", "a205", "a206", "a207", "a208", "a209", "a210", "a211", "a212", "a213", "a214", "a215", "a216", "a217", "a218", "a219", "a220", "a221", "a222", "a223", "a224", "a225", "a226", "a227", "a228", "a229", "a230", "a231", "a232", "a233", "a234", "a235", "a236", "a237", "a238", "a239", "a240", "a241", "a242", "a243", "a244", "a245", "a246", "a247", "a248", "a249", "a250", "a251", "a252", "a253", "a254", "a255", "s0", "s1", "s2", "s3", "s19", "s92", "s93", "s35", "s34", "s33", "s22", "s23", "s24", "s25", "s29", "s30", "s31", "s40", "s41", "s42", "s43", "s44", "s45", "s46", "s47", "s66", "s67", "s68", "s69", "s70", "s71", "s72", "s73", "s74", "s75", "s76", "s77", "s78", "s79", "s80", "s81", "s82", "s83", "s84", "s85", "s86", "s87", "s88", "s89", "s90", "s91", "memory", "scc", "vcc");
}

extern "C" void kernel_launch(int rank, int nranks, ncclComm_t comm, hipStream_t stream,
                              void* const* d_in, const int* in_sizes, int n_in,
                              void* d_out, int out_size, const int* in_dtypes, int out_dtype,
                              void* const* d_out_all, void* const* d_ws_all, size_t ws_bytes) {
  if (nranks != 8 || (unsigned)rank >= 8u || ws_bytes < ((size_t)104 << 20)) { fprintf(stderr, "unsupported configuration\n"); return; }
  HIP_CHECK(hipFuncSetAttribute((const void*)a2a_gemm, hipFuncAttributeMaxDynamicSharedMemorySize, LDS_BYTES));
  Args a;
  const unsigned long long px = (unsigned long long)d_in[0], pw = (unsigned long long)d_in[1] - 3072u;
  a.rX[0] = (unsigned)px; a.rX[1] = (unsigned)(px >> 32) & 0xffffu; a.rX[2] = 16u << 20; a.rX[3] = 0x00020000u;
  a.rW[0] = (unsigned)pw; a.rW[1] = (unsigned)(pw >> 32) & 0xffffu; a.rW[2] = (128u << 20) + 3072u; a.rW[3] = 0x00020000u;
  a.myws = (char*)d_ws_all[rank]; a.out = (char*)d_out; a.rank = (unsigned)rank; a.pad[0] = a.pad[1] = a.pad[2] = 0;
  for (int i = 0; i < 8; ++i) a.p[i] = (char*)d_ws_all[(rank - i) & 7] - 2 * i;
  a2a_gemm<<<256, 256, LDS_BYTES, stream>>>(a);
}
```
